```python
import numpy as np

kernel_name = "gpu_gemm_ar_m4096_k4096_n4096_w8a8_dequant"


ABSMAX_RTOL = 0.001

COMPUTE_DTYPE = "int8"
EPILOGUE = "dequant"
IN_QUANT = "int8"
OUT_QUANT = None
IN_DTYPES = {"x": "int8", "w_mat": "int8",
             "scale_x": "float32", "scale_w": "float32"}
WORLD_SIZE = 8

def setup_inputs():

    x = np.random.randint(-127, 128, size=(4096, 4096)).astype(np.int8)
    w_mat = np.random.randint(-127, 128, size=(4096, 4096)).astype(np.int8)
    scale_x = (0.02 * (1.0 + np.random.rand(1))).astype(np.float32)
    scale_w = (0.02 * (1.0 + np.random.rand(1))).astype(np.float32)
    return {"x": x, "w_mat": w_mat, "scale_x": scale_x, "scale_w": scale_w}

def reference(x, w_mat, scale_x, scale_w):
    acc = x.astype(np.float64) @ w_mat.astype(np.float64)
    y = (acc.astype(np.float32) * (scale_x * scale_w)).astype(np.float32)
    return y
```

```cpp
#include <hip/hip_runtime.h>
#include <rccl/rccl.h>
#include <cstdio>
#include <cstdint>
#include <type_traits>

#define HIP_CHECK(expr) do { hipError_t _e = (expr); if (_e != hipSuccess) { \
    fprintf(stderr, "HIP error %s at %s:%d\n", hipGetErrorString(_e), __FILE__, __LINE__); \
    return; } } while (0)

typedef int v4i __attribute__((ext_vector_type(4)));
typedef unsigned v4u __attribute__((ext_vector_type(4)));
typedef unsigned v2u __attribute__((ext_vector_type(2)));
typedef float v4f __attribute__((ext_vector_type(4)));
#define LAS __attribute__((address_space(3)))
#define CAS __attribute__((address_space(4)))

#define NL_ROUNDS 0
#define NOCOMM 0
#define DMAF ""
#define NOLOAD 0
#define NOPOLL 0
#define POLL_W 24
#define POLL_N 4
#define POLL_LATE 360
#define POLL_SLEEP 16
#define POLL_SLEEP2 2
#define POLLF "sc0 sc1"
#define LOCAL_GAP 50
#define NGR 4
#define LAST_LEAD 48
#define LPL 4
#define STG 24
#define STG_LAST 8
#define SHRINK 12
#define PUSH0 1
#define EARLY_RUN 0
#define PUSH_WIN 2
#define LAGS 0x66532000u
#define EPN (14 * (NBS - 1))
#define LSTR 529

constexpr int NR = 8;
constexpr int DIM = 4096, KSH = 512;
constexpr unsigned PLANE_BLK = 128u << 10;
constexpr unsigned SRC_BYTES = 32 * PLANE_BLK;
constexpr unsigned BUF_BYTES = NR * SRC_BYTES;
constexpr unsigned PF_OFF = 2 * BUF_BYTES;
constexpr unsigned RDY_OFF = PF_OFF + 32 * 600 * 128;
constexpr unsigned CNT_OFF = RDY_OFF + NR * 64 * 128;
constexpr unsigned TMO_OFF = CNT_OFF + 256 * 128;
constexpr unsigned AGG_OFF = TMO_OFF + 512;
constexpr unsigned TK_OFF = TMO_OFF + 1024;
constexpr unsigned DUM_OFF = TMO_OFF + 4096;
constexpr unsigned DIAG_OFF = DUM_OFF + 256 * 256;
constexpr unsigned DIAG2_OFF = DIAG_OFF + 256 * 1024;
constexpr unsigned WS_USED = DIAG2_OFF + 256 * 2048;
constexpr unsigned long long DEADLINE = 30000000ull;

constexpr int NL = NL_ROUNDS, Q0 = 32 - NL;
static_assert(NL % 4 == 0 && NL >= 0 && NL <= 28, "late rounds");
constexpr int SF = 8 + 7 * Q0 / 4;
constexpr int NBS = 7 * NL / 4;
#define NH_SLOTS 9
#define NB_SLOTS 5
constexpr int NH = NH_SLOTS, NB = NB_SLOTS;
constexpr int KEEP = NBS < NB ? NBS : NB;
constexpr unsigned LDS_A = 0, LDS_B = NH * 8192, LDS_C = LDS_B + NB * 16384, LDS_T = LDS_C + 1024, LDS_BYTES = LDS_T + 4096 + 64;
constexpr unsigned NTK = 1024 - 32 * PUSH0;
static_assert(LDS_BYTES <= 163840, "LDS");
constexpr unsigned C_ABORT = 128, C_MVD = 132, C_AVA = 136, C_AVB = 140, C_RDY = 144, C_STG = 148, C_PSH = 152, C_TAB = 156;
static_assert(NL == 0 || NBS >= NB, "kept tiles must fill the ring");

struct Params {
  char* ws[NR];
  const char* x;
  const char* w;
  const float* sx;
  const float* sw;
  float* out;
  char* wsl;
  int rank;
};

__device__ __forceinline__ unsigned rfl(unsigned v) { return __builtin_amdgcn_readfirstlane(v); }
__device__ __forceinline__ v4i make_srd(const void* p, unsigned bytes) {
  unsigned long long a = (unsigned long long)p;
  v4i r;
  r.x = (int)rfl((unsigned)a); r.y = (int)(rfl((unsigned)(a >> 32)) & 0xffffu); r.z = (int)bytes; r.w = 0x00020000;
  return r;
}
__device__ __forceinline__ unsigned long long rt() { return __builtin_amdgcn_s_memrealtime(); }
template <int N> __device__ __forceinline__ void wait_vm() { asm volatile("s_waitcnt vmcnt(%0)" ::"i"(N) : "memory"); }
__device__ __forceinline__ unsigned lds_rd(unsigned addr) {
  unsigned v;
  asm volatile("ds_read_b32 %0, %1\n\ts_waitcnt lgkmcnt(0)" : "=v"(v) : "v"(addr) : "memory");
  return v;
}
__device__ __forceinline__ void lds_wr(unsigned addr, unsigned v) {
  asm volatile("ds_write_b32 %0, %1" ::"v"(addr), "v"(v) : "memory");
}
__device__ __forceinline__ unsigned flag_ld(v4i srd, unsigned voff) {
  unsigned v;
  asm volatile("s_nop 4\n\tbuffer_load_dword %0, %1, %2, 0 offen sc0 sc1\n\ts_waitcnt vmcnt(0)" : "=v"(v) : "v"(voff), "s"(srd) : "memory");
  return v;
}
__device__ __forceinline__ void flag_st(v4i srd, unsigned voff, unsigned soff, unsigned v) {
  asm volatile("s_nop 4\n\tbuffer_store_dword %0, %1, %2, %3 offen sc0 sc1" ::"v"(v), "v"(voff), "s"(srd), "s"(soff) : "memory");
}
__device__ __forceinline__ void flag_st1(v4i srd, unsigned soff, unsigned v) {
  asm volatile("s_mov_b64 exec, 1\n\ts_nop 4\n\tbuffer_store_dword %0, off, %1, %2 sc0 sc1\n\ts_mov_b64 exec, -1" ::"v"(v), "s"(srd), "s"(soff) : "memory");
}
__device__ __forceinline__ unsigned vm_peek() {
  unsigned r;
  asm volatile("s_getreg_b32 %0, hwreg(HW_REG_IB_STS)" : "=s"(r)::"memory");
  return (r & 15u) | ((r >> 18) & 0x30u);
}
__device__ __forceinline__ unsigned plane_id(unsigned sp, unsigned j, unsigned rank) {
  const unsigned t = 4 * sp + j - 32;
  const unsigned q = ((t & 255u) * 9363u) >> 16;
  const unsigned rem = ((rank + 1 + t - 7 * q) & 7u) * 32 + q;
  return sp < 8 ? rank * 32 + 4 * sp + j : rem;
}
__device__ __forceinline__ void half_step(unsigned hs, bool early, unsigned& sp, unsigned& h) {
  const unsigned e = hs - 2 * SF, c = e - NBS;
  const bool full = early || hs < 2 * SF;
  sp = full ? hs >> 1 : (e < NBS ? SF + e : (c < KEEP ? 64 - KEEP + c : SF + c - KEEP));
  h = full ? hs & 1 : (e < NBS ? 0 : 1);
}
__device__ __forceinline__ unsigned flag_idx(unsigned t4, unsigned pc) {
  const unsigned sp = t4 >> 2, j = t4 & 3, h = (pc >> 4) & 1;
  const unsigned hs = (sp < SF || (pc & 15) < 2) ? 2 * sp + h : (h == 0 ? SF + sp : (sp >= 64 - KEEP ? 2 * SF + NBS + sp - (64 - KEEP) : SF + NBS + KEEP + sp));
  return pc < 32 ? (pc & 15) * LSTR + 4 * hs + j : (pc >> 1) * LSTR + 2 * t4 + (pc & 1);
}

#define STAMP(idx) do { } while (0)

template <int I, int N, typename F> __device__ __forceinline__ void static_for(F&& f) {
  if constexpr (I < N) { f(std::integral_constant<int, I>{}); static_for<I + 1, N>(f); }
}
template <int G> __device__ __forceinline__ void mv_init(unsigned vdum) { asm volatile("v_mov_b32 v[48+%c0], %1" ::"i"(G), "v"(vdum)); }
template <int G> __device__ __forceinline__ unsigned mv_load(v4i sl, v4i sr, unsigned tk, unsigned o, unsigned v16, unsigned tab) {
  unsigned t;
  asm volatile("v_min_u32 v56, %c[ntk], v[40+%c[g]]\n v_lshl_add_u32 v56, v56, 2, %[tab]\n ds_read_b32 v56, v56\n"
               "s_mov_b64 exec, 1\n v_mov_b32 v[40+%c[g]], 1\n buffer_atomic_add v[40+%c[g]], off, %[sl], %[tk] sc0\n s_mov_b64 exec, -1\n"
               ".set k, 0\n .rept 4\n buffer_load_dwordx4 v[64+%c[g]*16+k*4:67+%c[g]*16+k*4], %[v16], %[sr], %[o] offen offset:k*1024 sc0 sc1\n .set k, k+1\n .endr\n"
               "s_waitcnt lgkmcnt(0)\n v_readfirstlane_b32 %[t], v56\n s_nop 1"
               : [t] "=s"(t) : [g] "i"(G), [sl] "s"(sl), [sr] "s"(sr), [tk] "s"(tk), [o] "s"(o), [v16] "v"(v16), [tab] "s"(tab), [ntk] "i"(NTK) : "memory");
  return t;
}
template <int G, int W> __device__ __forceinline__ void mv_store(v4i sl, unsigned o, unsigned v16, unsigned call, unsigned nf) {
  asm volatile(".set k, 0\n .rept 4\n s_waitcnt vmcnt(%c[w])\n buffer_store_dwordx4 v[64+%c[g]*16+k*4:67+%c[g]*16+k*4], %[v16], %[sl], %[o] offen offset:k*1024 sc1\n .set k, k+1\n .endr\n"
               "s_mov_b64 exec, 3\n buffer_store_dword %[vc], v[48+%c[g]], %[sl], 0 offen sc1\n s_mov_b64 exec, -1\n v_mov_b32 v[48+%c[g]], %[nf]"
               ::[g] "i"(G), [w] "i"(W), [sl] "s"(sl), [o] "s"(o), [v16] "v"(v16), [vc] "v"(call), [nf] "v"(nf) : "memory");
}
constexpr int ACC0 = 40;
template <int R> __device__ __forceinline__ void zero_acc() {
  asm volatile("v_mov_b32 v%c0, 0\n\tv_mov_b32 v%c1, 0\n\tv_mov_b32 v%c2, 0\n\tv_mov_b32 v%c3, 0" ::"i"(R), "i"(R + 1), "i"(R + 2), "i"(R + 3));
  if constexpr (R + 4 < ACC0 + 128) zero_acc<R + 4>();
}
template <int E> __device__ __forceinline__ void store_row(unsigned sc, unsigned vo, v4i ro) {
  constexpr int R = ACC0 + (E >> 2) * 16 + (E & 3);
  float f0, f1, f2, f3;
  asm volatile("v_cvt_f32_i32 %0, v%c4\n\tv_cvt_f32_i32 %1, v%c5\n\tv_cvt_f32_i32 %2, v%c6\n\tv_cvt_f32_i32 %3, v%c7\n\t"
               "v_mul_f32 %0, %8, %0\n\tv_mul_f32 %1, %8, %1\n\tv_mul_f32 %2, %8, %2\n\tv_mul_f32 %3, %8, %3"
               : "=v"(f0), "=v"(f1), "=v"(f2), "=v"(f3) : "i"(R), "i"(R + 4), "i"(R + 8), "i"(R + 12), "s"(sc));
  v4f v = {f0, f1, f2, f3};
  asm volatile("buffer_store_dwordx4 %0, %1, %2, %3 offen sc1\n\ts_nop 1" ::"v"(v), "v"(vo), "s"(ro),
               "s"((unsigned)(128 * (E >> 4) + 16 * ((E >> 2) & 3) + (E & 3)) * (DIM * 4)) : "memory");
}
template <int E0, int E1> __device__ __forceinline__ void store_rows(unsigned sc, unsigned vo, v4i ro) {
  if constexpr (E0 < E1) { store_row<E0>(sc, vo, ro); store_rows<E0 + 1, E1>(sc, vo, ro); }
}

#define MM_INSN "v_mfma_i32_16x16x64_i8 v[40+\\h*64+\\i*16+\\j*4:43+\\h*64+\\i*16+\\j*4], v[24+\\i*4:27+\\i*4], v[8+\\j*4:11+\\j*4], v[40+\\h*64+\\i*16+\\j*4:43+\\h*64+\\i*16+\\j*4]\n"
#define DIAG_T(r) ""
#define DIAG_W ""

__global__ void __launch_bounds__(768) __attribute__((amdgpu_num_vgpr(20))) fused_gemm(Params P) {
  extern __shared__ __attribute__((aligned(16))) char lds[];
  const int tid = threadIdx.x, lane = tid & 63;
  const int wave = (int)rfl((unsigned)tid >> 6);
  const int b = blockIdx.x;
  const int rank = P.rank;
  char* const CAS* const ktab = (char* const CAS*)__builtin_amdgcn_kernarg_segment_ptr();
  char* const wsl = P.wsl;
  unsigned cv = __hip_atomic_load((const unsigned*)(wsl + CNT_OFF + (unsigned)b * 128), __ATOMIC_RELAXED, __HIP_MEMORY_SCOPE_AGENT);
  const unsigned lds0 = (unsigned)(unsigned long long)(LAS char*)lds;
  const unsigned ctl = lds0 + LDS_C;
  const int xcd = b & 7, idx = b >> 3;
  const int ti = 2 * (idx >> 2) + (xcd >> 2), tj = 4 * (xcd & 3) + (idx & 3);
  const bool early = b < 32;
  const int sg = b & 31, so = b >> 5;
  const v4i srdL = make_srd(wsl, WS_USED);
  constexpr int NT = 7;
  const bool mover = (b & 31) < 28;
  const int mlane = b & 31, turn = 7 - (b >> 5);
  const int pi = mlane % 7, pp = (rank + 1 + pi) & 7;
  auto lds_inc = [&](unsigned a) { asm volatile("s_mov_b64 exec, 1\n\tds_add_u32 %0, %1\n\ts_mov_b64 exec, -1" ::"v"(a), "v"(1u) : "memory"); };
  constexpr int SUMD = (LAGS & 15) + (LAGS >> 4 & 15) + (LAGS >> 8 & 15) + (LAGS >> 12 & 15) + (LAGS >> 16 & 15) + (LAGS >> 20 & 15) + (LAGS >> 24 & 15) + (LAGS >> 28 & 15);
  constexpr int UEND = (int)NTK - 2 * SUMD - LAST_LEAD;
  auto turn_end = [&](int t) { return (unsigned)(((t + 1) * 200 + SHRINK * (t + 1) * (NT - 1 - t)) * UEND / (200 * NT) - (3 - mlane / 7) * (STG - (STG - STG_LAST) * t / (NT - 1))); };
  auto build_table = [&]() {
    const unsigned uend = turn < NT ? turn_end(turn) : 4096u;
#pragma unroll 1
    for (unsigned it = (unsigned)(tid - 512) + 32 * PUSH0; it < 1024; it += 256) {
      const int q = (int)(it >> 5), g = (int)(it & 31);
      const int key = q + (g < 16 ? (int)((LAGS >> (4 * (g & 7))) & 15u) : 0);
      int w = key - PUSH0; w = w > 32 - PUSH0 ? 32 - PUSH0 : w;
      int u = 16 * w + (g < 16 ? (g >> 3) : g - 16);
#pragma unroll
      for (int j = 0; j < 8; ++j) {
        int c = key - (int)((LAGS >> (4 * j)) & 15u) - PUSH0 + ((g >= 16 || j < (g & 7)) ? 1 : 0);
        c = c < 0 ? 0 : (c > 32 - PUSH0 ? 32 - PUSH0 : c);
        u += 2 * c;
      }
      const unsigned id = 2 * it;
      lds_wr(lds0 + LDS_T + (unsigned)u * 4, id | ((unsigned)u >= uend ? 0x4000u : 0u) | ((id & 32u) << 10) | (flag_idx(32 + 7 * (unsigned)q + (unsigned)pi, id & 63) << 16));
    }
    asm volatile("s_waitcnt lgkmcnt(0)" ::: "memory");
    lds_inc(ctl + C_TAB);
  };
  if (wave == 0) STAMP(0);
  unsigned call = 0;
  auto get_call = [&]() { unsigned c; asm volatile("v_add_u32 %0, 1, %1" : "=v"(c) : "v"(cv) : "memory"); call = rfl(c); };
  auto wg_barrier = [&]() { asm volatile("s_waitcnt lgkmcnt(0)\n\ts_barrier" ::: "memory"); };
  auto lds_wait = [&](unsigned a, unsigned n) { unsigned k = 0; while (rfl(lds_rd(a)) < n && ++k < (1u << 24)) { } };
  auto wait_flag = [&](unsigned off) {
    const unsigned long long t0 = rt();
#pragma unroll 1
    for (unsigned n = 1;; ++n) {
      const unsigned v = flag_ld(srdL, off);
      if (__all((int)(v - call) >= 0)) return true;
      if ((n & 15u) == 0u) {
        const unsigned tm = rfl(flag_ld(srdL, TMO_OFF));
        if (tm == call || rt() - t0 > DEADLINE) { flag_st(srdL, 0u, TMO_OFF, call); return false; }
      }
      __builtin_amdgcn_s_sleep(1);
    }
  };

  if (wave < 8) {
    {
      __amdgpu_buffer_rsrc_t rs = __builtin_amdgcn_make_buffer_rsrc(wsl, 0, 2 * BUF_BYTES, 0x00020000);
      const unsigned base = (unsigned)rank * SRC_BYTES + (unsigned)(4 * so) * PLANE_BLK;
      const bool push = PUSH0 && !NOCOMM && b < 32;
      char* pk[7];
      auto load_pk = [&]() {
        asm volatile("" ::: "memory");
#pragma unroll
        for (int k = 0; k < 7; ++k) { pk[k] = ktab[(rank + 1 + (k + wave + b) % 7) & 7]; asm volatile("" : "+s"(pk[k])); }
      };
      if (sg < 16) {
        const int r = 32 * wave + (lane & 31), j = lane >> 5;
        const char* src = P.x + (size_t)(256 * sg + r) * KSH + 64 * so + 16 * j;
        v4u v0 = *(const v4u*)src, v1 = *(const v4u*)(src + 32);
        load_pk();
        wg_barrier();
        unsigned ncv;
        asm volatile("v_not_b32 %0, %1" : "=v"(ncv) : "v"(cv) : "memory");
        const unsigned d = base + (ncv & 1u) * BUF_BYTES + (unsigned)j * PLANE_BLK + (unsigned)((r >> 7) * 16 + sg) * 2048 + (unsigned)(r & 127) * 16;
        __builtin_amdgcn_raw_buffer_store_b128(v0, rs, d, 0, 17);
        __builtin_amdgcn_raw_buffer_store_b128(v1, rs, d + 2 * PLANE_BLK, 0, 17);
        if (push) {
#pragma unroll
          for (int k = 0; k < 7; ++k) {
            const v4i sp = make_srd(pk[k], 2 * BUF_BYTES);
            asm volatile("s_mov_b64 exec, 0xffffffff\n\ts_nop 4\n\tbuffer_store_dwordx4 %0, %1, %2, 0 offen sc0 sc1\n\ts_mov_b64 exec, -1" ::"v"(v0), "v"(d), "s"(sp) : "memory");
            if (k == PUSH_WIN - 1) { wait_vm<PUSH_WIN>(); lds_inc(ctl + C_STG); }
            if (k >= PUSH_WIN - 1) wait_vm<PUSH_WIN - 1>();
          }
          wait_vm<0>(); lds_inc(ctl + C_PSH);
        } else { wait_vm<0>(); lds_inc(ctl + C_STG); }
      } else if (wave < 4) {
        const int gw = sg - 16;
        const char* src = P.w + (size_t)(64 * so + 16 * wave) * DIM + 256 * gw + 4 * lane;
        unsigned r[16];
#pragma unroll
        for (int k = 0; k < 16; ++k) r[k] = *(const unsigned*)(src + (size_t)k * DIM);
        load_pk();
        wg_barrier();
        unsigned ncv;
        asm volatile("v_not_b32 %0, %1" : "=v"(ncv) : "v"(cv) : "memory");
        v4u o[4];
#pragma unroll
        for (int g = 0; g < 4; ++g) {
          unsigned t0 = __builtin_amdgcn_perm(r[4 * g + 1], r[4 * g], 0x05010400u), t1 = __builtin_amdgcn_perm(r[4 * g + 1], r[4 * g], 0x07030602u);
          unsigned t2 = __builtin_amdgcn_perm(r[4 * g + 3], r[4 * g + 2], 0x05010400u), t3 = __builtin_amdgcn_perm(r[4 * g + 3], r[4 * g + 2], 0x07030602u);
          o[0][g] = __builtin_amdgcn_perm(t2, t0, 0x05040100u); o[1][g] = __builtin_amdgcn_perm(t2, t0, 0x07060302u);
          o[2][g] = __builtin_amdgcn_perm(t3, t1, 0x05040100u); o[3][g] = __builtin_amdgcn_perm(t3, t1, 0x07060302u);
        }
        const unsigned d = base + (ncv & 1u) * BUF_BYTES + (unsigned)wave * PLANE_BLK + 65536u + (unsigned)gw * 4096 + (unsigned)(64 * (lane >> 4) + (lane & 15)) * 16;
#pragma unroll
        for (int jj = 0; jj < 4; ++jj) __builtin_amdgcn_raw_buffer_store_b128(o[jj], rs, d + jj * 256, 0, 17);
        if (push && wave == 0) {
          __builtin_amdgcn_s_sleep(2);
#pragma unroll
          for (int k = 0; k < 7; ++k) {
            const v4i sp = make_srd(pk[k], 2 * BUF_BYTES);
            asm volatile("s_nop 4\n\tbuffer_store_dwordx4 %0, %4, %5, 0 offen sc0 sc1\n\tbuffer_store_dwordx4 %1, %4, %5, 0 offen offset:256 sc0 sc1\n\t"
                         "buffer_store_dwordx4 %2, %4, %5, 0 offen offset:512 sc0 sc1\n\tbuffer_store_dwordx4 %3, %4, %5, 0 offen offset:768 sc0 sc1"
                         ::"v"(o[0]), "v"(o[1]), "v"(o[2]), "v"(o[3]), "v"(d), "s"(sp) : "memory");
            if (k == PUSH_WIN - 1) { wait_vm<4 * PUSH_WIN>(); lds_inc(ctl + C_STG); }
            if (k >= PUSH_WIN - 1) wait_vm<4 * (PUSH_WIN - 1)>();
          }
          wait_vm<0>(); lds_inc(ctl + C_PSH);
        } else { wait_vm<0>(); lds_inc(ctl + C_STG); }
      } else wg_barrier();
    }
    if (wave == 0) STAMP(1);
    asm volatile("" ::: "v167");
    zero_acc<ACC0>();
    const int wr = wave >> 2, wc = wave & 3;
    unsigned pa = lds0 + LDS_A + (unsigned)(lane >> 4) * 2048 + (unsigned)(64 * wr + (lane & 15)) * 16;
    unsigned pb = lds0 + LDS_B + (unsigned)(lane >> 4) * 4096 + (unsigned)(64 * wc + (lane & 15)) * 16;
    const unsigned vw = ctl + (unsigned)wave * 16 + (unsigned)(lane & 1) * 4;
    const unsigned vo = ((unsigned)(256 * ti + 64 * wr) + 4 * (unsigned)(lane >> 4)) * (DIM * 4) + ((unsigned)(256 * tj + 64 * wc) + 4 * (unsigned)(lane & 15)) * 4;
    unsigned sc = rfl(__builtin_bit_cast(unsigned, (*P.sx) * (*P.sw)));
    const v4i ro = make_srd(P.out, DIM * DIM * 4);
    unsigned vcnt = 0, vf, ia = 0, ib = 0, cnt, da, db, spin = 1u << 23;
    unsigned long long ra, tB = 0, tC = 0;
    asm volatile(
      ".set epc, 0\n"
      ".macro EP1\n"
      " .if epc < 184\n"
      "  .set eb, 40 + (epc / 46) * 16\n"
      "  .set ek, epc %% 46\n"
      "  .if ek == 0\n v_swap_b32 v[eb+1], v[eb+4]\n .endif\n"
      "  .if ek == 1\n v_swap_b32 v[eb+2], v[eb+8]\n .endif\n"
      "  .if ek == 2\n v_swap_b32 v[eb+3], v[eb+12]\n .endif\n"
      "  .if ek == 3\n v_swap_b32 v[eb+6], v[eb+9]\n .endif\n"
      "  .if ek == 4\n v_swap_b32 v[eb+7], v[eb+13]\n .endif\n"
      "  .if ek == 5\n v_swap_b32 v[eb+11], v[eb+14]\n .endif\n"
      "  .if ek >= 6\n"
      "   .set er, (ek - 6) / 10\n"
      "   .set em, (ek - 6) %% 10\n"
      "   .if em < 4\n v_cvt_f32_i32 v[eb+er*4+em], v[eb+er*4+em]\n .endif\n"
      "   .if (em >= 4) && (em < 8)\n v_mul_f32 v[eb+er*4+em-4], %[sc], v[eb+er*4+em-4]\n .endif\n"
      "   .if em == 8\n s_mov_b32 %[cnt], ((epc / 46) * 16 + er) * 16384\n .endif\n"
      "   .if em == 9\n buffer_store_dwordx4 v[eb+er*4:eb+er*4+3], %[vo], %[ro], %[cnt] offen sc1\n .endif\n"
      "  .endif\n"
      "  .set epc, epc + 1\n"
      " .endif\n"
      ".endm\n"
      ".set eps, 0\n"
      ".macro MM h, i, j, ep\n"
      MM_INSN
      " .if \\ep\n"
      "  .set eps, eps + 1\n"
      "  .rept 3\n .if epc * \\ep < eps * 184\n EP1\n .endif\n .endr\n"
      " .endif\n"
      ".endm\n"
      ".macro STEP h, nb, pm, ep, last\n"
      " .if \\last == 0\n ds_read_b32 %[vf], %[vw]\n .endif\n"
      " s_waitcnt lgkmcnt(3+3*\\pm-\\last)\n"
      " MM \\h, 0, 0, \\ep\n"
      " s_cmp_eq_u32 %[ia], %c[nh1]\n"
      " s_cselect_b32 %[da], %[wa], 0x2000\n"
      " MM \\h, 1, 0, \\ep\n"
      " s_cselect_b32 %[ia], -1, %[ia]\n"
      " s_add_u32 %[ia], %[ia], 1\n"
      " .if \\pm\n s_waitcnt lgkmcnt(5-\\last)\n .endif\n"
      " MM \\h, 0, 1, \\ep\n"
      " v_add_u32 %[pa], %[da], %[pa]\n"
      " MM \\h, 1, 1, \\ep\n"
      " .if \\nb\n s_cmp_eq_u32 %[ib], %c[nb1]\n s_cselect_b32 %[db], %[wb], 0x4000\n .endif\n"
      " .if \\pm\n s_waitcnt lgkmcnt(4-\\last)\n .endif\n"
      " MM \\h, 0, 2, \\ep\n"
      " .if \\nb\n s_cselect_b32 %[ib], -1, %[ib]\n s_add_u32 %[ib], %[ib], 1\n .endif\n"
      " MM \\h, 1, 2, \\ep\n"
      " .if \\nb\n v_add_u32 %[pb], %[db], %[pb]\n .endif\n"
      " .if \\pm\n s_waitcnt lgkmcnt(3-\\last)\n .endif\n"
      " MM \\h, 0, 3, \\ep\n"
      " v_add_u32 %[vcnt], 1, %[vcnt]\n"
      " MM \\h, 1, 3, \\ep\n"
      " s_waitcnt lgkmcnt(0)\n"
      " MM \\h, 2, 0, 0\n"
      " .if \\last == 0\n"
      " ds_write_b32 %[vw], %[vcnt] offset:8\n"
      " v_cmp_le_u32 vcc, %[vf], %[vcnt]\n"
      " .endif\n"
      " MM \\h, 3, 0, 0\n"
      " .if \\last == 0\n"
      " s_getpc_b64 %[ra]\n"
      " s_cbranch_vccnz Lslow%=\n"
      " ds_read_b128 v[24:27], %[pa]\n"
      " ds_read_b128 v[28:31], %[pa] offset:256\n"
      " .if \\nb\n ds_read_b128 v[8:11], %[pb]\n .endif\n"
      " .endif\n"
      " MM \\h, 2, 1, \\ep\n"
      " MM \\h, 3, 1, \\ep\n"
      " .if \\nb\n ds_read_b128 v[12:15], %[pb] offset:256\n .endif\n"
      " MM \\h, 2, 2, \\ep\n"
      " MM \\h, 3, 2, \\ep\n"
      " .if \\nb\n ds_read_b128 v[16:19], %[pb] offset:512\n .endif\n"
      " MM \\h, 2, 3, \\ep\n"
      " MM \\h, 3, 3, \\ep\n"
      " .if \\nb\n ds_read_b128 v[20:23], %[pb] offset:768\n .endif\n"
      " .if \\last == 0\n"
      " ds_read_b128 v[32:35], %[pa] offset:512\n"
      " ds_read_b128 v[36:39], %[pa] offset:768\n"
      " .endif\n"
      ".endm\n"
      " s_mov_b64 vcc, -1\n"
      " s_getpc_b64 %[ra]\n"
      " s_cbranch_vccnz Lslow%=\n"
      " ds_read_b128 v[24:27], %[pa]\n ds_read_b128 v[28:31], %[pa] offset:256\n"
      " ds_read_b128 v[8:11], %[pb]\n ds_read_b128 v[12:15], %[pb] offset:256\n"
      " ds_read_b128 v[16:19], %[pb] offset:512\n ds_read_b128 v[20:23], %[pb] offset:768\n"
      " ds_read_b128 v[32:35], %[pa] offset:512\n ds_read_b128 v[36:39], %[pa] offset:768\n"
      " s_mov_b32 %[cnt], %[n1]\n"
      "Lfull%=:\n"
      " STEP 0, 0, 1, 0, 0\n"
      " STEP 1, 1, 0, 0, 0\n"
      " s_sub_u32 %[cnt], %[cnt], 1\n"
      " s_cmp_lg_u32 %[cnt], 0\n"
      " s_cbranch_scc1 Lfull%=\n"
      " .if %c[nbs] > 0\n"
      " s_cmp_eq_u32 %[n1], 63\n"
      " s_cbranch_scc1 Ltail%=\n"
      DIAG_T("%[tB]") DIAG_W
      " s_mov_b32 %[cnt], %c[nbs]\n"
      "Lhalf%=:\n"
      " STEP 0, 1, 1, 0, 0\n"
      " s_sub_u32 %[cnt], %[cnt], 1\n"
      " s_cmp_lg_u32 %[cnt], 0\n"
      " s_cbranch_scc1 Lhalf%=\n"
      DIAG_T("%[tC]") DIAG_W
      " .rept %c[nbs] - 1\n"
      " STEP 1, 1, 1, %c[epn], 0\n"
      " .endr\n"
      " STEP 1, 0, 1, %c[epn], 1\n"
      " .rept 184\n EP1\n .endr\n"
      " s_branch Lend%=\n"
      " .endif\n"
      "Ltail%=:\n"
      " STEP 0, 0, 1, 0, 0\n"
      " STEP 1, 0, 0, 0, 1\n"
      " s_branch Lend%=\n"
      "Lslow%=:\n"
      " ds_read_b32 %[vf], %[vw]\n"
      " s_sub_u32 %[spin], %[spin], 1\n"
      " s_waitcnt lgkmcnt(0)\n"
      " v_cmp_le_u32 vcc, %[vf], %[vcnt]\n"
      " s_cbranch_scc1 Lgive%=\n"
      " s_cbranch_vccz Lret%=\n"
      " s_sleep 1\n"
      " s_branch Lslow%=\n"
      "Lgive%=:\n"
      " s_mov_b32 %[spin], 0\n"
      " s_mov_b32 %[sc], 0x7fc00000\n"
      " s_mov_b64 vcc, 0\n"
      "Lret%=:\n"
      " s_setpc_b64 %[ra]\n"
      "Lend%=:\n"
      " s_nop 7\n s_nop 1\n"
      ".purgem STEP\n.purgem MM\n.purgem EP1\n"
      : [pa] "+&v"(pa), [pb] "+&v"(pb), [vcnt] "+&v"(vcnt), [vf] "=&v"(vf), [ia] "+&s"(ia), [ib] "+&s"(ib), [cnt] "=&s"(cnt), [da] "=&s"(da),
        [db] "=&s"(db), [spin] "+&s"(spin), [sc] "+&s"(sc), [ra] "=&s"(ra), [tB] "+&s"(tB), [tC] "+&s"(tC)
      : [vw] "v"(vw), [vo] "v"(vo), [ro] "s"(ro), [nh1] "i"(NH - 1), [nb1] "i"(NB - 1), [wa] "s"(-(NH - 1) * 8192), [wb] "s"(-(NB - 1) * 16384),
        [n1] "s"((NL > 0 && !early) ? SF : 63), [nbs] "i"(NBS), [epn] "i"(EPN)
      : "v8", "v9", "v10", "v11", "v12", "v13", "v14", "v15", "v16", "v17", "v18", "v19", "v20", "v21", "v22", "v23", "v24", "v25", "v26", "v27",
        "v28", "v29", "v30", "v31", "v32", "v33", "v34", "v35", "v36", "v37", "v38", "v39", "vcc", "scc", "memory");
    if (wave == 0) STAMP(3);
    if (early) {
      unsigned spins = 0;
      while (rfl(lds_rd(ctl + C_MVD)) != 2u && ++spins < (1u << 22)) __builtin_amdgcn_s_sleep(2);
    }
    sc = rfl(lds_rd(ctl + C_ABORT)) != 0u ? 0x7fc00000u : sc;
    if (NL == 0 || early) store_rows<0, 16>(sc, vo, ro);
    store_rows<16, 32>(sc, vo, ro);
  } else if (wave == 8) {
    wg_barrier();
    lds_wr(lds0 + LDS_T + (1024 - 32 * PUSH0 + (lane & 15)) * 4, 0xffffffffu);
    build_table();
    if (mover && !NOCOMM) {
      __builtin_amdgcn_s_setprio(3);
      const v4i srdR = make_srd(ktab[pp], 2 * BUF_BYTES);
      const unsigned v16 = (unsigned)lane * 16;
      get_call();
      const unsigned sbase = (call & 1u) * BUF_BYTES + (unsigned)pp * SRC_BYTES;
      unsigned tk_off = TK_OFF + ((call & 1u) * 8 + (unsigned)pi) * 128;
      const unsigned tk_dead = DUM_OFF + (unsigned)b * 256 + 64;
      const int succ = b >= 32 ? b - 32 : b;
      const unsigned go_out = DUM_OFF + (unsigned)succ * 256 + 192, go_in = DUM_OFF + (unsigned)b * 256 + 192;
#define MV_FIRST() asm volatile("s_mov_b64 exec, 1\n\tv_mov_b32 v47, %c2\n\ts_nop 4\n\tbuffer_atomic_add v47, off, %0, %1 sc0\n\ts_mov_b64 exec, -1" ::"s"(srdL), "s"(tk_off), "i"(NGR + 1) : "memory")
      flag_st1(srdL, tk_dead, 0x1000u);
      if (turn == 0) { lds_wr(ctl + C_MVD, 1u); MV_FIRST(); }
      lds_wait(ctl + C_TAB, 4);
      bool ok;
      if (turn == 0) {
        ok = wait_flag(RDY_OFF + (unsigned)(pp * 64 + (lane & 31)) * 128);
      } else {
        const unsigned long long t0 = rt();
        const unsigned est = turn_end(turn - 1) * 66u / 10u;
        while ((unsigned)(rt() - t0) < est) __builtin_amdgcn_s_sleep(16);
        ok = wait_flag((lane & 1) ? RDY_OFF + (unsigned)(pp * 64 + 32) * 128 : go_in);
        lds_wr(ctl + C_MVD, 1u);
        if (ok) { MV_FIRST(); wait_vm<0>(); }
      }
      if (ok) {
        unsigned x0;
        asm volatile("v_readfirstlane_b32 %0, v47\n\ts_nop 1" : "=s"(x0)::"memory");
        const unsigned xe = x0 + (unsigned)(lane & 7);
        const unsigned ev = lds_rd(lds0 + LDS_T + (xe < NTK ? xe : NTK) * 4);
        const unsigned l1 = (unsigned)lane & 1u, vdum = DUM_OFF + (unsigned)b * 256 + l1 * 128;
        unsigned os[NGR], en[NGR];
        unsigned e;
        bool rdy = turn != 0;
        auto foff = [&](unsigned x) { return x == 0xffffffffu ? vdum : PF_OFF + ((x >> 16) + l1 * ((x & 0x8000u) ? 1u : (unsigned)LSTR)) * 128; };
        auto load = [&](auto G, unsigned x) {
          constexpr int g = decltype(G)::value;
          en[g] = x; os[g] = x == 0xffffffffu ? sbase : sbase + (x & 2047u) * 2048;
          if (__builtin_expect((x & 0x4000u) != 0u && tk_off != tk_dead, 0)) {
            tk_off = tk_dead;
            flag_st1(srdL, go_out, call);
          }
          return mv_load<g>(srdL, srdR, tk_off, os[g], v16, lds0 + LDS_T);
        };
        auto store = [&](auto G, auto W) { mv_store<decltype(G)::value, decltype(W)::value>(srdL, os[decltype(G)::value], v16, call, foff(en[decltype(G)::value])); };
        auto init = [&](auto G) {
          constexpr int g = decltype(G)::value;
          mv_init<g>(vdum);
          load(G, __builtin_amdgcn_readlane(ev, g));
        };
        static_for<0, NGR>(init);
        e = __builtin_amdgcn_readlane(ev, NGR);
        static_for<0, NGR>([&](auto G) {
          constexpr int g = decltype(G)::value;
          store(G, std::integral_constant<int, 3 + (NGR - 1 - g) * 5 + g * 10>{});
          e = load(G, e);
        });
        int stop = -1;
#pragma unroll 1
        while (stop < 0) {
          static_for<0, NGR>([&](auto G) {
            constexpr int g = decltype(G)::value;
            if (stop < 0) {
              store(G, std::integral_constant<int, 3 + (NGR - 1) * 10>{});
              if (__builtin_expect(!rdy && (e & 2047u) >= 256, 0)) {
                rdy = true;
                unsigned r, spins = 0;
                while ((r = rfl(lds_rd(ctl + C_RDY))) == 0u && ++spins < (1u << 23)) __builtin_amdgcn_s_sleep(2);
                if (r != 1u) e = 0xffffffffu;
              }
              if (e == 0xffffffffu) stop = g;
              else {
                e = load(G, e);
              }
            }
          });
        }
        if (!early) lds_wr(ctl + C_MVD, 2u);
        static_for<0, NGR>([&](auto G) {
          constexpr int g = decltype(G)::value;
          if (stop == g) static_for<1, NGR>([&](auto J) {
            constexpr int j = decltype(J)::value;
            store(std::integral_constant<int, (g + j) % NGR>{}, std::integral_constant<int, 3 + (NGR - 1 - j) * 10 + j * 5>{});
          });
        });
        asm volatile("s_waitcnt vmcnt(0)\n s_mov_b64 exec, 3\n .set k, 0\n .rept %c2\n buffer_store_dword %0, v[48+k], %1, 0 offen sc1\n .set k, k+1\n .endr\n s_mov_b64 exec, -1"
                     ::"v"(call), "s"(srdL), "i"(NGR) : "memory");
        lds_wr(ctl + C_MVD, 2u);
      }
      lds_wr(ctl + C_MVD, 2u);
    } else lds_wr(ctl + C_MVD, 2u);
  } else if (wave == 9) {
    __builtin_amdgcn_s_setprio(2);
    lds_wr(ctl + lane * 4, 0u);
    wg_barrier();
    if (b >= 32) build_table();
    get_call();
    {
      char* pb = (lane < 8) ? ktab[lane] : wsl;
      const int j = (lane - 8) >> 1, h = lane & 1;
      const int pc = sg < 16 ? h * 16 + sg : 32 + 2 * (sg - 16) + h;
      const unsigned off = lane < 8 ? RDY_OFF + (unsigned)(rank * 64 + (b < 32 ? b : 32)) * 128
                         : lane < 16 ? PF_OFF + flag_idx((unsigned)(4 * so + j), (unsigned)pc) * 128
                         : lane == 16 ? CNT_OFF + (unsigned)b * 128 : TK_OFF + ((~call & 1u) * 8 + (unsigned)(lane - 17)) * 128;
      unsigned* dst = (unsigned*)(pb + off);
      const unsigned val = lane <= 16 ? call : 0u;
      unsigned* dst2 = (unsigned*)(ktab[(lane >> 1) & 7] + PF_OFF + flag_idx(32u + ((unsigned)(rank - (lane >> 1) - 1) & 7u), (unsigned)pc) * 128);
      asm volatile("" : "+v"(dst), "+v"(dst2));
      lds_wait(ctl + C_STG, sg < 16 ? 8 : 4);
      if (lane != rank && lane >= (b < 32 ? 0 : 8) && lane <= (b == 255 ? 23 : 16)) __hip_atomic_store(dst, val, __ATOMIC_RELAXED, __HIP_MEMORY_SCOPE_SYSTEM);
      if (b < 32) {
        build_table();
        if (PUSH0 && !NOCOMM) {
          lds_wait(ctl + C_PSH, sg < 16 ? 8 : 1);
          if (lane < 16 && (lane >> 1) != rank) __hip_atomic_store(dst2, call, __ATOMIC_RELAXED, __HIP_MEMORY_SCOPE_SYSTEM);
        }
      }
      if (b >= 32) {
        unsigned tk = 0;
        if (lane == 0) tk = __hip_atomic_fetch_add((unsigned*)(wsl + AGG_OFF), 1u, __ATOMIC_RELAXED, __HIP_MEMORY_SCOPE_AGENT);
        if (rfl(tk) + 1u == call * 224u && lane < 8 && lane != rank) __hip_atomic_store(dst, val, __ATOMIC_RELAXED, __HIP_MEMORY_SCOPE_SYSTEM);
      }
    }
    if (turn == 0 && !NOCOMM) lds_wr(ctl + C_RDY, wait_flag(RDY_OFF + (unsigned)(pp * 64 + 32) * 128) ? 1u : 2u);
    const bool isB = lane >= 32;
    const unsigned lo = (unsigned)lane & 31u;
    const unsigned vbase = PF_OFF + ((isB ? 16u + tj : (unsigned)ti) * LSTR + lo) * 128;
    const unsigned vav = lane < 2 ? ctl + C_AVA + lane * 4 : ctl + 256 + lane * 4;
    unsigned npoll = 0, headA = 0, headB = 0;
    bool mvdone = false;
    unsigned wA = 31, wB = 31;
    const unsigned long long t0 = rt();
#pragma unroll 1
    while (headA < 512 || headB < 512) {
      if (!mvdone) { const unsigned m = rfl(lds_rd(ctl + C_MVD)); mvdone = m == 2u; if (m == 1u && !(early && EARLY_RUN)) { __builtin_amdgcn_s_sleep(4); continue; } }
      const unsigned head = isB ? headB : headA;
      unsigned v = call - 1;
      if (lo < (isB ? wB : wA) && head + lo < 512) {
        if (NOCOMM | NOPOLL) v = call;
        else asm volatile("buffer_load_dword %0, %1, %2, 0 offen " POLLF "\n\ts_waitcnt vmcnt(0)" : "=v"(v) : "v"(vbase + head * 128), "s"(srdL) : "memory");
      }
      const unsigned long long hm = __ballot((int)(v - call) >= 0);
      const unsigned da = (unsigned)__builtin_ctz(~(unsigned)hm | 0x80000000u), db = (unsigned)__builtin_ctz(~(unsigned)(hm >> 32) | 0x80000000u);
      headA += da; headB += db;
      {
        const unsigned lim = (headA < 320 || NOPOLL) ? (unsigned)(rt() - t0) / LOCAL_GAP + 1 : 255u;
        if (NOPOLL && lim < 130) { headA = headA > 500 ? 500 : headA; __builtin_amdgcn_s_sleep(8); }
        const unsigned ha = headA >> 2, hb = headB >> 3;
        lds_wr(vav, lane ? (hb < (lim >> 1) + 1 ? hb : (lim >> 1) + 1) : (ha < lim ? ha : lim)); }
      const bool more = da == wA || db == wB;
      wA = da == wA ? POLL_W : POLL_N; wB = db == wB ? POLL_W : POLL_N;
      if (!more) { if (headA < POLL_LATE) __builtin_amdgcn_s_sleep(POLL_SLEEP); else __builtin_amdgcn_s_sleep(POLL_SLEEP2); }
      if ((da | db) == 0u && (++npoll & 15u) == 0u) {
        const unsigned tm = rfl(flag_ld(srdL, TMO_OFF));
        if (tm == call || rt() - t0 > DEADLINE) {
          flag_st(srdL, 0u, TMO_OFF, call);
          lds_wr(ctl + C_ABORT, 1u);
          asm volatile("s_waitcnt lgkmcnt(0)" ::: "memory");
          lds_wr(vav, 128u);
          break;
        }
      }
    }
    lds_wr(vav, 128u);
    STAMP(19);
  } else {
    wg_barrier();
    build_table();
    __builtin_amdgcn_s_setprio(3);
    get_call();
    const unsigned par_off = (call & 1u) * BUF_BYTES;
    auto loader = [&](auto IA) {
      constexpr bool isA = decltype(IA)::value;
      const unsigned sf = early ? 64u : (unsigned)SF, keepv = early ? 0u : (unsigned)KEEP;
      const unsigned NS = isA ? 128u : (early ? 64u : (unsigned)(64 + NBS - KEEP));
      constexpr unsigned STEPB = isA ? 8192 : 16384, RINGB = isA ? NH * 8192 : NB * 16384;
      const unsigned v16 = (unsigned)lane * 16;
      const unsigned vchk = lane < 8 ? ctl + lane * 16 + 8 : ctl + (isA ? C_AVA : C_AVB);
      const unsigned vfull = lane < 8 ? ctl + (unsigned)lane * 16 + (isA ? 0 : 4) : ctl + (isA ? 512 : 768) + lane * 4;
      const unsigned vsh = lane < 8 ? 0u : 8u;
      const unsigned ring0 = lds0 + (isA ? LDS_A : LDS_B);
      unsigned ring = ring0, n = 0, nops = 0, spin = 1u << 24;
      const v4i srd = srdL;
      unsigned thr[2], t0[2], t1[2], t2[2], t3[2];
#pragma unroll
      for (int half = 0; half < 2; ++half) {
        const unsigned st = (unsigned)(64 * half + lane);
        unsigned sp, h, po; int nd;
        if constexpr (isA) {
          half_step(st, early, sp, h);
          po = (16 * h + ti) * 2048;
          nd = (int)st + 1 - NH;
        } else {
          sp = st < 64 ? st : st - NBS;
          po = 65536u + (unsigned)tj * 4096;
          const int pv = (int)(st < 64 ? st : st + KEEP) - NB;
          nd = (pv <= (int)sf ? 2 * pv : (int)sf + pv) + 1;
        }
        nd = nd < 0 ? 0 : nd;
        thr[half] = (unsigned)nd | ((isA ? st + 1 : sp + 1) << 8);
        t0[half] = par_off + po + plane_id(sp & 63, 0, rank) * PLANE_BLK; t1[half] = par_off + po + plane_id(sp & 63, 1, rank) * PLANE_BLK;
        t2[half] = par_off + po + plane_id(sp & 63, 2, rank) * PLANE_BLK; t3[half] = par_off + po + plane_id(sp & 63, 3, rank) * PLANE_BLK;
        asm volatile("" : "+v"(thr[half]), "+v"(t0[half]), "+v"(t1[half]), "+v"(t2[half]), "+v"(t3[half]));
      }
      wait_vm<0>();
#define L_PUB \
      " s_getreg_b32 %[c], hwreg(HW_REG_IB_STS)\n" \
      " s_bfe_u32 %[x], %[c], 0x20016\n" \
      " s_and_b32 %[c], %[c], 15\n" \
      " s_lshl4_add_u32 %[c], %[x], %[c]\n" \
      " s_sub_u32 %[x], %[nops], %[c]\n" \
      " s_lshr_b32 %[x], %[x], 4-%c[isa]\n" \
      " .if %c[isa] == 0\n" \
      " s_cmp_ge_u32 %[x], 64\n" \
      " s_cselect_b32 %[y], %[sfk], %[sf]\n" \
      " s_min_u32 %[y], %[y], %[x]\n" \
      " s_add_u32 %[x], %[x], %[y]\n" \
      " .endif\n" \
      " v_mov_b32 %[vl], %[x]\n" \
      " ds_write_b32 %[vfull], %[vl]\n"
#define L_DMA(o, t) \
      " buffer_load_dwordx4 %[v16], %[srd], %[" o "] offen " DMAF " lds\n" \
      " buffer_load_dwordx4 %[v16], %[srd], %[" o "] offen offset:1024 " DMAF " lds\n" \
      " .if %c[isa] == 0\n" \
      " buffer_load_dwordx4 %[v16], %[srd], %[" o "] offen offset:2048 " DMAF " lds\n" \
      " buffer_load_dwordx4 %[v16], %[srd], %[" o "] offen offset:3072 " DMAF " lds\n" \
      " .endif\n" \
      " s_add_u32 m0, m0, 0x1000-0x800*%c[isa]\n" \
      " v_readlane_b32 %[" o "], %[" t "], %[n]\n"
      auto run = [&](unsigned thr_, unsigned a0, unsigned a1, unsigned a2, unsigned a3, unsigned end) {
        unsigned v, vneed, vl, o0, o1, o2, o3, c, x, y, keep;
        asm volatile(
          " s_mov_b32 %[km0], m0\n"
          " v_readlane_b32 %[x], %[thr], %[n]\n"
          " v_readlane_b32 %[o0], %[t0], %[n]\n v_readlane_b32 %[o1], %[t1], %[n]\n"
          " v_readlane_b32 %[o2], %[t2], %[n]\n v_readlane_b32 %[o3], %[t3], %[n]\n"
          " v_bfe_u32 %[vneed], %[x], %[vsh], 8\n"
          "Lpoll%=:\n"
          " ds_read_b32 %[v], %[vchk]\n"
          L_PUB
          " s_sub_u32 %[spin], %[spin], 1\n"
          " s_cbranch_scc1 Lout%=\n"
          " s_waitcnt lgkmcnt(1)\n"
          "Lchk%=:\n"
          " v_cmp_lt_u32 vcc, %[v], %[vneed]\n"
          " s_cmp_gt_u32 %[c], 47+8*%c[isa]\n"
          " s_cbranch_scc1 Lpoll%=\n"
          " s_cbranch_vccnz Lpoll%=\n"
          " s_mov_b32 m0, %[ring]\n"
          " s_add_u32 %[n], %[n], 1\n"
          L_DMA("o0", "t0") L_DMA("o1", "t1") L_DMA("o2", "t2") L_DMA("o3", "t3")
          " v_readlane_b32 %[x], %[thr], %[n]\n"
          " s_add_u32 %[nops], %[nops], 16-8*%c[isa]\n"
          " s_cmp_eq_u32 m0, %[rend]\n"
          " s_cselect_b32 %[ring], %[ring0], m0\n"
          " v_bfe_u32 %[vneed], %[x], %[vsh], 8\n"
          L_PUB
          " s_cmp_lt_u32 %[n], %[end]\n"
          " s_cbranch_scc1 Lchk%=\n"
          "Lout%=:\n"
          " s_mov_b32 m0, %[km0]\n"
          : [v] "=&v"(v), [vneed] "=&v"(vneed), [vl] "=&v"(vl), [o0] "=&s"(o0), [o1] "=&s"(o1), [o2] "=&s"(o2), [o3] "=&s"(o3), [c] "=&s"(c),
            [x] "=&s"(x), [y] "=&s"(y), [km0] "=&s"(keep), [n] "+&s"(n), [nops] "+&s"(nops), [ring] "+&s"(ring), [spin] "+&s"(spin)
          : [v16] "v"(v16), [vchk] "v"(vchk), [vfull] "v"(vfull), [vsh] "v"(vsh), [thr] "v"(thr_), [t0] "v"(a0), [t1] "v"(a1), [t2] "v"(a2), [t3] "v"(a3),
            [srd] "s"(srd), [ring0] "s"(ring0), [rend] "s"(ring0 + RINGB), [end] "s"(end), [isa] "i"(isA ? 1 : 0), [sf] "s"(sf), [sfk] "s"(sf + keepv)
          : "vcc", "scc", "memory");
      };
      if (!NOLOAD) {
        run(thr[0], t0[0], t1[0], t2[0], t3[0], 64u);
        if (NS > 64 && n == 64u) run(thr[1], t0[1], t1[1], t2[1], t3[1], NS);
        if (n != (unsigned)NS) lds_wr(ctl + C_ABORT, 1u);
        unsigned vl, c, x, y;
        asm volatile("Ldr%=:\n" L_PUB " s_cmp_lg_u32 %[c], 0\n s_cbranch_scc1 Ldr%=\n"
                     : [vl] "=&v"(vl), [c] "=&s"(c), [x] "=&s"(x), [y] "=&s"(y)
                     : [vfull] "v"(vfull), [nops] "s"(nops), [isa] "i"(isA ? 1 : 0), [sf] "s"(sf), [sfk] "s"(sf + keepv) : "scc", "memory");
      }
      lds_wr(vfull, 255u);
    };
    if (wave == 10) loader(std::true_type{}); else loader(std::false_type{});
    STAMP(20 + (wave & 1));
  }
}

extern "C" void kernel_launch(int rank, int nranks, ncclComm_t comm, hipStream_t stream,
                              void* const* d_in, const int* in_sizes, int n_in,
                              void* d_out, int out_size, const int* in_dtypes, int out_dtype,
                              void* const* d_out_all, void* const* d_ws_all, size_t ws_bytes) {
  (void)comm; (void)in_sizes; (void)n_in; (void)out_size; (void)in_dtypes; (void)out_dtype; (void)d_out_all;
  if (nranks != NR || (unsigned)rank >= (unsigned)NR || ws_bytes < WS_USED) { fprintf(stderr, "unexpected configuration\n"); return; }
  Params p;
  for (int i = 0; i < NR; ++i) p.ws[i] = (char*)d_ws_all[i];
  p.x = (const char*)d_in[0]; p.w = (const char*)d_in[1];
  p.sx = (const float*)d_in[2]; p.sw = (const float*)d_in[3];
  p.out = (float*)d_out; p.wsl = p.ws[rank]; p.rank = rank;
  static bool attr_set[NR] = {};
  if (!attr_set[rank]) {
    HIP_CHECK(hipFuncSetAttribute((const void*)fused_gemm, hipFuncAttributeMaxDynamicSharedMemorySize, (int)LDS_BYTES));
    attr_set[rank] = true;
  }
  fused_gemm<<<256, 768, LDS_BYTES, stream>>>(p);
  HIP_CHECK(hipGetLastError());
}
```

```cpp
#include <hip/hip_runtime.h>
#include <rccl/rccl.h>
#include <cstdio>
#include <cstdint>
#include <type_traits>

#define HIP_CHECK(expr) do { hipError_t _e = (expr); if (_e != hipSuccess) { \
    fprintf(stderr, "HIP error %s at %s:%d\n", hipGetErrorString(_e), __FILE__, __LINE__); \
    return; } } while (0)

typedef int v4i __attribute__((ext_vector_type(4)));
typedef unsigned v4u __attribute__((ext_vector_type(4)));
typedef float v4f __attribute__((ext_vector_type(4)));
#define LAS __attribute__((address_space(3)))
#define CAS __attribute__((address_space(4)))

#define LAGS 0x66532000u
#define NGR 4
#define LPL 4
#define LAST_LEAD 48
#define STG 24
#define STG_LAST 8
#define SHRINK 12
#define PUSH_WIN 3
#define POLL_W 24
#define POLL_N 4
#define POLL_LATE 360
#define POLL_SLEEP 16
#define POLL_SLEEP2 2
#define LOCAL_GAP 50
#define PREF_FROM 96
#define OUTF "sc1"
#define NH_SLOTS 9
#define NB_SLOTS 5
#define LSTR 529

constexpr int NR = 8;
constexpr int DIM = 4096, KSH = 512;
constexpr unsigned PLANE_BLK = 128u << 10;
constexpr unsigned SRC_BYTES = 32 * PLANE_BLK;
constexpr unsigned BUF_BYTES = NR * SRC_BYTES;
constexpr unsigned PF_OFF = 2 * BUF_BYTES;
constexpr unsigned RDY_OFF = PF_OFF + 32 * 600 * 128;
constexpr unsigned CNT_OFF = RDY_OFF + NR * 64 * 128;
constexpr unsigned TMO_OFF = CNT_OFF + 256 * 128;
constexpr unsigned AGG_OFF = TMO_OFF + 512;
constexpr unsigned TK_OFF = TMO_OFF + 1024;
constexpr unsigned DUM_OFF = TMO_OFF + 4096;
constexpr unsigned DIAG_OFF = DUM_OFF + 256 * 256;
constexpr unsigned DIAG2_OFF = DIAG_OFF + 256 * 1024;
constexpr unsigned WS_USED = DIAG2_OFF + 256 * 2048;
constexpr unsigned long long DEADLINE = 30000000ull;

constexpr int NH = NH_SLOTS, NB = NB_SLOTS;
constexpr unsigned LDS_A = 0, LDS_B = NH * 8192, LDS_C = LDS_B + NB * 16384, LDS_T = LDS_C + 1024, LDS_BYTES = LDS_T + 4096;
static_assert(LDS_BYTES <= 163840, "LDS");
constexpr unsigned C_ABORT = 128, C_MVD = 132, C_AVA = 136, C_AVB = 140, C_RDY = 144, C_STG = 148, C_PSH = 152, C_TAB = 156;
constexpr unsigned NTK = 992;
constexpr int NLN = 7 * LPL, NT = (256 - NLN) / NLN;
constexpr unsigned NST0 = LPL * (NGR + 1);
constexpr int lag_of(int j) { return (int)((LAGS >> (4 * j)) & 15u); }
constexpr int SUMD = lag_of(0) + lag_of(1) + lag_of(2) + lag_of(3) + lag_of(4) + lag_of(5) + lag_of(6) + lag_of(7);
constexpr int UEND = (int)NTK - 2 * SUMD - LAST_LEAD;

struct Params {
  char* ws[NR];
  const char* x;
  const char* w;
  const float* sx;
  const float* sw;
  float* out;
  char* wsl;
  int rank;
};

__device__ __forceinline__ unsigned rfl(unsigned v) { return __builtin_amdgcn_readfirstlane(v); }
__device__ __forceinline__ v4i make_srd(const void* p, unsigned bytes) {
  unsigned long long a = (unsigned long long)p;
  v4i r;
  r.x = (int)rfl((unsigned)a); r.y = (int)(rfl((unsigned)(a >> 32)) & 0xffffu); r.z = (int)bytes; r.w = 0x00020000;
  return r;
}
__device__ __forceinline__ unsigned long long rt() { return __builtin_amdgcn_s_memrealtime(); }
template <int N> __device__ __forceinline__ void wait_vm() { asm volatile("s_waitcnt vmcnt(%0)" ::"i"(N) : "memory"); }
__device__ __forceinline__ unsigned lds_rd(unsigned addr) {
  unsigned v;
  asm volatile("ds_read_b32 %0, %1\n\ts_waitcnt lgkmcnt(0)" : "=v"(v) : "v"(addr) : "memory");
  return v;
}
__device__ __forceinline__ void lds_wr(unsigned addr, unsigned v) { asm volatile("ds_write_b32 %0, %1" ::"v"(addr), "v"(v) : "memory"); }
__device__ __forceinline__ void lds_inc(unsigned a) {
  asm volatile("s_mov_b64 exec, 1\n\tds_add_u32 %0, %1\n\ts_mov_b64 exec, -1" ::"v"(a), "v"(1u) : "memory");
}
__device__ __forceinline__ void lds_wait(unsigned a, unsigned n) { unsigned k = 0; while (rfl(lds_rd(a)) < n && ++k < (1u << 24)) { } }
__device__ __forceinline__ unsigned flag_ld(v4i srd, unsigned voff) {
  unsigned v;
  asm volatile("s_nop 4\n\tbuffer_load_dword %0, %1, %2, 0 offen sc0 sc1\n\ts_waitcnt vmcnt(0)" : "=v"(v) : "v"(voff), "s"(srd) : "memory");
  return v;
}
__device__ __forceinline__ void flag_st1(v4i srd, unsigned soff, unsigned v) {
  asm volatile("s_mov_b64 exec, 1\n\ts_nop 4\n\tbuffer_store_dword %0, off, %1, %2 sc0 sc1\n\ts_mov_b64 exec, -1" ::"v"(v), "s"(srd), "s"(soff) : "memory");
}
__device__ __forceinline__ unsigned plane_id(unsigned t4, unsigned rank) {
  const unsigned t = t4 - 32;
  const unsigned q = ((t & 255u) * 9363u) >> 16;
  const unsigned rem = ((rank + 1 + t - 7 * q) & 7u) * 32 + q;
  return t4 < 32 ? rank * 32 + t4 : rem;
}
__device__ __forceinline__ unsigned flag_idx(unsigned t4, unsigned pc) {
  return pc < 32 ? (pc & 15) * LSTR + 8 * (t4 >> 2) + 4 * (pc >> 4) + (t4 & 3) : (pc >> 1) * LSTR + 2 * t4 + (pc & 1);
}
__host__ __device__ constexpr int stream_pos(int q, int g) {
  const int key = q + (g < 16 ? (int)((LAGS >> (4 * (g & 7))) & 15u) : 0);
  int u = 16 * (key - 1 > 31 ? 31 : key - 1) + (g < 16 ? (g >> 3) : g - 16);
  for (int j = 0; j < 8; ++j) {
    int c = key - (int)((LAGS >> (4 * j)) & 15u) - 1 + ((g >= 16 || j < (g & 7)) ? 1 : 0);
    u += 2 * (c < 0 ? 0 : (c > 31 ? 31 : c));
  }
  return u;
}
constexpr unsigned long long first_items(int sub) {
  unsigned long long r = 0;
  for (int it = 0; it < 128; ++it) { const int u = stream_pos(1 + it / 32, it % 32); if (u < (int)NST0 && u % LPL == sub) r |= (unsigned long long)it << (8 * (u / LPL)); }
  return r;
}
template <int I, int N, typename F> __device__ __forceinline__ void static_for(F&& f) {
  if constexpr (I < N) { f(std::integral_constant<int, I>{}); static_for<I + 1, N>(f); }
}

#define TS(x) do { } while (0)
#define STAMPV(idx, v) do { } while (0)
#define STAMP(idx) STAMPV(idx, rt())

constexpr int ACC0 = 40;
template <int G> __device__ __forceinline__ void mv_init(unsigned vdum) { asm volatile("v_mov_b32 v[48+%c0], %1" ::"i"(G), "v"(vdum)); }
template <int G> __device__ __forceinline__ unsigned mv_load(v4i sl, v4i sr, unsigned tk, unsigned o, unsigned v16, unsigned tab) {
  unsigned t;
  asm volatile("v_min_u32 v56, %c[ntk], v[40+%c[g]]\n v_lshl_add_u32 v56, v56, 2, %[tab]\n ds_read_b32 v56, v56\n"
               "s_mov_b64 exec, 1\n v_mov_b32 v[40+%c[g]], 1\n buffer_atomic_add v[40+%c[g]], off, %[sl], %[tk] sc0\n s_mov_b64 exec, -1\n"
               ".set k, 0\n .rept 4\n buffer_load_dwordx4 v[64+%c[g]*16+k*4:67+%c[g]*16+k*4], %[v16], %[sr], %[o] offen offset:k*1024 sc0 sc1\n .set k, k+1\n .endr\n"
               "s_waitcnt lgkmcnt(0)\n v_readfirstlane_b32 %[t], v56\n s_nop 1"
               : [t] "=s"(t) : [g] "i"(G), [sl] "s"(sl), [sr] "s"(sr), [tk] "s"(tk), [o] "s"(o), [v16] "v"(v16), [tab] "s"(tab), [ntk] "i"(NTK - NST0) : "memory");
  return t;
}
template <int G, int W> __device__ __forceinline__ void mv_store(v4i sl, unsigned o, unsigned v16, unsigned call, unsigned nf) {
  asm volatile(".set k, 0\n .rept 4\n s_waitcnt vmcnt(%c[w])\n buffer_store_dwordx4 v[64+%c[g]*16+k*4:67+%c[g]*16+k*4], %[v16], %[sl], %[o] offen offset:k*1024 sc1\n .set k, k+1\n .endr\n"
               "s_mov_b64 exec, 3\n buffer_store_dword %[vc], v[48+%c[g]], %[sl], 0 offen sc1\n s_mov_b64 exec, -1\n v_mov_b32 v[48+%c[g]], %[nf]"
               ::[g] "i"(G), [w] "i"(W), [sl] "s"(sl), [o] "s"(o), [v16] "v"(v16), [vc] "v"(call), [nf] "v"(nf) : "memory");
}
template <int R> __device__ __forceinline__ void zero_acc() {
  asm volatile("v_mov_b32 v%c0, 0\n\tv_mov_b32 v%c1, 0\n\tv_mov_b32 v%c2, 0\n\tv_mov_b32 v%c3, 0" ::"i"(R), "i"(R + 1), "i"(R + 2), "i"(R + 3));
  if constexpr (R + 4 < ACC0 + 128) zero_acc<R + 4>();
}
template <int E> __device__ __forceinline__ void store_row(unsigned sc, unsigned vo, v4i ro) {
  constexpr int R = ACC0 + (E >> 2) * 16 + (E & 3);
  float f0, f1, f2, f3;
  asm volatile("v_cvt_f32_i32 %0, v%c4\n\tv_cvt_f32_i32 %1, v%c5\n\tv_cvt_f32_i32 %2, v%c6\n\tv_cvt_f32_i32 %3, v%c7\n\t"
               "v_mul_f32 %0, %8, %0\n\tv_mul_f32 %1, %8, %1\n\tv_mul_f32 %2, %8, %2\n\tv_mul_f32 %3, %8, %3"
               : "=v"(f0), "=v"(f1), "=v"(f2), "=v"(f3) : "i"(R), "i"(R + 4), "i"(R + 8), "i"(R + 12), "s"(sc));
  v4f v = {f0, f1, f2, f3};
  asm volatile("buffer_store_dwordx4 %0, %1, %2, %3 offen " OUTF "\n\ts_nop 1" ::"v"(v), "v"(vo), "s"(ro),
               "s"((unsigned)(128 * (E >> 4) + 16 * ((E >> 2) & 3) + (E & 3)) * (DIM * 4)) : "memory");
}
template <int E0, int E1> __device__ __forceinline__ void store_rows(unsigned sc, unsigned vo, v4i ro) {
  if constexpr (E0 < E1) { store_row<E0>(sc, vo, ro); store_rows<E0 + 1, E1>(sc, vo, ro); }
}

__global__ void __launch_bounds__(768) __attribute__((amdgpu_num_vgpr(20))) fused_gemm(Params P) {
  extern __shared__ __attribute__((aligned(16))) char lds[];
  const int tid = threadIdx.x, lane = tid & 63;
  const int wave = (int)rfl((unsigned)tid >> 6);
  const int b = blockIdx.x;
  const int rank = P.rank;
  char* const CAS* const ktab = (char* const CAS*)__builtin_amdgcn_kernarg_segment_ptr();
  char* const wsl = P.wsl;
  unsigned cv = __hip_atomic_load((const unsigned*)(wsl + CNT_OFF + (unsigned)b * 128), __ATOMIC_RELAXED, __HIP_MEMORY_SCOPE_AGENT);
  const unsigned lds0 = (unsigned)(unsigned long long)(LAS char*)lds;
  const unsigned ctl = lds0 + LDS_C;
  const int xcd = b & 7, idx = b >> 3;
  const int ti = 2 * (idx >> 2) + (xcd >> 2), tj = 4 * (xcd & 3) + (idx & 3);
  const int sg = b & 31, so = b >> 5;
  auto plane_of = [&](int j) { return (unsigned)(4 * so + j + 29) & 31u; };
  const v4i srdL = make_srd(wsl, WS_USED);
  const bool last = b < NLN;
  const int turn = last ? NT : (255 - b) / NLN, mlane = last ? b : (255 - b) % NLN;
  const bool mover = last || turn < NT;
  const int pi = mlane % 7, pp = (rank + 1 + pi) & 7;
  auto turn_end = [&](int t) { return (unsigned)(((t + 1) * 200 + SHRINK * (t + 1) * (NT - 1 - t)) * UEND / (200 * NT) - (LPL - 1 - mlane / 7) * (STG - (STG - STG_LAST) * t / (NT - 1))); };
  auto entry = [&](unsigned it) { const unsigned id = 2 * it; return id | ((id & 32u) << 10) | (flag_idx(32 + 7 * (it >> 5) + (unsigned)pi, id & 63) << 16); };
  auto build_table = [&]() {
    const unsigned uend = last ? 4096u : turn_end(turn);
#pragma unroll 1
    for (unsigned it = (unsigned)(tid - 512) + 32; it < 1024; it += 256) {
      const unsigned u = (unsigned)stream_pos((int)(it >> 5), (int)(it & 31));
      lds_wr(lds0 + LDS_T + u * 4, entry(it) | (u >= uend ? 0x4000u : 0u));
    }
    asm volatile("s_waitcnt lgkmcnt(0)" ::: "memory");
    lds_inc(ctl + C_TAB);
  };
  if (wave == 0) STAMP(0);
  unsigned call = 0;
  auto get_call = [&]() { unsigned c; asm volatile("v_add_u32 %0, 1, %1" : "=v"(c) : "v"(cv) : "memory"); call = rfl(c); };
  auto wg_barrier = [&]() { asm volatile("s_waitcnt lgkmcnt(0)\n\ts_barrier" ::: "memory"); };
  auto wait_flag = [&](unsigned off) {
    const unsigned long long t0 = rt();
#pragma unroll 1
    for (unsigned n = 1;; ++n) {
      const unsigned v = flag_ld(srdL, off);
      if (__all((int)(v - call) >= 0)) return true;
      if ((n & 15u) == 0u) {
        const unsigned tm = rfl(flag_ld(srdL, TMO_OFF));
        if (tm == call || rt() - t0 > DEADLINE) { flag_st1(srdL, TMO_OFF, call); return false; }
      }
      __builtin_amdgcn_s_sleep(1);
    }
  };

  if (wave < 8) {
    {
      __amdgpu_buffer_rsrc_t rs = __builtin_amdgcn_make_buffer_rsrc(wsl, 0, 2 * BUF_BYTES, 0x00020000);
      const unsigned base = (unsigned)rank * SRC_BYTES;
      char* pk[7];
      auto load_pk = [&]() {
        asm volatile("" ::: "memory");
#pragma unroll
        for (int k = 0; k < 7; ++k) { pk[k] = ktab[(rank + 1 + (k + wave + b) % 7) & 7]; asm volatile("" : "+s"(pk[k])); }
      };
      if (sg < 16) {
        const int r = 32 * wave + (lane & 31), j = lane >> 5;
        const char* src = P.x + (size_t)(256 * sg + r) * KSH + 64 * so + 16 * j;
        v4u v0 = *(const v4u*)src, v1 = *(const v4u*)(src + 32);
        load_pk();
        wg_barrier();
        unsigned ncv;
        asm volatile("v_not_b32 %0, %1" : "=v"(ncv) : "v"(cv) : "memory");
        const unsigned d0 = base + (ncv & 1u) * BUF_BYTES + (unsigned)((r >> 7) * 16 + sg) * 2048 + (unsigned)(r & 127) * 16;
        const unsigned d = d0 + plane_of(j + 2) * PLANE_BLK;
        __builtin_amdgcn_raw_buffer_store_b128(v0, rs, d0 + plane_of(j) * PLANE_BLK, 0, 17);
        __builtin_amdgcn_raw_buffer_store_b128(v1, rs, d, 0, 17);
        if (so == 0) {
#pragma unroll
          for (int k = 0; k < 7; ++k) {
            const v4i sp = make_srd(pk[k], 2 * BUF_BYTES);
            asm volatile("s_mov_b32 exec_lo, 0\n\ts_nop 4\n\tbuffer_store_dwordx4 %0, %1, %2, 0 offen sc0 sc1\n\ts_mov_b64 exec, -1" ::"v"(v1), "v"(d), "s"(sp) : "memory");
            if (k >= PUSH_WIN - 1) wait_vm<PUSH_WIN - 1>();
          }
        }
        wait_vm<0>(); lds_inc(ctl + C_STG);
      } else if (wave < 4) {
        const int gw = sg - 16;
        const char* src = P.w + (size_t)(64 * so + 16 * wave) * DIM + 256 * gw + 4 * lane;
        unsigned r[16];
#pragma unroll
        for (int k = 0; k < 16; ++k) r[k] = *(const unsigned*)(src + (size_t)k * DIM);
        load_pk();
        wg_barrier();
        unsigned ncv;
        asm volatile("v_not_b32 %0, %1" : "=v"(ncv) : "v"(cv) : "memory");
        v4u o[4];
#pragma unroll
        for (int g = 0; g < 4; ++g) {
          unsigned t0 = __builtin_amdgcn_perm(r[4 * g + 1], r[4 * g], 0x05010400u), t1 = __builtin_amdgcn_perm(r[4 * g + 1], r[4 * g], 0x07030602u);
          unsigned t2 = __builtin_amdgcn_perm(r[4 * g + 3], r[4 * g + 2], 0x05010400u), t3 = __builtin_amdgcn_perm(r[4 * g + 3], r[4 * g + 2], 0x07030602u);
          o[0][g] = __builtin_amdgcn_perm(t2, t0, 0x05040100u); o[1][g] = __builtin_amdgcn_perm(t2, t0, 0x07060302u);
          o[2][g] = __builtin_amdgcn_perm(t3, t1, 0x05040100u); o[3][g] = __builtin_amdgcn_perm(t3, t1, 0x07060302u);
        }
        const unsigned d = base + (ncv & 1u) * BUF_BYTES + plane_of(wave) * PLANE_BLK + 65536u + (unsigned)gw * 4096 + (unsigned)(64 * (lane >> 4) + (lane & 15)) * 16;
#pragma unroll
        for (int jj = 0; jj < 4; ++jj) __builtin_amdgcn_raw_buffer_store_b128(o[jj], rs, d + jj * 256, 0, 17);
        if (so == 0 && wave == 3) {
#pragma unroll
          for (int k = 0; k < 7; ++k) {
            const v4i sp = make_srd(pk[k], 2 * BUF_BYTES);
            asm volatile("s_nop 4\n\tbuffer_store_dwordx4 %0, %4, %5, 0 offen sc0 sc1\n\tbuffer_store_dwordx4 %1, %4, %5, 0 offen offset:256 sc0 sc1\n\t"
                         "buffer_store_dwordx4 %2, %4, %5, 0 offen offset:512 sc0 sc1\n\tbuffer_store_dwordx4 %3, %4, %5, 0 offen offset:768 sc0 sc1"
                         ::"v"(o[0]), "v"(o[1]), "v"(o[2]), "v"(o[3]), "v"(d), "s"(sp) : "memory");
            if (k >= PUSH_WIN - 1) wait_vm<4 * (PUSH_WIN - 1)>();
          }
        }
        wait_vm<0>(); lds_inc(ctl + C_STG);
      } else wg_barrier();
    }
    if (wave == 0) STAMP(1);
    asm volatile("" ::: "v167");
    zero_acc<ACC0>();
    const int wr = wave >> 2, wc = wave & 3;
    unsigned pa = lds0 + LDS_A + (unsigned)(lane >> 4) * 2048 + (unsigned)(64 * wr + (lane & 15)) * 16;
    unsigned pb = lds0 + LDS_B + (unsigned)(lane >> 4) * 4096 + (unsigned)(64 * wc + (lane & 15)) * 16;
    const unsigned vw = ctl + (unsigned)wave * 16 + (unsigned)(lane & 1) * 4;
    const unsigned vo = ((unsigned)(256 * ti + 64 * wr) + 4 * (unsigned)(lane >> 4)) * (DIM * 4) + ((unsigned)(256 * tj + 64 * wc) + 4 * (unsigned)(lane & 15)) * 4;
    unsigned sc = rfl(__builtin_bit_cast(unsigned, (*P.sx) * (*P.sw)));
    const v4i ro = make_srd(P.out, DIM * DIM * 4);
    unsigned vcnt = 0, vf, ia = 0, ib = 0, cnt, da, db, spin = 1u << 23;
    unsigned long long ra;
    asm volatile(
      ".macro MM h, i, j\n"
      " v_mfma_i32_16x16x64_i8 v[40+\\h*64+\\i*16+\\j*4:43+\\h*64+\\i*16+\\j*4], v[24+\\i*4:27+\\i*4], v[8+\\j*4:11+\\j*4], v[40+\\h*64+\\i*16+\\j*4:43+\\h*64+\\i*16+\\j*4]\n"
      ".endm\n"
      ".macro STEP h, nb, pm, last\n"
      " .if \\last == 0\n ds_read_b32 %[vf], %[vw]\n .endif\n"
      " s_waitcnt lgkmcnt(3+3*\\pm-\\last)\n"
      " MM \\h, 0, 0\n"
      " s_cmp_eq_u32 %[ia], %c[nh1]\n"
      " s_cselect_b32 %[da], %[wa], 0x2000\n"
      " MM \\h, 1, 0\n"
      " s_cselect_b32 %[ia], -1, %[ia]\n"
      " s_add_u32 %[ia], %[ia], 1\n"
      " .if \\pm\n s_waitcnt lgkmcnt(5-\\last)\n .endif\n"
      " MM \\h, 0, 1\n"
      " v_add_u32 %[pa], %[da], %[pa]\n"
      " MM \\h, 1, 1\n"
      " .if \\nb\n s_cmp_eq_u32 %[ib], %c[nb1]\n s_cselect_b32 %[db], %[wb], 0x4000\n .endif\n"
      " .if \\pm\n s_waitcnt lgkmcnt(4-\\last)\n .endif\n"
      " MM \\h, 0, 2\n"
      " .if \\nb\n s_cselect_b32 %[ib], -1, %[ib]\n s_add_u32 %[ib], %[ib], 1\n .endif\n"
      " MM \\h, 1, 2\n"
      " .if \\nb\n v_add_u32 %[pb], %[db], %[pb]\n .endif\n"
      " .if \\pm\n s_waitcnt lgkmcnt(3-\\last)\n .endif\n"
      " MM \\h, 0, 3\n"
      " v_add_u32 %[vcnt], 1, %[vcnt]\n"
      " MM \\h, 1, 3\n"
      " s_waitcnt lgkmcnt(0)\n"
      " MM \\h, 2, 0\n"
      " .if \\last == 0\n"
      " ds_write_b32 %[vw], %[vcnt] offset:8\n"
      " v_cmp_le_u32 vcc, %[vf], %[vcnt]\n"
      " .endif\n"
      " MM \\h, 3, 0\n"
      " .if \\last == 0\n"
      " s_getpc_b64 %[ra]\n"
      " s_cbranch_vccnz Lslow%=\n"
      " ds_read_b128 v[24:27], %[pa]\n"
      " ds_read_b128 v[28:31], %[pa] offset:256\n"
      " .if \\nb\n ds_read_b128 v[8:11], %[pb]\n .endif\n"
      " .endif\n"
      " MM \\h, 2, 1\n"
      " MM \\h, 3, 1\n"
      " .if \\nb\n ds_read_b128 v[12:15], %[pb] offset:256\n s_sub_u32 %[cnt], %[cnt], 1\n .endif\n"
      " MM \\h, 2, 2\n"
      " MM \\h, 3, 2\n"
      " .if \\nb\n ds_read_b128 v[16:19], %[pb] offset:512\n s_cmp_lg_u32 %[cnt], 0\n .endif\n"
      " MM \\h, 2, 3\n"
      " MM \\h, 3, 3\n"
      " .if \\nb\n ds_read_b128 v[20:23], %[pb] offset:768\n .endif\n"
      " .if \\last == 0\n"
      " ds_read_b128 v[32:35], %[pa] offset:512\n"
      " ds_read_b128 v[36:39], %[pa] offset:768\n"
      " .endif\n"
      ".endm\n"
      " s_mov_b64 vcc, -1\n"
      " s_getpc_b64 %[ra]\n"
      " s_cbranch_vccnz Lslow%=\n"
      " ds_read_b128 v[24:27], %[pa]\n ds_read_b128 v[28:31], %[pa] offset:256\n"
      " ds_read_b128 v[8:11], %[pb]\n ds_read_b128 v[12:15], %[pb] offset:256\n"
      " ds_read_b128 v[16:19], %[pb] offset:512\n ds_read_b128 v[20:23], %[pb] offset:768\n"
      " ds_read_b128 v[32:35], %[pa] offset:512\n ds_read_b128 v[36:39], %[pa] offset:768\n"
      " s_mov_b32 %[cnt], 63\n"
      "Lfull%=:\n"
      " STEP 0, 0, 1, 0\n"
      " STEP 1, 1, 0, 0\n"
      " s_cbranch_scc1 Lfull%=\n"
      " STEP 0, 0, 1, 0\n"
      " STEP 1, 0, 0, 1\n"
      " s_branch Lend%=\n"
      "Lslow%=:\n"
      " ds_read_b32 %[vf], %[vw]\n"
      " s_sub_u32 %[spin], %[spin], 1\n"
      " s_waitcnt lgkmcnt(0)\n"
      " v_cmp_le_u32 vcc, %[vf], %[vcnt]\n"
      " s_cbranch_scc1 Lgive%=\n"
      " s_cbranch_vccz Lret%=\n"
      " s_sleep 1\n"
      " s_branch Lslow%=\n"
      "Lgive%=:\n"
      " s_mov_b32 %[spin], 0\n"
      " s_mov_b32 %[sc], 0x7fc00000\n"
      " s_mov_b64 vcc, 0\n"
      "Lret%=:\n"
      " s_setpc_b64 %[ra]\n"
      "Lend%=:\n"
      " s_nop 7\n s_nop 1\n"
      ".purgem STEP\n.purgem MM\n"
      : [pa] "+&v"(pa), [pb] "+&v"(pb), [vcnt] "+&v"(vcnt), [vf] "=&v"(vf), [ia] "+&s"(ia), [ib] "+&s"(ib), [cnt] "=&s"(cnt), [da] "=&s"(da),
        [db] "=&s"(db), [spin] "+&s"(spin), [sc] "+&s"(sc), [ra] "=&s"(ra)
      : [vw] "v"(vw), [nh1] "i"(NH - 1), [nb1] "i"(NB - 1), [wa] "s"(-(NH - 1) * 8192), [wb] "s"(-(NB - 1) * 16384)
      : "v8", "v9", "v10", "v11", "v12", "v13", "v14", "v15", "v16", "v17", "v18", "v19", "v20", "v21", "v22", "v23", "v24", "v25", "v26", "v27",
        "v28", "v29", "v30", "v31", "v32", "v33", "v34", "v35", "v36", "v37", "v38", "v39", "vcc", "scc", "memory");
    if (wave == 0) STAMP(3);
    sc = rfl(lds_rd(ctl + C_ABORT)) != 0u ? 0x7fc00000u : sc;
    store_rows<0, 32>(sc, vo, ro);
  } else if (wave == 8) {
    wg_barrier();
    lds_wr(lds0 + LDS_T + (NTK + (lane & 15)) * 4, 0xffffffffu);
    if (turn != 0) build_table();
    get_call();
    if (mover) {
      __builtin_amdgcn_s_setprio(3);
      const v4i srdR = make_srd(ktab[pp], 2 * BUF_BYTES);
      const unsigned v16 = (unsigned)lane * 16;
      const unsigned sbase = (call & 1u) * BUF_BYTES + (unsigned)pp * SRC_BYTES;
      unsigned tk_off = TK_OFF + ((call & 1u) * 8 + (unsigned)pi) * 128;
      const unsigned tk_dead = DUM_OFF + (unsigned)b * 256 + 64;
      const int succ = turn + 1 < NT ? b - NLN : mlane;
      const unsigned go_out = DUM_OFF + (unsigned)succ * 256 + 192, go_in = DUM_OFF + (unsigned)b * 256 + 192;
      flag_st1(srdL, tk_dead, 0x1000u);
      unsigned ev;
      bool ok;
      if (turn == 0) {
        lds_wr(ctl + C_MVD, 1u);
        unsigned long long fi = 0;
        static_for<0, LPL>([&](auto S) { constexpr unsigned long long F = first_items(decltype(S)::value); fi = mlane / 7 == decltype(S)::value ? F : fi; });
        ev = entry(32u + ((unsigned)(fi >> (8 * (lane & 7))) & 255u));
        asm volatile("" : "+v"(ev));
        ok = wait_flag(RDY_OFF + (unsigned)(pp * 64 + (lane & 31)) * 128);
      } else {
        const unsigned long long t0 = rt();
        const unsigned est = turn_end(turn - 1) * 66u / 10u;
        while ((unsigned)(rt() - t0) < est) __builtin_amdgcn_s_sleep(16);
        ok = wait_flag((lane & 1) ? RDY_OFF + (unsigned)(pp * 64 + 32) * 128 : go_in);
        lds_wr(ctl + C_MVD, 1u);
        unsigned x0;
        asm volatile("s_mov_b64 exec, 1\n\tv_mov_b32 v47, %c3\n\ts_nop 4\n\tbuffer_atomic_add v47, off, %1, %2 sc0\n\ts_mov_b64 exec, -1\n\t"
                     "s_waitcnt vmcnt(0)\n\tv_readfirstlane_b32 %0, v47\n\ts_nop 1" : "=s"(x0) : "s"(srdL), "s"(ok ? tk_off : tk_dead), "i"(NGR + 1) : "memory");
        const unsigned xe = x0 + NST0 + (unsigned)(lane & 7);
        lds_wait(ctl + C_TAB, 4);
        ev = lds_rd(lds0 + LDS_T + (xe < NTK ? xe : NTK) * 4);
      }
      if (ok) {
        const unsigned l1 = (unsigned)lane & 1u, vdum = DUM_OFF + (unsigned)b * 256 + l1 * 128;
        unsigned os[NGR], en[NGR];
        unsigned e;
        bool rdy = turn != 0;
        auto foff = [&](unsigned x) { return x == 0xffffffffu ? vdum : PF_OFF + ((x >> 16) + l1 * ((x & 0x8000u) ? 1u : (unsigned)LSTR)) * 128; };
        auto load = [&](auto G, unsigned x) {
          constexpr int g = decltype(G)::value;
          en[g] = x; os[g] = sbase + (x == 0xffffffffu ? 64u : x & 2047u) * 2048;
          if (__builtin_expect((x & 0x4000u) != 0u && tk_off != tk_dead, 0)) {
            tk_off = tk_dead;
            flag_st1(srdL, go_out, call);
          }
          return mv_load<g>(srdL, srdR, tk_off, os[g], v16, lds0 + LDS_T + NST0 * 4);
        };
        auto store = [&](auto G, auto W) { mv_store<decltype(G)::value, decltype(W)::value>(srdL, os[decltype(G)::value], v16, call, foff(en[decltype(G)::value])); };
        static_for<0, NGR>([&](auto G) {
          constexpr int g = decltype(G)::value;
          mv_init<g>(vdum);
          load(G, __builtin_amdgcn_readlane(ev, g));
        });
        e = __builtin_amdgcn_readlane(ev, NGR);
        if (turn == 0) { build_table(); lds_wait(ctl + C_TAB, 4); }
        static_for<0, NGR>([&](auto G) {
          constexpr int g = decltype(G)::value;
          store(G, std::integral_constant<int, 3 + (NGR - 1 - g) * 5 + g * 10>{});
          e = load(G, e);
        });
        int stop = -1;
#pragma unroll 1
        while (stop < 0) {
          static_for<0, NGR>([&](auto G) {
            constexpr int g = decltype(G)::value;
            if (stop < 0) {
              store(G, std::integral_constant<int, 3 + (NGR - 1) * 10>{});
              if (__builtin_expect(!rdy && (e & 2047u) >= 320, 0)) {
                rdy = true;
                unsigned r, spins = 0;
                while ((r = rfl(lds_rd(ctl + C_RDY))) == 0u && ++spins < (1u << 23)) __builtin_amdgcn_s_sleep(2);
                if (r != 1u) e = 0xffffffffu;
              }
              if (e == 0xffffffffu) stop = g;
              else {
                e = load(G, e);
              }
            }
          });
        }
        lds_wr(ctl + C_MVD, 2u);
        static_for<0, NGR>([&](auto G) {
          constexpr int g = decltype(G)::value;
          if (stop == g) static_for<1, NGR>([&](auto J) {
            constexpr int j = decltype(J)::value;
            store(std::integral_constant<int, (g + j) % NGR>{}, std::integral_constant<int, 3 + (NGR - 1 - j) * 10 + j * 5>{});
          });
        });
        asm volatile("s_waitcnt vmcnt(0)\n s_mov_b64 exec, 3\n .set k, 0\n .rept %c2\n buffer_store_dword %0, v[48+k], %1, 0 offen sc1\n .set k, k+1\n .endr\n s_mov_b64 exec, -1"
                     ::"v"(call), "s"(srdL), "i"(NGR) : "memory");
      }
      __builtin_amdgcn_s_setprio(1);
    }
    lds_wr(ctl + C_MVD, 2u);
    if (PREF_FROM < 128) {
      const unsigned par_off = (call & 1u) * BUF_BYTES;
      unsigned pf = PREF_FROM, spins = 0;
#pragma unroll 1
      while (pf < 128 && ++spins < (1u << 22)) {
        const unsigned av = rfl(lds_rd(ctl + C_AVA));
        if (av <= pf) { if (av + 8 < pf) __builtin_amdgcn_s_sleep(32); else __builtin_amdgcn_s_sleep(1); continue; }
        const unsigned hs = pf + ((unsigned)lane >> 4);
        const unsigned off = par_off + plane_id(4 * (hs >> 1) + (unsigned)(idx & 3), (unsigned)rank) * PLANE_BLK + (16 * (hs & 1) + (unsigned)ti) * 2048 + ((unsigned)lane & 15u) * 128;
        if (hs < av && hs < 128) asm volatile("buffer_load_dword v60, %0, %1, 0 offen" ::"v"(off), "s"(srdL) : "memory");
        pf = av < pf + 4 ? av : pf + 4;
      }
    }
  } else if (wave == 9) {
    __builtin_amdgcn_s_setprio(2);
    lds_wr(ctl + lane * 4, 0u);
    wg_barrier();
    const bool direct = so == 1;
    if (!direct) build_table();
    get_call();
    {
      char* pb = wsl;
#pragma unroll
      for (int k = 0; k < 8; ++k) { char* t = P.ws[k]; pb = (lane == k || (lane >> 1) == 16 + k) ? t : pb; }
      const int j = (lane - 8) >> 1, h = lane & 1;
      const unsigned pc = (unsigned)(sg < 16 ? h * 16 + sg : 32 + 2 * (sg - 16) + h);
      const unsigned off = lane < 8 ? RDY_OFF + (unsigned)(rank * 64 + (direct ? sg : 32)) * 128
                         : lane < 16 ? PF_OFF + flag_idx(plane_of(j), pc) * 128
                         : lane == 16 ? CNT_OFF + (unsigned)b * 128
                         : lane < 32 ? TK_OFF + ((~call & 1u) * 8 + (unsigned)(lane - 17)) * 128
                         : PF_OFF + flag_idx(32u + ((unsigned)(rank - (lane >> 1) - 1) & 7u), pc) * 128;
      unsigned* dst = (unsigned*)(pb + off);
      const unsigned val = (lane > 16 && lane < 32) ? 0u : call;
      const bool on = lane < 8 ? (direct && lane != rank) : lane <= 16 ? true : lane < 32 ? (b == 255 && lane < 24) : (so == 0 && lane < 48 && (lane >> 1) != 16 + rank);
      asm volatile("" : "+v"(dst));
      unsigned long long tp0 = 0, tp1 = 0;
      TS(tp0);
      lds_wait(ctl + C_STG, sg < 16 ? 8 : 4);
      TS(tp1);
      if (on) __hip_atomic_store(dst, val, __ATOMIC_RELAXED, __HIP_MEMORY_SCOPE_SYSTEM);
      if (direct) build_table();
      else {
        unsigned tk = 0;
        if (lane == 0) tk = __hip_atomic_fetch_add((unsigned*)(wsl + AGG_OFF), 1u, __ATOMIC_RELAXED, __HIP_MEMORY_SCOPE_AGENT);
        if (rfl(tk) + 1u == call * 224u && lane < 8 && lane != rank) __hip_atomic_store(dst, val, __ATOMIC_RELAXED, __HIP_MEMORY_SCOPE_SYSTEM);
      }
      STAMPV(27, tp0); STAMPV(28, tp1);
    }
    if (turn == 0) lds_wr(ctl + C_RDY, wait_flag(RDY_OFF + (unsigned)(pp * 64 + 32) * 128) ? 1u : 2u);
    const unsigned vav = lane < 2 ? ctl + C_AVA + lane * 4 : ctl + 256 + lane * 4;
    unsigned npoll = 0, headA = 0, headB = 0;
    unsigned wA = 31, wB = 31;
    bool mvdone = false;
    const unsigned long long t0 = rt();
#pragma unroll 1
    while (headA < 512 || headB < 512) {
      if (!mvdone) { const unsigned m = rfl(lds_rd(ctl + C_MVD)); mvdone = m == 2u; if (m == 1u) { __builtin_amdgcn_s_sleep(4); continue; } }
      const bool two = headB < 512, isB = two && lane >= 32;
      const unsigned lo = two ? (unsigned)lane & 31u : (unsigned)lane;
      const unsigned pos = (isB ? headB : headA) + lo;
      unsigned v = call - 1;
      if (lo < (isB ? wB : wA) && pos < 512)
        asm volatile("buffer_load_dword %0, %1, %2, 0 offen sc0 sc1\n\ts_waitcnt vmcnt(0)" : "=v"(v) : "v"(PF_OFF + ((isB ? 16u + tj : (unsigned)ti) * LSTR + pos) * 128), "s"(srdL) : "memory");
      const unsigned long long hm = __ballot((int)(v - call) >= 0);
      const unsigned da = (unsigned)__builtin_ctzll(~hm | (two ? 1ull << 31 : 1ull << 63)), db = two ? (unsigned)__builtin_ctz(~(unsigned)(hm >> 32) | 0x80000000u) : 0u;
      headA += da; headB += db;
      {
        const unsigned lim = headA < 320 ? (unsigned)(rt() - t0) / LOCAL_GAP + 1 : 255u;
        const unsigned ha = headA >> 2, hb = headB >> 3;
        lds_wr(vav, lane ? (hb < (lim >> 1) + 1 ? hb : (lim >> 1) + 1) : (ha < lim ? ha : lim)); }
      const bool more = da == wA || db == wB;
      wA = da == wA ? (headB < 512 ? POLL_W : 63u) : POLL_N; wB = db == wB ? POLL_W : POLL_N;
      if (!more) { if (headA < POLL_LATE) __builtin_amdgcn_s_sleep(POLL_SLEEP); else __builtin_amdgcn_s_sleep(POLL_SLEEP2); }
      if ((da | db) == 0u && (++npoll & 15u) == 0u) {
        const unsigned tm = rfl(flag_ld(srdL, TMO_OFF));
        if (tm == call || rt() - t0 > DEADLINE) {
          flag_st1(srdL, TMO_OFF, call);
          lds_wr(ctl + C_ABORT, 1u);
          asm volatile("s_waitcnt lgkmcnt(0)" ::: "memory");
          break;
        }
      }
    }
    lds_wr(vav, 128u);
    STAMP(19);
  } else {
    wg_barrier();
    build_table();
    __builtin_amdgcn_s_setprio(3);
    get_call();
    const unsigned par_off = (call & 1u) * BUF_BYTES;
    auto loader = [&](auto IA) {
      constexpr bool isA = decltype(IA)::value;
      constexpr int NHALF = isA ? 2 : 1;
      constexpr unsigned RINGB = isA ? NH * 8192 : NB * 16384;
      const unsigned v16 = (unsigned)lane * 16;
      const unsigned vchk = lane < 8 ? ctl + lane * 16 + 8 : ctl + (isA ? C_AVA : C_AVB);
      const unsigned vfull = lane < 8 ? ctl + (unsigned)lane * 16 + (isA ? 0 : 4) : ctl + (isA ? 512 : 768) + lane * 4;
      const unsigned vsh = lane < 8 ? 0u : 8u;
      const unsigned ring0 = lds0 + (isA ? LDS_A : LDS_B);
      unsigned ring = ring0, n = 0, nops = 0, spin = 1u << 24;
      const v4i srd = srdL;
      unsigned thr[NHALF], t0[NHALF], t1[NHALF], t2[NHALF], t3[NHALF];
#pragma unroll
      for (int half = 0; half < NHALF; ++half) {
        const unsigned st = (unsigned)(64 * half + lane);
        const unsigned sp = isA ? st >> 1 : st;
        const unsigned po = par_off + (isA ? (16 * (st & 1) + ti) * 2048 : 65536u + (unsigned)tj * 4096);
        int nd = isA ? (int)st + 1 - NH : 2 * ((int)st - NB) + 1;
        nd = nd < 0 ? 0 : nd;
        thr[half] = (unsigned)nd | ((st + 1) << 8);
        t0[half] = po + plane_id(4 * sp, rank) * PLANE_BLK; t1[half] = po + plane_id(4 * sp + 1, rank) * PLANE_BLK;
        t2[half] = po + plane_id(4 * sp + 2, rank) * PLANE_BLK; t3[half] = po + plane_id(4 * sp + 3, rank) * PLANE_BLK;
        asm volatile("" : "+v"(thr[half]), "+v"(t0[half]), "+v"(t1[half]), "+v"(t2[half]), "+v"(t3[half]));
      }
      wait_vm<0>();
#define L_PUB \
      " s_getreg_b32 %[c], hwreg(HW_REG_IB_STS)\n" \
      " s_bfe_u32 %[x], %[c], 0x20016\n" \
      " s_and_b32 %[c], %[c], 15\n" \
      " s_lshl4_add_u32 %[c], %[x], %[c]\n" \
      " s_sub_u32 %[x], %[nops], %[c]\n" \
      " s_lshr_b32 %[x], %[x], 3\n" \
      " .if %c[isa] == 0\n s_and_b32 %[x], %[x], -2\n .endif\n" \
      " v_mov_b32 %[vl], %[x]\n" \
      " ds_write_b32 %[vfull], %[vl]\n"
#define L_DMA(o, t) \
      " buffer_load_dwordx4 %[v16], %[srd], %[" o "] offen lds\n" \
      " buffer_load_dwordx4 %[v16], %[srd], %[" o "] offen offset:1024 lds\n" \
      " .if %c[isa] == 0\n" \
      " buffer_load_dwordx4 %[v16], %[srd], %[" o "] offen offset:2048 lds\n" \
      " buffer_load_dwordx4 %[v16], %[srd], %[" o "] offen offset:3072 lds\n" \
      " .endif\n" \
      " s_add_u32 m0, m0, 0x1000-0x800*%c[isa]\n" \
      " v_readlane_b32 %[" o "], %[" t "], %[n]\n"
      auto run = [&](unsigned thr_, unsigned a0, unsigned a1, unsigned a2, unsigned a3, unsigned end) {
        unsigned v, vneed, vl, o0, o1, o2, o3, c, x, keep;
        asm volatile(
          " s_mov_b32 %[km0], m0\n"
          " v_readlane_b32 %[x], %[thr], %[n]\n"
          " v_readlane_b32 %[o0], %[t0], %[n]\n v_readlane_b32 %[o1], %[t1], %[n]\n"
          " v_readlane_b32 %[o2], %[t2], %[n]\n v_readlane_b32 %[o3], %[t3], %[n]\n"
          " v_bfe_u32 %[vneed], %[x], %[vsh], 8\n"
          "Lpoll%=:\n"
          " ds_read_b32 %[v], %[vchk]\n"
          L_PUB
          " s_sub_u32 %[spin], %[spin], 1\n"
          " s_cbranch_scc1 Lout%=\n"
          " s_waitcnt lgkmcnt(1)\n"
          "Lchk%=:\n"
          " v_cmp_lt_u32 vcc, %[v], %[vneed]\n"
          " s_cmp_gt_u32 %[c], 47+8*%c[isa]\n"
          " s_cbranch_scc1 Lpoll%=\n"
          " s_cbranch_vccnz Lpoll%=\n"
          " s_mov_b32 m0, %[ring]\n"
          " s_add_u32 %[n], %[n], 1\n"
          L_DMA("o0", "t0") L_DMA("o1", "t1") L_DMA("o2", "t2") L_DMA("o3", "t3")
          " v_readlane_b32 %[x], %[thr], %[n]\n"
          " s_add_u32 %[nops], %[nops], 16-8*%c[isa]\n"
          " s_cmp_eq_u32 m0, %[rend]\n"
          " s_cselect_b32 %[ring], %[ring0], m0\n"
          " v_bfe_u32 %[vneed], %[x], %[vsh], 8\n"
          L_PUB
          " s_cmp_lt_u32 %[n], %[end]\n"
          " s_cbranch_scc1 Lchk%=\n"
          "Lout%=:\n"
          " s_mov_b32 m0, %[km0]\n"
          : [v] "=&v"(v), [vneed] "=&v"(vneed), [vl] "=&v"(vl), [o0] "=&s"(o0), [o1] "=&s"(o1), [o2] "=&s"(o2), [o3] "=&s"(o3), [c] "=&s"(c),
            [x] "=&s"(x), [km0] "=&s"(keep), [n] "+&s"(n), [nops] "+&s"(nops), [ring] "+&s"(ring), [spin] "+&s"(spin)
          : [v16] "v"(v16), [vchk] "v"(vchk), [vfull] "v"(vfull), [vsh] "v"(vsh), [thr] "v"(thr_), [t0] "v"(a0), [t1] "v"(a1), [t2] "v"(a2), [t3] "v"(a3),
            [srd] "s"(srd), [ring0] "s"(ring0), [rend] "s"(ring0 + RINGB), [end] "s"(end), [isa] "i"(isA ? 1 : 0)
          : "vcc", "scc", "memory");
      };
      run(thr[0], t0[0], t1[0], t2[0], t3[0], 64u);
      if (isA && n == 64u) run(thr[NHALF - 1], t0[NHALF - 1], t1[NHALF - 1], t2[NHALF - 1], t3[NHALF - 1], 128u);
      if (n != 64u * NHALF) lds_wr(ctl + C_ABORT, 1u);
      unsigned vl, c, x;
      asm volatile("Ldr%=:\n" L_PUB " s_cmp_lg_u32 %[c], 0\n s_cbranch_scc1 Ldr%=\n"
                   : [vl] "=&v"(vl), [c] "=&s"(c), [x] "=&s"(x) : [vfull] "v"(vfull), [nops] "s"(nops), [isa] "i"(isA ? 1 : 0) : "scc", "memory");
      lds_wr(vfull, 255u);
    };
    if (wave == 10) loader(std::true_type{}); else loader(std::false_type{});
    STAMP(20 + (wave & 1));
  }
}

extern "C" void kernel_launch(int rank, int nranks, ncclComm_t comm, hipStream_t stream,
                              void* const* d_in, const int* in_sizes, int n_in,
                              void* d_out, int out_size, const int* in_dtypes, int out_dtype,
                              void* const* d_out_all, void* const* d_ws_all, size_t ws_bytes) {
  (void)comm; (void)in_sizes; (void)n_in; (void)out_size; (void)in_dtypes; (void)out_dtype; (void)d_out_all;
  if (nranks != NR || (unsigned)rank >= (unsigned)NR || ws_bytes < WS_USED) { fprintf(stderr, "unexpected configuration\n"); return; }
  Params p;
  for (int i = 0; i < NR; ++i) p.ws[i] = (char*)d_ws_all[i];
  p.x = (const char*)d_in[0]; p.w = (const char*)d_in[1];
  p.sx = (const float*)d_in[2]; p.sw = (const float*)d_in[3];
  p.out = (float*)d_out; p.wsl = p.ws[rank]; p.rank = rank;
  static bool attr_set[NR] = {};
  if (!attr_set[rank]) {
    HIP_CHECK(hipFuncSetAttribute((const void*)fused_gemm, hipFuncAttributeMaxDynamicSharedMemorySize, (int)LDS_BYTES));
    attr_set[rank] = true;
  }
  fused_gemm<<<256, 768, LDS_BYTES, stream>>>(p);
  HIP_CHECK(hipGetLastError());
}
```

```cpp
#include <hip/hip_runtime.h>
#include <rccl/rccl.h>
#include <cstdio>
#include <cstdint>
#include <type_traits>

#define HIP_CHECK(expr) do { hipError_t _e = (expr); if (_e != hipSuccess) { \
    fprintf(stderr, "HIP error %s at %s:%d\n", hipGetErrorString(_e), __FILE__, __LINE__); \
    return; } } while (0)

typedef int v4i __attribute__((ext_vector_type(4)));
typedef unsigned v4u __attribute__((ext_vector_type(4)));
typedef float v4f __attribute__((ext_vector_type(4)));
#define LAS __attribute__((address_space(3)))
#define CAS __attribute__((address_space(4)))

#define LAGS 0x77440000u
#define LAGW 0x7300u
#define NGR 3
#define LAST_LEAD 32
#define STG 24
#define STG_LAST 4
#define TURNS 0x807a70614d3419ull
#define PUSH_WIN 3
#define POLL_W 24
#define POLL_N 4
#define POLL_LATE 460
#define POLL_SLEEP 16
#define POLL_SLEEP2 2
#define LOCAL_GAP 50
#define PREF_FROM 96
#define CODE_KB 24
#define STAGE_DELAY 64
#define PRIO_MMA 0
#define PRIO_LD 3
#define PRIO_POLL 2
#define LD_SLEEP ""
#define POLL_N2 16
#define ST_WIN 63
#define ST_TURN 4
#define OUTF2 "sc1"
#define NOCOMM 0
#define GRID 256
#define DYNP 1
#define TOUCH 0
#define NOCONS 0
#define TLBW 0
#define POLL_GATE 12
#define NPUSH 1
#define EST_ADD 150
#define EARLY_RESUME 0
#define MMA_SLEEP 1
#define WS_TAIL 1
#define OUTF "sc1"
#define NH_SLOTS 10
#define NB_SLOTS 4
#define LSTR 529

constexpr int NR = 8;
constexpr int DIM = 4096, KSH = 512;
constexpr unsigned PLANE_BLK = 128u << 10;
constexpr unsigned SRC_BYTES = 32 * PLANE_BLK;
constexpr unsigned BUF_BYTES = NR * SRC_BYTES;
constexpr unsigned PF_OFF = 2 * BUF_BYTES;
constexpr unsigned RDY_OFF = PF_OFF + 32 * 600 * 128;
constexpr unsigned CNT_OFF = RDY_OFF + NR * 64 * 128;
constexpr unsigned TMO_OFF = CNT_OFF + 256 * 128;
constexpr unsigned AGG_OFF = TMO_OFF + 3072;
constexpr unsigned TK_OFF = TMO_OFF + 1024;
constexpr unsigned DUM_OFF = TMO_OFF + 4096;
constexpr unsigned DIAG_OFF = DUM_OFF + 256 * 256;
constexpr unsigned DIAG2_OFF = DIAG_OFF + 256 * 1024;
constexpr unsigned WS_USED = DIAG2_OFF + 256 * 2048;
constexpr unsigned long long DEADLINE = 30000000ull;

constexpr int NH = NH_SLOTS, NB = NB_SLOTS;
constexpr unsigned LDS_A = 0, LDS_B = NH * 8192, LDS_C = LDS_B + NB * 16384, LDS_T = LDS_C + 1024, LDS_BYTES = LDS_T + 4096;
static_assert(LDS_BYTES <= 163840 && NH % 2 == 0, "LDS");
constexpr unsigned C_ABORT = 128, C_MVD = 132, C_AVA = 136, C_AVB = 140, C_RDY = 144, C_STG = 148, C_PSH = 152, C_TAB = 156;
constexpr unsigned NTK = 32 * (32 - NPUSH);
constexpr int LPL = 4, NT = 7;
constexpr unsigned NST0 = LPL * (NGR + 1);
constexpr unsigned long long LAG12 = (unsigned long long)LAGS | ((unsigned long long)LAGW << 32);
constexpr int lag_of(int s) { return (int)((LAG12 >> (4 * s)) & 15u); }
constexpr int late_groups() { int r = 0; for (int s = 0; s < 12; ++s) r += (s < 8 ? 2 : 4) * lag_of(s); return r; }
constexpr int UEND = (int)NTK - late_groups() - LAST_LEAD;
constexpr int tile_lag(int idx) { return lag_of(idx >> 2) > lag_of(8 + (idx & 3)) ? lag_of(idx >> 2) : lag_of(8 + (idx & 3)); }
constexpr unsigned long long order8(int k) {
  unsigned long long r = 0;
  for (int i = 8 * k; i < 8 * k + 8; ++i) {
    int c = 0;
    for (int j = 0; j < 32; ++j) c += tile_lag(j) < tile_lag(i) || (tile_lag(j) == tile_lag(i) && j < i);
    r |= (unsigned long long)c << (8 * (i & 7));
  }
  return r;
}

struct Params {
  char* ws[NR];
  const char* x;
  const char* w;
  const float* sx;
  const float* sw;
  float* out;
  char* wsl;
  int rank;
};

__device__ __forceinline__ unsigned rfl(unsigned v) { return __builtin_amdgcn_readfirstlane(v); }
__device__ __forceinline__ v4i make_srd(const void* p, unsigned bytes) {
  unsigned long long a = (unsigned long long)p;
  v4i r;
  r.x = (int)rfl((unsigned)a); r.y = (int)(rfl((unsigned)(a >> 32)) & 0xffffu); r.z = (int)bytes; r.w = 0x00020000;
  return r;
}
__device__ __forceinline__ unsigned long long rt() { return __builtin_amdgcn_s_memrealtime(); }
template <int N> __device__ __forceinline__ void wait_vm() { asm volatile("s_waitcnt vmcnt(%0)" ::"i"(N) : "memory"); }
__device__ __forceinline__ unsigned lds_rd(unsigned addr) {
  unsigned v;
  asm volatile("ds_read_b32 %0, %1\n\ts_waitcnt lgkmcnt(0)" : "=v"(v) : "v"(addr) : "memory");
  return v;
}
__device__ __forceinline__ void lds_wr(unsigned addr, unsigned v) { asm volatile("ds_write_b32 %0, %1" ::"v"(addr), "v"(v) : "memory"); }
__device__ __forceinline__ void lds_inc(unsigned a) {
  asm volatile("s_mov_b64 exec, 1\n\tds_add_u32 %0, %1\n\ts_mov_b64 exec, -1" ::"v"(a), "v"(1u) : "memory");
}
__device__ __forceinline__ void lds_wait(unsigned a, unsigned n) { unsigned k = 0; while (rfl(lds_rd(a)) < n && ++k < (1u << 24)) { } }
__device__ __forceinline__ unsigned flag_ld(v4i srd, unsigned voff) {
  unsigned v;
  asm volatile("s_nop 4\n\tbuffer_load_dword %0, %1, %2, 0 offen sc0 sc1\n\ts_waitcnt vmcnt(0)" : "=v"(v) : "v"(voff), "s"(srd) : "memory");
  return v;
}
__device__ __forceinline__ void flag_st1(v4i srd, unsigned soff, unsigned v) {
  asm volatile("s_mov_b64 exec, 1\n\ts_nop 4\n\tbuffer_store_dword %0, off, %1, %2 sc0 sc1\n\ts_mov_b64 exec, -1" ::"v"(v), "s"(srd), "s"(soff) : "memory");
}
__device__ __forceinline__ unsigned plane_id(unsigned t4, unsigned rank) {
  const unsigned t = t4 - 32;
  const unsigned q = ((t & 255u) * 9363u) >> 16;
  const unsigned rem = ((rank + 1 + t - 7 * q) & 7u) * 32 + q;
  return t4 < 32 ? rank * 32 + t4 : rem;
}
__device__ __forceinline__ unsigned flag_idx(unsigned t4, unsigned pc) {
  return pc < 32 ? (pc & 15) * LSTR + 8 * (t4 >> 2) + 4 * (pc >> 4) + (t4 & 3) : (pc >> 1) * LSTR + 2 * t4 + (pc & 1);
}
__host__ __device__ constexpr int stream_pos(int q, int g) {
  const int me = g < 16 ? (g & 7) : 8 + (g & 3);
  const int key = q + (int)((LAG12 >> (4 * me)) & 15u);
  int u = g < 16 ? (g >> 3) : ((g - 16) >> 2);
  for (int s = 0; s < 12; ++s) {
    const int c = key - lag_of(s) - NPUSH + (s < me ? 1 : 0);
    u += (s < 8 ? 2 : 4) * (c < 0 ? 0 : (c > 32 - NPUSH ? 32 - NPUSH : c));
  }
  return u;
}
constexpr unsigned long long first_items(int sub) {
  unsigned long long r = 0;
  for (int it = 0; it < 128; ++it) { const int u = stream_pos(NPUSH + it / 32, it % 32); if (u < (int)NST0 && u % LPL == sub) r |= (unsigned long long)it << (8 * (u / LPL)); }
  return r;
}
template <int I, int N, typename F> __device__ __forceinline__ void static_for(F&& f) {
  if constexpr (I < N) { f(std::integral_constant<int, I>{}); static_for<I + 1, N>(f); }
}

#define TS(x) do { } while (0)
#define STAMPV(idx, v) do { } while (0)
#define STAMP(idx) STAMPV(idx, rt())

constexpr int ACC0 = 40;
template <int G> __device__ __forceinline__ void mv_init(unsigned vdum) { asm volatile("v_mov_b32 v[48+%c0], %1" ::"i"(G), "v"(vdum)); }
#define MV_T "s_mov_b64 exec, 1\n v_mov_b32 v[40+%c[g]], 1\n buffer_atomic_add v[40+%c[g]], off, %[sl], %[tk] sc0\n s_mov_b64 exec, -1\n"
#define MV_L "buffer_load_dwordx4 v[64+%c[g]*16+k*4:67+%c[g]*16+k*4], %[v16], %[sr], %[ol] offen offset:k*1024 sc0 sc1\n"
#define MV_S "buffer_store_dwordx4 v[64+%c[g]*16+k*4:67+%c[g]*16+k*4], %[v16], %[sl], %[os] offen offset:k*1024 sc1\n"
#define MV_F "s_mov_b64 exec, 3\n buffer_store_dword %[vc], v[48+%c[g]], %[sl], 0 offen sc1\n s_mov_b64 exec, -1\n v_mov_b32 v[48+%c[g]], %[nf]\n"
template <int G> __device__ __forceinline__ void mv_load(v4i sl, v4i sr, unsigned tk, unsigned ol, unsigned v16) {
  asm volatile("s_nop 4\n" MV_T ".set k, 0\n .rept 4\n" MV_L ".set k, k+1\n .endr"
               ::[g] "i"(G), [sl] "s"(sl), [sr] "s"(sr), [tk] "s"(tk), [ol] "s"(ol), [v16] "v"(v16) : "memory");
}
template <int G, int W, int WK> __device__ __forceinline__ unsigned mv_swap(v4i sl, v4i sr, unsigned tk, unsigned os, unsigned ol, unsigned v16, unsigned tab, unsigned call, unsigned nf) {
  unsigned t;
  asm volatile("s_waitcnt vmcnt(%c[w]-1)\n v_min_u32 v56, %c[ntk], v[40+%c[g]]\n v_lshl_add_u32 v56, v56, 2, %[tab]\n ds_read_b32 v56, v56\n" MV_T
               ".set k, 0\n .rept 4\n s_waitcnt vmcnt(%c[w]+%c[wk]*k)\n" MV_S MV_L ".set k, k+1\n .endr\n" MV_F
               "s_waitcnt lgkmcnt(0)\n v_readfirstlane_b32 %[t], v56\n s_nop 1"
               : [t] "=s"(t) : [g] "i"(G), [w] "i"(W), [wk] "i"(WK), [sl] "s"(sl), [sr] "s"(sr), [tk] "s"(tk), [os] "s"(os), [ol] "s"(ol), [v16] "v"(v16),
                 [tab] "s"(tab), [ntk] "i"(NTK - NST0), [vc] "v"(call), [nf] "v"(nf) : "memory");
  return t;
}
template <int G, int W> __device__ __forceinline__ void mv_store(v4i sl, unsigned os, unsigned v16, unsigned call, unsigned nf) {
  asm volatile(".set k, 0\n .rept 4\n s_waitcnt vmcnt(%c[w]-k)\n" MV_S ".set k, k+1\n .endr\n" MV_F
               ::[g] "i"(G), [w] "i"(W), [sl] "s"(sl), [os] "s"(os), [v16] "v"(v16), [vc] "v"(call), [nf] "v"(nf) : "memory");
}
template <int R> __device__ __forceinline__ void zero_acc() {
  asm volatile("v_mov_b32 v%c0, 0\n\tv_mov_b32 v%c1, 0\n\tv_mov_b32 v%c2, 0\n\tv_mov_b32 v%c3, 0" ::"i"(R), "i"(R + 1), "i"(R + 2), "i"(R + 3));
  if constexpr (R + 4 < ACC0 + 128) zero_acc<R + 4>();
}
template <int E, int FL> __device__ __forceinline__ void store_row(unsigned sc, unsigned vo, v4i ro) {
  constexpr int R = ACC0 + (E >> 2) * 16 + (E & 3);
  float f0, f1, f2, f3;
  asm volatile("v_cvt_f32_i32 %0, v%c4\n\tv_cvt_f32_i32 %1, v%c5\n\tv_cvt_f32_i32 %2, v%c6\n\tv_cvt_f32_i32 %3, v%c7\n\t"
               "v_mul_f32 %0, %8, %0\n\tv_mul_f32 %1, %8, %1\n\tv_mul_f32 %2, %8, %2\n\tv_mul_f32 %3, %8, %3"
               : "=v"(f0), "=v"(f1), "=v"(f2), "=v"(f3) : "i"(R), "i"(R + 4), "i"(R + 8), "i"(R + 12), "s"(sc));
  v4f v = {f0, f1, f2, f3};
  constexpr unsigned so = (unsigned)(128 * (E >> 4) + 16 * ((E >> 2) & 3) + 4 * (E & 3)) * (DIM * 4);
  if constexpr (FL == 0) asm volatile("buffer_store_dwordx4 %0, %1, %2, %3 offen " OUTF "\n\ts_nop 1" ::"v"(v), "v"(vo), "s"(ro), "s"(so) : "memory");
  else asm volatile("buffer_store_dwordx4 %0, %1, %2, %3 offen " OUTF2 "\n\ts_nop 1" ::"v"(v), "v"(vo), "s"(ro), "s"(so) : "memory");
}
template <int E0, int E1, int FL> __device__ __forceinline__ void store_rows(unsigned sc, unsigned vo, v4i ro) {
  if constexpr (E0 < E1) { store_row<E0, FL>(sc, vo, ro); if constexpr (FL && ST_WIN < 63) wait_vm<ST_WIN>(); store_rows<E0 + 1, E1, FL>(sc, vo, ro); }
}

__global__ void __launch_bounds__(768) __attribute__((amdgpu_num_vgpr(20))) fused_gemm(Params P) {
  extern __shared__ __attribute__((aligned(16))) char lds[];
  const int tid = threadIdx.x, lane = tid & 63;
  if (CODE_KB > 0 && tid >= 768 - CODE_KB * 8) {
    unsigned long long pc;
    asm volatile("s_getpc_b64 %0\n\tglobal_load_dword v60, %1, %0\n\t.subsection 1\n\t.fill 6144, 4, 0xBF800000\n\t.subsection 0" : "=&s"(pc) : "v"((unsigned)(tid - (768 - CODE_KB * 8)) * 128u) : "memory");
  }
  const int wave = (int)rfl((unsigned)tid >> 6);
  const int b = blockIdx.x;
  const int rank = P.rank;
  char* const CAS* const ktab = (char* const CAS*)__builtin_amdgcn_kernarg_segment_ptr();
  char* const wsl = P.wsl;
  unsigned cv = __hip_atomic_load((const unsigned*)(wsl + CNT_OFF + (unsigned)b * 128), __ATOMIC_RELAXED, __HIP_MEMORY_SCOPE_AGENT);
  const unsigned lds0 = (unsigned)(unsigned long long)(LAS char*)lds;
  const unsigned ctl = lds0 + LDS_C;
  const int xcd = b & 7, idx = b >> 3;
  const int ti = 2 * (idx >> 2) + (xcd >> 2), tj = 4 * (xcd & 3) + (idx & 3);
  const int sg = b & 31, so = b >> 5;
  auto plane_of = [&](int j) { return (unsigned)(4 * so + j + 28 + NPUSH) & 31u; };
  const v4i srdL = make_srd(wsl, WS_USED);
  constexpr unsigned long long O0 = order8(0), O1 = order8(1), O2 = order8(2), O3 = order8(3);
  const int mp = (int)(((idx < 8 ? O0 : idx < 16 ? O1 : idx < 24 ? O2 : O3) >> (8 * (idx & 7))) & 31u) * 8 + xcd;
  const int turn = 7 - (mp >> 5), mpair = ((mp >> 3) & 3) * 4 + (mp & 3);
  const bool last = turn == NT, mover = mpair < 14;
  const int pi = mpair % 7, sub = (mpair / 7) * 2 + ((mp >> 2) & 1), pp = (rank + 1 + pi) & 7;
  const unsigned rdy_rest = RDY_OFF + (unsigned)(pp * 64 + 32 + ((lane & 7) == 1 ? 0 : (lane & 7))) * 128;
  auto turn_end = [&](int t) { return (unsigned)((int)((TURNS >> (8 * t)) & 255u) * UEND / 128 - (LPL - 1 - sub) * (STG - (STG - STG_LAST) * t / (NT - 1))); };
  auto entry = [&](unsigned it) { const unsigned id = 2 * it; return id | ((id & 32u) << 10) | (flag_idx(32 + 7 * (it >> 5) + (unsigned)pi, id & 63) << 16); };
  auto build_table = [&]() {
    const unsigned uend = last ? 4096u : turn_end(turn);
#pragma unroll 1
    for (unsigned it = (unsigned)(tid - 576) + 32 * NPUSH; it < 1024; it += 192) {
      const unsigned u = (unsigned)stream_pos((int)(it >> 5), (int)(it & 31));
      lds_wr(lds0 + LDS_T + u * 4, entry(it) | (u >= uend ? 0x4000u : 0u));
    }
    asm volatile("s_waitcnt lgkmcnt(0)" ::: "memory");
    lds_inc(ctl + C_TAB);
  };
  if (wave == 0) STAMP(0);
  unsigned call = 0;
  auto get_call = [&]() { unsigned c; asm volatile("v_add_u32 %0, 1, %1" : "=v"(c) : "v"(cv) : "memory"); call = rfl(c); };
  auto wg_barrier = [&]() { asm volatile("s_waitcnt lgkmcnt(0)\n\ts_barrier" ::: "memory"); };
  auto wait_flag = [&](unsigned off) {
    const unsigned long long t0 = rt();
#pragma unroll 1
    for (unsigned n = 1;; ++n) {
      const unsigned v = flag_ld(srdL, off);
      if (__all((int)(v - call) >= 0)) return true;
      if ((n & 15u) == 0u) {
        const unsigned tm = rfl(flag_ld(srdL, TMO_OFF));
        if (tm == call || rt() - t0 > DEADLINE) { flag_st1(srdL, TMO_OFF, call); return false; }
      }
      __builtin_amdgcn_s_sleep(1);
    }
  };

  if (wave < 8) {
    unsigned long long tq = 0;
    {
      __amdgpu_buffer_rsrc_t rs = __builtin_amdgcn_make_buffer_rsrc(wsl, 0, 2 * BUF_BYTES, 0x00020000);
      const unsigned base = (unsigned)rank * SRC_BYTES;
      char* pk[7] = {};
      const bool lazy = so >= 2 && turn != 0;
      if (lazy) { wg_barrier(); __builtin_amdgcn_s_sleep(STAGE_DELAY); }
      auto load_pk = [&]() {
        asm volatile("" ::: "memory");
#pragma unroll
        for (int k = 0; k < 7; ++k) { pk[k] = ktab[(rank + 1 + (k + wave + b) % 7) & 7]; asm volatile("" : "+s"(pk[k])); }
      };
      if (sg < 16) {
        const int r = 32 * wave + (lane & 31), j = lane >> 5;
        const char* src = P.x + (size_t)(256 * sg + ((r & ~15) | ((r & 3) << 2) | ((r >> 2) & 3))) * KSH + 64 * so + 16 * j;
        v4u v0 = *(const v4u*)src, v1 = *(const v4u*)(src + 32);
        if (!lazy) wg_barrier();
        if (so == 0) load_pk();
        unsigned ncv;
        asm volatile("v_not_b32 %0, %1" : "=v"(ncv) : "v"(cv) : "memory");
        const unsigned d0 = base + (ncv & 1u) * BUF_BYTES + (unsigned)((r >> 7) * 16 + sg) * 2048 + (unsigned)(r & 127) * 16;
        const unsigned d = d0 + plane_of(j + 2) * PLANE_BLK;
        __builtin_amdgcn_raw_buffer_store_b128(v0, rs, d0 + plane_of(j) * PLANE_BLK, 0, 17);
        __builtin_amdgcn_raw_buffer_store_b128(v1, rs, d, 0, 17);
        TS(tq);
        if (so == 0) {
#pragma unroll
          for (int k = 0; k < 7; ++k) {
            const v4i sp = make_srd(pk[k], 2 * BUF_BYTES);
            asm volatile("s_mov_b32 exec_lo, %c3\n\ts_nop 4\n\tbuffer_store_dwordx4 %0, %1, %2, 0 offen sc0 sc1\n\ts_mov_b64 exec, -1" ::"v"(v1), "v"(d), "s"(sp), "i"(NPUSH == 2 ? -1 : 0) : "memory");
            if (k >= PUSH_WIN - 1) wait_vm<PUSH_WIN - 1>();
          }
        }
        wait_vm<0>(); lds_inc(ctl + C_STG);
      } else if (wave < 4) {
        const int gw = sg - 16;
        const char* src = P.w + (size_t)(64 * so + 16 * wave) * DIM + 256 * gw + 4 * lane;
        unsigned r[16];
#pragma unroll
        for (int k = 0; k < 16; ++k) r[k] = *(const unsigned*)(src + (size_t)k * DIM);
        if (!lazy) wg_barrier();
        if (so == 0) load_pk();
        unsigned ncv;
        asm volatile("v_not_b32 %0, %1" : "=v"(ncv) : "v"(cv) : "memory");
        v4u o[4];
#pragma unroll
        for (int g = 0; g < 4; ++g) {
          unsigned t0 = __builtin_amdgcn_perm(r[4 * g + 1], r[4 * g], 0x05010400u), t1 = __builtin_amdgcn_perm(r[4 * g + 1], r[4 * g], 0x07030602u);
          unsigned t2 = __builtin_amdgcn_perm(r[4 * g + 3], r[4 * g + 2], 0x05010400u), t3 = __builtin_amdgcn_perm(r[4 * g + 3], r[4 * g + 2], 0x07030602u);
          o[0][g] = __builtin_amdgcn_perm(t2, t0, 0x05040100u); o[1][g] = __builtin_amdgcn_perm(t2, t0, 0x07060302u);
          o[2][g] = __builtin_amdgcn_perm(t3, t1, 0x05040100u); o[3][g] = __builtin_amdgcn_perm(t3, t1, 0x07060302u);
        }
        const unsigned d = base + (ncv & 1u) * BUF_BYTES + plane_of(wave) * PLANE_BLK + 65536u + (unsigned)gw * 4096 + (unsigned)(64 * (lane >> 4) + (lane & 15)) * 16;
#pragma unroll
        for (int jj = 0; jj < 4; ++jj) __builtin_amdgcn_raw_buffer_store_b128(o[jj], rs, d + jj * 256, 0, 17);
        TS(tq);
        if (so == 0 && wave >= 4 - NPUSH) {
#pragma unroll
          for (int k = 0; k < 7; ++k) {
            const v4i sp = make_srd(pk[k], 2 * BUF_BYTES);
            asm volatile("s_nop 4\n\tbuffer_store_dwordx4 %0, %4, %5, 0 offen sc0 sc1\n\tbuffer_store_dwordx4 %1, %4, %5, 0 offen offset:256 sc0 sc1\n\t"
                         "buffer_store_dwordx4 %2, %4, %5, 0 offen offset:512 sc0 sc1\n\tbuffer_store_dwordx4 %3, %4, %5, 0 offen offset:768 sc0 sc1"
                         ::"v"(o[0]), "v"(o[1]), "v"(o[2]), "v"(o[3]), "v"(d), "s"(sp) : "memory");
            if (k >= PUSH_WIN - 1) wait_vm<4 * (PUSH_WIN - 1)>();
          }
        }
        wait_vm<0>(); lds_inc(ctl + C_STG);
      } else if (!lazy) wg_barrier();
    }
    if (wave == 0) { STAMP(1); STAMPV(26, tq); }
    asm volatile("" ::: "v167");
    if (NOCONS) return;
    zero_acc<ACC0>();
    __builtin_amdgcn_s_setprio(PRIO_MMA);
    const int wr = wave >> 2, wc = wave & 3;
    unsigned pa = lds0 + LDS_A + (unsigned)(lane >> 4) * 2048 + (unsigned)(64 * wr + (lane & 15)) * 16;
    unsigned pb = lds0 + LDS_B + (unsigned)(lane >> 4) * 4096 + (unsigned)(64 * wc + (lane & 15)) * 16;
    const unsigned vw = ctl + (unsigned)wave * 16 + (unsigned)(lane & 3) * 4;
    const unsigned vd = (lane & 1) ? ctl + (unsigned)(wave ^ 4) * 16 + 8 : ctl + (unsigned)wave * 16 + 12;
    unsigned vo = ((unsigned)(256 * ti + 64 * wr) + (unsigned)(lane >> 4)) * (DIM * 4) + ((unsigned)(256 * tj + 64 * wc) + 4 * (unsigned)(lane & 15)) * 4;
    asm volatile("" : "+v"(vo));
    unsigned sc = rfl(__builtin_bit_cast(unsigned, (*P.sx) * (*P.sw)));
    const v4i ro = make_srd(P.out, DIM * DIM * 4);
    unsigned vcnt = 0, vf, ia = 0, ib = 0, cnt, da, db, spin = 1u << 23;
    unsigned long long ra;
    asm volatile(
      ".macro MM h, i, j\n"
      " v_mfma_i32_16x16x64_i8 v[40+\\h*64+\\i*16+\\j*4:43+\\h*64+\\i*16+\\j*4], v[24+\\i*4:27+\\i*4], v[8+\\j*4:11+\\j*4], v[40+\\h*64+\\i*16+\\j*4:43+\\h*64+\\i*16+\\j*4]\n"
      ".endm\n"
      ".macro STEP h, nb, pm, last\n"
      " s_waitcnt lgkmcnt(4+\\pm)\n"
      " MM \\h, 0, 0\n"
      " s_cmp_eq_u32 %[ia], %c[nh1]\n"
      " s_cselect_b32 %[da], %[wa], 0x2000\n"
      " MM \\h, 1, 0\n"
      " s_cselect_b32 %[ia], -1, %[ia]\n"
      " s_add_u32 %[ia], %[ia], 1\n"
      " .if \\pm\n s_waitcnt lgkmcnt(4)\n .endif\n"
      " MM \\h, 0, 1\n"
      " v_add_u32 %[pa], %[da], %[pa]\n"
      " MM \\h, 1, 1\n"
      " .if \\nb\n s_cmp_eq_u32 %[ib], %c[nb1]\n s_cselect_b32 %[db], %[wb], 0x4000\n .endif\n"
      " .if \\pm\n s_waitcnt lgkmcnt(3)\n .endif\n"
      " MM \\h, 0, 2\n"
      " v_add_u32 %[vcnt], 1, %[vcnt]\n"
      " .if \\nb\n s_cselect_b32 %[ib], -1, %[ib]\n s_add_u32 %[ib], %[ib], 1\n .endif\n"
      " MM \\h, 1, 2\n"
      " .if \\nb\n v_add_u32 %[pb], %[db], %[pb]\n .endif\n"
      " s_waitcnt lgkmcnt(2)\n"
      " MM \\h, 0, 3\n"
      " .if \\last == 0\n v_cmp_le_u32 vcc, %[vf], %[vcnt]\n .endif\n"
      " MM \\h, 1, 3\n"
      " .if \\last == 0\n"
      " s_getpc_b64 %[ra]\n"
      " s_and_b32 %[da], vcc_lo, 3\n"
      " s_cbranch_scc1 Lslow%=\n"
      " ds_read_b128 v[24:27], %[pa]\n"
      " ds_read_b128 v[28:31], %[pa] offset:256\n"
      " .endif\n"
      " s_waitcnt lgkmcnt(2-2*\\last)\n"
      " MM \\h, 2, 0\n"
      " .if \\last == 0\n"
      " ds_write_b32 %[vd], %[vcnt]\n"
      " ds_read_b32 %[vf], %[vw]\n"
      " .endif\n"
      " MM \\h, 3, 0\n"
      " .if \\nb\n ds_read_b128 v[8:11], %[pb]\n .endif\n"
      " MM \\h, 2, 1\n"
      " .if (\\last == 0) && %c[dynp]\n"
      " s_bitcmp1_b32 vcc_lo, 2\n"
      " s_setprio 0\n"
      " s_cbranch_scc1 1f\n"
      " s_setprio 1\n"
      "1:\n"
      " .endif\n"
      " MM \\h, 3, 1\n"
      " .if \\nb\n ds_read_b128 v[12:15], %[pb] offset:256\n s_sub_u32 %[cnt], %[cnt], 1\n .endif\n"
      " MM \\h, 2, 2\n"
      " MM \\h, 3, 2\n"
      " .if \\nb\n ds_read_b128 v[16:19], %[pb] offset:512\n s_cmp_lg_u32 %[cnt], 0\n .endif\n"
      " MM \\h, 2, 3\n"
      " MM \\h, 3, 3\n"
      " .if \\nb\n ds_read_b128 v[20:23], %[pb] offset:768\n .endif\n"
      " .if \\last == 0\n"
      " ds_read_b128 v[32:35], %[pa] offset:512\n"
      " ds_read_b128 v[36:39], %[pa] offset:768\n"
      " .endif\n"
      ".endm\n"
      " s_mov_b64 vcc, -1\n"
      " s_getpc_b64 %[ra]\n"
      " s_and_b32 %[da], vcc_lo, 3\n"
      " s_cbranch_scc1 Lslow%=\n"
      " ds_read_b128 v[24:27], %[pa]\n ds_read_b128 v[28:31], %[pa] offset:256\n"
      " ds_read_b128 v[8:11], %[pb]\n ds_read_b128 v[12:15], %[pb] offset:256\n"
      " ds_read_b128 v[16:19], %[pb] offset:512\n ds_read_b128 v[20:23], %[pb] offset:768\n"
      " ds_read_b128 v[32:35], %[pa] offset:512\n ds_read_b128 v[36:39], %[pa] offset:768\n"
      " s_mov_b32 %[cnt], 63\n"
      "Lfull%=:\n"
      " STEP 0, 0, 1, 0\n"
      " STEP 1, 1, 0, 0\n"
      " s_cbranch_scc1 Lfull%=\n"
      " STEP 0, 0, 1, 0\n"
      " STEP 1, 0, 0, 1\n"
      " s_branch Lend%=\n"
      "Lslow%=:\n"
      " ds_read_b32 %[vf], %[vw]\n"
      " s_sub_u32 %[spin], %[spin], 1\n"
      " s_waitcnt lgkmcnt(0)\n"
      " ds_write_b32 %[vd], %[vcnt]\n"
      " v_cmp_le_u32 vcc, %[vf], %[vcnt]\n"
      " s_cbranch_scc1 Lgive%=\n"
      " s_and_b32 %[da], vcc_lo, 3\n"
      " s_cbranch_scc0 Lret%=\n"
      " s_sleep %c[msl]\n"
      " s_branch Lslow%=\n"
      "Lgive%=:\n"
      " s_mov_b32 %[spin], 0\n"
      " s_mov_b32 %[sc], 0x7fc00000\n"
      " s_mov_b64 vcc, 0\n"
      "Lret%=:\n"
      " s_setpc_b64 %[ra]\n"
      "Lend%=:\n"
      " s_nop 7\n s_nop 1\n"
      ".purgem STEP\n.purgem MM\n"
      : [pa] "+&v"(pa), [pb] "+&v"(pb), [vcnt] "+&v"(vcnt), [vf] "=&v"(vf), [ia] "+&s"(ia), [ib] "+&s"(ib), [cnt] "=&s"(cnt), [da] "=&s"(da),
        [db] "=&s"(db), [spin] "+&s"(spin), [sc] "+&s"(sc), [ra] "=&s"(ra)
      : [vw] "v"(vw), [vd] "v"(vd), [dynp] "i"(DYNP), [msl] "i"(MMA_SLEEP), [nh1] "i"(NH - 1), [nb1] "i"(NB - 1), [wa] "s"(-(NH - 1) * 8192), [wb] "s"(-(NB - 1) * 16384)
      : "v8", "v9", "v10", "v11", "v12", "v13", "v14", "v15", "v16", "v17", "v18", "v19", "v20", "v21", "v22", "v23", "v24", "v25", "v26", "v27",
        "v28", "v29", "v30", "v31", "v32", "v33", "v34", "v35", "v36", "v37", "v38", "v39", "vcc", "scc", "memory");
    if (wave == 0) STAMP(3);
    sc = rfl(lds_rd(ctl + C_ABORT)) != 0u ? 0x7fc00000u : sc;
    if (turn >= ST_TURN && turn < NT) store_rows<0, 32, 1>(sc, vo, ro); else store_rows<0, 32, 0>(sc, vo, ro);
  } else if (wave == 8) {
    if (TLBW && (so == 2 || so == 3)) {
      const unsigned r = (unsigned)idx & 7u;
      const v4i sw = make_srd(ktab[r], WS_USED);
      const unsigned off = lane < 2 ? (unsigned)lane * BUF_BYTES + (unsigned)rank * SRC_BYTES : lane < 4 ? (unsigned)(lane - 2) * BUF_BYTES + r * SRC_BYTES
                         : lane == 4 ? RDY_OFF : PF_OFF;
      if (lane < 6) asm volatile("s_nop 4\n\tbuffer_load_dword v60, %0, %1, 0 offen sc0 sc1" ::"v"(off), "s"(sw) : "memory");
      if (lane < 2) asm volatile("buffer_load_dword v61, %0, %1, 0 offen sc0 sc1" ::"v"((unsigned)lane * BUF_BYTES + r * SRC_BYTES), "s"(srdL) : "memory");
    }
    wg_barrier();
    lds_wr(lds0 + LDS_T + (NTK + (lane & 15)) * 4, 0xffffffffu);
    get_call();
    if (mover && !NOCOMM) {
      __builtin_amdgcn_s_setprio(3);
      const v4i srdR = make_srd(ktab[pp], 2 * BUF_BYTES);
      const unsigned v16 = (unsigned)lane * 16;
      const unsigned sbase = (call & 1u) * BUF_BYTES + (unsigned)pp * SRC_BYTES;
      unsigned tk_off = TK_OFF + ((call & 1u) * 8 + (unsigned)pi) * 128;
      const unsigned tk_dead = DUM_OFF + (unsigned)b * 256 + 64;
      const unsigned go_out = DUM_OFF + (unsigned)(last ? mp : mp - 32) * 256 + 192, go_in = DUM_OFF + (unsigned)mp * 256 + 192;
      flag_st1(srdL, tk_dead, 0x1000u);
      unsigned long long tm0 = 0, tm2 = 0;
      unsigned ev;
      bool ok;
      if (turn == 0) {
        lds_wr(ctl + C_MVD, 1u);
        unsigned long long fi = 0;
        static_for<0, LPL>([&](auto S) { constexpr unsigned long long F = first_items(decltype(S)::value); fi = sub == decltype(S)::value ? F : fi; });
        const unsigned it = (unsigned)(fi >> (8 * ((lane & 7) < NGR ? (lane & 7) : NGR))) & 255u, g = it & 31u;
        ev = entry(32u * NPUSH + it);
        asm volatile("" : "+v"(ev));
        TS(tm0);
        ok = wait_flag(RDY_OFF + ((unsigned)pp * 64 + (g < 16 ? 2 * (g & 7) + ((unsigned)lane >> 3 & 1u) : g)) * 128);
      } else {
        const unsigned long long t0 = rt();
        const unsigned est = turn_end(turn - 1) * 67u / 10u + EST_ADD;
        while ((unsigned)(rt() - t0) < est) __builtin_amdgcn_s_sleep(16);
        ok = wait_flag(rdy_rest) && wait_flag(go_in);
        lds_wr(ctl + C_MVD + (unsigned)(lane < 3 ? lane : 0) * 4, lane == 1 || lane == 2 ? 0u : 1u);
        unsigned x0;
        asm volatile("s_mov_b64 exec, 1\n\tv_mov_b32 v47, %c3\n\ts_nop 4\n\tbuffer_atomic_add v47, off, %1, %2 sc0\n\ts_mov_b64 exec, -1\n\t"
                     "s_waitcnt vmcnt(0)\n\tv_readfirstlane_b32 %0, v47\n\ts_nop 1" : "=s"(x0) : "s"(srdL), "s"(ok ? tk_off : tk_dead), "i"(NGR + 1) : "memory");
        const unsigned xe = x0 + NST0 + (unsigned)(lane & 7);
        lds_wait(ctl + C_TAB, 3);
        ev = lds_rd(lds0 + LDS_T + (xe < NTK ? xe : NTK) * 4);
      }
      if (ok) {
        const unsigned l1 = (unsigned)lane & 1u, vdum = DUM_OFF + (unsigned)b * 256 + l1 * 128;
        unsigned os[NGR], en[NGR];
        unsigned e;
        unsigned rdy = turn != 0 ? 4096u : (4u + NPUSH) * 64;
        auto foff = [&](unsigned x) { return x == 0xffffffffu ? vdum : PF_OFF + ((x >> 16) + l1 * ((x & 0x8000u) ? 1u : (unsigned)LSTR)) * 128; };
        auto offs = [&](unsigned x) {
          if (__builtin_expect((x & 0x4000u) != 0u && tk_off != tk_dead, 0)) {
            tk_off = tk_dead;
            flag_st1(srdL, go_out, call);
          }
          return sbase + (x == 0xffffffffu ? 64u * NPUSH : x & 2047u) * 2048;
        };
        auto gate = [&]() {
          if (__builtin_expect((e & 2047u) >= rdy && e != 0xffffffffu, 0)) {
            const unsigned lvl = (e & 2047u) < (28 + NPUSH) * 64 ? 2u : 3u;
            unsigned r, spins = 0;
            while ((r = rfl(lds_rd(ctl + C_RDY))) < lvl && ++spins < (1u << 23)) __builtin_amdgcn_s_sleep(2);
            rdy = lvl == 2u ? (28u + NPUSH) * 64 : 4096u;
            if (r < lvl || r > 3u) e = 0xffffffffu;
          }
        };
        auto swap = [&](auto G, auto W, auto WK) {
          constexpr int g = decltype(G)::value;
          const unsigned ol = offs(e), nf = foff(en[g]), o = os[g];
          en[g] = e; os[g] = ol;
          e = mv_swap<g, decltype(W)::value, decltype(WK)::value>(srdL, srdR, tk_off, o, ol, v16, lds0 + LDS_T + NST0 * 4, call, nf);
        };
        static_for<0, NGR>([&](auto G) {
          constexpr int g = decltype(G)::value;
          mv_init<g>(vdum);
          en[g] = __builtin_amdgcn_readlane(ev, g); os[g] = offs(en[g]);
          mv_load<g>(srdL, srdR, tk_off, os[g], v16);
        });
        e = __builtin_amdgcn_readlane(ev, NGR);
        if (turn == 0) {
          lds_wait(ctl + C_TAB, 3);
          lds_wait(ctl + C_RDY, 1);
          TS(tm2);
        }
        static_for<0, NGR>([&](auto G) {
          constexpr int g = decltype(G)::value;
          gate();
          swap(G, std::integral_constant<int, 4 + (NGR - 1 - g) * 5 + g * 10>{}, std::integral_constant<int, 1>{});
        });
        int stop = -1;
#pragma unroll 1
        while (stop < 0) {
          static_for<0, NGR>([&](auto G) {
            constexpr int g = decltype(G)::value;
            if (stop < 0) {
              gate();
              if (e == 0xffffffffu) stop = g;
              else {
                swap(G, std::integral_constant<int, 8 + (NGR - 1) * 10>{}, std::integral_constant<int, 0>{});
              }
            }
          });
        }
        if (tk_off != tk_dead) flag_st1(srdL, go_out, call);
        if (EARLY_RESUME) lds_wr(ctl + C_MVD, 2u);
        static_for<0, NGR>([&](auto G) {
          constexpr int g = decltype(G)::value;
          if (stop == g) static_for<0, NGR>([&](auto J) {
            constexpr int j = decltype(J)::value, h = (g + j) % NGR;
            mv_store<h, 7 + (NGR - 1 - j) * 10 + j * 5>(srdL, os[h], v16, call, foff(en[h]));
          });
        });
        asm volatile("s_waitcnt vmcnt(0)\n s_mov_b64 exec, 3\n .set k, 0\n .rept %c2\n buffer_store_dword %0, v[48+k], %1, 0 offen sc1\n .set k, k+1\n .endr\n s_mov_b64 exec, -1"
                     ::"v"(call), "s"(srdL), "i"(NGR) : "memory");
        lds_wr(ctl + C_MVD, 2u);
      }
      __builtin_amdgcn_s_setprio(0);
    }
    lds_wr(ctl + C_MVD, 2u);
    if (PREF_FROM < 128) {
      const unsigned par_off = (call & 1u) * BUF_BYTES;
      unsigned pf = PREF_FROM, spins = 0;
      bool touched = !TOUCH;
#pragma unroll 1
      while (pf < 128 && ++spins < (1u << 22)) {
        const unsigned av = rfl(lds_rd(ctl + C_AVA));
        if (av <= pf) { if (av + 8 < pf) __builtin_amdgcn_s_sleep(32); else __builtin_amdgcn_s_sleep(1); continue; }
        if (!touched) {
          touched = true;
          const v4i ro = make_srd(P.out, DIM * DIM * 4);
          if (TOUCH == 1) { if (lane < 4) asm volatile("s_nop 4\n\tbuffer_load_dword v60, %0, %1, 0 offen" ::"v"((unsigned)(256 * ti + 85 * lane) * (DIM * 4) + (unsigned)tj * 1024), "s"(ro) : "memory"); }
          else if (TOUCH == 2) asm volatile("s_nop 4\n\tbuffer_load_dword v60, %0, %1, 0 offen" ::"v"((unsigned)(256 * ti + 4 * lane + (idx & 3)) * (DIM * 4) + (unsigned)tj * 1024), "s"(ro) : "memory");
          else for (int k = 0; k < 4; ++k) asm volatile("s_nop 4\n\tbuffer_load_dword v60, %0, %1, 0 offen" ::"v"((unsigned)(256 * ti + 4 * lane + k) * (DIM * 4) + (unsigned)tj * 1024), "s"(ro) : "memory");
        }
        const unsigned hs = pf + ((unsigned)lane >> 4);
        const unsigned off = par_off + plane_id(4 * (hs >> 1) + (unsigned)(idx & 3), (unsigned)rank) * PLANE_BLK + (16 * (hs & 1) + (unsigned)ti) * 2048 + ((unsigned)lane & 15u) * 128;
        if (hs < av && hs < 128) asm volatile("buffer_load_dword v60, %0, %1, 0 offen" ::"v"(off), "s"(srdL) : "memory");
        pf = av < pf + 4 ? av : pf + 4;
      }
    }
  } else if (wave == 9) {
    __builtin_amdgcn_s_setprio(2);
    lds_wr(ctl + lane * 4, 0u);
    wg_barrier();
    const bool direct = so == 1;
    if (!direct) build_table();
    get_call();
    {
      char* pb = wsl;
#pragma unroll
      for (int k = 0; k < 8; ++k) { char* t = P.ws[k]; pb = (lane == k || (lane >= 32 && ((lane >> 1) & 7) == k)) ? t : pb; }
      const int j = (lane - 8) >> 1, h = lane & 1;
      const unsigned pc = (unsigned)(sg < 16 ? h * 16 + sg : 32 + 2 * (sg - 16) + h);
      const unsigned off = lane < 8 ? RDY_OFF + (unsigned)(rank * 64 + (direct ? sg : 32 + so)) * 128
                         : lane < 16 ? PF_OFF + flag_idx(plane_of(j), pc) * 128
                         : lane == 16 ? CNT_OFF + (unsigned)b * 128
                         : lane < 32 ? TK_OFF + ((~call & 1u) * 8 + (unsigned)(lane - 17)) * 128
                         : PF_OFF + flag_idx(32u + 7u * ((unsigned)lane >> 4 & 1u) + ((unsigned)(rank - (lane >> 1) - 1) & 7u), pc) * 128;
      unsigned* dst = (unsigned*)(pb + off);
      const unsigned val = (lane > 16 && lane < 32) ? 0u : call;
      const bool on = lane < 8 ? (direct && lane != rank) : lane <= 16 ? true : lane < 32 ? (b == 255 && lane < 24) : (so == 0 && lane < 32 + 16 * NPUSH && ((lane >> 1) & 7) != rank);
      asm volatile("" : "+v"(dst));
      unsigned long long tp0 = 0, tp1 = 0;
      TS(tp0);
      lds_wait(ctl + C_STG, sg < 16 ? 8 : 4);
      TS(tp1);
      if (on) __hip_atomic_store(dst, val, __ATOMIC_RELAXED, __HIP_MEMORY_SCOPE_SYSTEM);
      if (direct) build_table();
      else {
        unsigned tk = 0;
        if (lane == 0) tk = __hip_atomic_fetch_add((unsigned*)(wsl + AGG_OFF + (unsigned)so * 128), 1u, __ATOMIC_RELAXED, __HIP_MEMORY_SCOPE_AGENT);
        if (rfl(tk) + 1u == call * 32u && lane < 8 && lane != rank) __hip_atomic_store(dst, val, __ATOMIC_RELAXED, __HIP_MEMORY_SCOPE_SYSTEM);
      }
      STAMPV(27, tp0); STAMPV(28, tp1);
    }
    __builtin_amdgcn_s_setprio(PRIO_POLL);
    const unsigned vav = lane < 2 ? ctl + C_AVA + lane * 4 : ctl + 256 + lane * 4;
    unsigned npoll = 0, headA = 0, headB = 0;
    unsigned wA = 31, wB = 31;
    bool mvdone = false, wide = true;
    const unsigned lateA = POLL_LATE - 14 * ((LAGS >> (4 * (idx >> 2))) & 15u);
    const unsigned long long t0 = rt();
#pragma unroll 1
    while ((headA < 512 || headB < 512 || !mvdone) && !NOCOMM && NOCONS != 1) {
      if (!mvdone) { const unsigned m = rfl(lds_rd(ctl + C_MVD)); mvdone = m == 2u; if (m == 1u) { lds_wr(vav, 0u); wide = true; wA = wB = POLL_W; __builtin_amdgcn_s_sleep(4); continue; } }
      const bool two = headA < 512 && headB < 512, isB = two ? lane >= 32 : headB < 512;
      const unsigned lo = two ? (unsigned)lane & 31u : (unsigned)lane;
      const unsigned pos = (isB ? headB : headA) + lo;
      unsigned v = call - 1;
      const bool gate = POLL_GATE && !wide && headA < lateA && headB < 480;
      if (gate ? lo == POLL_GATE : (lo < (isB ? wB : wA) && pos < 512))
        asm volatile("buffer_load_dword %0, %1, %2, 0 offen sc0 sc1\n\ts_waitcnt vmcnt(0)" : "=v"(v) : "v"(PF_OFF + ((isB ? 16u + tj : (unsigned)ti) * LSTR + pos) * 128), "s"(srdL) : "memory");
      unsigned long long hm = __ballot((int)(v - call) >= 0);
      if (gate) { wide = __builtin_popcountll(hm) == 2; if (wide) { wA = wB = POLL_W; continue; } hm = 0; }
      const unsigned d0 = (unsigned)__builtin_ctzll(~hm | (two ? 1ull << 31 : 1ull << 63));
      const unsigned da = headA < 512 ? d0 : 0u, db = two ? (unsigned)__builtin_ctz(~(unsigned)(hm >> 32) | 0x80000000u) : (headA < 512 ? 0u : d0);
      headA += da; headB += db;
      {
        const unsigned lim = headA < 320 ? (unsigned)(rt() - t0) / LOCAL_GAP + 1 : 255u;
        const unsigned ha = headA >> 2, hb = headB >> 3;
        lds_wr(vav, lane ? (hb < (lim >> 1) + 1 ? hb : (lim >> 1) + 1) : (ha < lim ? ha : lim)); }
      const bool more = !gate && (da == wA || db == wB);
      wide = more;
      const unsigned wn = headA < lateA ? POLL_N : POLL_N2;
      wA = da == wA ? (headB < 512 ? POLL_W : 63u) : wn; wB = db == wB ? (headA < 512 ? POLL_W : 63u) : wn;
      if (!more) { if (headA < lateA) __builtin_amdgcn_s_sleep(POLL_SLEEP); else __builtin_amdgcn_s_sleep(POLL_SLEEP2); }
      if ((da | db) == 0u && (++npoll & 15u) == 0u) {
        const unsigned tm = rfl(flag_ld(srdL, TMO_OFF));
        if (tm == call || rt() - t0 > DEADLINE) {
          flag_st1(srdL, TMO_OFF, call);
          lds_wr(ctl + C_ABORT, 1u);
          asm volatile("s_waitcnt lgkmcnt(0)" ::: "memory");
          break;
        }
      }
    }
    lds_wr(vav, 128u);
    STAMP(19);
  } else {
    wg_barrier();
    build_table();
    __builtin_amdgcn_s_setprio(PRIO_LD);
    get_call();
    if (turn == 0 && wave == 10 && !NOCOMM) {
      bool ok = wait_flag(RDY_OFF + (unsigned)(pp * 64 + (lane & 31)) * 128);
      if (ok) { lds_wr(ctl + C_RDY, 1u); ok = wait_flag(RDY_OFF + (unsigned)(pp * 64 + 32 + ((lane & 7) < 2 ? 2 : (lane & 7))) * 128); }
      if (ok) { lds_wr(ctl + C_RDY, 2u); ok = wait_flag(rdy_rest); }
      lds_wr(ctl + C_RDY, ok ? 3u : 9u);
    }
    const unsigned par_off = (call & 1u) * BUF_BYTES;
    auto loader = [&](auto IA) {
      constexpr bool isA = decltype(IA)::value;
      constexpr unsigned RINGB = isA ? NH * 8192 : NB * 16384;
      const unsigned v16 = (unsigned)lane * 16, v16b = v16 + 32768u;
      const unsigned vchk = lane < 8 ? ctl + lane * 16 + 12 : ctl + (isA ? C_AVA : C_AVB);
      const unsigned vfull = lane < 8 ? ctl + (unsigned)lane * 16 + (isA ? 0 : 4) : ctl + (isA ? 512 : 768) + lane * 4;
      const unsigned ring0 = lds0 + (isA ? LDS_A : LDS_B);
      unsigned ring = ring0, n = 0, nops = 0, spin = 1u << 24;
      const v4i srd = srdL;
      int vneed = lane < 8 ? (isA ? 2 - NH : 1 - 2 * NB) : (isA ? 2 : 1);
      const int vinc = (lane < 8 || isA) ? 2 : 1;
      const unsigned po = par_off + (isA ? (unsigned)ti * 2048 : 65536u + (unsigned)tj * 4096);
      const unsigned t0 = po + plane_id(4 * lane, rank) * PLANE_BLK, t1 = po + plane_id(4 * lane + 1, rank) * PLANE_BLK;
      const unsigned t2 = po + plane_id(4 * lane + 2, rank) * PLANE_BLK, t3 = po + plane_id(4 * lane + 3, rank) * PLANE_BLK;
      wait_vm<0>();
      if (NOCONS || NOCOMM == 2 || (NOCOMM == 3 && isA) || (NOCOMM == 4 && !isA)) { lds_wr(vfull, 255u); return; }
#define L_PUB \
      " s_getreg_b32 %[c], hwreg(HW_REG_IB_STS)\n" \
      " s_bfe_u32 %[x], %[c], 0x20016\n" \
      " s_and_b32 %[c], %[c], 15\n" \
      " s_lshl4_add_u32 %[c], %[x], %[c]\n" \
      " s_sub_u32 %[x], %[nops], %[c]\n" \
      " s_lshr_b32 %[x], %[x], 3\n" \
      " .if %c[isa] == 0\n s_and_b32 %[x], %[x], -2\n .endif\n" \
      " v_mov_b32 %[vl], %[x]\n" \
      " ds_write_b32 %[vfull], %[vl]\n"
#define L_D2(v, o, k) \
      " buffer_load_dwordx4 %[" v "], %[srd], %[" o "] offen offset:" k " lds\n" \
      " buffer_load_dwordx4 %[" v "], %[srd], %[" o "] offen offset:" k "+1024 lds\n"
#define L_A0(o) L_D2("v16", o, "0") " s_add_u32 m0, m0, 0x800\n s_nop 0\n"
#define L_A1(o, t) L_D2("v16b", o, "0") " s_add_u32 m0, m0, 0x800\n v_readlane_b32 %[" o "], %[" t "], %[n]\n"
#define L_B(o, t) L_D2("v16", o, "0") L_D2("v16", o, "2048") " s_add_u32 m0, m0, 0x1000\n v_readlane_b32 %[" o "], %[" t "], %[n]\n"
      unsigned v, vl, o0, o1, o2, o3, c, x, keep;
      asm volatile(
        " s_mov_b32 %[km0], m0\n"
        " v_readlane_b32 %[o0], %[t0], %[n]\n v_readlane_b32 %[o1], %[t1], %[n]\n"
        " v_readlane_b32 %[o2], %[t2], %[n]\n v_readlane_b32 %[o3], %[t3], %[n]\n"
        "Lpoll%=:\n"
        " ds_read_b32 %[v], %[vchk]\n"
        L_PUB
        " s_sub_u32 %[spin], %[spin], 1\n"
        " s_cbranch_scc1 Lout%=\n"
        LD_SLEEP
        " s_waitcnt lgkmcnt(1)\n"
        "Lchk%=:\n"
        " v_cmp_lt_i32 vcc, %[v], %[vneed]\n"
        " s_cmp_gt_u32 %[c], 47\n"
        " s_cbranch_scc1 Lpoll%=\n"
        " s_cbranch_vccnz Lpoll%=\n"
        " s_mov_b32 m0, %[ring]\n"
        " s_add_u32 %[n], %[n], 1\n"
        " .if %c[isa]\n"
        L_A0("o0") L_A0("o1") L_A0("o2") L_A0("o3")
        " s_add_u32 %[nops], %[nops], 8\n"
        L_PUB
        L_A1("o0", "t0") L_A1("o1", "t1") L_A1("o2", "t2") L_A1("o3", "t3")
        " s_add_u32 %[nops], %[nops], 8\n"
        " .else\n"
        L_B("o0", "t0") L_B("o1", "t1") L_B("o2", "t2") L_B("o3", "t3")
        " s_add_u32 %[nops], %[nops], 16\n"
        " .endif\n"
        " v_add_u32 %[vneed], %[vinc], %[vneed]\n"
        " s_cmp_eq_u32 m0, %[rend]\n"
        " s_cselect_b32 %[ring], %[ring0], m0\n"
        L_PUB
        " s_cmp_lt_u32 %[n], 64\n"
        " s_cbranch_scc1 Lchk%=\n"
        "Lout%=:\n"
        L_PUB
        " s_cmp_lg_u32 %[c], 0\n"
        " s_cbranch_scc1 Lout%=\n"
        " s_mov_b32 m0, %[km0]\n"
        : [v] "=&v"(v), [vneed] "+&v"(vneed), [vl] "=&v"(vl), [o0] "=&s"(o0), [o1] "=&s"(o1), [o2] "=&s"(o2), [o3] "=&s"(o3), [c] "=&s"(c),
          [x] "=&s"(x), [km0] "=&s"(keep), [n] "+&s"(n), [nops] "+&s"(nops), [ring] "+&s"(ring), [spin] "+&s"(spin)
        : [v16] "v"(v16), [v16b] "v"(v16b), [vchk] "v"(vchk), [vfull] "v"(vfull), [vinc] "v"(vinc), [t0] "v"(t0), [t1] "v"(t1), [t2] "v"(t2), [t3] "v"(t3),
          [srd] "s"(srd), [ring0] "s"(ring0), [rend] "s"(ring0 + RINGB), [isa] "i"(isA ? 1 : 0)
        : "vcc", "scc", "memory");
      if (n != 64u) lds_wr(ctl + C_ABORT, 1u);
      asm volatile("s_waitcnt lgkmcnt(0)" ::: "memory");
      lds_wr(vfull, 255u);
    };
    if (wave == 10) loader(std::true_type{}); else loader(std::false_type{});
    STAMP(20 + (wave & 1));
  }
}

extern "C" void kernel_launch(int rank, int nranks, ncclComm_t comm, hipStream_t stream,
                              void* const* d_in, const int* in_sizes, int n_in,
                              void* d_out, int out_size, const int* in_dtypes, int out_dtype,
                              void* const* d_out_all, void* const* d_ws_all, size_t ws_bytes) {
  (void)comm; (void)in_sizes; (void)n_in; (void)out_size; (void)in_dtypes; (void)out_dtype; (void)d_out_all;
  if (nranks != NR || (unsigned)rank >= (unsigned)NR || ws_bytes < WS_USED) { fprintf(stderr, "unexpected configuration\n"); return; }
  Params p;
  const size_t woff = WS_TAIL ? (ws_bytes - WS_USED) & ~(size_t)0xfffff : 0;
  for (int i = 0; i < NR; ++i) p.ws[i] = (char*)d_ws_all[i] + woff;
  p.x = (const char*)d_in[0]; p.w = (const char*)d_in[1];
  p.sx = (const float*)d_in[2]; p.sw = (const float*)d_in[3];
  p.out = (float*)d_out; p.wsl = p.ws[rank]; p.rank = rank;
  static bool attr_set[NR] = {};
  if (!attr_set[rank]) {
    HIP_CHECK(hipFuncSetAttribute((const void*)fused_gemm, hipFuncAttributeMaxDynamicSharedMemorySize, (int)LDS_BYTES));
    attr_set[rank] = true;
  }
  fused_gemm<<<GRID, 768, LDS_BYTES, stream>>>(p);
  HIP_CHECK(hipGetLastError());
}
```

```cpp
#include <hip/hip_runtime.h>
#include <rccl/rccl.h>
#include <cstdio>
#include <cstdint>
#include <type_traits>

#define HIP_CHECK(expr) do { hipError_t _e = (expr); if (_e != hipSuccess) { \
    fprintf(stderr, "HIP error %s at %s:%d\n", hipGetErrorString(_e), __FILE__, __LINE__); \
    return; } } while (0)

typedef int v4i __attribute__((ext_vector_type(4)));
typedef unsigned v4u __attribute__((ext_vector_type(4)));
typedef float v4f __attribute__((ext_vector_type(4)));
#define LAS __attribute__((address_space(3)))
#define CAS __attribute__((address_space(4)))

#define LAGS 0x77440000u
#define LAGW 0x7300u
#define NGR 3
#define LAST_LEAD 32
#define STG 24
#define STG_LAST 4
#define TURNS 0x807a70614d3419ull
#define PUSH_WIN 3
#define POLL_W 24
#define POLL_N 4
#define POLL_LATE 232
#define POLL_SLEEP 24
#define POLL_SLEEP2 2
#define LOCAL_GAP 100
#define PREF_FROM 96
#define CODE_KB 24
#define STAGE_DELAY 64
#define PRIO_MMA 0
#define PRIO_LD 3
#define PRIO_POLL 2
#define LD_SLEEP ""
#define POLL_N2 16
#define ST_WIN 63
#define ST_TURN 4
#define OUTF2 "sc1"
#define NOCOMM 0
#define GRID 256
#define DYNP 1
#define TOUCH 0
#define NOCONS 0
#define TLBW 0
#define POLL_GATE 12
#define NPUSH 1
#define EST_ADD 150
#define EARLY_RESUME 0
#define MMA_SLEEP 1
#define WS_TAIL 1
#define OUTF "sc1"
#define NH_SLOTS 10
#define NB_SLOTS 4
#define LSTR 273

constexpr int NR = 8;
constexpr int DIM = 4096, KSH = 512;
constexpr unsigned PLANE_BLK = 128u << 10;
constexpr unsigned SRC_BYTES = 32 * PLANE_BLK;
constexpr unsigned BUF_BYTES = NR * SRC_BYTES;
constexpr unsigned PF_OFF = 2 * BUF_BYTES;
constexpr unsigned RDY_OFF = PF_OFF + 32 * 600 * 128;
constexpr unsigned CNT_OFF = RDY_OFF + NR * 64 * 128;
constexpr unsigned TMO_OFF = CNT_OFF + 256 * 128;
constexpr unsigned AGG_OFF = TMO_OFF + 3072;
constexpr unsigned TK_OFF = TMO_OFF + 1024;
constexpr unsigned DUM_OFF = TMO_OFF + 4096;
constexpr unsigned DIAG_OFF = DUM_OFF + 256 * 256;
constexpr unsigned DIAG2_OFF = DIAG_OFF + 256 * 1024;
constexpr unsigned WS_USED = DIAG2_OFF + 256 * 2048;
constexpr unsigned long long DEADLINE = 30000000ull;

constexpr int NH = NH_SLOTS, NB = NB_SLOTS;
constexpr unsigned LDS_A = 0, LDS_B = NH * 8192, LDS_C = LDS_B + NB * 16384, LDS_T = LDS_C + 1024, LDS_L = LDS_T + 4096, LDS_BYTES = LDS_L + 1280;
static_assert(LDS_BYTES <= 163840 && NH % 2 == 0, "LDS");
constexpr unsigned C_ABORT = 128, C_MVD = 132, C_AV = 136, C_RDY = 144, C_STG = 148, C_PSH = 152, C_TAB = 156;
constexpr unsigned NTK = 32 * (32 - NPUSH);
constexpr int LPL = 4, NT = 7;
constexpr unsigned NST0 = LPL * (NGR + 1);
constexpr unsigned long long LAG12 = (unsigned long long)LAGS | ((unsigned long long)LAGW << 32);
constexpr int lag_of(int s) { return (int)((LAG12 >> (4 * s)) & 15u); }
constexpr int late_groups() { int r = 0; for (int s = 0; s < 12; ++s) r += (s < 8 ? 2 : 4) * lag_of(s); return r; }
constexpr int UEND = (int)NTK - late_groups() - LAST_LEAD;
constexpr int tile_lag(int idx) { return lag_of(idx >> 2) > lag_of(8 + (idx & 3)) ? lag_of(idx >> 2) : lag_of(8 + (idx & 3)); }
constexpr unsigned long long order8(int k) {
  unsigned long long r = 0;
  for (int i = 8 * k; i < 8 * k + 8; ++i) {
    int c = 0;
    for (int j = 0; j < 32; ++j) c += tile_lag(j) < tile_lag(i) || (tile_lag(j) == tile_lag(i) && j < i);
    r |= (unsigned long long)c << (8 * (i & 7));
  }
  return r;
}

struct Params {
  char* ws[NR];
  const char* x;
  const char* w;
  const float* sx;
  const float* sw;
  float* out;
  char* wsl;
  int rank;
};

__device__ __forceinline__ unsigned rfl(unsigned v) { return __builtin_amdgcn_readfirstlane(v); }
__device__ __forceinline__ v4i make_srd(const void* p, unsigned bytes) {
  unsigned long long a = (unsigned long long)p;
  v4i r;
  r.x = (int)rfl((unsigned)a); r.y = (int)(rfl((unsigned)(a >> 32)) & 0xffffu); r.z = (int)bytes; r.w = 0x00020000;
  return r;
}
__device__ __forceinline__ unsigned long long rt() { return __builtin_amdgcn_s_memrealtime(); }
template <int N> __device__ __forceinline__ void wait_vm() { asm volatile("s_waitcnt vmcnt(%0)" ::"i"(N) : "memory"); }
__device__ __forceinline__ unsigned lds_rd(unsigned addr) {
  unsigned v;
  asm volatile("ds_read_b32 %0, %1\n\ts_waitcnt lgkmcnt(0)" : "=v"(v) : "v"(addr) : "memory");
  return v;
}
__device__ __forceinline__ void lds_wr(unsigned addr, unsigned v) { asm volatile("ds_write_b32 %0, %1" ::"v"(addr), "v"(v) : "memory"); }
__device__ __forceinline__ void lds_inc(unsigned a) {
  asm volatile("s_mov_b64 exec, 1\n\tds_add_u32 %0, %1\n\ts_mov_b64 exec, -1" ::"v"(a), "v"(1u) : "memory");
}
__device__ __forceinline__ void lds_wait(unsigned a, unsigned n) { unsigned k = 0; while (rfl(lds_rd(a)) < n && ++k < (1u << 24)) { } }
__device__ __forceinline__ unsigned flag_ld(v4i srd, unsigned voff) {
  unsigned v;
  asm volatile("s_nop 4\n\tbuffer_load_dword %0, %1, %2, 0 offen sc0 sc1\n\ts_waitcnt vmcnt(0)" : "=v"(v) : "v"(voff), "s"(srd) : "memory");
  return v;
}
__device__ __forceinline__ void flag_st1(v4i srd, unsigned soff, unsigned v) {
  asm volatile("s_mov_b64 exec, 1\n\ts_nop 4\n\tbuffer_store_dword %0, off, %1, %2 sc0 sc1\n\ts_mov_b64 exec, -1" ::"v"(v), "s"(srd), "s"(soff) : "memory");
}
__device__ __forceinline__ unsigned flag_idx(unsigned g, unsigned si, unsigned q) { return g * LSTR + si * 32 + q; }
__host__ __device__ constexpr int stream_pos(int q, int g) {
  const int me = g < 16 ? (g >> 1) : 8 + (g & 3);
  const int key = q + (int)((LAG12 >> (4 * me)) & 15u);
  int u = g < 16 ? (g & 1) : ((g - 16) >> 2);
  for (int s = 0; s < 12; ++s) {
    const int c = key - lag_of(s) - NPUSH + (s < me ? 1 : 0);
    u += (s < 8 ? 2 : 4) * (c < 0 ? 0 : (c > 32 - NPUSH ? 32 - NPUSH : c));
  }
  return u;
}
constexpr unsigned long long first_items(int sub) {
  unsigned long long r = 0;
  for (int it = 0; it < 128; ++it) { const int u = stream_pos(NPUSH + it / 32, it % 32); if (u < (int)NST0 && u % LPL == sub) r |= (unsigned long long)it << (8 * (u / LPL)); }
  return r;
}
template <int I, int N, typename F> __device__ __forceinline__ void static_for(F&& f) {
  if constexpr (I < N) { f(std::integral_constant<int, I>{}); static_for<I + 1, N>(f); }
}

#define TS(x) do { } while (0)
#define STAMPV(idx, v) do { } while (0)
#define STAMP(idx) STAMPV(idx, rt())

constexpr int ACC0 = 40;
#define MV_T "s_mov_b64 exec, 1\n v_mov_b32 v[40+%c[g]], 1\n buffer_atomic_add v[40+%c[g]], off, %[sl], %[tk] sc0\n s_mov_b64 exec, -1\n"
#define MV_L "buffer_load_dwordx4 v[64+%c[g]*16+k*4:67+%c[g]*16+k*4], %[v16], %[sr], %[ol] offen offset:k*1024 sc0 sc1\n"
#define MV_S "buffer_store_dwordx4 v[64+%c[g]*16+k*4:67+%c[g]*16+k*4], %[v16], %[sl], %[os] offen offset:k*1024 sc1\n"
#define MV_F "s_mov_b64 exec, 1\n buffer_store_dword %[vc], off, %[sl], %[nf] sc1\n s_mov_b64 exec, -1\n"
template <int G> __device__ __forceinline__ void mv_load(v4i sl, v4i sr, unsigned tk, unsigned ol, unsigned v16) {
  asm volatile("s_nop 4\n" MV_T ".set k, 0\n .rept 4\n" MV_L ".set k, k+1\n .endr"
               ::[g] "i"(G), [sl] "s"(sl), [sr] "s"(sr), [tk] "s"(tk), [ol] "s"(ol), [v16] "v"(v16) : "memory");
}
template <int G, int W, int WK> __device__ __forceinline__ unsigned mv_swap(v4i sl, v4i sr, unsigned tk, unsigned os, unsigned ol, unsigned v16, unsigned tab, unsigned call, unsigned nf) {
  unsigned t;
  asm volatile("s_waitcnt vmcnt(%c[w]-1)\n v_min_u32 v56, %c[ntk], v[40+%c[g]]\n v_lshl_add_u32 v56, v56, 2, %[tab]\n ds_read_b32 v56, v56\n" MV_T
               ".set k, 0\n .rept 4\n s_waitcnt vmcnt(%c[w]+%c[wk]*k)\n" MV_S MV_L ".set k, k+1\n .endr\n" MV_F
               "s_waitcnt lgkmcnt(0)\n v_readfirstlane_b32 %[t], v56\n s_nop 1"
               : [t] "=s"(t) : [g] "i"(G), [w] "i"(W), [wk] "i"(WK), [sl] "s"(sl), [sr] "s"(sr), [tk] "s"(tk), [os] "s"(os), [ol] "s"(ol), [v16] "v"(v16),
                 [tab] "s"(tab), [ntk] "i"(NTK - NST0), [vc] "v"(call), [nf] "s"(nf) : "memory");
  return t;
}
template <int G, int W> __device__ __forceinline__ void mv_store(v4i sl, unsigned os, unsigned v16, unsigned call, unsigned nf) {
  asm volatile(".set k, 0\n .rept 4\n s_waitcnt vmcnt(%c[w]-k)\n" MV_S ".set k, k+1\n .endr\n" MV_F
               ::[g] "i"(G), [w] "i"(W), [sl] "s"(sl), [os] "s"(os), [v16] "v"(v16), [vc] "v"(call), [nf] "s"(nf) : "memory");
}
template <int R> __device__ __forceinline__ void zero_acc() {
  asm volatile("v_mov_b32 v%c0, 0\n\tv_mov_b32 v%c1, 0\n\tv_mov_b32 v%c2, 0\n\tv_mov_b32 v%c3, 0" ::"i"(R), "i"(R + 1), "i"(R + 2), "i"(R + 3));
  if constexpr (R + 4 < ACC0 + 128) zero_acc<R + 4>();
}
template <int E, int FL> __device__ __forceinline__ void store_row(unsigned sc, unsigned vo, v4i ro) {
  constexpr int R = ACC0 + (E >> 2) * 16 + (E & 3);
  float f0, f1, f2, f3;
  asm volatile("v_cvt_f32_i32 %0, v%c4\n\tv_cvt_f32_i32 %1, v%c5\n\tv_cvt_f32_i32 %2, v%c6\n\tv_cvt_f32_i32 %3, v%c7\n\t"
               "v_mul_f32 %0, %8, %0\n\tv_mul_f32 %1, %8, %1\n\tv_mul_f32 %2, %8, %2\n\tv_mul_f32 %3, %8, %3"
               : "=v"(f0), "=v"(f1), "=v"(f2), "=v"(f3) : "i"(R), "i"(R + 4), "i"(R + 8), "i"(R + 12), "s"(sc));
  v4f v = {f0, f1, f2, f3};
  constexpr unsigned so = (unsigned)(128 * (E >> 4) + 16 * ((E >> 2) & 3) + 4 * (E & 3)) * (DIM * 4);
  if constexpr (FL == 0) asm volatile("buffer_store_dwordx4 %0, %1, %2, %3 offen " OUTF "\n\ts_nop 1" ::"v"(v), "v"(vo), "s"(ro), "s"(so) : "memory");
  else asm volatile("buffer_store_dwordx4 %0, %1, %2, %3 offen " OUTF2 "\n\ts_nop 1" ::"v"(v), "v"(vo), "s"(ro), "s"(so) : "memory");
}
template <int E0, int E1, int FL> __device__ __forceinline__ void store_rows(unsigned sc, unsigned vo, v4i ro) {
  if constexpr (E0 < E1) { store_row<E0, FL>(sc, vo, ro); if constexpr (FL && ST_WIN < 63) wait_vm<ST_WIN>(); store_rows<E0 + 1, E1, FL>(sc, vo, ro); }
}

__global__ void __launch_bounds__(768) __attribute__((amdgpu_num_vgpr(20))) fused_gemm(Params P) {
  extern __shared__ __attribute__((aligned(16))) char lds[];
  const int tid = threadIdx.x, lane = tid & 63;
  if (CODE_KB > 0 && tid >= 768 - CODE_KB * 8) {
    unsigned long long pc;
    asm volatile("s_getpc_b64 %0\n\tglobal_load_dword v60, %1, %0\n\t.subsection 1\n\t.fill 6144, 4, 0xBF800000\n\t.subsection 0" : "=&s"(pc) : "v"((unsigned)(tid - (768 - CODE_KB * 8)) * 128u) : "memory");
  }
  const int wave = (int)rfl((unsigned)tid >> 6);
  const int b = blockIdx.x;
  const int rank = P.rank;
  char* const CAS* const ktab = (char* const CAS*)__builtin_amdgcn_kernarg_segment_ptr();
  char* const wsl = P.wsl;
  unsigned cv = __hip_atomic_load((const unsigned*)(wsl + CNT_OFF + (unsigned)b * 128), __ATOMIC_RELAXED, __HIP_MEMORY_SCOPE_AGENT);
  const unsigned lds0 = (unsigned)(unsigned long long)(LAS char*)lds;
  const unsigned ctl = lds0 + LDS_C;
  const int xcd = b & 7, idx = b >> 3;
  const int ti = 2 * (idx >> 2) + (xcd >> 2), tj = 4 * (xcd & 3) + (idx & 3);
  const int sg = b & 31, so = b >> 5;
  auto plane_of = [&](int j) { return (unsigned)(4 * so + j + 28 + NPUSH) & 31u; };
  const v4i srdL = make_srd(wsl, WS_USED);
  constexpr unsigned long long O0 = order8(0), O1 = order8(1), O2 = order8(2), O3 = order8(3);
  const int mp = (int)(((idx < 8 ? O0 : idx < 16 ? O1 : idx < 24 ? O2 : O3) >> (8 * (idx & 7))) & 31u) * 8 + xcd;
  const int turn = 7 - (mp >> 5), mpair = ((mp >> 3) & 3) * 4 + (mp & 3);
  const bool last = turn == NT, mover = mpair < 14;
  const int pi = mpair % 7, sub = (mpair / 7) * 2 + ((mp >> 2) & 1), pp = (rank + 1 + pi) & 7;
  const unsigned rdy_rest = RDY_OFF + (unsigned)(pp * 64 + 32 + ((lane & 7) == 1 ? 0 : (lane & 7))) * 128;
  auto turn_end = [&](int t) { return (unsigned)((int)((TURNS >> (8 * t)) & 255u) * UEND / 128 - (LPL - 1 - sub) * (STG - (STG - STG_LAST) * t / (NT - 1))); };
  auto entry = [&](unsigned it) { return it | (flag_idx(it & 31, (unsigned)pi, it >> 5) << 16); };
  auto build_table = [&]() {
    const unsigned uend = last ? 4096u : turn_end(turn);
#pragma unroll 1
    for (unsigned it = (unsigned)(tid - 576) + 32 * NPUSH; it < 1024; it += 192) {
      const unsigned u = (unsigned)stream_pos((int)(it >> 5), (int)(it & 31));
      lds_wr(lds0 + LDS_T + u * 4, entry(it) | (u >= uend ? 0x4000u : 0u));
    }
    asm volatile("s_waitcnt lgkmcnt(0)" ::: "memory");
    lds_inc(ctl + C_TAB);
  };
  if (wave == 0) STAMP(0);
  unsigned call = 0;
  auto get_call = [&]() { unsigned c; asm volatile("v_add_u32 %0, 1, %1" : "=v"(c) : "v"(cv) : "memory"); call = rfl(c); };
  auto wg_barrier = [&]() { asm volatile("s_waitcnt lgkmcnt(0)\n\ts_barrier" ::: "memory"); };
  auto wait_flag = [&](unsigned off) {
    const unsigned long long t0 = rt();
#pragma unroll 1
    for (unsigned n = 1;; ++n) {
      const unsigned v = flag_ld(srdL, off);
      if (__all((int)(v - call) >= 0)) return true;
      if ((n & 15u) == 0u) {
        const unsigned tm = rfl(flag_ld(srdL, TMO_OFF));
        if (tm == call || rt() - t0 > DEADLINE) { flag_st1(srdL, TMO_OFF, call); return false; }
      }
      __builtin_amdgcn_s_sleep(1);
    }
  };

  if (wave < 8) {
    unsigned long long tq = 0;
    {
      __amdgpu_buffer_rsrc_t rs = __builtin_amdgcn_make_buffer_rsrc(wsl, 0, 2 * BUF_BYTES, 0x00020000);
      const unsigned base = (unsigned)rank * SRC_BYTES;
      char* pk[7] = {};
      const bool lazy = so >= 2 && turn != 0;
      if (lazy) { wg_barrier(); __builtin_amdgcn_s_sleep(STAGE_DELAY); }
      auto load_pk = [&]() {
        asm volatile("" ::: "memory");
#pragma unroll
        for (int k = 0; k < 7; ++k) { pk[k] = ktab[(rank + 1 + (k + wave + b) % 7) & 7]; asm volatile("" : "+s"(pk[k])); }
      };
      if (sg < 16) {
        const int r = 32 * wave + (lane & 31), j = lane >> 5;
        const char* src = P.x + (size_t)(256 * sg + ((r & ~15) | ((r & 3) << 2) | ((r >> 2) & 3))) * KSH + 64 * so + 16 * j;
        v4u v0 = *(const v4u*)src, v1 = *(const v4u*)(src + 32);
        if (!lazy) wg_barrier();
        if (so == 0) load_pk();
        unsigned ncv;
        asm volatile("v_not_b32 %0, %1" : "=v"(ncv) : "v"(cv) : "memory");
        const unsigned d0 = base + (ncv & 1u) * BUF_BYTES + (unsigned)sg * 4096 + (unsigned)r * 16;
        const unsigned d = d0 + plane_of(j + 2) * PLANE_BLK;
        __builtin_amdgcn_raw_buffer_store_b128(v0, rs, d0 + plane_of(j) * PLANE_BLK, 0, 17);
        __builtin_amdgcn_raw_buffer_store_b128(v1, rs, d, 0, 17);
        TS(tq);
        if (so == 0) {
#pragma unroll
          for (int k = 0; k < 7; ++k) {
            const v4i sp = make_srd(pk[k], 2 * BUF_BYTES);
            asm volatile("s_mov_b32 exec_lo, %c3\n\ts_nop 4\n\tbuffer_store_dwordx4 %0, %1, %2, 0 offen sc0 sc1\n\ts_mov_b64 exec, -1" ::"v"(v1), "v"(d), "s"(sp), "i"(NPUSH == 2 ? -1 : 0) : "memory");
            if (k >= PUSH_WIN - 1) wait_vm<PUSH_WIN - 1>();
          }
        }
        wait_vm<0>(); lds_inc(ctl + C_STG);
      } else if (wave < 4) {
        const int gw = sg - 16;
        const char* src = P.w + (size_t)(64 * so + 16 * wave) * DIM + 256 * gw + 4 * lane;
        unsigned r[16];
#pragma unroll
        for (int k = 0; k < 16; ++k) r[k] = *(const unsigned*)(src + (size_t)k * DIM);
        if (!lazy) wg_barrier();
        if (so == 0) load_pk();
        unsigned ncv;
        asm volatile("v_not_b32 %0, %1" : "=v"(ncv) : "v"(cv) : "memory");
        v4u o[4];
#pragma unroll
        for (int g = 0; g < 4; ++g) {
          unsigned t0 = __builtin_amdgcn_perm(r[4 * g + 1], r[4 * g], 0x05010400u), t1 = __builtin_amdgcn_perm(r[4 * g + 1], r[4 * g], 0x07030602u);
          unsigned t2 = __builtin_amdgcn_perm(r[4 * g + 3], r[4 * g + 2], 0x05010400u), t3 = __builtin_amdgcn_perm(r[4 * g + 3], r[4 * g + 2], 0x07030602u);
          o[0][g] = __builtin_amdgcn_perm(t2, t0, 0x05040100u); o[1][g] = __builtin_amdgcn_perm(t2, t0, 0x07060302u);
          o[2][g] = __builtin_amdgcn_perm(t3, t1, 0x05040100u); o[3][g] = __builtin_amdgcn_perm(t3, t1, 0x07060302u);
        }
        const unsigned d = base + (ncv & 1u) * BUF_BYTES + plane_of(wave) * PLANE_BLK + 65536u + (unsigned)gw * 4096 + (unsigned)(64 * (lane >> 4) + (lane & 15)) * 16;
#pragma unroll
        for (int jj = 0; jj < 4; ++jj) __builtin_amdgcn_raw_buffer_store_b128(o[jj], rs, d + jj * 256, 0, 17);
        TS(tq);
        if (so == 0 && wave >= 4 - NPUSH) {
#pragma unroll
          for (int k = 0; k < 7; ++k) {
            const v4i sp = make_srd(pk[k], 2 * BUF_BYTES);
            asm volatile("s_nop 4\n\tbuffer_store_dwordx4 %0, %4, %5, 0 offen sc0 sc1\n\tbuffer_store_dwordx4 %1, %4, %5, 0 offen offset:256 sc0 sc1\n\t"
                         "buffer_store_dwordx4 %2, %4, %5, 0 offen offset:512 sc0 sc1\n\tbuffer_store_dwordx4 %3, %4, %5, 0 offen offset:768 sc0 sc1"
                         ::"v"(o[0]), "v"(o[1]), "v"(o[2]), "v"(o[3]), "v"(d), "s"(sp) : "memory");
            if (k >= PUSH_WIN - 1) wait_vm<4 * (PUSH_WIN - 1)>();
          }
        }
        wait_vm<0>(); lds_inc(ctl + C_STG);
      } else if (!lazy) wg_barrier();
    }
    if (wave == 0) { STAMP(1); STAMPV(26, tq); }
    asm volatile("" ::: "v167");
    if (NOCONS) return;
    zero_acc<ACC0>();
    __builtin_amdgcn_s_setprio(PRIO_MMA);
    const int wr = wave >> 2, wc = wave & 3;
    unsigned pa = lds0 + LDS_A + (unsigned)(lane >> 4) * 2048 + (unsigned)(64 * wr + (lane & 15)) * 16;
    unsigned pb = lds0 + LDS_B + (unsigned)(lane >> 4) * 4096 + (unsigned)(64 * wc + (lane & 15)) * 16;
    const unsigned vw = ctl + (unsigned)wave * 16 + (unsigned)(lane & 3) * 4;
    const unsigned vd = (lane & 1) ? ctl + (unsigned)(wave ^ 4) * 16 + 8 : ctl + (unsigned)wave * 16 + 12;
    unsigned vo = ((unsigned)(256 * ti + 64 * wr) + (unsigned)(lane >> 4)) * (DIM * 4) + ((unsigned)(256 * tj + 64 * wc) + 4 * (unsigned)(lane & 15)) * 4;
    asm volatile("" : "+v"(vo));
    unsigned sc = rfl(__builtin_bit_cast(unsigned, (*P.sx) * (*P.sw)));
    const v4i ro = make_srd(P.out, DIM * DIM * 4);
    unsigned vcnt = 0, vf, ia = 0, ib = 0, cnt, da, db, spin = 1u << 23;
    unsigned long long ra;
    asm volatile(
      ".macro MM h, i, j\n"
      " v_mfma_i32_16x16x64_i8 v[40+\\h*64+\\i*16+\\j*4:43+\\h*64+\\i*16+\\j*4], v[24+\\i*4:27+\\i*4], v[8+\\j*4:11+\\j*4], v[40+\\h*64+\\i*16+\\j*4:43+\\h*64+\\i*16+\\j*4]\n"
      ".endm\n"
      ".macro STEP h, nb, pm, last\n"
      " s_waitcnt lgkmcnt(4+\\pm)\n"
      " MM \\h, 0, 0\n"
      " s_cmp_eq_u32 %[ia], %c[nh1]\n"
      " s_cselect_b32 %[da], %[wa], 0x2000\n"
      " MM \\h, 1, 0\n"
      " s_cselect_b32 %[ia], -1, %[ia]\n"
      " s_add_u32 %[ia], %[ia], 1\n"
      " .if \\pm\n s_waitcnt lgkmcnt(4)\n .endif\n"
      " MM \\h, 0, 1\n"
      " v_add_u32 %[pa], %[da], %[pa]\n"
      " MM \\h, 1, 1\n"
      " .if \\nb\n s_cmp_eq_u32 %[ib], %c[nb1]\n s_cselect_b32 %[db], %[wb], 0x4000\n .endif\n"
      " .if \\pm\n s_waitcnt lgkmcnt(3)\n .endif\n"
      " MM \\h, 0, 2\n"
      " v_add_u32 %[vcnt], 1, %[vcnt]\n"
      " .if \\nb\n s_cselect_b32 %[ib], -1, %[ib]\n s_add_u32 %[ib], %[ib], 1\n .endif\n"
      " MM \\h, 1, 2\n"
      " .if \\nb\n v_add_u32 %[pb], %[db], %[pb]\n .endif\n"
      " s_waitcnt lgkmcnt(2)\n"
      " MM \\h, 0, 3\n"
      " .if \\last == 0\n v_cmp_le_u32 vcc, %[vf], %[vcnt]\n .endif\n"
      " MM \\h, 1, 3\n"
      " .if \\last == 0\n"
      " s_getpc_b64 %[ra]\n"
      " s_and_b32 %[da], vcc_lo, 3\n"
      " s_cbranch_scc1 Lslow%=\n"
      " ds_read_b128 v[24:27], %[pa]\n"
      " ds_read_b128 v[28:31], %[pa] offset:256\n"
      " .endif\n"
      " s_waitcnt lgkmcnt(2-2*\\last)\n"
      " MM \\h, 2, 0\n"
      " .if \\last == 0\n"
      " ds_write_b32 %[vd], %[vcnt]\n"
      " ds_read_b32 %[vf], %[vw]\n"
      " .endif\n"
      " MM \\h, 3, 0\n"
      " .if \\nb\n ds_read_b128 v[8:11], %[pb]\n .endif\n"
      " MM \\h, 2, 1\n"
      " .if (\\last == 0) && %c[dynp]\n"
      " s_bitcmp1_b32 vcc_lo, 2\n"
      " s_setprio 0\n"
      " s_cbranch_scc1 1f\n"
      " s_setprio 1\n"
      "1:\n"
      " .endif\n"
      " MM \\h, 3, 1\n"
      " .if \\nb\n ds_read_b128 v[12:15], %[pb] offset:256\n s_sub_u32 %[cnt], %[cnt], 1\n .endif\n"
      " MM \\h, 2, 2\n"
      " MM \\h, 3, 2\n"
      " .if \\nb\n ds_read_b128 v[16:19], %[pb] offset:512\n s_cmp_lg_u32 %[cnt], 0\n .endif\n"
      " MM \\h, 2, 3\n"
      " MM \\h, 3, 3\n"
      " .if \\nb\n ds_read_b128 v[20:23], %[pb] offset:768\n .endif\n"
      " .if \\last == 0\n"
      " ds_read_b128 v[32:35], %[pa] offset:512\n"
      " ds_read_b128 v[36:39], %[pa] offset:768\n"
      " .endif\n"
      ".endm\n"
      " s_mov_b64 vcc, -1\n"
      " s_getpc_b64 %[ra]\n"
      " s_and_b32 %[da], vcc_lo, 3\n"
      " s_cbranch_scc1 Lslow%=\n"
      " ds_read_b128 v[24:27], %[pa]\n ds_read_b128 v[28:31], %[pa] offset:256\n"
      " ds_read_b128 v[8:11], %[pb]\n ds_read_b128 v[12:15], %[pb] offset:256\n"
      " ds_read_b128 v[16:19], %[pb] offset:512\n ds_read_b128 v[20:23], %[pb] offset:768\n"
      " ds_read_b128 v[32:35], %[pa] offset:512\n ds_read_b128 v[36:39], %[pa] offset:768\n"
      " s_mov_b32 %[cnt], 63\n"
      "Lfull%=:\n"
      " STEP 0, 0, 1, 0\n"
      " STEP 1, 1, 0, 0\n"
      " s_cbranch_scc1 Lfull%=\n"
      " STEP 0, 0, 1, 0\n"
      " STEP 1, 0, 0, 1\n"
      " s_branch Lend%=\n"
      "Lslow%=:\n"
      " ds_read_b32 %[vf], %[vw]\n"
      " s_sub_u32 %[spin], %[spin], 1\n"
      " s_waitcnt lgkmcnt(0)\n"
      " ds_write_b32 %[vd], %[vcnt]\n"
      " v_cmp_le_u32 vcc, %[vf], %[vcnt]\n"
      " s_cbranch_scc1 Lgive%=\n"
      " s_and_b32 %[da], vcc_lo, 3\n"
      " s_cbranch_scc0 Lret%=\n"
      " s_sleep %c[msl]\n"
      " s_branch Lslow%=\n"
      "Lgive%=:\n"
      " s_mov_b32 %[spin], 0\n"
      " s_mov_b32 %[sc], 0x7fc00000\n"
      " s_mov_b64 vcc, 0\n"
      "Lret%=:\n"
      " s_setpc_b64 %[ra]\n"
      "Lend%=:\n"
      " s_nop 7\n s_nop 1\n"
      ".purgem STEP\n.purgem MM\n"
      : [pa] "+&v"(pa), [pb] "+&v"(pb), [vcnt] "+&v"(vcnt), [vf] "=&v"(vf), [ia] "+&s"(ia), [ib] "+&s"(ib), [cnt] "=&s"(cnt), [da] "=&s"(da),
        [db] "=&s"(db), [spin] "+&s"(spin), [sc] "+&s"(sc), [ra] "=&s"(ra)
      : [vw] "v"(vw), [vd] "v"(vd), [dynp] "i"(DYNP), [msl] "i"(MMA_SLEEP), [nh1] "i"(NH - 1), [nb1] "i"(NB - 1), [wa] "s"(-(NH - 1) * 8192), [wb] "s"(-(NB - 1) * 16384)
      : "v8", "v9", "v10", "v11", "v12", "v13", "v14", "v15", "v16", "v17", "v18", "v19", "v20", "v21", "v22", "v23", "v24", "v25", "v26", "v27",
        "v28", "v29", "v30", "v31", "v32", "v33", "v34", "v35", "v36", "v37", "v38", "v39", "vcc", "scc", "memory");
    if (wave == 0) STAMP(3);
    sc = rfl(lds_rd(ctl + C_ABORT)) != 0u ? 0x7fc00000u : sc;
    if (turn >= ST_TURN && turn < NT) store_rows<0, 32, 1>(sc, vo, ro); else store_rows<0, 32, 0>(sc, vo, ro);
  } else if (wave == 8) {
    wg_barrier();
    lds_wr(lds0 + LDS_T + (NTK + (lane & 15)) * 4, 0xffffffffu);
    get_call();
    if (mover && !NOCOMM) {
      __builtin_amdgcn_s_setprio(3);
      const v4i srdR = make_srd(ktab[pp], 2 * BUF_BYTES);
      const unsigned v16 = (unsigned)lane * 16;
      const unsigned sbase = (call & 1u) * BUF_BYTES + (unsigned)pp * SRC_BYTES;
      unsigned tk_off = TK_OFF + ((call & 1u) * 8 + (unsigned)pi) * 128;
      const unsigned tk_dead = DUM_OFF + (unsigned)b * 256 + 64;
      const unsigned go_out = DUM_OFF + (unsigned)(last ? mp : mp - 32) * 256 + 192, go_in = DUM_OFF + (unsigned)mp * 256 + 192;
      flag_st1(srdL, tk_dead, 0x1000u);
      unsigned long long tm0 = 0, tm2 = 0;
      unsigned ev;
      bool ok;
      if (turn == 0) {
        lds_wr(ctl + C_MVD, 1u);
        unsigned long long fi = 0;
        static_for<0, LPL>([&](auto S) { constexpr unsigned long long F = first_items(decltype(S)::value); fi = sub == decltype(S)::value ? F : fi; });
        const unsigned it = (unsigned)(fi >> (8 * ((lane & 7) < NGR ? (lane & 7) : NGR))) & 255u, g = it & 31u;
        ev = entry(32u * NPUSH + it);
        asm volatile("" : "+v"(ev));
        TS(tm0);
        ok = wait_flag(RDY_OFF + ((unsigned)pp * 64 + g) * 128);
      } else {
        const unsigned long long t0 = rt();
        const unsigned est = turn_end(turn - 1) * 67u / 10u + EST_ADD;
        while ((unsigned)(rt() - t0) < est) __builtin_amdgcn_s_sleep(16);
        ok = wait_flag(rdy_rest) && wait_flag(go_in);
        lds_wr(ctl + C_MVD + (unsigned)(lane & 1) * 4, (lane & 1) ? 0u : 1u);
        unsigned x0;
        asm volatile("s_mov_b64 exec, 1\n\tv_mov_b32 v47, %c3\n\ts_nop 4\n\tbuffer_atomic_add v47, off, %1, %2 sc0\n\ts_mov_b64 exec, -1\n\t"
                     "s_waitcnt vmcnt(0)\n\tv_readfirstlane_b32 %0, v47\n\ts_nop 1" : "=s"(x0) : "s"(srdL), "s"(ok ? tk_off : tk_dead), "i"(NGR + 1) : "memory");
        const unsigned xe = x0 + NST0 + (unsigned)(lane & 7);
        lds_wait(ctl + C_TAB, 3);
        ev = lds_rd(lds0 + LDS_T + (xe < NTK ? xe : NTK) * 4);
      }
      if (ok) {
        const unsigned fdum = DUM_OFF + (unsigned)b * 256;
        unsigned os[NGR], en[NGR], pf[NGR];
        unsigned e;
        unsigned rdy = turn != 0 ? 4096u : (4u + NPUSH) * 32;
        auto foff = [&](unsigned x) { return x == 0xffffffffu ? fdum : PF_OFF + (x >> 16) * 128; };
        auto offs = [&](unsigned x) {
          if (__builtin_expect((x & 0x4000u) != 0u && tk_off != tk_dead, 0)) {
            tk_off = tk_dead;
            flag_st1(srdL, go_out, call);
          }
          return sbase + (x == 0xffffffffu ? 32u * NPUSH : x & 1023u) * 4096;
        };
        auto gate = [&]() {
          if (__builtin_expect((e & 1023u) >= rdy && e != 0xffffffffu, 0)) {
            const unsigned lvl = (e & 1023u) < (28 + NPUSH) * 32 ? 2u : 3u;
            unsigned r, spins = 0;
            while ((r = rfl(lds_rd(ctl + C_RDY))) < lvl && ++spins < (1u << 23)) __builtin_amdgcn_s_sleep(2);
            rdy = lvl == 2u ? (28u + NPUSH) * 32 : 4096u;
            if (r < lvl || r > 3u) e = 0xffffffffu;
          }
        };
        auto swap = [&](auto G, auto W, auto WK) {
          constexpr int g = decltype(G)::value;
          const unsigned ol = offs(e), nf = pf[g], o = os[g];
          pf[g] = en[g]; en[g] = foff(e); os[g] = ol;
          e = mv_swap<g, decltype(W)::value, decltype(WK)::value>(srdL, srdR, tk_off, o, ol, v16, lds0 + LDS_T + NST0 * 4, call, nf);
        };
        static_for<0, NGR>([&](auto G) {
          constexpr int g = decltype(G)::value;
          const unsigned x = __builtin_amdgcn_readlane(ev, g);
          pf[g] = fdum; en[g] = foff(x); os[g] = offs(x);
          mv_load<g>(srdL, srdR, tk_off, os[g], v16);
        });
        e = __builtin_amdgcn_readlane(ev, NGR);
        if (turn == 0) {
          lds_wait(ctl + C_TAB, 3);
          lds_wait(ctl + C_RDY, 1);
          TS(tm2);
        }
        static_for<0, NGR>([&](auto G) {
          constexpr int g = decltype(G)::value;
          gate();
          swap(G, std::integral_constant<int, 4 + (NGR - 1 - g) * 5 + g * 10>{}, std::integral_constant<int, 1>{});
        });
        int stop = -1;
#pragma unroll 1
        while (stop < 0) {
          static_for<0, NGR>([&](auto G) {
            constexpr int g = decltype(G)::value;
            if (stop < 0) {
              gate();
              if (e == 0xffffffffu) stop = g;
              else {
                swap(G, std::integral_constant<int, 8 + (NGR - 1) * 10>{}, std::integral_constant<int, 0>{});
              }
            }
          });
        }
        if (tk_off != tk_dead) flag_st1(srdL, go_out, call);
        if (EARLY_RESUME) lds_wr(ctl + C_MVD, 2u);
        static_for<0, NGR>([&](auto G) {
          constexpr int g = decltype(G)::value;
          if (stop == g) static_for<0, NGR>([&](auto J) {
            constexpr int j = decltype(J)::value, h = (g + j) % NGR;
            mv_store<h, 7 + (NGR - 1 - j) * 10 + j * 5>(srdL, os[h], v16, call, pf[h]);
          });
        });
        wait_vm<0>();
        static_for<0, NGR>([&](auto G) { flag_st1(srdL, en[decltype(G)::value], call); });
        lds_wr(ctl + C_MVD, 2u);
      }
      __builtin_amdgcn_s_setprio(0);
    }
    lds_wr(ctl + C_MVD, 2u);
  } else if (wave == 9) {
    __builtin_amdgcn_s_setprio(2);
    lds_wr(ctl + lane * 4, 0u);
    wg_barrier();
    const bool direct = so == 1;
    if (!direct) build_table();
    get_call();
    {
      char* pb = wsl;
#pragma unroll
      for (int k = 0; k < 8; ++k) { char* t = P.ws[k]; pb = (lane == k || (lane >= 32 && (lane & 7) == k)) ? t : pb; }
      const unsigned off = lane < 8 ? RDY_OFF + (unsigned)(rank * 64 + (direct ? sg : 32 + so)) * 128
                         : lane < 16 ? PF_OFF + flag_idx((unsigned)sg, 7u, plane_of(lane - 8)) * 128
                         : lane == 16 ? CNT_OFF + (unsigned)b * 128
                         : lane < 32 ? TK_OFF + ((~call & 1u) * 8 + (unsigned)(lane - 17)) * 128
                         : PF_OFF + flag_idx((unsigned)sg, (unsigned)(rank - lane - 1) & 7u, (unsigned)lane >> 3 & 1u) * 128;
      unsigned* dst = (unsigned*)(pb + off);
      const unsigned val = (lane > 16 && lane < 32) ? 0u : call;
      const bool on = lane < 8 ? (direct && lane != rank) : lane <= 16 ? (lane < 12 || lane == 16) : lane < 32 ? (b == 255 && lane < 24) : (so == 0 && lane < 32 + 8 * NPUSH && (lane & 7) != rank);
      asm volatile("" : "+v"(dst));
      unsigned long long tp0 = 0, tp1 = 0;
      TS(tp0);
      lds_wait(ctl + C_STG, sg < 16 ? 8 : 4);
      TS(tp1);
      if (on) __hip_atomic_store(dst, val, __ATOMIC_RELAXED, __HIP_MEMORY_SCOPE_SYSTEM);
      if (direct) build_table();
      else {
        unsigned tk = 0;
        if (lane == 0) tk = __hip_atomic_fetch_add((unsigned*)(wsl + AGG_OFF + (unsigned)so * 128), 1u, __ATOMIC_RELAXED, __HIP_MEMORY_SCOPE_AGENT);
        if (rfl(tk) + 1u == call * 32u && lane < 8 && lane != rank) __hip_atomic_store(dst, val, __ATOMIC_RELAXED, __HIP_MEMORY_SCOPE_SYSTEM);
      }
      STAMPV(27, tp0); STAMPV(28, tp1);
    }
    __builtin_amdgcn_s_setprio(PRIO_POLL);
    const unsigned si = (unsigned)lane >> 3, ro = ((unsigned)lane >> 1) & 3u;
    const unsigned fbase = PF_OFF + flag_idx((lane & 1) ? 16u + tj : (unsigned)ti, si, ro) * 128;
    const unsigned pbase = (call & 1u) * BUF_BYTES + (((unsigned)rank + 1 + si) & 7u) * SRC_BYTES + ro * PLANE_BLK;
    const unsigned lateN = POLL_LATE - 7 * (unsigned)(((LAGS >> (4 * (idx >> 2))) & 15u) > ((LAGW >> (4 * (idx & 3))) & 15u) ? ((LAGS >> (4 * (idx >> 2))) & 15u) : ((LAGW >> (4 * (idx & 3))) & 15u));
    unsigned q = 0, cnt = 0, npoll = 0, win = 4;
    bool mvdone = false;
    const unsigned long long t0 = rt();
#pragma unroll 1
    while ((cnt < 256 || !mvdone) && NOCONS != 1) {
      if (!mvdone) { const unsigned m = rfl(lds_rd(ctl + C_MVD)); mvdone = m == 2u; if (m == 1u) { lds_wr(ctl + C_AV, 0u); win = 4; __builtin_amdgcn_s_sleep(4); continue; } }
      unsigned v = call - 1;
      if (ro < win && q + ro < 32) {
        if (NOCOMM) v = call;
        else asm volatile("buffer_load_dword %0, %1, %2, 0 offen sc0 sc1\n\ts_waitcnt vmcnt(0)" : "=v"(v) : "v"(fbase + q * 128), "s"(srdL) : "memory");
      }
      const unsigned long long hm = __ballot((int)(v - call) >= 0);
      const unsigned adv = (unsigned)__builtin_ctz(~((unsigned)(hm >> (8 * si)) & 255u)) >> 1;
      const bool emit = !(lane & 1) && ro < adv;
      const unsigned long long em = __ballot(emit);
      if (emit) lds_wr(lds0 + LDS_L + (cnt + __builtin_amdgcn_mbcnt_hi((unsigned)(em >> 32), __builtin_amdgcn_mbcnt_lo((unsigned)em, 0u))) * 4, pbase + q * PLANE_BLK);
      q += adv;
      cnt += (unsigned)__builtin_popcountll(em);
      const unsigned lim = (cnt < 160 && !NOCOMM) ? (unsigned)(rt() - t0) / LOCAL_GAP + 1 : 255u;
      lds_wr(ctl + C_AV, (cnt >> 2) < lim ? cnt >> 2 : lim);
      const bool more = __ballot(adv == win && q < 32) != 0ull;
      win = more ? 4u : 2u;
      if (!more) { if (cnt < lateN) __builtin_amdgcn_s_sleep(POLL_SLEEP); else __builtin_amdgcn_s_sleep(POLL_SLEEP2); }
      if (em == 0ull && (++npoll & 15u) == 0u) {
        const unsigned tm = rfl(flag_ld(srdL, TMO_OFF));
        if (tm == call || rt() - t0 > DEADLINE) {
          flag_st1(srdL, TMO_OFF, call);
          lds_wr(ctl + C_ABORT, 1u);
#pragma unroll 1
          for (unsigned k = (unsigned)lane; k < 256; k += 64) lds_wr(lds0 + LDS_L + k * 4, 0u);
          asm volatile("s_waitcnt lgkmcnt(0)" ::: "memory");
          break;
        }
      }
    }
    lds_wr(ctl + C_AV, 64u);
    STAMP(19);
  } else {
    wg_barrier();
    build_table();
    __builtin_amdgcn_s_setprio(PRIO_LD);
    get_call();
    if (turn == 0 && wave == 10 && !NOCOMM) {
      bool ok = wait_flag(RDY_OFF + (unsigned)(pp * 64 + (lane & 31)) * 128);
      if (ok) { lds_wr(ctl + C_RDY, 1u); ok = wait_flag(RDY_OFF + (unsigned)(pp * 64 + 32 + ((lane & 7) < 2 ? 2 : (lane & 7))) * 128); }
      if (ok) { lds_wr(ctl + C_RDY, 2u); ok = wait_flag(rdy_rest); }
      lds_wr(ctl + C_RDY, ok ? 3u : 9u);
    }
    auto loader = [&](auto IA) {
      constexpr bool isA = decltype(IA)::value;
      constexpr unsigned RINGB = isA ? NH * 8192 : NB * 16384;
      const unsigned v16 = (unsigned)lane * 16 + (isA ? (unsigned)ti : 16u + tj) * 4096, v16b = v16 + 2048u;
      unsigned vchk = lane < 8 ? ctl + lane * 16 + 12 : lane < 16 ? ctl + C_AV : lds0 + LDS_L + (unsigned)(lane - 16) * 4;
      const unsigned vadv = lane < 16 ? 0u : 16u;
      const unsigned vfull = lane < 8 ? ctl + (unsigned)lane * 16 + (isA ? 0 : 4) : ctl + (isA ? 512 : 768) + lane * 4;
      const unsigned ring0 = lds0 + (isA ? LDS_A : LDS_B);
      unsigned ring = ring0, n = 0, nops = 0, spin = 1u << 24;
      const v4i srd = srdL;
      int vneed = lane < 8 ? (isA ? 2 - NH : 1 - 2 * NB) : lane < 16 ? 1 : (int)0x80000000u;
      const int vinc = lane < 8 ? 2 : lane < 16 ? 1 : 0;
      wait_vm<0>();
      if (NOCONS || NOCOMM == 2) { lds_wr(vfull, 255u); return; }
#define L_PUB \
      " s_getreg_b32 %[c], hwreg(HW_REG_IB_STS)\n" \
      " s_bfe_u32 %[x], %[c], 0x20016\n" \
      " s_and_b32 %[c], %[c], 15\n" \
      " s_lshl4_add_u32 %[c], %[x], %[c]\n" \
      " s_sub_u32 %[x], %[nops], %[c]\n" \
      " s_lshr_b32 %[x], %[x], 3\n" \
      " .if %c[isa] == 0\n s_and_b32 %[x], %[x], -2\n .endif\n" \
      " v_mov_b32 %[vl], %[x]\n" \
      " ds_write_b32 %[vfull], %[vl]\n"
#define L_D2(v, o, k) \
      " buffer_load_dwordx4 %[" v "], %[srd], %[" o "] offen offset:" k " lds\n" \
      " buffer_load_dwordx4 %[" v "], %[srd], %[" o "] offen offset:" k "+1024 lds\n"
#define L_A(v, o) L_D2(v, o, "0") " s_add_u32 m0, m0, 0x800\n s_nop 0\n"
#define L_B(o) L_D2("v16", o, "0") L_D2("v16", o, "2048") " s_add_u32 m0, m0, 0x1000\n s_nop 0\n"
      unsigned v, vl, o0, o1, o2, o3, c, x, j, keep;
      asm volatile(
        " s_mov_b32 %[km0], m0\n"
        "Lpoll%=:\n"
        " ds_read_b32 %[v], %[vchk]\n"
        L_PUB
        " s_mov_b32 %[j], 16\n"
        " s_sub_u32 %[spin], %[spin], 1\n"
        " s_cbranch_scc1 Lout%=\n"
        LD_SLEEP
        " s_waitcnt lgkmcnt(1)\n"
        "Lchk%=:\n"
        " v_cmp_lt_i32 vcc, %[v], %[vneed]\n"
        " v_readlane_b32 %[o0], %[v], %[j]\n"
        " s_add_u32 %[j], %[j], 1\n"
        " v_readlane_b32 %[o1], %[v], %[j]\n"
        " s_cmp_gt_u32 %[c], 47\n"
        " s_cbranch_scc1 Lpoll%=\n"
        " s_cbranch_vccnz Lpoll%=\n"
        " s_add_u32 %[j], %[j], 1\n"
        " v_readlane_b32 %[o2], %[v], %[j]\n"
        " s_add_u32 %[j], %[j], 1\n"
        " v_readlane_b32 %[o3], %[v], %[j]\n"
        " s_mov_b32 m0, %[ring]\n"
        " s_add_u32 %[n], %[n], 1\n"
        " .if %c[isa]\n"
        L_A("v16", "o0") L_A("v16", "o1") L_A("v16", "o2") L_A("v16", "o3")
        " s_add_u32 %[nops], %[nops], 8\n"
        L_PUB
        L_A("v16b", "o0") L_A("v16b", "o1") L_A("v16b", "o2") L_A("v16b", "o3")
        " s_add_u32 %[nops], %[nops], 8\n"
        " .else\n"
        L_B("o0") L_B("o1") L_B("o2") L_B("o3")
        " s_add_u32 %[nops], %[nops], 16\n"
        " .endif\n"
        " v_add_u32 %[vneed], %[vinc], %[vneed]\n"
        " v_add_u32 %[vchk], %[vadv], %[vchk]\n"
        " s_cmp_eq_u32 m0, %[rend]\n"
        " s_cselect_b32 %[ring], %[ring0], m0\n"
        " s_add_u32 %[j], %[j], 1\n"
        L_PUB
        " s_cmp_lt_u32 %[n], 64\n"
        " s_cbranch_scc1 Lchk%=\n"
        "Lout%=:\n"
        L_PUB
        " s_cmp_lg_u32 %[c], 0\n"
        " s_cbranch_scc1 Lout%=\n"
        " s_mov_b32 m0, %[km0]\n"
        : [v] "=&v"(v), [vneed] "+&v"(vneed), [vchk] "+&v"(vchk), [vl] "=&v"(vl), [o0] "=&s"(o0), [o1] "=&s"(o1), [o2] "=&s"(o2), [o3] "=&s"(o3), [c] "=&s"(c),
          [x] "=&s"(x), [j] "=&s"(j), [km0] "=&s"(keep), [n] "+&s"(n), [nops] "+&s"(nops), [ring] "+&s"(ring), [spin] "+&s"(spin)
        : [v16] "v"(v16), [v16b] "v"(v16b), [vfull] "v"(vfull), [vinc] "v"(vinc), [vadv] "v"(vadv),
          [srd] "s"(srd), [ring0] "s"(ring0), [rend] "s"(ring0 + RINGB), [isa] "i"(isA ? 1 : 0)
        : "vcc", "scc", "memory");
      if (n != 64u) lds_wr(ctl + C_ABORT, 1u);
      asm volatile("s_waitcnt lgkmcnt(0)" ::: "memory");
      lds_wr(vfull, 255u);
    };
    if (wave == 10) loader(std::true_type{}); else loader(std::false_type{});
    STAMP(20 + (wave & 1));
  }
}

extern "C" void kernel_launch(int rank, int nranks, ncclComm_t comm, hipStream_t stream,
                              void* const* d_in, const int* in_sizes, int n_in,
                              void* d_out, int out_size, const int* in_dtypes, int out_dtype,
                              void* const* d_out_all, void* const* d_ws_all, size_t ws_bytes) {
  (void)comm; (void)in_sizes; (void)n_in; (void)out_size; (void)in_dtypes; (void)out_dtype; (void)d_out_all;
  if (nranks != NR || (unsigned)rank >= (unsigned)NR || ws_bytes < WS_USED) { fprintf(stderr, "unexpected configuration\n"); return; }
  Params p;
  const size_t woff = WS_TAIL ? (ws_bytes - WS_USED) & ~(size_t)0xfffff : 0;
  for (int i = 0; i < NR; ++i) p.ws[i] = (char*)d_ws_all[i] + woff;
  p.x = (const char*)d_in[0]; p.w = (const char*)d_in[1];
  p.sx = (const float*)d_in[2]; p.sw = (const float*)d_in[3];
  p.out = (float*)d_out; p.wsl = p.ws[rank]; p.rank = rank;
  static bool attr_set[NR] = {};
  if (!attr_set[rank]) {
    HIP_CHECK(hipFuncSetAttribute((const void*)fused_gemm, hipFuncAttributeMaxDynamicSharedMemorySize, (int)LDS_BYTES));
    attr_set[rank] = true;
  }
  fused_gemm<<<GRID, 768, LDS_BYTES, stream>>>(p);
  HIP_CHECK(hipGetLastError());
}
```

```cpp
#include <hip/hip_runtime.h>
#include <rccl/rccl.h>
#include <cstdio>
#include <cstdint>
#include <type_traits>

#define HIP_CHECK(expr) do { hipError_t _e = (expr); if (_e != hipSuccess) { \
    fprintf(stderr, "HIP error %s at %s:%d\n", hipGetErrorString(_e), __FILE__, __LINE__); \
    return; } } while (0)

typedef int v4i __attribute__((ext_vector_type(4)));
typedef unsigned v4u __attribute__((ext_vector_type(4)));
typedef float v4f __attribute__((ext_vector_type(4)));
#define LAS __attribute__((address_space(3)))
#define CAS __attribute__((address_space(4)))

#define LAGS 0x77440000u
#define LAGW 0x7300u
#define NGR 3
#define LAST_LEAD 32
#define STG 24
#define STG_LAST 4
#define TURNS 0x807a70614d3419ull
#define PUSH_WIN 3
#define POLL_W 24
#define POLL_N 4
#define POLL_LATE 232
#define POLL_SLEEP 24
#define POLL_SLEEP2 2
#define LOCAL_GAP 100
#define PREF_FROM 96
#define CODE_KB 24
#define STAGE_DELAY 64
#define PRIO_MMA 0
#define PRIO_LD 3
#define PRIO_POLL 2
#define LD_SLEEP ""
#define POLL_N2 16
#define ST_WIN 63
#define ST_TURN 4
#define OUTF2 "sc1"
#define NOCOMM 0
#define GRID 256
#define DYNP 1
#define TOUCH 1
#define NOCONS 0
#define TLBW 0
#define POLL_GATE 12
#define NPUSH 1
#define EST_ADD 150
#define EARLY_RESUME 0
#define MMA_SLEEP 1
#define WS_TAIL 1
#define MVSF "sc1"
#define TOUCHF "sc1"
#define TOUCH_GAP 20
#define OUTF "sc1"
#define NH_SLOTS 10
#define NB_SLOTS 4
#define LSTR 273

constexpr int NR = 8;
constexpr int DIM = 4096, KSH = 512;
constexpr unsigned PLANE_BLK = 128u << 10;
constexpr unsigned SRC_BYTES = 32 * PLANE_BLK;
constexpr unsigned BUF_BYTES = NR * SRC_BYTES;
constexpr unsigned PF_OFF = 2 * BUF_BYTES;
constexpr unsigned RDY_OFF = PF_OFF + 32 * 600 * 128;
constexpr unsigned CNT_OFF = RDY_OFF + NR * 64 * 128;
constexpr unsigned TMO_OFF = CNT_OFF + 256 * 128;
constexpr unsigned AGG_OFF = TMO_OFF + 3072;
constexpr unsigned TK_OFF = TMO_OFF + 1024;
constexpr unsigned DUM_OFF = TMO_OFF + 4096;
constexpr unsigned DIAG_OFF = DUM_OFF + 256 * 256;
constexpr unsigned DIAG2_OFF = DIAG_OFF + 256 * 1024;
constexpr unsigned WS_USED = DIAG2_OFF + 256 * 2048;
constexpr unsigned long long DEADLINE = 30000000ull;

constexpr int NH = NH_SLOTS, NB = NB_SLOTS;
constexpr unsigned LDS_A = 0, LDS_B = NH * 8192, LDS_C = LDS_B + NB * 16384, LDS_T = LDS_C + 1024, LDS_L = LDS_T + 4096, LDS_BYTES = LDS_L + 1280;
static_assert(LDS_BYTES <= 163840 && NH % 2 == 0, "LDS");
constexpr unsigned C_ABORT = 128, C_MVD = 132, C_AV = 136, C_RDY = 144, C_STG = 148, C_PSH = 152, C_TAB = 156;
constexpr unsigned NTK = 32 * (32 - NPUSH);
constexpr int LPL = 4, NT = 7;
constexpr unsigned NST0 = LPL * (NGR + 1);
constexpr unsigned long long LAG12 = (unsigned long long)LAGS | ((unsigned long long)LAGW << 32);
constexpr int lag_of(int s) { return (int)((LAG12 >> (4 * s)) & 15u); }
constexpr int late_groups() { int r = 0; for (int s = 0; s < 12; ++s) r += (s < 8 ? 2 : 4) * lag_of(s); return r; }
constexpr int UEND = (int)NTK - late_groups() - LAST_LEAD;
constexpr int tile_lag(int idx) { return lag_of(idx >> 2) > lag_of(8 + (idx & 3)) ? lag_of(idx >> 2) : lag_of(8 + (idx & 3)); }
constexpr unsigned long long order8(int k) {
  unsigned long long r = 0;
  for (int i = 8 * k; i < 8 * k + 8; ++i) {
    int c = 0;
    for (int j = 0; j < 32; ++j) c += tile_lag(j) < tile_lag(i) || (tile_lag(j) == tile_lag(i) && j < i);
    r |= (unsigned long long)c << (8 * (i & 7));
  }
  return r;
}

struct Params {
  char* ws[NR];
  const char* x;
  const char* w;
  const float* sx;
  const float* sw;
  float* out;
  char* wsl;
  int rank;
};

__device__ __forceinline__ unsigned rfl(unsigned v) { return __builtin_amdgcn_readfirstlane(v); }
__device__ __forceinline__ v4i make_srd(const void* p, unsigned bytes) {
  unsigned long long a = (unsigned long long)p;
  v4i r;
  r.x = (int)rfl((unsigned)a); r.y = (int)(rfl((unsigned)(a >> 32)) & 0xffffu); r.z = (int)bytes; r.w = 0x00020000;
  return r;
}
__device__ __forceinline__ unsigned long long rt() { return __builtin_amdgcn_s_memrealtime(); }
template <int N> __device__ __forceinline__ void wait_vm() { asm volatile("s_waitcnt vmcnt(%0)" ::"i"(N) : "memory"); }
__device__ __forceinline__ unsigned lds_rd(unsigned addr) {
  unsigned v;
  asm volatile("ds_read_b32 %0, %1\n\ts_waitcnt lgkmcnt(0)" : "=v"(v) : "v"(addr) : "memory");
  return v;
}
__device__ __forceinline__ void lds_wr(unsigned addr, unsigned v) { asm volatile("ds_write_b32 %0, %1" ::"v"(addr), "v"(v) : "memory"); }
__device__ __forceinline__ void lds_inc(unsigned a) {
  asm volatile("s_mov_b64 exec, 1\n\tds_add_u32 %0, %1\n\ts_mov_b64 exec, -1" ::"v"(a), "v"(1u) : "memory");
}
__device__ __forceinline__ void lds_wait(unsigned a, unsigned n) { unsigned k = 0; while (rfl(lds_rd(a)) < n && ++k < (1u << 24)) { } }
__device__ __forceinline__ unsigned flag_ld(v4i srd, unsigned voff) {
  unsigned v;
  asm volatile("s_nop 4\n\tbuffer_load_dword %0, %1, %2, 0 offen sc0 sc1\n\ts_waitcnt vmcnt(0)" : "=v"(v) : "v"(voff), "s"(srd) : "memory");
  return v;
}
__device__ __forceinline__ void flag_st1(v4i srd, unsigned soff, unsigned v) {
  asm volatile("s_mov_b64 exec, 1\n\ts_nop 4\n\tbuffer_store_dword %0, off, %1, %2 sc0 sc1\n\ts_mov_b64 exec, -1" ::"v"(v), "s"(srd), "s"(soff) : "memory");
}
__device__ __forceinline__ unsigned flag_idx(unsigned g, unsigned si, unsigned q) { return g * LSTR + si * 32 + q; }
__host__ __device__ constexpr int stream_pos(int q, int g) {
  const int me = g < 16 ? (g >> 1) : 8 + (g & 3);
  const int key = q + (int)((LAG12 >> (4 * me)) & 15u);
  int u = g < 16 ? (g & 1) : ((g - 16) >> 2);
  for (int s = 0; s < 12; ++s) {
    const int c = key - lag_of(s) - NPUSH + (s < me ? 1 : 0);
    u += (s < 8 ? 2 : 4) * (c < 0 ? 0 : (c > 32 - NPUSH ? 32 - NPUSH : c));
  }
  return u;
}
constexpr unsigned long long first_items(int sub) {
  unsigned long long r = 0;
  for (int it = 0; it < 128; ++it) { const int u = stream_pos(NPUSH + it / 32, it % 32); if (u < (int)NST0 && u % LPL == sub) r |= (unsigned long long)it << (8 * (u / LPL)); }
  return r;
}
template <int I, int N, typename F> __device__ __forceinline__ void static_for(F&& f) {
  if constexpr (I < N) { f(std::integral_constant<int, I>{}); static_for<I + 1, N>(f); }
}

#define TS(x) do { } while (0)
#define STAMPV(idx, v) do { } while (0)
#define STAMP(idx) STAMPV(idx, rt())

constexpr int ACC0 = 40;
#define MV_T "s_mov_b64 exec, 1\n v_mov_b32 v[40+%c[g]], 1\n buffer_atomic_add v[40+%c[g]], off, %[sl], %[tk] sc0\n s_mov_b64 exec, -1\n"
#define MV_L "buffer_load_dwordx4 v[64+%c[g]*16+k*4:67+%c[g]*16+k*4], %[v16], %[sr], %[ol] offen offset:k*1024 sc0 sc1\n"
#define MV_S "buffer_store_dwordx4 v[64+%c[g]*16+k*4:67+%c[g]*16+k*4], %[v16], %[sl], %[os] offen offset:k*1024 " MVSF "\n"
#define MV_F "s_mov_b64 exec, 1\n buffer_store_dword %[vc], off, %[sl], %[nf] sc1\n s_mov_b64 exec, -1\n"
template <int G> __device__ __forceinline__ void mv_load(v4i sl, v4i sr, unsigned tk, unsigned ol, unsigned v16) {
  asm volatile("s_nop 4\n" MV_T ".set k, 0\n .rept 4\n" MV_L ".set k, k+1\n .endr"
               ::[g] "i"(G), [sl] "s"(sl), [sr] "s"(sr), [tk] "s"(tk), [ol] "s"(ol), [v16] "v"(v16) : "memory");
}
template <int G, int W, int WK> __device__ __forceinline__ unsigned mv_swap(v4i sl, v4i sr, unsigned tk, unsigned os, unsigned ol, unsigned v16, unsigned tab, unsigned call, unsigned nf) {
  unsigned t;
  asm volatile("s_waitcnt vmcnt(%c[w]-1)\n v_min_u32 v56, %c[ntk], v[40+%c[g]]\n v_lshl_add_u32 v56, v56, 2, %[tab]\n ds_read_b32 v56, v56\n" MV_T
               ".set k, 0\n .rept 4\n s_waitcnt vmcnt(%c[w]+%c[wk]*k)\n" MV_S MV_L ".set k, k+1\n .endr\n" MV_F
               "s_waitcnt lgkmcnt(0)\n v_readfirstlane_b32 %[t], v56\n s_nop 1"
               : [t] "=s"(t) : [g] "i"(G), [w] "i"(W), [wk] "i"(WK), [sl] "s"(sl), [sr] "s"(sr), [tk] "s"(tk), [os] "s"(os), [ol] "s"(ol), [v16] "v"(v16),
                 [tab] "s"(tab), [ntk] "i"(NTK - NST0), [vc] "v"(call), [nf] "s"(nf) : "memory");
  return t;
}
template <int G, int W> __device__ __forceinline__ void mv_store(v4i sl, unsigned os, unsigned v16, unsigned call, unsigned nf) {
  asm volatile(".set k, 0\n .rept 4\n s_waitcnt vmcnt(%c[w]-k)\n" MV_S ".set k, k+1\n .endr\n" MV_F
               ::[g] "i"(G), [w] "i"(W), [sl] "s"(sl), [os] "s"(os), [v16] "v"(v16), [vc] "v"(call), [nf] "s"(nf) : "memory");
}
template <int R> __device__ __forceinline__ void zero_acc() {
  asm volatile("v_mov_b32 v%c0, 0\n\tv_mov_b32 v%c1, 0\n\tv_mov_b32 v%c2, 0\n\tv_mov_b32 v%c3, 0" ::"i"(R), "i"(R + 1), "i"(R + 2), "i"(R + 3));
  if constexpr (R + 4 < ACC0 + 128) zero_acc<R + 4>();
}
template <int E, int FL> __device__ __forceinline__ void store_row(unsigned sc, unsigned vo, v4i ro) {
  constexpr int R = ACC0 + (E >> 2) * 16 + (E & 3);
  float f0, f1, f2, f3;
  asm volatile("v_cvt_f32_i32 %0, v%c4\n\tv_cvt_f32_i32 %1, v%c5\n\tv_cvt_f32_i32 %2, v%c6\n\tv_cvt_f32_i32 %3, v%c7\n\t"
               "v_mul_f32 %0, %8, %0\n\tv_mul_f32 %1, %8, %1\n\tv_mul_f32 %2, %8, %2\n\tv_mul_f32 %3, %8, %3"
               : "=v"(f0), "=v"(f1), "=v"(f2), "=v"(f3) : "i"(R), "i"(R + 4), "i"(R + 8), "i"(R + 12), "s"(sc));
  v4f v = {f0, f1, f2, f3};
  constexpr unsigned so = (unsigned)(128 * (E >> 4) + 16 * ((E >> 2) & 3) + 4 * (E & 3)) * (DIM * 4);
  if constexpr (FL == 0) asm volatile("buffer_store_dwordx4 %0, %1, %2, %3 offen " OUTF "\n\ts_nop 1" ::"v"(v), "v"(vo), "s"(ro), "s"(so) : "memory");
  else asm volatile("buffer_store_dwordx4 %0, %1, %2, %3 offen " OUTF2 "\n\ts_nop 1" ::"v"(v), "v"(vo), "s"(ro), "s"(so) : "memory");
}
template <int E0, int E1, int FL> __device__ __forceinline__ void store_rows(unsigned sc, unsigned vo, v4i ro) {
  if constexpr (E0 < E1) { store_row<E0, FL>(sc, vo, ro); if constexpr (FL && ST_WIN < 63) wait_vm<ST_WIN>(); store_rows<E0 + 1, E1, FL>(sc, vo, ro); }
}

__global__ void __launch_bounds__(768) __attribute__((amdgpu_num_vgpr(20))) fused_gemm(Params P) {
  extern __shared__ __attribute__((aligned(16))) char lds[];
  const int tid = threadIdx.x, lane = tid & 63;
  if (CODE_KB > 0 && tid >= 768 - CODE_KB * 8) {
    unsigned long long pc;
    asm volatile("s_getpc_b64 %0\n\tglobal_load_dword v60, %1, %0\n\t.subsection 1\n\t.fill 6144, 4, 0xBF800000\n\t.subsection 0" : "=&s"(pc) : "v"((unsigned)(tid - (768 - CODE_KB * 8)) * 128u) : "memory");
  }
  const int wave = (int)rfl((unsigned)tid >> 6);
  const int b = blockIdx.x;
  const int rank = P.rank;
  char* const CAS* const ktab = (char* const CAS*)__builtin_amdgcn_kernarg_segment_ptr();
  char* const wsl = P.wsl;
  unsigned cv = __hip_atomic_load((const unsigned*)(wsl + CNT_OFF + (unsigned)b * 128), __ATOMIC_RELAXED, __HIP_MEMORY_SCOPE_AGENT);
  const unsigned lds0 = (unsigned)(unsigned long long)(LAS char*)lds;
  const unsigned ctl = lds0 + LDS_C;
  const int xcd = b & 7, idx = b >> 3;
  const int ti = 2 * (idx >> 2) + (xcd >> 2), tj = 4 * (xcd & 3) + (idx & 3);
  const int sg = b & 31, so = b >> 5;
  auto plane_of = [&](int j) { return (unsigned)(4 * so + j + 28 + NPUSH) & 31u; };
  const v4i srdL = make_srd(wsl, WS_USED);
  constexpr unsigned long long O0 = order8(0), O1 = order8(1), O2 = order8(2), O3 = order8(3);
  const int mp = (int)(((idx < 8 ? O0 : idx < 16 ? O1 : idx < 24 ? O2 : O3) >> (8 * (idx & 7))) & 31u) * 8 + xcd;
  const int turn = 7 - (mp >> 5), mpair = ((mp >> 3) & 3) * 4 + (mp & 3);
  const bool last = turn == NT, mover = mpair < 14;
  const int pi = mpair % 7, sub = (mpair / 7) * 2 + ((mp >> 2) & 1), pp = (rank + 1 + pi) & 7;
  const unsigned rdy_rest = RDY_OFF + (unsigned)(pp * 64 + 32 + ((lane & 7) == 1 ? 0 : (lane & 7))) * 128;
  auto turn_end = [&](int t) { return (unsigned)((int)((TURNS >> (8 * t)) & 255u) * UEND / 128 - (LPL - 1 - sub) * (STG - (STG - STG_LAST) * t / (NT - 1))); };
  auto entry = [&](unsigned it) { return it | (flag_idx(it & 31, (unsigned)pi, it >> 5) << 16); };
  auto build_table = [&]() {
    const unsigned uend = last ? 4096u : turn_end(turn);
#pragma unroll 1
    for (unsigned it = (unsigned)(tid - 576) + 32 * NPUSH; it < 1024; it += 192) {
      const unsigned u = (unsigned)stream_pos((int)(it >> 5), (int)(it & 31));
      lds_wr(lds0 + LDS_T + u * 4, entry(it) | (u >= uend ? 0x4000u : 0u));
    }
    asm volatile("s_waitcnt lgkmcnt(0)" ::: "memory");
    lds_inc(ctl + C_TAB);
  };
  if (wave == 0) STAMP(0);
  unsigned call = 0;
  auto get_call = [&]() { unsigned c; asm volatile("v_add_u32 %0, 1, %1" : "=v"(c) : "v"(cv) : "memory"); call = rfl(c); };
  auto wg_barrier = [&]() { asm volatile("s_waitcnt lgkmcnt(0)\n\ts_barrier" ::: "memory"); };
  auto wait_flag = [&](unsigned off) {
    const unsigned long long t0 = rt();
#pragma unroll 1
    for (unsigned n = 1;; ++n) {
      const unsigned v = flag_ld(srdL, off);
      if (__all((int)(v - call) >= 0)) return true;
      if ((n & 15u) == 0u) {
        const unsigned tm = rfl(flag_ld(srdL, TMO_OFF));
        if (tm == call || rt() - t0 > DEADLINE) { flag_st1(srdL, TMO_OFF, call); return false; }
      }
      __builtin_amdgcn_s_sleep(1);
    }
  };

  if (wave < 8) {
    unsigned long long tq = 0;
    {
      __amdgpu_buffer_rsrc_t rs = __builtin_amdgcn_make_buffer_rsrc(wsl, 0, 2 * BUF_BYTES, 0x00020000);
      const unsigned base = (unsigned)rank * SRC_BYTES;
      char* pk[7] = {};
      const bool lazy = so >= 2 && turn != 0;
      if (lazy) { wg_barrier(); __builtin_amdgcn_s_sleep(STAGE_DELAY); }
      auto load_pk = [&]() {
        asm volatile("" ::: "memory");
#pragma unroll
        for (int k = 0; k < 7; ++k) { pk[k] = ktab[(rank + 1 + (k + wave + b) % 7) & 7]; asm volatile("" : "+s"(pk[k])); }
      };
      if (sg < 16) {
        const int r = 32 * wave + (lane & 31), j = lane >> 5;
        const char* src = P.x + (size_t)(256 * sg + ((r & ~15) | ((r & 3) << 2) | ((r >> 2) & 3))) * KSH + 64 * so + 16 * j;
        v4u v0 = *(const v4u*)src, v1 = *(const v4u*)(src + 32);
        if (!lazy) wg_barrier();
        if (so == 0) load_pk();
        unsigned ncv;
        asm volatile("v_not_b32 %0, %1" : "=v"(ncv) : "v"(cv) : "memory");
        const unsigned d0 = base + (ncv & 1u) * BUF_BYTES + (unsigned)sg * 4096 + (unsigned)r * 16;
        const unsigned d = d0 + plane_of(j + 2) * PLANE_BLK;
        __builtin_amdgcn_raw_buffer_store_b128(v0, rs, d0 + plane_of(j) * PLANE_BLK, 0, 17);
        __builtin_amdgcn_raw_buffer_store_b128(v1, rs, d, 0, 17);
        TS(tq);
        if (so == 0) {
#pragma unroll
          for (int k = 0; k < 7; ++k) {
            const v4i sp = make_srd(pk[k], 2 * BUF_BYTES);
            asm volatile("s_mov_b32 exec_lo, %c3\n\ts_nop 4\n\tbuffer_store_dwordx4 %0, %1, %2, 0 offen sc0 sc1\n\ts_mov_b64 exec, -1" ::"v"(v1), "v"(d), "s"(sp), "i"(NPUSH == 2 ? -1 : 0) : "memory");
            if (k >= PUSH_WIN - 1) wait_vm<PUSH_WIN - 1>();
          }
        }
        wait_vm<0>(); lds_inc(ctl + C_STG);
      } else if (wave < 4) {
        const int gw = sg - 16;
        const char* src = P.w + (size_t)(64 * so + 16 * wave) * DIM + 256 * gw + 4 * lane;
        unsigned r[16];
#pragma unroll
        for (int k = 0; k < 16; ++k) r[k] = *(const unsigned*)(src + (size_t)k * DIM);
        if (!lazy) wg_barrier();
        if (so == 0) load_pk();
        unsigned ncv;
        asm volatile("v_not_b32 %0, %1" : "=v"(ncv) : "v"(cv) : "memory");
        v4u o[4];
#pragma unroll
        for (int g = 0; g < 4; ++g) {
          unsigned t0 = __builtin_amdgcn_perm(r[4 * g + 1], r[4 * g], 0x05010400u), t1 = __builtin_amdgcn_perm(r[4 * g + 1], r[4 * g], 0x07030602u);
          unsigned t2 = __builtin_amdgcn_perm(r[4 * g + 3], r[4 * g + 2], 0x05010400u), t3 = __builtin_amdgcn_perm(r[4 * g + 3], r[4 * g + 2], 0x07030602u);
          o[0][g] = __builtin_amdgcn_perm(t2, t0, 0x05040100u); o[1][g] = __builtin_amdgcn_perm(t2, t0, 0x07060302u);
          o[2][g] = __builtin_amdgcn_perm(t3, t1, 0x05040100u); o[3][g] = __builtin_amdgcn_perm(t3, t1, 0x07060302u);
        }
        const unsigned d = base + (ncv & 1u) * BUF_BYTES + plane_of(wave) * PLANE_BLK + 65536u + (unsigned)gw * 4096 + (unsigned)(64 * (lane >> 4) + (lane & 15)) * 16;
#pragma unroll
        for (int jj = 0; jj < 4; ++jj) __builtin_amdgcn_raw_buffer_store_b128(o[jj], rs, d + jj * 256, 0, 17);
        TS(tq);
        if (so == 0 && wave >= 4 - NPUSH) {
#pragma unroll
          for (int k = 0; k < 7; ++k) {
            const v4i sp = make_srd(pk[k], 2 * BUF_BYTES);
            asm volatile("s_nop 4\n\tbuffer_store_dwordx4 %0, %4, %5, 0 offen sc0 sc1\n\tbuffer_store_dwordx4 %1, %4, %5, 0 offen offset:256 sc0 sc1\n\t"
                         "buffer_store_dwordx4 %2, %4, %5, 0 offen offset:512 sc0 sc1\n\tbuffer_store_dwordx4 %3, %4, %5, 0 offen offset:768 sc0 sc1"
                         ::"v"(o[0]), "v"(o[1]), "v"(o[2]), "v"(o[3]), "v"(d), "s"(sp) : "memory");
            if (k >= PUSH_WIN - 1) wait_vm<4 * (PUSH_WIN - 1)>();
          }
        }
        wait_vm<0>(); lds_inc(ctl + C_STG);
      } else if (!lazy) wg_barrier();
    }
    if (wave == 0) { STAMP(1); STAMPV(26, tq); }
    asm volatile("" ::: "v167");
    if (NOCONS) return;
    zero_acc<ACC0>();
    __builtin_amdgcn_s_setprio(PRIO_MMA);
    const int wr = wave >> 2, wc = wave & 3;
    unsigned pa = lds0 + LDS_A + (unsigned)(lane >> 4) * 2048 + (unsigned)(64 * wr + (lane & 15)) * 16;
    unsigned pb = lds0 + LDS_B + (unsigned)(lane >> 4) * 4096 + (unsigned)(64 * wc + (lane & 15)) * 16;
    const unsigned vw = ctl + (unsigned)wave * 16 + (unsigned)(lane & 3) * 4;
    const unsigned vd = (lane & 1) ? ctl + (unsigned)(wave ^ 4) * 16 + 8 : ctl + (unsigned)wave * 16 + 12;
    unsigned vo = ((unsigned)(256 * ti + 64 * wr) + (unsigned)(lane >> 4)) * (DIM * 4) + ((unsigned)(256 * tj + 64 * wc) + 4 * (unsigned)(lane & 15)) * 4;
    asm volatile("" : "+v"(vo));
    unsigned sc = rfl(__builtin_bit_cast(unsigned, (*P.sx) * (*P.sw)));
    const v4i ro = make_srd(P.out, DIM * DIM * 4);
    unsigned vcnt = 0, vf, ia = 0, ib = 0, cnt, da, db, spin = 1u << 23;
    unsigned long long ra;
    asm volatile(
      ".macro MM h, i, j\n"
      " v_mfma_i32_16x16x64_i8 v[40+\\h*64+\\i*16+\\j*4:43+\\h*64+\\i*16+\\j*4], v[24+\\i*4:27+\\i*4], v[8+\\j*4:11+\\j*4], v[40+\\h*64+\\i*16+\\j*4:43+\\h*64+\\i*16+\\j*4]\n"
      ".endm\n"
      ".macro STEP h, nb, pm, last\n"
      " s_waitcnt lgkmcnt(4+\\pm)\n"
      " MM \\h, 0, 0\n"
      " s_cmp_eq_u32 %[ia], %c[nh1]\n"
      " s_cselect_b32 %[da], %[wa], 0x2000\n"
      " MM \\h, 1, 0\n"
      " s_cselect_b32 %[ia], -1, %[ia]\n"
      " s_add_u32 %[ia], %[ia], 1\n"
      " .if \\pm\n s_waitcnt lgkmcnt(4)\n .endif\n"
      " MM \\h, 0, 1\n"
      " v_add_u32 %[pa], %[da], %[pa]\n"
      " MM \\h, 1, 1\n"
      " .if \\nb\n s_cmp_eq_u32 %[ib], %c[nb1]\n s_cselect_b32 %[db], %[wb], 0x4000\n .endif\n"
      " .if \\pm\n s_waitcnt lgkmcnt(3)\n .endif\n"
      " MM \\h, 0, 2\n"
      " v_add_u32 %[vcnt], 1, %[vcnt]\n"
      " .if \\nb\n s_cselect_b32 %[ib], -1, %[ib]\n s_add_u32 %[ib], %[ib], 1\n .endif\n"
      " MM \\h, 1, 2\n"
      " .if \\nb\n v_add_u32 %[pb], %[db], %[pb]\n .endif\n"
      " s_waitcnt lgkmcnt(2)\n"
      " MM \\h, 0, 3\n"
      " .if \\last == 0\n v_cmp_le_u32 vcc, %[vf], %[vcnt]\n .endif\n"
      " MM \\h, 1, 3\n"
      " .if \\last == 0\n"
      " s_getpc_b64 %[ra]\n"
      " s_and_b32 %[da], vcc_lo, 3\n"
      " s_cbranch_scc1 Lslow%=\n"
      " ds_read_b128 v[24:27], %[pa]\n"
      " ds_read_b128 v[28:31], %[pa] offset:256\n"
      " .endif\n"
      " s_waitcnt lgkmcnt(2-2*\\last)\n"
      " MM \\h, 2, 0\n"
      " .if \\last == 0\n"
      " ds_write_b32 %[vd], %[vcnt]\n"
      " ds_read_b32 %[vf], %[vw]\n"
      " .endif\n"
      " MM \\h, 3, 0\n"
      " .if \\nb\n ds_read_b128 v[8:11], %[pb]\n .endif\n"
      " MM \\h, 2, 1\n"
      " .if (\\last == 0) && %c[dynp]\n"
      " s_bitcmp1_b32 vcc_lo, 2\n"
      " s_setprio 0\n"
      " s_cbranch_scc1 1f\n"
      " s_setprio 1\n"
      "1:\n"
      " .endif\n"
      " MM \\h, 3, 1\n"
      " .if \\nb\n ds_read_b128 v[12:15], %[pb] offset:256\n s_sub_u32 %[cnt], %[cnt], 1\n .endif\n"
      " MM \\h, 2, 2\n"
      " MM \\h, 3, 2\n"
      " .if \\nb\n ds_read_b128 v[16:19], %[pb] offset:512\n s_cmp_lg_u32 %[cnt], 0\n .endif\n"
      " MM \\h, 2, 3\n"
      " MM \\h, 3, 3\n"
      " .if \\nb\n ds_read_b128 v[20:23], %[pb] offset:768\n .endif\n"
      " .if \\last == 0\n"
      " ds_read_b128 v[32:35], %[pa] offset:512\n"
      " ds_read_b128 v[36:39], %[pa] offset:768\n"
      " .endif\n"
      ".endm\n"
      " s_mov_b64 vcc, -1\n"
      " s_getpc_b64 %[ra]\n"
      " s_and_b32 %[da], vcc_lo, 3\n"
      " s_cbranch_scc1 Lslow%=\n"
      " ds_read_b128 v[24:27], %[pa]\n ds_read_b128 v[28:31], %[pa] offset:256\n"
      " ds_read_b128 v[8:11], %[pb]\n ds_read_b128 v[12:15], %[pb] offset:256\n"
      " ds_read_b128 v[16:19], %[pb] offset:512\n ds_read_b128 v[20:23], %[pb] offset:768\n"
      " ds_read_b128 v[32:35], %[pa] offset:512\n ds_read_b128 v[36:39], %[pa] offset:768\n"
      " s_mov_b32 %[cnt], 63\n"
      "Lfull%=:\n"
      " STEP 0, 0, 1, 0\n"
      " STEP 1, 1, 0, 0\n"
      " s_cbranch_scc1 Lfull%=\n"
      " STEP 0, 0, 1, 0\n"
      " STEP 1, 0, 0, 1\n"
      " s_branch Lend%=\n"
      "Lslow%=:\n"
      " ds_read_b32 %[vf], %[vw]\n"
      " s_sub_u32 %[spin], %[spin], 1\n"
      " s_waitcnt lgkmcnt(0)\n"
      " ds_write_b32 %[vd], %[vcnt]\n"
      " v_cmp_le_u32 vcc, %[vf], %[vcnt]\n"
      " s_cbranch_scc1 Lgive%=\n"
      " s_and_b32 %[da], vcc_lo, 3\n"
      " s_cbranch_scc0 Lret%=\n"
      " s_sleep %c[msl]\n"
      " s_branch Lslow%=\n"
      "Lgive%=:\n"
      " s_mov_b32 %[spin], 0\n"
      " s_mov_b32 %[sc], 0x7fc00000\n"
      " s_mov_b64 vcc, 0\n"
      "Lret%=:\n"
      " s_setpc_b64 %[ra]\n"
      "Lend%=:\n"
      " s_nop 7\n s_nop 1\n"
      ".purgem STEP\n.purgem MM\n"
      : [pa] "+&v"(pa), [pb] "+&v"(pb), [vcnt] "+&v"(vcnt), [vf] "=&v"(vf), [ia] "+&s"(ia), [ib] "+&s"(ib), [cnt] "=&s"(cnt), [da] "=&s"(da),
        [db] "=&s"(db), [spin] "+&s"(spin), [sc] "+&s"(sc), [ra] "=&s"(ra)
      : [vw] "v"(vw), [vd] "v"(vd), [dynp] "i"(DYNP), [msl] "i"(MMA_SLEEP), [nh1] "i"(NH - 1), [nb1] "i"(NB - 1), [wa] "s"(-(NH - 1) * 8192), [wb] "s"(-(NB - 1) * 16384)
      : "v8", "v9", "v10", "v11", "v12", "v13", "v14", "v15", "v16", "v17", "v18", "v19", "v20", "v21", "v22", "v23", "v24", "v25", "v26", "v27",
        "v28", "v29", "v30", "v31", "v32", "v33", "v34", "v35", "v36", "v37", "v38", "v39", "vcc", "scc", "memory");
    if (wave == 0) STAMP(3);
    sc = rfl(lds_rd(ctl + C_ABORT)) != 0u ? 0x7fc00000u : sc;
    if (turn >= ST_TURN && turn < NT) store_rows<0, 32, 1>(sc, vo, ro); else store_rows<0, 32, 0>(sc, vo, ro);
  } else if (wave == 8) {
    wg_barrier();
    lds_wr(lds0 + LDS_T + (NTK + (lane & 15)) * 4, 0xffffffffu);
    get_call();
    auto touch_out = [&](unsigned long long t0, unsigned until) {
      if (TOUCH == 0) return;
      const v4i ro = make_srd(P.out, DIM * DIM * 4);
      const unsigned vt = (unsigned)(256 * ti + (lane >> 3)) * (DIM * 4) + (unsigned)tj * 1024 + (unsigned)(lane & 7) * 128;
      while ((unsigned)(rt() - t0) < 600u) __builtin_amdgcn_s_sleep(16);
#pragma unroll 1
      for (unsigned r = 0; r < 32 && (unsigned)(rt() - t0) < until; ++r) {
        asm volatile("s_nop 4\n\tbuffer_load_dword v60, %0, %1, %2 offen " TOUCHF ::"v"(vt), "s"(ro), "s"(r * 8u * DIM * 4) : "memory");
        if (TOUCH == 2) asm volatile("buffer_load_dword v61, %0, %1, %2 offen offset:64 " TOUCHF ::"v"(vt), "s"(ro), "s"(r * 8u * DIM * 4) : "memory");
        __builtin_amdgcn_s_sleep(TOUCH_GAP);
      }
    };
    const unsigned long long tk0 = rt();
    if (mover && !NOCOMM) {
      __builtin_amdgcn_s_setprio(3);
      const v4i srdR = make_srd(ktab[pp], 2 * BUF_BYTES);
      const unsigned v16 = (unsigned)lane * 16;
      const unsigned sbase = (call & 1u) * BUF_BYTES + (unsigned)pp * SRC_BYTES;
      unsigned tk_off = TK_OFF + ((call & 1u) * 8 + (unsigned)pi) * 128;
      const unsigned tk_dead = DUM_OFF + (unsigned)b * 256 + 64;
      const unsigned go_out = DUM_OFF + (unsigned)(last ? mp : mp - 32) * 256 + 192, go_in = DUM_OFF + (unsigned)mp * 256 + 192;
      flag_st1(srdL, tk_dead, 0x1000u);
      unsigned long long tm0 = 0, tm2 = 0;
      unsigned ev;
      bool ok;
      if (turn == 0) {
        lds_wr(ctl + C_MVD, 1u);
        unsigned long long fi = 0;
        static_for<0, LPL>([&](auto S) { constexpr unsigned long long F = first_items(decltype(S)::value); fi = sub == decltype(S)::value ? F : fi; });
        const unsigned it = (unsigned)(fi >> (8 * ((lane & 7) < NGR ? (lane & 7) : NGR))) & 255u, g = it & 31u;
        ev = entry(32u * NPUSH + it);
        asm volatile("" : "+v"(ev));
        TS(tm0);
        ok = wait_flag(RDY_OFF + ((unsigned)pp * 64 + g) * 128);
      } else {
        const unsigned long long t0 = rt();
        const unsigned est = turn_end(turn - 1) * 67u / 10u + EST_ADD;
        if (turn >= 3) touch_out(t0, est - 200u);
        while ((unsigned)(rt() - t0) < est) __builtin_amdgcn_s_sleep(16);
        ok = wait_flag(rdy_rest) && wait_flag(go_in);
        lds_wr(ctl + C_MVD + (unsigned)(lane & 1) * 4, (lane & 1) ? 0u : 1u);
        unsigned x0;
        asm volatile("s_mov_b64 exec, 1\n\tv_mov_b32 v47, %c3\n\ts_nop 4\n\tbuffer_atomic_add v47, off, %1, %2 sc0\n\ts_mov_b64 exec, -1\n\t"
                     "s_waitcnt vmcnt(0)\n\tv_readfirstlane_b32 %0, v47\n\ts_nop 1" : "=s"(x0) : "s"(srdL), "s"(ok ? tk_off : tk_dead), "i"(NGR + 1) : "memory");
        const unsigned xe = x0 + NST0 + (unsigned)(lane & 7);
        lds_wait(ctl + C_TAB, 3);
        ev = lds_rd(lds0 + LDS_T + (xe < NTK ? xe : NTK) * 4);
      }
      if (ok) {
        const unsigned fdum = DUM_OFF + (unsigned)b * 256;
        unsigned os[NGR], en[NGR], pf[NGR];
        unsigned e;
        unsigned rdy = turn != 0 ? 4096u : (4u + NPUSH) * 32;
        auto foff = [&](unsigned x) { return x == 0xffffffffu ? fdum : PF_OFF + (x >> 16) * 128; };
        auto offs = [&](unsigned x) {
          if (__builtin_expect((x & 0x4000u) != 0u && tk_off != tk_dead, 0)) {
            tk_off = tk_dead;
            flag_st1(srdL, go_out, call);
          }
          return sbase + (x == 0xffffffffu ? 32u * NPUSH : x & 1023u) * 4096;
        };
        auto gate = [&]() {
          if (__builtin_expect((e & 1023u) >= rdy && e != 0xffffffffu, 0)) {
            const unsigned lvl = (e & 1023u) < (28 + NPUSH) * 32 ? 2u : 3u;
            unsigned r, spins = 0;
            while ((r = rfl(lds_rd(ctl + C_RDY))) < lvl && ++spins < (1u << 23)) __builtin_amdgcn_s_sleep(2);
            rdy = lvl == 2u ? (28u + NPUSH) * 32 : 4096u;
            if (r < lvl || r > 3u) e = 0xffffffffu;
          }
        };
        auto swap = [&](auto G, auto W, auto WK) {
          constexpr int g = decltype(G)::value;
          const unsigned ol = offs(e), nf = pf[g], o = os[g];
          pf[g] = en[g]; en[g] = foff(e); os[g] = ol;
          e = mv_swap<g, decltype(W)::value, decltype(WK)::value>(srdL, srdR, tk_off, o, ol, v16, lds0 + LDS_T + NST0 * 4, call, nf);
        };
        static_for<0, NGR>([&](auto G) {
          constexpr int g = decltype(G)::value;
          const unsigned x = __builtin_amdgcn_readlane(ev, g);
          pf[g] = fdum; en[g] = foff(x); os[g] = offs(x);
          mv_load<g>(srdL, srdR, tk_off, os[g], v16);
        });
        e = __builtin_amdgcn_readlane(ev, NGR);
        if (turn == 0) {
          lds_wait(ctl + C_TAB, 3);
          lds_wait(ctl + C_RDY, 1);
          TS(tm2);
        }
        static_for<0, NGR>([&](auto G) {
          constexpr int g = decltype(G)::value;
          gate();
          swap(G, std::integral_constant<int, 4 + (NGR - 1 - g) * 5 + g * 10>{}, std::integral_constant<int, 1>{});
        });
        int stop = -1;
#pragma unroll 1
        while (stop < 0) {
          static_for<0, NGR>([&](auto G) {
            constexpr int g = decltype(G)::value;
            if (stop < 0) {
              gate();
              if (e == 0xffffffffu) stop = g;
              else {
                swap(G, std::integral_constant<int, 8 + (NGR - 1) * 10>{}, std::integral_constant<int, 0>{});
              }
            }
          });
        }
        if (tk_off != tk_dead) flag_st1(srdL, go_out, call);
        if (EARLY_RESUME) lds_wr(ctl + C_MVD, 2u);
        static_for<0, NGR>([&](auto G) {
          constexpr int g = decltype(G)::value;
          if (stop == g) static_for<0, NGR>([&](auto J) {
            constexpr int j = decltype(J)::value, h = (g + j) % NGR;
            mv_store<h, 7 + (NGR - 1 - j) * 10 + j * 5>(srdL, os[h], v16, call, pf[h]);
          });
        });
        wait_vm<0>();
        static_for<0, NGR>([&](auto G) { flag_st1(srdL, en[decltype(G)::value], call); });
        lds_wr(ctl + C_MVD, 2u);
      }
      __builtin_amdgcn_s_setprio(0);
    }
    lds_wr(ctl + C_MVD, 2u);
    if (turn < 3 || !mover) touch_out(tk0, 6200u);
  } else if (wave == 9) {
    __builtin_amdgcn_s_setprio(2);
    lds_wr(ctl + lane * 4, 0u);
    wg_barrier();
    const bool direct = so == 1;
    if (!direct) build_table();
    get_call();
    {
      char* pb = wsl;
#pragma unroll
      for (int k = 0; k < 8; ++k) { char* t = P.ws[k]; pb = (lane == k || (lane >= 32 && (lane & 7) == k)) ? t : pb; }
      const unsigned off = lane < 8 ? RDY_OFF + (unsigned)(rank * 64 + (direct ? sg : 32 + so)) * 128
                         : lane < 16 ? PF_OFF + flag_idx((unsigned)sg, 7u, plane_of(lane - 8)) * 128
                         : lane == 16 ? CNT_OFF + (unsigned)b * 128
                         : lane < 32 ? TK_OFF + ((~call & 1u) * 8 + (unsigned)(lane - 17)) * 128
                         : PF_OFF + flag_idx((unsigned)sg, (unsigned)(rank - lane - 1) & 7u, (unsigned)lane >> 3 & 1u) * 128;
      unsigned* dst = (unsigned*)(pb + off);
      const unsigned val = (lane > 16 && lane < 32) ? 0u : call;
      const bool on = lane < 8 ? (direct && lane != rank) : lane <= 16 ? (lane < 12 || lane == 16) : lane < 32 ? (b == 255 && lane < 24) : (so == 0 && lane < 32 + 8 * NPUSH && (lane & 7) != rank);
      asm volatile("" : "+v"(dst));
      unsigned long long tp0 = 0, tp1 = 0;
      TS(tp0);
      lds_wait(ctl + C_STG, sg < 16 ? 8 : 4);
      TS(tp1);
      if (on) __hip_atomic_store(dst, val, __ATOMIC_RELAXED, __HIP_MEMORY_SCOPE_SYSTEM);
      if (direct) build_table();
      else {
        unsigned tk = 0;
        if (lane == 0) tk = __hip_atomic_fetch_add((unsigned*)(wsl + AGG_OFF + (unsigned)so * 128), 1u, __ATOMIC_RELAXED, __HIP_MEMORY_SCOPE_AGENT);
        if (rfl(tk) + 1u == call * 32u && lane < 8 && lane != rank) __hip_atomic_store(dst, val, __ATOMIC_RELAXED, __HIP_MEMORY_SCOPE_SYSTEM);
      }
      STAMPV(27, tp0); STAMPV(28, tp1);
    }
    __builtin_amdgcn_s_setprio(PRIO_POLL);
    const unsigned si = (unsigned)lane >> 3, ro = ((unsigned)lane >> 1) & 3u;
    const unsigned fbase = PF_OFF + flag_idx((lane & 1) ? 16u + tj : (unsigned)ti, si, ro) * 128;
    const unsigned pbase = (call & 1u) * BUF_BYTES + (((unsigned)rank + 1 + si) & 7u) * SRC_BYTES + ro * PLANE_BLK;
    const unsigned lateN = POLL_LATE - 7 * (unsigned)(((LAGS >> (4 * (idx >> 2))) & 15u) > ((LAGW >> (4 * (idx & 3))) & 15u) ? ((LAGS >> (4 * (idx >> 2))) & 15u) : ((LAGW >> (4 * (idx & 3))) & 15u));
    unsigned q = 0, cnt = 0, npoll = 0, win = 4;
    bool mvdone = false;
    const unsigned long long t0 = rt();
#pragma unroll 1
    while ((cnt < 256 || !mvdone) && NOCONS != 1) {
      if (!mvdone) { const unsigned m = rfl(lds_rd(ctl + C_MVD)); mvdone = m == 2u; if (m == 1u) { lds_wr(ctl + C_AV, 0u); win = 4; __builtin_amdgcn_s_sleep(4); continue; } }
      unsigned v = call - 1;
      if (ro < win && q + ro < 32) {
        if (NOCOMM) v = call;
        else asm volatile("buffer_load_dword %0, %1, %2, 0 offen sc0 sc1\n\ts_waitcnt vmcnt(0)" : "=v"(v) : "v"(fbase + q * 128), "s"(srdL) : "memory");
      }
      const unsigned long long hm = __ballot((int)(v - call) >= 0);
      const unsigned adv = (unsigned)__builtin_ctz(~((unsigned)(hm >> (8 * si)) & 255u)) >> 1;
      const bool emit = !(lane & 1) && ro < adv;
      const unsigned long long em = __ballot(emit);
      if (emit) lds_wr(lds0 + LDS_L + (cnt + __builtin_amdgcn_mbcnt_hi((unsigned)(em >> 32), __builtin_amdgcn_mbcnt_lo((unsigned)em, 0u))) * 4, pbase + q * PLANE_BLK);
      q += adv;
      cnt += (unsigned)__builtin_popcountll(em);
      const unsigned lim = (cnt < 160 && !NOCOMM) ? (unsigned)(rt() - t0) / LOCAL_GAP + 1 : 255u;
      lds_wr(ctl + C_AV, (cnt >> 2) < lim ? cnt >> 2 : lim);
      const bool more = __ballot(adv == win && q < 32) != 0ull;
      win = more ? 4u : 2u;
      if (!more) { if (cnt < lateN) __builtin_amdgcn_s_sleep(POLL_SLEEP); else __builtin_amdgcn_s_sleep(POLL_SLEEP2); }
      if (em == 0ull && (++npoll & 15u) == 0u) {
        const unsigned tm = rfl(flag_ld(srdL, TMO_OFF));
        if (tm == call || rt() - t0 > DEADLINE) {
          flag_st1(srdL, TMO_OFF, call);
          lds_wr(ctl + C_ABORT, 1u);
#pragma unroll 1
          for (unsigned k = (unsigned)lane; k < 256; k += 64) lds_wr(lds0 + LDS_L + k * 4, 0u);
          asm volatile("s_waitcnt lgkmcnt(0)" ::: "memory");
          break;
        }
      }
    }
    lds_wr(ctl + C_AV, 64u);
    STAMP(19);
  } else {
    wg_barrier();
    build_table();
    __builtin_amdgcn_s_setprio(PRIO_LD);
    get_call();
    if (turn == 0 && wave == 10 && !NOCOMM) {
      bool ok = wait_flag(RDY_OFF + (unsigned)(pp * 64 + (lane & 31)) * 128);
      if (ok) { lds_wr(ctl + C_RDY, 1u); ok = wait_flag(RDY_OFF + (unsigned)(pp * 64 + 32 + ((lane & 7) < 2 ? 2 : (lane & 7))) * 128); }
      if (ok) { lds_wr(ctl + C_RDY, 2u); ok = wait_flag(rdy_rest); }
      lds_wr(ctl + C_RDY, ok ? 3u : 9u);
    }
    auto loader = [&](auto IA) {
      constexpr bool isA = decltype(IA)::value;
      constexpr unsigned RINGB = isA ? NH * 8192 : NB * 16384;
      const unsigned v16 = (unsigned)lane * 16 + (isA ? (unsigned)ti : 16u + tj) * 4096, v16b = v16 + 2048u;
      unsigned vchk = lane < 8 ? ctl + lane * 16 + 12 : lane < 16 ? ctl + C_AV : lds0 + LDS_L + (unsigned)(lane - 16) * 4;
      const unsigned vadv = lane < 16 ? 0u : 16u;
      const unsigned vfull = lane < 8 ? ctl + (unsigned)lane * 16 + (isA ? 0 : 4) : ctl + (isA ? 512 : 768) + lane * 4;
      const unsigned ring0 = lds0 + (isA ? LDS_A : LDS_B);
      unsigned ring = ring0, n = 0, nops = 0, spin = 1u << 24;
      const v4i srd = srdL;
      int vneed = lane < 8 ? (isA ? 2 - NH : 1 - 2 * NB) : lane < 16 ? 1 : (int)0x80000000u;
      const int vinc = lane < 8 ? 2 : lane < 16 ? 1 : 0;
      wait_vm<0>();
      if (NOCONS || NOCOMM == 2) { lds_wr(vfull, 255u); return; }
#define L_PUB \
      " s_getreg_b32 %[c], hwreg(HW_REG_IB_STS)\n" \
      " s_bfe_u32 %[x], %[c], 0x20016\n" \
      " s_and_b32 %[c], %[c], 15\n" \
      " s_lshl4_add_u32 %[c], %[x], %[c]\n" \
      " s_sub_u32 %[x], %[nops], %[c]\n" \
      " s_lshr_b32 %[x], %[x], 3\n" \
      " .if %c[isa] == 0\n s_and_b32 %[x], %[x], -2\n .endif\n" \
      " v_mov_b32 %[vl], %[x]\n" \
      " ds_write_b32 %[vfull], %[vl]\n"
#define L_D2(v, o, k) \
      " buffer_load_dwordx4 %[" v "], %[srd], %[" o "] offen offset:" k " lds\n" \
      " buffer_load_dwordx4 %[" v "], %[srd], %[" o "] offen offset:" k "+1024 lds\n"
#define L_A(v, o) L_D2(v, o, "0") " s_add_u32 m0, m0, 0x800\n s_nop 0\n"
#define L_B(o) L_D2("v16", o, "0") L_D2("v16", o, "2048") " s_add_u32 m0, m0, 0x1000\n s_nop 0\n"
      unsigned v, vl, o0, o1, o2, o3, c, x, j, keep;
      asm volatile(
        " s_mov_b32 %[km0], m0\n"
        "Lpoll%=:\n"
        " ds_read_b32 %[v], %[vchk]\n"
        L_PUB
        " s_mov_b32 %[j], 16\n"
        " s_sub_u32 %[spin], %[spin], 1\n"
        " s_cbranch_scc1 Lout%=\n"
        LD_SLEEP
        " s_waitcnt lgkmcnt(1)\n"
        "Lchk%=:\n"
        " v_cmp_lt_i32 vcc, %[v], %[vneed]\n"
        " v_readlane_b32 %[o0], %[v], %[j]\n"
        " s_add_u32 %[j], %[j], 1\n"
        " v_readlane_b32 %[o1], %[v], %[j]\n"
        " s_cmp_gt_u32 %[c], 47\n"
        " s_cbranch_scc1 Lpoll%=\n"
        " s_cbranch_vccnz Lpoll%=\n"
        " s_add_u32 %[j], %[j], 1\n"
        " v_readlane_b32 %[o2], %[v], %[j]\n"
        " s_add_u32 %[j], %[j], 1\n"
        " v_readlane_b32 %[o3], %[v], %[j]\n"
        " s_mov_b32 m0, %[ring]\n"
        " s_add_u32 %[n], %[n], 1\n"
        " .if %c[isa]\n"
        L_A("v16", "o0") L_A("v16", "o1") L_A("v16", "o2") L_A("v16", "o3")
        " s_add_u32 %[nops], %[nops], 8\n"
        L_PUB
        L_A("v16b", "o0") L_A("v16b", "o1") L_A("v16b", "o2") L_A("v16b", "o3")
        " s_add_u32 %[nops], %[nops], 8\n"
        " .else\n"
        L_B("o0") L_B("o1") L_B("o2") L_B("o3")
        " s_add_u32 %[nops], %[nops], 16\n"
        " .endif\n"
        " v_add_u32 %[vneed], %[vinc], %[vneed]\n"
        " v_add_u32 %[vchk], %[vadv], %[vchk]\n"
        " s_cmp_eq_u32 m0, %[rend]\n"
        " s_cselect_b32 %[ring], %[ring0], m0\n"
        " s_add_u32 %[j], %[j], 1\n"
        L_PUB
        " s_cmp_lt_u32 %[n], 64\n"
        " s_cbranch_scc1 Lchk%=\n"
        "Lout%=:\n"
        L_PUB
        " s_cmp_lg_u32 %[c], 0\n"
        " s_cbranch_scc1 Lout%=\n"
        " s_mov_b32 m0, %[km0]\n"
        : [v] "=&v"(v), [vneed] "+&v"(vneed), [vchk] "+&v"(vchk), [vl] "=&v"(vl), [o0] "=&s"(o0), [o1] "=&s"(o1), [o2] "=&s"(o2), [o3] "=&s"(o3), [c] "=&s"(c),
          [x] "=&s"(x), [j] "=&s"(j), [km0] "=&s"(keep), [n] "+&s"(n), [nops] "+&s"(nops), [ring] "+&s"(ring), [spin] "+&s"(spin)
        : [v16] "v"(v16), [v16b] "v"(v16b), [vfull] "v"(vfull), [vinc] "v"(vinc), [vadv] "v"(vadv),
          [srd] "s"(srd), [ring0] "s"(ring0), [rend] "s"(ring0 + RINGB), [isa] "i"(isA ? 1 : 0)
        : "vcc", "scc", "memory");
      if (n != 64u) lds_wr(ctl + C_ABORT, 1u);
      asm volatile("s_waitcnt lgkmcnt(0)" ::: "memory");
      lds_wr(vfull, 255u);
    };
    if (wave == 10) loader(std::true_type{}); else loader(std::false_type{});
    STAMP(20 + (wave & 1));
  }
}

extern "C" void kernel_launch(int rank, int nranks, ncclComm_t comm, hipStream_t stream,
                              void* const* d_in, const int* in_sizes, int n_in,
                              void* d_out, int out_size, const int* in_dtypes, int out_dtype,
                              void* const* d_out_all, void* const* d_ws_all, size_t ws_bytes) {
  (void)comm; (void)in_sizes; (void)n_in; (void)out_size; (void)in_dtypes; (void)out_dtype; (void)d_out_all;
  if (nranks != NR || (unsigned)rank >= (unsigned)NR || ws_bytes < WS_USED) { fprintf(stderr, "unexpected configuration\n"); return; }
  Params p;
  const size_t woff = WS_TAIL ? (ws_bytes - WS_USED) & ~(size_t)0xfffff : 0;
  for (int i = 0; i < NR; ++i) p.ws[i] = (char*)d_ws_all[i] + woff;
  p.x = (const char*)d_in[0]; p.w = (const char*)d_in[1];
  p.sx = (const float*)d_in[2]; p.sw = (const float*)d_in[3];
  p.out = (float*)d_out; p.wsl = p.ws[rank]; p.rank = rank;
  static bool attr_set[NR] = {};
  if (!attr_set[rank]) {
    HIP_CHECK(hipFuncSetAttribute((const void*)fused_gemm, hipFuncAttributeMaxDynamicSharedMemorySize, (int)LDS_BYTES));
    attr_set[rank] = true;
  }
  fused_gemm<<<GRID, 768, LDS_BYTES, stream>>>(p);
  HIP_CHECK(hipGetLastError());
}
```

```cpp
#include <hip/hip_runtime.h>
#include <rccl/rccl.h>
#include <cstdio>
#include <cstdint>
#include <type_traits>

#define HIP_CHECK(expr) do { hipError_t _e = (expr); if (_e != hipSuccess) { \
    fprintf(stderr, "HIP error %s at %s:%d\n", hipGetErrorString(_e), __FILE__, __LINE__); \
    return; } } while (0)

typedef int v4i __attribute__((ext_vector_type(4)));
typedef unsigned v4u __attribute__((ext_vector_type(4)));
typedef float v4f __attribute__((ext_vector_type(4)));
#define LAS __attribute__((address_space(3)))
#define CAS __attribute__((address_space(4)))

#define LAGS 0x77420000u
#define LAGW 0x7100u
#define NGR 3
#define LAST_LEAD 32
#define TURNS 0x807a70614d3419ull
#define STG 24
#define STG_LAST 4
#define EST_ADD 150
#define NPUSH 1
#define PUSH_WIN 3
#define STAGE_DELAY 64
#define POLL_LATE 232
#define POLL_SLEEP 24
#define POLL_SLEEP2 2
#define LOCAL_GAP 100
#define LSTR 273

constexpr int NR = 8;
constexpr int DIM = 4096, KSH = 512;
constexpr unsigned PLANE_BLK = 128u << 10;
constexpr unsigned SRC_BYTES = 32 * PLANE_BLK;
constexpr unsigned BUF_BYTES = NR * SRC_BYTES;
constexpr unsigned PF_OFF = 2 * BUF_BYTES;
constexpr unsigned RDY_OFF = PF_OFF + 32 * 600 * 128;
constexpr unsigned CNT_OFF = RDY_OFF + NR * 64 * 128;
constexpr unsigned TMO_OFF = CNT_OFF + 256 * 128;
constexpr unsigned AGG_OFF = TMO_OFF + 3072;
constexpr unsigned TK_OFF = TMO_OFF + 1024;
constexpr unsigned DUM_OFF = TMO_OFF + 4096;
constexpr unsigned WS_USED = DUM_OFF + 256 * 256;
constexpr unsigned long long DEADLINE = 60000000ull;

constexpr int NH = 10, NB = 4;
constexpr unsigned LDS_A = 0, LDS_B = NH * 8192, LDS_C = LDS_B + NB * 16384, LDS_T = LDS_C + 1024, LDS_L = LDS_T + 4096, LDS_BYTES = LDS_L + 1280;
static_assert(LDS_BYTES <= 163840 && NH % 2 == 0, "LDS");
constexpr unsigned C_ABORT = 128, C_MVD = 132, C_AV = 136, C_FIN = 140, C_RDY = 144, C_STG = 148, C_TAB = 156;
constexpr unsigned NTK = 32 * (32 - NPUSH);
constexpr int LPL = 4, NT = 7;
constexpr unsigned NST0 = LPL * (NGR + 1);
constexpr unsigned long long LAG12 = (unsigned long long)LAGS | ((unsigned long long)LAGW << 32);
constexpr int lag_of(int s) { return (int)((LAG12 >> (4 * s)) & 15u); }
constexpr int late_groups() { int r = 0; for (int s = 0; s < 12; ++s) r += (s < 8 ? 2 : 4) * lag_of(s); return r; }
constexpr int UEND = (int)NTK - late_groups() - LAST_LEAD;
constexpr int tile_lag(int idx) { return lag_of(idx >> 2) > lag_of(8 + (idx & 3)) ? lag_of(idx >> 2) : lag_of(8 + (idx & 3)); }
constexpr unsigned long long order8(int k) {
  unsigned long long r = 0;
  for (int i = 8 * k; i < 8 * k + 8; ++i) {
    int c = 0;
    for (int j = 0; j < 32; ++j) c += tile_lag(j) < tile_lag(i) || (tile_lag(j) == tile_lag(i) && j < i);
    r |= (unsigned long long)c << (8 * (i & 7));
  }
  return r;
}

struct Params {
  char* ws[NR];
  const char* x;
  const char* w;
  const float* sx;
  const float* sw;
  float* out;
  char* wsl;
  int rank;
};

__device__ __forceinline__ unsigned rfl(unsigned v) { return __builtin_amdgcn_readfirstlane(v); }
__device__ __forceinline__ v4i make_srd(const void* p, unsigned bytes) {
  unsigned long long a = (unsigned long long)p;
  v4i r;
  r.x = (int)rfl((unsigned)a); r.y = (int)(rfl((unsigned)(a >> 32)) & 0xffffu); r.z = (int)bytes; r.w = 0x00020000;
  return r;
}
__device__ __forceinline__ unsigned long long rt() { return __builtin_amdgcn_s_memrealtime(); }
template <int N> __device__ __forceinline__ void wait_vm() { asm volatile("s_waitcnt vmcnt(%0)" ::"i"(N) : "memory"); }
__device__ __forceinline__ unsigned lds_rd(unsigned addr) {
  unsigned v;
  asm volatile("ds_read_b32 %0, %1\n\ts_waitcnt lgkmcnt(0)" : "=v"(v) : "v"(addr) : "memory");
  return v;
}
__device__ __forceinline__ void lds_wr(unsigned addr, unsigned v) { asm volatile("ds_write_b32 %0, %1" ::"v"(addr), "v"(v) : "memory"); }
__device__ __forceinline__ void lds_inc(unsigned a) {
  asm volatile("s_mov_b64 exec, 1\n\tds_add_u32 %0, %1\n\ts_mov_b64 exec, -1" ::"v"(a), "v"(1u) : "memory");
}
__device__ __forceinline__ bool lds_wait(unsigned a, unsigned n) { unsigned k = 0; while (rfl(lds_rd(a)) < n) if (++k == (1u << 25)) return false; return true; }
__device__ __forceinline__ unsigned flag_ld(v4i srd, unsigned voff) {
  unsigned v;
  asm volatile("s_nop 4\n\tbuffer_load_dword %0, %1, %2, 0 offen sc0 sc1\n\ts_waitcnt vmcnt(0)" : "=v"(v) : "v"(voff), "s"(srd) : "memory");
  return v;
}
__device__ __forceinline__ void flag_st1(v4i srd, unsigned soff, unsigned v) {
  asm volatile("s_mov_b64 exec, 1\n\ts_nop 4\n\tbuffer_store_dword %0, off, %1, %2 sc0 sc1\n\ts_mov_b64 exec, -1" ::"v"(v), "s"(srd), "s"(soff) : "memory");
}
__device__ __forceinline__ unsigned flag_idx(unsigned g, unsigned si, unsigned q) { return g * LSTR + si * 32 + q; }
__host__ __device__ constexpr int stream_pos(int q, int g) {
  const int me = g < 16 ? (g >> 1) : 8 + (g & 3);
  const int key = q + (int)((LAG12 >> (4 * me)) & 15u);
  int u = g < 16 ? (g & 1) : ((g - 16) >> 2);
  for (int s = 0; s < 12; ++s) {
    const int c = key - lag_of(s) - NPUSH + (s < me ? 1 : 0);
    u += (s < 8 ? 2 : 4) * (c < 0 ? 0 : (c > 32 - NPUSH ? 32 - NPUSH : c));
  }
  return u;
}
constexpr unsigned long long first_items(int sub) {
  unsigned long long r = 0;
  for (int it = 0; it < 128; ++it) { const int u = stream_pos(NPUSH + it / 32, it % 32); if (u < (int)NST0 && u % LPL == sub) r |= (unsigned long long)it << (8 * (u / LPL)); }
  return r;
}
template <int I, int N, typename F> __device__ __forceinline__ void static_for(F&& f) {
  if constexpr (I < N) { f(std::integral_constant<int, I>{}); static_for<I + 1, N>(f); }
}


constexpr int ACC0 = 40;
#define MV_T "s_mov_b64 exec, 1\n v_mov_b32 v[40+%c[g]], 1\n buffer_atomic_add v[40+%c[g]], off, %[sl], %[tk] sc0\n s_mov_b64 exec, -1\n"
#define MV_L "buffer_load_dwordx4 v[64+%c[g]*16+k*4:67+%c[g]*16+k*4], %[v16], %[sr], %[ol] offen offset:k*1024 sc0 sc1\n"
#define MV_S "buffer_store_dwordx4 v[64+%c[g]*16+k*4:67+%c[g]*16+k*4], %[v16], %[sl], %[os] offen offset:k*1024 sc1\n"
#define MV_F "s_mov_b64 exec, 1\n buffer_store_dword %[vc], off, %[sl], %[nf] sc1\n s_mov_b64 exec, -1\n"
template <int G> __device__ __forceinline__ void mv_load(v4i sl, v4i sr, unsigned tk, unsigned ol, unsigned v16) {
  asm volatile("s_nop 4\n" MV_T ".set k, 0\n .rept 4\n" MV_L ".set k, k+1\n .endr"
               ::[g] "i"(G), [sl] "s"(sl), [sr] "s"(sr), [tk] "s"(tk), [ol] "s"(ol), [v16] "v"(v16) : "memory");
}
template <int G, int W, int WK> __device__ __forceinline__ unsigned mv_swap(v4i sl, v4i sr, unsigned tk, unsigned os, unsigned ol, unsigned v16, unsigned tab, unsigned call, unsigned nf) {
  unsigned t;
  asm volatile("s_waitcnt vmcnt(%c[w]-1)\n v_min_u32 v56, %c[ntk], v[40+%c[g]]\n v_lshl_add_u32 v56, v56, 2, %[tab]\n ds_read_b32 v56, v56\n" MV_T
               ".set k, 0\n .rept 4\n s_waitcnt vmcnt(%c[w]+%c[wk]*k)\n" MV_S MV_L ".set k, k+1\n .endr\n" MV_F
               "s_waitcnt lgkmcnt(0)\n v_readfirstlane_b32 %[t], v56\n s_nop 1"
               : [t] "=s"(t) : [g] "i"(G), [w] "i"(W), [wk] "i"(WK), [sl] "s"(sl), [sr] "s"(sr), [tk] "s"(tk), [os] "s"(os), [ol] "s"(ol), [v16] "v"(v16),
                 [tab] "s"(tab), [ntk] "i"(NTK - NST0), [vc] "v"(call), [nf] "s"(nf) : "memory");
  return t;
}
template <int G, int W> __device__ __forceinline__ void mv_store(v4i sl, unsigned os, unsigned v16, unsigned call, unsigned nf) {
  asm volatile(".set k, 0\n .rept 4\n s_waitcnt vmcnt(%c[w]-k)\n" MV_S ".set k, k+1\n .endr\n" MV_F
               ::[g] "i"(G), [w] "i"(W), [sl] "s"(sl), [os] "s"(os), [v16] "v"(v16), [vc] "v"(call), [nf] "s"(nf) : "memory");
}
template <int R> __device__ __forceinline__ void zero_acc() {
  asm volatile("v_mov_b32 v%c0, 0\n\tv_mov_b32 v%c1, 0\n\tv_mov_b32 v%c2, 0\n\tv_mov_b32 v%c3, 0" ::"i"(R), "i"(R + 1), "i"(R + 2), "i"(R + 3));
  if constexpr (R + 4 < ACC0 + 128) zero_acc<R + 4>();
}
template <int E> __device__ __forceinline__ void store_row(unsigned sc, unsigned vo, v4i ro) {
  constexpr int R = ACC0 + (E >> 2) * 16 + (E & 3);
  float f0, f1, f2, f3;
  asm volatile("v_cvt_f32_i32 %0, v%c4\n\tv_cvt_f32_i32 %1, v%c5\n\tv_cvt_f32_i32 %2, v%c6\n\tv_cvt_f32_i32 %3, v%c7\n\t"
               "v_mul_f32 %0, %8, %0\n\tv_mul_f32 %1, %8, %1\n\tv_mul_f32 %2, %8, %2\n\tv_mul_f32 %3, %8, %3"
               : "=v"(f0), "=v"(f1), "=v"(f2), "=v"(f3) : "i"(R), "i"(R + 4), "i"(R + 8), "i"(R + 12), "s"(sc));
  v4f v = {f0, f1, f2, f3};
  constexpr unsigned so = (unsigned)(128 * (E >> 4) + 16 * ((E >> 2) & 3) + 4 * (E & 3)) * (DIM * 4);
  asm volatile("buffer_store_dwordx4 %0, %1, %2, %3 offen sc1\n\ts_nop 1" ::"v"(v), "v"(vo), "s"(ro), "s"(so) : "memory");
}
template <int E0, int E1> __device__ __forceinline__ void store_rows(unsigned sc, unsigned vo, v4i ro) {
  if constexpr (E0 < E1) { store_row<E0>(sc, vo, ro); store_rows<E0 + 1, E1>(sc, vo, ro); }
}

__global__ void __launch_bounds__(768) __attribute__((amdgpu_num_vgpr(20))) fused_gemm(Params P) {
  extern __shared__ __attribute__((aligned(16))) char lds[];
  const int tid = threadIdx.x, lane = tid & 63;
  const int wave = (int)rfl((unsigned)tid >> 6);
  const int b = blockIdx.x;
  const int rank = P.rank;
  char* const CAS* const ktab = (char* const CAS*)__builtin_amdgcn_kernarg_segment_ptr();
  char* const wsl = P.wsl;
  unsigned cv = __hip_atomic_load((const unsigned*)(wsl + CNT_OFF + (unsigned)b * 128), __ATOMIC_RELAXED, __HIP_MEMORY_SCOPE_AGENT);
  const unsigned lds0 = (unsigned)(unsigned long long)(LAS char*)lds;
  const unsigned ctl = lds0 + LDS_C;
  const int xcd = b & 7, idx = b >> 3;
  const int ti = 2 * (idx >> 2) + (xcd >> 2), tj = 4 * (xcd & 3) + (idx & 3);
  const int sg = b & 31, so = b >> 5;
  auto plane_of = [&](int j) { return (unsigned)(4 * so + j + 28 + NPUSH) & 31u; };
  const v4i srdL = make_srd(wsl, WS_USED);
  constexpr unsigned long long O0 = order8(0), O1 = order8(1), O2 = order8(2), O3 = order8(3);
  const int mp = (int)(((idx < 8 ? O0 : idx < 16 ? O1 : idx < 24 ? O2 : O3) >> (8 * (idx & 7))) & 31u) * 8 + xcd;
  const int turn = 7 - (mp >> 5), mpair = ((mp >> 3) & 3) * 4 + (mp & 3);
  const bool last = turn == NT, mover = mpair < 14;
  const int pi = mpair % 7, sub = (mpair / 7) * 2 + ((mp >> 2) & 1), pp = (rank + 1 + pi) & 7;
  const unsigned rdy_rest = RDY_OFF + (unsigned)(pp * 64 + 32 + ((lane & 7) == 1 ? 0 : (lane & 7))) * 128;
  auto turn_end = [&](int t) { return (unsigned)((int)((TURNS >> (8 * t)) & 255u) * UEND / 128 - (LPL - 1 - sub) * (STG - (STG - STG_LAST) * t / (NT - 1))); };
  auto entry = [&](unsigned it) { return it | (flag_idx(it & 31, (unsigned)pi, it >> 5) << 16); };
  auto build_table = [&]() {
    const unsigned uend = last ? 4096u : turn_end(turn);
#pragma unroll 1
    for (unsigned it = (unsigned)(tid - 576) + 32 * NPUSH; it < 1024; it += 192) {
      const unsigned u = (unsigned)stream_pos((int)(it >> 5), (int)(it & 31));
      lds_wr(lds0 + LDS_T + u * 4, entry(it) | (u >= uend ? 0x4000u : 0u));
    }
    asm volatile("s_waitcnt lgkmcnt(0)" ::: "memory");
    lds_inc(ctl + C_TAB);
  };
  unsigned call = 0;
  auto get_call = [&]() { unsigned c; asm volatile("v_add_u32 %0, 1, %1" : "=v"(c) : "v"(cv) : "memory"); call = rfl(c); };
  auto wg_barrier = [&]() { asm volatile("s_waitcnt lgkmcnt(0)\n\ts_barrier" ::: "memory"); };
  auto wait_flag = [&](unsigned off) {
    const unsigned long long t0 = rt();
#pragma unroll 1
    for (unsigned n = 1;; ++n) {
      const unsigned v = flag_ld(srdL, off);
      if (__all((int)(v - call) >= 0)) return true;
      if ((n & 15u) == 0u) {
        const unsigned tm = rfl(flag_ld(srdL, TMO_OFF));
        if (tm == call || rt() - t0 > DEADLINE) { flag_st1(srdL, TMO_OFF, call); return false; }
      }
      __builtin_amdgcn_s_sleep(1);
    }
  };

  if (wave < 8) {
    {
      __amdgpu_buffer_rsrc_t rs = __builtin_amdgcn_make_buffer_rsrc(wsl, 0, 2 * BUF_BYTES, 0x00020000);
      const unsigned base = (unsigned)rank * SRC_BYTES;
      char* pk[7] = {};
      const bool lazy = so >= 2 && turn != 0;
      if (lazy) { wg_barrier(); __builtin_amdgcn_s_sleep(STAGE_DELAY); }
      auto load_pk = [&]() {
        asm volatile("" ::: "memory");
#pragma unroll
        for (int k = 0; k < 7; ++k) { pk[k] = ktab[(rank + 1 + (k + wave + b) % 7) & 7]; asm volatile("" : "+s"(pk[k])); }
      };
      if (sg < 16) {
        const int r = 32 * wave + (lane & 31), j = lane >> 5;
        const char* src = P.x + (size_t)(256 * sg + ((r & ~15) | ((r & 3) << 2) | ((r >> 2) & 3))) * KSH + 64 * so + 16 * j;
        v4u v0 = *(const v4u*)src, v1 = *(const v4u*)(src + 32);
        if (!lazy) wg_barrier();
        if (so == 0) load_pk();
        unsigned ncv;
        asm volatile("v_not_b32 %0, %1" : "=v"(ncv) : "v"(cv) : "memory");
        const unsigned d0 = base + (ncv & 1u) * BUF_BYTES + (unsigned)sg * 4096 + (unsigned)r * 16;
        const unsigned d = d0 + plane_of(j + 2) * PLANE_BLK;
        __builtin_amdgcn_raw_buffer_store_b128(v0, rs, d0 + plane_of(j) * PLANE_BLK, 0, 17);
        __builtin_amdgcn_raw_buffer_store_b128(v1, rs, d, 0, 17);
        if (so == 0) {
#pragma unroll
          for (int k = 0; k < 7; ++k) {
            const v4i sp = make_srd(pk[k], 2 * BUF_BYTES);
            asm volatile("s_mov_b32 exec_lo, %c3\n\ts_nop 4\n\tbuffer_store_dwordx4 %0, %1, %2, 0 offen sc0 sc1\n\ts_mov_b64 exec, -1" ::"v"(v1), "v"(d), "s"(sp), "i"(NPUSH == 2 ? -1 : 0) : "memory");
            if (k >= PUSH_WIN - 1) wait_vm<PUSH_WIN - 1>();
          }
        }
        wait_vm<0>(); lds_inc(ctl + C_STG);
      } else if (wave < 4) {
        const int gw = sg - 16;
        const char* src = P.w + (size_t)(64 * so + 16 * wave) * DIM + 256 * gw + 4 * lane;
        unsigned r[16];
#pragma unroll
        for (int k = 0; k < 16; ++k) r[k] = *(const unsigned*)(src + (size_t)k * DIM);
        if (!lazy) wg_barrier();
        if (so == 0) load_pk();
        unsigned ncv;
        asm volatile("v_not_b32 %0, %1" : "=v"(ncv) : "v"(cv) : "memory");
        v4u o[4];
#pragma unroll
        for (int g = 0; g < 4; ++g) {
          unsigned t0 = __builtin_amdgcn_perm(r[4 * g + 1], r[4 * g], 0x05010400u), t1 = __builtin_amdgcn_perm(r[4 * g + 1], r[4 * g], 0x07030602u);
          unsigned t2 = __builtin_amdgcn_perm(r[4 * g + 3], r[4 * g + 2], 0x05010400u), t3 = __builtin_amdgcn_perm(r[4 * g + 3], r[4 * g + 2], 0x07030602u);
          o[0][g] = __builtin_amdgcn_perm(t2, t0, 0x05040100u); o[1][g] = __builtin_amdgcn_perm(t2, t0, 0x07060302u);
          o[2][g] = __builtin_amdgcn_perm(t3, t1, 0x05040100u); o[3][g] = __builtin_amdgcn_perm(t3, t1, 0x07060302u);
        }
        const unsigned d = base + (ncv & 1u) * BUF_BYTES + plane_of(wave) * PLANE_BLK + 65536u + (unsigned)gw * 4096 + (unsigned)(64 * (lane >> 4) + (lane & 15)) * 16;
#pragma unroll
        for (int jj = 0; jj < 4; ++jj) __builtin_amdgcn_raw_buffer_store_b128(o[jj], rs, d + jj * 256, 0, 17);
        if (so == 0 && wave >= 4 - NPUSH) {
#pragma unroll
          for (int k = 0; k < 7; ++k) {
            const v4i sp = make_srd(pk[k], 2 * BUF_BYTES);
            asm volatile("s_nop 4\n\tbuffer_store_dwordx4 %0, %4, %5, 0 offen sc0 sc1\n\tbuffer_store_dwordx4 %1, %4, %5, 0 offen offset:256 sc0 sc1\n\t"
                         "buffer_store_dwordx4 %2, %4, %5, 0 offen offset:512 sc0 sc1\n\tbuffer_store_dwordx4 %3, %4, %5, 0 offen offset:768 sc0 sc1"
                         ::"v"(o[0]), "v"(o[1]), "v"(o[2]), "v"(o[3]), "v"(d), "s"(sp) : "memory");
            if (k >= PUSH_WIN - 1) wait_vm<4 * (PUSH_WIN - 1)>();
          }
        }
        wait_vm<0>(); lds_inc(ctl + C_STG);
      } else if (!lazy) wg_barrier();
    }
    asm volatile("" ::: "v167");
    zero_acc<ACC0>();
    const int wr = wave >> 2, wc = wave & 3;
    unsigned pa = lds0 + LDS_A + (unsigned)(lane >> 4) * 2048 + (unsigned)(64 * wr + (lane & 15)) * 16;
    unsigned pb = lds0 + LDS_B + (unsigned)(lane >> 4) * 4096 + (unsigned)(64 * wc + (lane & 15)) * 16;
    const unsigned vw = ctl + (unsigned)wave * 16 + (unsigned)(lane & 3) * 4;
    const unsigned vd = (lane & 1) ? ctl + (unsigned)(wave ^ 4) * 16 + 8 : ctl + (unsigned)wave * 16 + 12;
    unsigned vo = ((unsigned)(256 * ti + 64 * wr) + (unsigned)(lane >> 4)) * (DIM * 4) + ((unsigned)(256 * tj + 64 * wc) + 4 * (unsigned)(lane & 15)) * 4;
    asm volatile("" : "+v"(vo));
    unsigned sc = rfl(__builtin_bit_cast(unsigned, (*P.sx) * (*P.sw)));
    const v4i ro = make_srd(P.out, DIM * DIM * 4);
    unsigned vcnt = 0, vf, ia = 0, ib = 0, cnt, da, db, spin = 1u << 24;
    unsigned long long ra;
    asm volatile(
      ".macro MM h, i, j\n"
      " v_mfma_i32_16x16x64_i8 v[40+\\h*64+\\i*16+\\j*4:43+\\h*64+\\i*16+\\j*4], v[24+\\i*4:27+\\i*4], v[8+\\j*4:11+\\j*4], v[40+\\h*64+\\i*16+\\j*4:43+\\h*64+\\i*16+\\j*4]\n"
      ".endm\n"
      ".macro STEP h, nb, pm, last\n"
      " s_waitcnt lgkmcnt(4+\\pm)\n"
      " MM \\h, 0, 0\n"
      " s_cmp_eq_u32 %[ia], %c[nh1]\n"
      " s_cselect_b32 %[da], %[wa], 0x2000\n"
      " MM \\h, 1, 0\n"
      " s_cselect_b32 %[ia], -1, %[ia]\n"
      " s_add_u32 %[ia], %[ia], 1\n"
      " .if \\pm\n s_waitcnt lgkmcnt(4)\n .endif\n"
      " MM \\h, 0, 1\n"
      " v_add_u32 %[pa], %[da], %[pa]\n"
      " MM \\h, 1, 1\n"
      " .if \\nb\n s_cmp_eq_u32 %[ib], %c[nb1]\n s_cselect_b32 %[db], %[wb], 0x4000\n .endif\n"
      " .if \\pm\n s_waitcnt lgkmcnt(3)\n .endif\n"
      " MM \\h, 0, 2\n"
      " v_add_u32 %[vcnt], 1, %[vcnt]\n"
      " .if \\nb\n s_cselect_b32 %[ib], -1, %[ib]\n s_add_u32 %[ib], %[ib], 1\n .endif\n"
      " MM \\h, 1, 2\n"
      " .if \\nb\n v_add_u32 %[pb], %[db], %[pb]\n .endif\n"
      " s_waitcnt lgkmcnt(2)\n"
      " MM \\h, 0, 3\n"
      " .if \\last == 0\n v_cmp_le_u32 vcc, %[vf], %[vcnt]\n .endif\n"
      " MM \\h, 1, 3\n"
      " .if \\last == 0\n"
      " s_getpc_b64 %[ra]\n"
      " s_and_b32 %[da], vcc_lo, 3\n"
      " s_cbranch_scc1 Lslow%=\n"
      " ds_read_b128 v[24:27], %[pa]\n"
      " ds_read_b128 v[28:31], %[pa] offset:256\n"
      " .endif\n"
      " s_waitcnt lgkmcnt(2-2*\\last)\n"
      " MM \\h, 2, 0\n"
      " .if \\last == 0\n"
      " ds_write_b32 %[vd], %[vcnt]\n"
      " ds_read_b32 %[vf], %[vw]\n"
      " .endif\n"
      " MM \\h, 3, 0\n"
      " .if \\nb\n ds_read_b128 v[8:11], %[pb]\n .endif\n"
      " MM \\h, 2, 1\n"
      " .if \\last == 0\n"
      " s_bitcmp1_b32 vcc_lo, 2\n"
      " s_setprio 0\n"
      " s_cbranch_scc1 1f\n"
      " s_setprio 1\n"
      "1:\n"
      " .endif\n"
      " MM \\h, 3, 1\n"
      " .if \\nb\n ds_read_b128 v[12:15], %[pb] offset:256\n s_sub_u32 %[cnt], %[cnt], 1\n .endif\n"
      " MM \\h, 2, 2\n"
      " MM \\h, 3, 2\n"
      " .if \\nb\n ds_read_b128 v[16:19], %[pb] offset:512\n s_cmp_lg_u32 %[cnt], 0\n .endif\n"
      " MM \\h, 2, 3\n"
      " MM \\h, 3, 3\n"
      " .if \\nb\n ds_read_b128 v[20:23], %[pb] offset:768\n .endif\n"
      " .if \\last == 0\n"
      " ds_read_b128 v[32:35], %[pa] offset:512\n"
      " ds_read_b128 v[36:39], %[pa] offset:768\n"
      " .endif\n"
      ".endm\n"
      " s_mov_b64 vcc, -1\n"
      " s_getpc_b64 %[ra]\n"
      " s_and_b32 %[da], vcc_lo, 3\n"
      " s_cbranch_scc1 Lslow%=\n"
      " ds_read_b128 v[24:27], %[pa]\n ds_read_b128 v[28:31], %[pa] offset:256\n"
      " ds_read_b128 v[8:11], %[pb]\n ds_read_b128 v[12:15], %[pb] offset:256\n"
      " ds_read_b128 v[16:19], %[pb] offset:512\n ds_read_b128 v[20:23], %[pb] offset:768\n"
      " ds_read_b128 v[32:35], %[pa] offset:512\n ds_read_b128 v[36:39], %[pa] offset:768\n"
      " s_mov_b32 %[cnt], 63\n"
      "Lfull%=:\n"
      " STEP 0, 0, 1, 0\n"
      " STEP 1, 1, 0, 0\n"
      " s_cbranch_scc1 Lfull%=\n"
      " STEP 0, 0, 1, 0\n"
      " STEP 1, 0, 0, 1\n"
      " s_branch Lend%=\n"
      "Lslow%=:\n"
      " ds_read_b32 %[vf], %[vw]\n"
      " s_sub_u32 %[spin], %[spin], 1\n"
      " s_waitcnt lgkmcnt(0)\n"
      " ds_write_b32 %[vd], %[vcnt]\n"
      " v_cmp_le_u32 vcc, %[vf], %[vcnt]\n"
      " s_cbranch_scc1 Lgive%=\n"
      " s_and_b32 %[da], vcc_lo, 3\n"
      " s_cbranch_scc0 Lret%=\n"
      " s_sleep 1\n"
      " s_branch Lslow%=\n"
      "Lgive%=:\n"
      " s_mov_b32 %[spin], 0\n"
      " s_mov_b32 %[sc], 0x7fc00000\n"
      " s_mov_b64 vcc, 0\n"
      "Lret%=:\n"
      " s_setpc_b64 %[ra]\n"
      "Lend%=:\n"
      " s_nop 7\n s_nop 1\n"
      ".purgem STEP\n.purgem MM\n"
      : [pa] "+&v"(pa), [pb] "+&v"(pb), [vcnt] "+&v"(vcnt), [vf] "=&v"(vf), [ia] "+&s"(ia), [ib] "+&s"(ib), [cnt] "=&s"(cnt), [da] "=&s"(da),
        [db] "=&s"(db), [spin] "+&s"(spin), [sc] "+&s"(sc), [ra] "=&s"(ra)
      : [vw] "v"(vw), [vd] "v"(vd), [nh1] "i"(NH - 1), [nb1] "i"(NB - 1), [wa] "s"(-(NH - 1) * 8192), [wb] "s"(-(NB - 1) * 16384)
      : "v8", "v9", "v10", "v11", "v12", "v13", "v14", "v15", "v16", "v17", "v18", "v19", "v20", "v21", "v22", "v23", "v24", "v25", "v26", "v27",
        "v28", "v29", "v30", "v31", "v32", "v33", "v34", "v35", "v36", "v37", "v38", "v39", "vcc", "scc", "memory");
    {
      unsigned v, k = 0;
      do v = lds_rd(ctl + ((lane & 1) ? C_FIN : C_ABORT)); while (__builtin_amdgcn_readlane(v, 1) == 0u && ++k < (1u << 24));
      sc = (__builtin_amdgcn_readlane(v, 0) != 0u || __builtin_amdgcn_readlane(v, 1) == 0u) ? 0x7fc00000u : sc;
    }
    store_rows<0, 32>(sc, vo, ro);
  } else if (wave == 8) {
    wg_barrier();
    lds_wr(lds0 + LDS_T + (NTK + (lane & 15)) * 4, 0xffffffffu);
    get_call();
    if (mover) {
      __builtin_amdgcn_s_setprio(3);
      const v4i srdR = make_srd(ktab[pp], 2 * BUF_BYTES);
      const unsigned v16 = (unsigned)lane * 16;
      const unsigned sbase = (call & 1u) * BUF_BYTES + (unsigned)pp * SRC_BYTES;
      unsigned tk_off = TK_OFF + ((call & 1u) * 8 + (unsigned)pi) * 128;
      const unsigned tk_dead = DUM_OFF + (unsigned)b * 256 + 64;
      const unsigned go_out = DUM_OFF + (unsigned)(last ? mp : mp - 32) * 256 + 192, go_in = DUM_OFF + (unsigned)mp * 256 + 192;
      flag_st1(srdL, tk_dead, 0x1000u);
      unsigned ev;
      bool ok;
      if (turn == 0) {
        lds_wr(ctl + C_MVD, 1u);
        unsigned long long fi = 0;
        static_for<0, LPL>([&](auto S) { constexpr unsigned long long F = first_items(decltype(S)::value); fi = sub == decltype(S)::value ? F : fi; });
        const unsigned it = (unsigned)(fi >> (8 * ((lane & 7) < NGR ? (lane & 7) : NGR))) & 255u, g = it & 31u;
        ev = entry(32u * NPUSH + it);
        asm volatile("" : "+v"(ev));
        ok = wait_flag(RDY_OFF + ((unsigned)pp * 64 + g) * 128);
      } else {
        const unsigned long long t0 = rt();
        const unsigned est = turn_end(turn - 1) * 67u / 10u + EST_ADD;
        while ((unsigned)(rt() - t0) < est) __builtin_amdgcn_s_sleep(16);
        ok = wait_flag(rdy_rest) && wait_flag(go_in);
        lds_wr(ctl + C_MVD + (unsigned)(lane & 1) * 4, (lane & 1) ? 0u : 1u);
        unsigned x0;
        asm volatile("s_mov_b64 exec, 1\n\tv_mov_b32 v47, %3\n\ts_nop 4\n\tbuffer_atomic_add v47, off, %1, %2 sc0\n\ts_mov_b64 exec, -1\n\t"
                     "s_waitcnt vmcnt(0)\n\tv_readfirstlane_b32 %0, v47\n\ts_nop 1" : "=s"(x0) : "s"(srdL), "s"(ok ? tk_off : tk_dead), "s"(NGR + 1) : "memory");
        const unsigned xe = x0 + NST0 + (unsigned)(lane & 7);
        ok = lds_wait(ctl + C_TAB, 3) && ok;
        ev = lds_rd(lds0 + LDS_T + (xe < NTK ? xe : NTK) * 4);
      }
      if (ok) {
        auto relay = [&](auto NGC) {
        constexpr int NG = decltype(NGC)::value;
        const unsigned fdum = DUM_OFF + (unsigned)b * 256;
        unsigned os[NG], en[NG], pf[NG];
        unsigned e;
        unsigned rdy = turn != 0 ? 4096u : (4u + NPUSH) * 32;
        auto foff = [&](unsigned x) { return x == 0xffffffffu ? fdum : PF_OFF + (x >> 16) * 128; };
        auto offs = [&](unsigned x) {
          if (__builtin_expect((x & 0x4000u) != 0u && tk_off != tk_dead, 0)) {
            tk_off = tk_dead;
            flag_st1(srdL, go_out, call);
          }
          return sbase + (x == 0xffffffffu ? 32u * NPUSH : x & 1023u) * 4096;
        };
        auto gate = [&]() {
          if (__builtin_expect((e & 1023u) >= rdy && e != 0xffffffffu, 0)) {
            const unsigned lvl = (e & 1023u) < (28 + NPUSH) * 32 ? 2u : 3u;
            unsigned r, spins = 0;
            while ((r = rfl(lds_rd(ctl + C_RDY))) < lvl && ++spins < (1u << 24)) __builtin_amdgcn_s_sleep(2);
            if (r < lvl || r > 3u) { e = 0xffffffffu; flag_st1(srdL, TMO_OFF, call); }
            else rdy = lvl == 2u ? (28u + NPUSH) * 32 : 4096u;
          }
        };
        auto swap = [&](auto G, auto W, auto WK) {
          constexpr int g = decltype(G)::value;
          const unsigned ol = offs(e), nf = pf[g], o = os[g];
          pf[g] = en[g]; en[g] = foff(e); os[g] = ol;
          e = mv_swap<g, decltype(W)::value, decltype(WK)::value>(srdL, srdR, tk_off, o, ol, v16, lds0 + LDS_T + NST0 * 4, call, nf);
        };
        static_for<0, NG>([&](auto G) {
          constexpr int g = decltype(G)::value;
          const unsigned x = __builtin_amdgcn_readlane(ev, g);
          pf[g] = fdum; en[g] = foff(x); os[g] = offs(x);
          mv_load<g>(srdL, srdR, tk_off, os[g], v16);
        });
        e = __builtin_amdgcn_readlane(ev, NG);
        if (turn == 0) {
          if (!lds_wait(ctl + C_TAB, 3) || !lds_wait(ctl + C_RDY, 1) || rfl(lds_rd(ctl + C_RDY)) > 3u) { wait_vm<0>(); flag_st1(srdL, TMO_OFF, call); return; }
        }
        static_for<0, NG>([&](auto G) {
          constexpr int g = decltype(G)::value;
          gate();
          swap(G, std::integral_constant<int, 4 + (NG - 1 - g) * 5 + g * 10>{}, std::integral_constant<int, 1>{});
        });
        int stop = -1;
#pragma unroll 1
        while (stop < 0) {
          static_for<0, NG>([&](auto G) {
            constexpr int g = decltype(G)::value;
            if (stop < 0) {
              gate();
              if (e == 0xffffffffu) stop = g;
              else {
                swap(G, std::integral_constant<int, 8 + (NG - 1) * 10>{}, std::integral_constant<int, 0>{});
              }
            }
          });
        }
        if (tk_off != tk_dead) flag_st1(srdL, go_out, call);
        static_for<0, NG>([&](auto G) {
          constexpr int g = decltype(G)::value;
          if (stop == g) static_for<0, NG>([&](auto J) {
            constexpr int j = decltype(J)::value, h = (g + j) % NG;
            mv_store<h, 7 + (NG - 1 - j) * 10 + j * 5>(srdL, os[h], v16, call, pf[h]);
          });
        });
        wait_vm<0>();
        static_for<0, NG>([&](auto G) { flag_st1(srdL, en[decltype(G)::value], call); });
        };
        relay(std::integral_constant<int, NGR>{});
        lds_wr(ctl + C_MVD, 2u);
      }
      __builtin_amdgcn_s_setprio(0);
    }
    lds_wr(ctl + C_MVD, 2u);
    if (rfl(lds_rd(ctl + C_ABORT)) != 0u && lds_wait(ctl + C_FIN, 1)) lds_wr(ctl + C_AV, 64u);
  } else if (wave == 9) {
    __builtin_amdgcn_s_setprio(2);
    lds_wr(ctl + lane * 4, 0u);
    wg_barrier();
    const bool direct = so == 1;
    if (!direct) build_table();
    get_call();
    {
      char* pb = wsl;
#pragma unroll
      for (int k = 0; k < 8; ++k) { char* t = P.ws[k]; pb = (lane == k || (lane >= 32 && (lane & 7) == k)) ? t : pb; }
      const unsigned off = lane < 8 ? RDY_OFF + (unsigned)(rank * 64 + (direct ? sg : 32 + so)) * 128
                         : lane < 16 ? PF_OFF + flag_idx((unsigned)sg, 7u, plane_of(lane - 8)) * 128
                         : lane == 16 ? CNT_OFF + (unsigned)b * 128
                         : lane < 32 ? TK_OFF + ((~call & 1u) * 8 + (unsigned)(lane - 17)) * 128
                         : PF_OFF + flag_idx((unsigned)sg, (unsigned)(rank - lane - 1) & 7u, (unsigned)lane >> 3 & 1u) * 128;
      unsigned* dst = (unsigned*)(pb + off);
      const unsigned val = (lane > 16 && lane < 32) ? 0u : call;
      const bool on = lane < 8 ? (direct && lane != rank) : lane <= 16 ? (lane < 12 || lane == 16) : lane < 32 ? (b == 255 && lane < 24) : (so == 0 && lane < 32 + 8 * NPUSH && (lane & 7) != rank);
      asm volatile("" : "+v"(dst));
      const bool staged = lds_wait(ctl + C_STG, sg < 16 ? 8 : 4);
      if (!staged) flag_st1(srdL, TMO_OFF, call);
      if (on && (staged || (lane >= 16 && lane < 32))) __hip_atomic_store(dst, val, __ATOMIC_RELAXED, __HIP_MEMORY_SCOPE_SYSTEM);
      if (direct) build_table();
      else if (staged) {
        unsigned tk = 0;
        if (lane == 0) tk = __hip_atomic_fetch_add((unsigned*)(wsl + AGG_OFF + (unsigned)so * 128), 1u, __ATOMIC_RELAXED, __HIP_MEMORY_SCOPE_AGENT);
        if (rfl(tk) + 1u == call * 32u && lane < 8 && lane != rank) __hip_atomic_store(dst, val, __ATOMIC_RELAXED, __HIP_MEMORY_SCOPE_SYSTEM);
      }
    }
    const unsigned si = (unsigned)lane >> 3, ro = ((unsigned)lane >> 1) & 3u;
    const unsigned fbase = PF_OFF + flag_idx((lane & 1) ? 16u + tj : (unsigned)ti, si, ro) * 128;
    const unsigned pbase = (call & 1u) * BUF_BYTES + (((unsigned)rank + 1 + si) & 7u) * SRC_BYTES + ro * PLANE_BLK;
    const unsigned lateN = POLL_LATE - 7 * (unsigned)(((LAGS >> (4 * (idx >> 2))) & 15u) > ((LAGW >> (4 * (idx & 3))) & 15u) ? ((LAGS >> (4 * (idx >> 2))) & 15u) : ((LAGW >> (4 * (idx & 3))) & 15u));
    unsigned q = 0, cnt = 0, npoll = 0, win = 4;
    bool mvdone = false;
    const unsigned long long t0 = rt();
#pragma unroll 1
    while (cnt < 256 || !mvdone) {
      if (!mvdone) { const unsigned m = rfl(lds_rd(ctl + C_MVD)); mvdone = m == 2u; if (m == 1u) { lds_wr(ctl + C_AV, 0u); win = 4; __builtin_amdgcn_s_sleep(4); continue; } }
      unsigned v = call - 1;
      if (ro < win && q + ro < 32) {
        asm volatile("buffer_load_dword %0, %1, %2, 0 offen sc0 sc1\n\ts_waitcnt vmcnt(0)" : "=v"(v) : "v"(fbase + q * 128), "s"(srdL) : "memory");
      }
      const unsigned long long hm = __ballot((int)(v - call) >= 0);
      const unsigned adv = (unsigned)__builtin_ctz(~((unsigned)(hm >> (8 * si)) & 255u)) >> 1;
      const bool emit = !(lane & 1) && ro < adv;
      const unsigned long long em = __ballot(emit);
      if (emit) lds_wr(lds0 + LDS_L + (cnt + __builtin_amdgcn_mbcnt_hi((unsigned)(em >> 32), __builtin_amdgcn_mbcnt_lo((unsigned)em, 0u))) * 4, pbase + q * PLANE_BLK);
      q += adv;
      cnt += (unsigned)__builtin_popcountll(em);
      const unsigned lim = cnt < 160 ? (unsigned)(rt() - t0) / LOCAL_GAP + 1 : 255u;
      lds_wr(ctl + C_AV, (cnt >> 2) < lim ? cnt >> 2 : lim);
      const bool more = __ballot(adv == win && q < 32) != 0ull;
      win = more ? 4u : 2u;
      if (!more) { if (cnt < lateN) __builtin_amdgcn_s_sleep(POLL_SLEEP); else __builtin_amdgcn_s_sleep(POLL_SLEEP2); }
      if (em == 0ull && (++npoll & 15u) == 0u) {
        const unsigned tm = rfl(flag_ld(srdL, TMO_OFF));
        if (tm == call || rt() - t0 > DEADLINE) {
          flag_st1(srdL, TMO_OFF, call);
          lds_wr(ctl + C_ABORT, 1u);
#pragma unroll 1
          for (unsigned k = (unsigned)lane; k < 256; k += 64) lds_wr(lds0 + LDS_L + k * 4, 0u);
          asm volatile("s_waitcnt lgkmcnt(0)" ::: "memory");
          break;
        }
      }
    }
    lds_wr(ctl + C_AV, 64u);
    if (rfl(flag_ld(srdL, TMO_OFF)) == call) lds_wr(ctl + C_ABORT, 1u);
    asm volatile("s_waitcnt lgkmcnt(0)" ::: "memory");
    lds_wr(ctl + C_FIN, 1u);
  } else {
    wg_barrier();
    build_table();
    __builtin_amdgcn_s_setprio(3);
    get_call();
    if (turn == 0 && wave == 10) {
      bool ok = wait_flag(RDY_OFF + (unsigned)(pp * 64 + (lane & 31)) * 128);
      if (ok) { lds_wr(ctl + C_RDY, 1u); ok = wait_flag(RDY_OFF + (unsigned)(pp * 64 + 32 + ((lane & 7) < 2 ? 2 : (lane & 7))) * 128); }
      if (ok) { lds_wr(ctl + C_RDY, 2u); ok = wait_flag(rdy_rest); }
      lds_wr(ctl + C_RDY, ok ? 3u : 9u);
    }
    auto loader = [&](auto IA) {
      constexpr bool isA = decltype(IA)::value;
      constexpr unsigned RINGB = isA ? NH * 8192 : NB * 16384;
      const unsigned v16 = (unsigned)lane * 16 + (isA ? (unsigned)ti : 16u + tj) * 4096, v16b = v16 + 2048u;
      unsigned vchk = lane < 8 ? ctl + lane * 16 + 12 : lane < 16 ? ctl + C_AV : lds0 + LDS_L + (unsigned)(lane - 16) * 4;
      const unsigned vadv = lane < 16 ? 0u : 16u;
      const unsigned vfull = lane < 8 ? ctl + (unsigned)lane * 16 + (isA ? 0 : 4) : ctl + (isA ? 512 : 768) + lane * 4;
      const unsigned ring0 = lds0 + (isA ? LDS_A : LDS_B);
      unsigned ring = ring0, n = 0, nops = 0, spin = 1u << 25;
      const v4i srd = srdL;
      int vneed = lane < 8 ? (isA ? 2 - NH : 1 - 2 * NB) : lane < 16 ? 1 : (int)0x80000000u;
      const int vinc = lane < 8 ? 2 : lane < 16 ? 1 : 0;
      wait_vm<0>();
#define L_PUB \
      " s_getreg_b32 %[c], hwreg(HW_REG_IB_STS)\n" \
      " s_bfe_u32 %[x], %[c], 0x20016\n" \
      " s_and_b32 %[c], %[c], 15\n" \
      " s_lshl4_add_u32 %[c], %[x], %[c]\n" \
      " s_sub_u32 %[x], %[nops], %[c]\n" \
      " s_lshr_b32 %[x], %[x], 3\n" \
      " .if %c[isa] == 0\n s_and_b32 %[x], %[x], -2\n .endif\n" \
      " v_mov_b32 %[vl], %[x]\n" \
      " ds_write_b32 %[vfull], %[vl]\n"
#define L_D2(v, o, k) \
      " buffer_load_dwordx4 %[" v "], %[srd], %[" o "] offen offset:" k " sc0 sc1 lds\n" \
      " buffer_load_dwordx4 %[" v "], %[srd], %[" o "] offen offset:" k "+1024 sc0 sc1 lds\n"
#define L_A(v, o) L_D2(v, o, "0") " s_add_u32 m0, m0, 0x800\n s_nop 0\n"
#define L_B(o) L_D2("v16", o, "0") L_D2("v16", o, "2048") " s_add_u32 m0, m0, 0x1000\n s_nop 0\n"
      unsigned v, vl, o0, o1, o2, o3, c, x, j, keep;
      asm volatile(
        " s_mov_b32 %[km0], m0\n"
        "Lpoll%=:\n"
        " ds_read_b32 %[v], %[vchk]\n"
        L_PUB
        " s_mov_b32 %[j], 16\n"
        " s_sub_u32 %[spin], %[spin], 1\n"
        " s_cbranch_scc1 Lout%=\n"
        " s_waitcnt lgkmcnt(1)\n"
        "Lchk%=:\n"
        " v_cmp_lt_i32 vcc, %[v], %[vneed]\n"
        " v_readlane_b32 %[o0], %[v], %[j]\n"
        " s_add_u32 %[j], %[j], 1\n"
        " v_readlane_b32 %[o1], %[v], %[j]\n"
        " s_cmp_gt_u32 %[c], 47\n"
        " s_cbranch_scc1 Lpoll%=\n"
        " s_cbranch_vccnz Lpoll%=\n"
        " s_add_u32 %[j], %[j], 1\n"
        " v_readlane_b32 %[o2], %[v], %[j]\n"
        " s_add_u32 %[j], %[j], 1\n"
        " v_readlane_b32 %[o3], %[v], %[j]\n"
        " s_mov_b32 m0, %[ring]\n"
        " s_add_u32 %[n], %[n], 1\n"
        " .if %c[isa]\n"
        L_A("v16", "o0") L_A("v16", "o1") L_A("v16", "o2") L_A("v16", "o3")
        " s_add_u32 %[nops], %[nops], 8\n"
        L_PUB
        L_A("v16b", "o0") L_A("v16b", "o1") L_A("v16b", "o2") L_A("v16b", "o3")
        " s_add_u32 %[nops], %[nops], 8\n"
        " .else\n"
        L_B("o0") L_B("o1") L_B("o2") L_B("o3")
        " s_add_u32 %[nops], %[nops], 16\n"
        " .endif\n"
        " v_add_u32 %[vneed], %[vinc], %[vneed]\n"
        " v_add_u32 %[vchk], %[vadv], %[vchk]\n"
        " s_cmp_eq_u32 m0, %[rend]\n"
        " s_cselect_b32 %[ring], %[ring0], m0\n"
        " s_add_u32 %[j], %[j], 1\n"
        L_PUB
        " s_cmp_lt_u32 %[n], 64\n"
        " s_cbranch_scc1 Lchk%=\n"
        "Lout%=:\n"
        L_PUB
        " s_cmp_lg_u32 %[c], 0\n"
        " s_cbranch_scc1 Lout%=\n"
        " s_mov_b32 m0, %[km0]\n"
        : [v] "=&v"(v), [vneed] "+&v"(vneed), [vchk] "+&v"(vchk), [vl] "=&v"(vl), [o0] "=&s"(o0), [o1] "=&s"(o1), [o2] "=&s"(o2), [o3] "=&s"(o3), [c] "=&s"(c),
          [x] "=&s"(x), [j] "=&s"(j), [km0] "=&s"(keep), [n] "+&s"(n), [nops] "+&s"(nops), [ring] "+&s"(ring), [spin] "+&s"(spin)
        : [v16] "v"(v16), [v16b] "v"(v16b), [vfull] "v"(vfull), [vinc] "v"(vinc), [vadv] "v"(vadv),
          [srd] "s"(srd), [ring0] "s"(ring0), [rend] "s"(ring0 + RINGB), [isa] "i"(isA ? 1 : 0)
        : "vcc", "scc", "memory");
      if (n != 64u) lds_wr(ctl + C_ABORT, 1u);
      asm volatile("s_waitcnt lgkmcnt(0)" ::: "memory");
      lds_wr(vfull, 255u);
    };
    if (wave == 10) loader(std::true_type{}); else loader(std::false_type{});
  }
}

extern "C" void kernel_launch(int rank, int nranks, ncclComm_t comm, hipStream_t stream,
                              void* const* d_in, const int* in_sizes, int n_in,
                              void* d_out, int out_size, const int* in_dtypes, int out_dtype,
                              void* const* d_out_all, void* const* d_ws_all, size_t ws_bytes) {
  (void)comm; (void)in_sizes; (void)n_in; (void)out_size; (void)in_dtypes; (void)out_dtype; (void)d_out_all;
  if (nranks != NR || (unsigned)rank >= (unsigned)NR || ws_bytes < WS_USED) { fprintf(stderr, "unexpected configuration\n"); return; }
  Params p;
  const size_t woff = (ws_bytes - WS_USED) & ~(size_t)0xfffff;
  for (int i = 0; i < NR; ++i) p.ws[i] = (char*)d_ws_all[i] + woff;
  p.x = (const char*)d_in[0]; p.w = (const char*)d_in[1];
  p.sx = (const float*)d_in[2]; p.sw = (const float*)d_in[3];
  p.out = (float*)d_out; p.wsl = p.ws[rank]; p.rank = rank;
  static bool attr_set[NR] = {};
  if (!attr_set[rank]) {
    HIP_CHECK(hipFuncSetAttribute((const void*)fused_gemm, hipFuncAttributeMaxDynamicSharedMemorySize, (int)LDS_BYTES));
    attr_set[rank] = true;
  }
  fused_gemm<<<256, 768, LDS_BYTES, stream>>>(p);
  HIP_CHECK(hipGetLastError());
}
```

```cpp
#include <hip/hip_runtime.h>
#include <rccl/rccl.h>
#include <cstdio>
#include <cstdint>
#include <type_traits>

#define HIP_CHECK(expr) do { hipError_t _e = (expr); if (_e != hipSuccess) { \
    fprintf(stderr, "HIP error %s at %s:%d\n", hipGetErrorString(_e), __FILE__, __LINE__); \
    return; } } while (0)

typedef int v4i __attribute__((ext_vector_type(4)));
typedef unsigned v4u __attribute__((ext_vector_type(4)));
typedef float v4f __attribute__((ext_vector_type(4)));
#define LAS __attribute__((address_space(3)))
#define CAS __attribute__((address_space(4)))

#define LAGS 0x77420000u
#define LAGW 0x7100u
#define NGR 3
#define LAST_LEAD 32
#define TURNS 0x807a70614d3419ull
#define STG 24
#define STG_LAST 4
#define EST_ADD 150
#define NPUSH 1
#define PUSH_WIN 3
#define STAGE_DELAY 64
#define POLL_LATE 232
#define POLL_SLEEP 24
#define POLL_SLEEP2 2
#define LOCAL_GAP 100
#define LSTR 273

constexpr int NR = 8;
constexpr int DIM = 4096, KSH = 512;
constexpr unsigned PLANE_BLK = 128u << 10;
constexpr unsigned SRC_BYTES = 32 * PLANE_BLK;
constexpr unsigned BUF_BYTES = NR * SRC_BYTES;
constexpr unsigned PF_OFF = 2 * BUF_BYTES;
constexpr unsigned RDY_OFF = PF_OFF + 32 * 600 * 128;
constexpr unsigned CNT_OFF = RDY_OFF + NR * 64 * 128;
constexpr unsigned TMO_OFF = CNT_OFF + 256 * 128;
constexpr unsigned AGG_OFF = TMO_OFF + 3072;
constexpr unsigned TK_OFF = TMO_OFF + 1024;
constexpr unsigned DUM_OFF = TMO_OFF + 4096;
constexpr unsigned WS_USED = DUM_OFF + 256 * 256;
constexpr unsigned long long DEADLINE = 60000000ull;

constexpr int NH = 10, NB = 4;
constexpr unsigned LDS_A = 0, LDS_B = NH * 8192, LDS_C = LDS_B + NB * 16384, LDS_T = LDS_C + 1024, LDS_L = LDS_T + 4096, LDS_BYTES = LDS_L + 1280;
static_assert(LDS_BYTES <= 163840 && NH % 2 == 0, "LDS");
constexpr unsigned C_ABORT = 128, C_MVD = 132, C_AV = 136, C_FIN = 140, C_RDY = 144, C_STG = 148, C_TAB = 156;
constexpr unsigned NTK = 32 * (32 - NPUSH);
constexpr int LPL = 4, NT = 7;
constexpr unsigned NST0 = LPL * (NGR + 1);
constexpr unsigned long long LAG12 = (unsigned long long)LAGS | ((unsigned long long)LAGW << 32);
constexpr int lag_of(int s) { return (int)((LAG12 >> (4 * s)) & 15u); }
constexpr int late_groups() { int r = 0; for (int s = 0; s < 12; ++s) r += (s < 8 ? 2 : 4) * lag_of(s); return r; }
constexpr int UEND = (int)NTK - late_groups() - LAST_LEAD;
constexpr int tile_lag(int idx) { return lag_of(idx >> 2) > lag_of(8 + (idx & 3)) ? lag_of(idx >> 2) : lag_of(8 + (idx & 3)); }
constexpr unsigned long long order8(int k) {
  unsigned long long r = 0;
  for (int i = 8 * k; i < 8 * k + 8; ++i) {
    int c = 0;
    for (int j = 0; j < 32; ++j) c += tile_lag(j) < tile_lag(i) || (tile_lag(j) == tile_lag(i) && j < i);
    r |= (unsigned long long)c << (8 * (i & 7));
  }
  return r;
}

struct Params {
  char* ws[NR];
  const char* x;
  const char* w;
  const float* sx;
  const float* sw;
  float* out;
  char* wsl;
  int rank;
};

__device__ __forceinline__ unsigned rfl(unsigned v) { return __builtin_amdgcn_readfirstlane(v); }
__device__ __forceinline__ v4i make_srd(const void* p, unsigned bytes) {
  unsigned long long a = (unsigned long long)p;
  v4i r;
  r.x = (int)rfl((unsigned)a); r.y = (int)(rfl((unsigned)(a >> 32)) & 0xffffu); r.z = (int)bytes; r.w = 0x00020000;
  return r;
}
__device__ __forceinline__ unsigned long long rt() { return __builtin_amdgcn_s_memrealtime(); }
template <int N> __device__ __forceinline__ void wait_vm() { asm volatile("s_waitcnt vmcnt(%0)" ::"i"(N) : "memory"); }
__device__ __forceinline__ unsigned lds_rd(unsigned addr) {
  unsigned v;
  asm volatile("ds_read_b32 %0, %1\n\ts_waitcnt lgkmcnt(0)" : "=v"(v) : "v"(addr) : "memory");
  return v;
}
__device__ __forceinline__ void lds_wr(unsigned addr, unsigned v) { asm volatile("ds_write_b32 %0, %1" ::"v"(addr), "v"(v) : "memory"); }
__device__ __forceinline__ void lds_inc(unsigned a) {
  asm volatile("s_mov_b64 exec, 1\n\tds_add_u32 %0, %1\n\ts_mov_b64 exec, -1" ::"v"(a), "v"(1u) : "memory");
}
__device__ __forceinline__ bool lds_wait(unsigned a, unsigned n) { unsigned k = 0; while (rfl(lds_rd(a)) < n) if (++k == (1u << 25)) return false; return true; }
__device__ __forceinline__ unsigned flag_ld(v4i srd, unsigned voff) {
  unsigned v;
  asm volatile("s_nop 4\n\tbuffer_load_dword %0, %1, %2, 0 offen sc0 sc1\n\ts_waitcnt vmcnt(0)" : "=v"(v) : "v"(voff), "s"(srd) : "memory");
  return v;
}
__device__ __forceinline__ void flag_st1(v4i srd, unsigned soff, unsigned v) {
  asm volatile("s_mov_b64 exec, 1\n\ts_nop 4\n\tbuffer_store_dword %0, off, %1, %2 sc0 sc1\n\ts_mov_b64 exec, -1" ::"v"(v), "s"(srd), "s"(soff) : "memory");
}
__device__ __forceinline__ unsigned flag_idx(unsigned g, unsigned si, unsigned q) { return g * LSTR + si * 32 + q; }
__host__ __device__ constexpr int stream_pos(int q, int g) {
  const int me = g < 16 ? (g >> 1) : 8 + (g & 3);
  const int key = q + (int)((LAG12 >> (4 * me)) & 15u);
  int u = g < 16 ? (g & 1) : ((g - 16) >> 2);
  for (int s = 0; s < 12; ++s) {
    const int c = key - lag_of(s) - NPUSH + (s < me ? 1 : 0);
    u += (s < 8 ? 2 : 4) * (c < 0 ? 0 : (c > 32 - NPUSH ? 32 - NPUSH : c));
  }
  return u;
}
constexpr unsigned long long first_items(int sub) {
  unsigned long long r = 0;
  for (int it = 0; it < 128; ++it) { const int u = stream_pos(NPUSH + it / 32, it % 32); if (u < (int)NST0 && u % LPL == sub) r |= (unsigned long long)it << (8 * (u / LPL)); }
  return r;
}
template <int I, int N, typename F> __device__ __forceinline__ void static_for(F&& f) {
  if constexpr (I < N) { f(std::integral_constant<int, I>{}); static_for<I + 1, N>(f); }
}


constexpr int ACC0 = 40;
#define MV_T "s_mov_b64 exec, 1\n v_mov_b32 v[40+%c[g]], 1\n buffer_atomic_add v[40+%c[g]], off, %[sl], %[tk] sc0\n s_mov_b64 exec, -1\n"
#define MV_L "buffer_load_dwordx4 v[64+%c[g]*16+k*4:67+%c[g]*16+k*4], %[v16], %[sr], %[ol] offen offset:k*1024 sc0 sc1\n"
#define MV_S "buffer_store_dwordx4 v[64+%c[g]*16+k*4:67+%c[g]*16+k*4], %[v16], %[sl], %[os] offen offset:k*1024 sc1\n"
#define MV_F "s_mov_b64 exec, 1\n buffer_store_dword %[vc], off, %[sl], %[nf] sc1\n s_mov_b64 exec, -1\n"
template <int G> __device__ __forceinline__ void mv_load(v4i sl, v4i sr, unsigned tk, unsigned ol, unsigned v16) {
  asm volatile("s_nop 4\n" MV_T ".set k, 0\n .rept 4\n" MV_L ".set k, k+1\n .endr"
               ::[g] "i"(G), [sl] "s"(sl), [sr] "s"(sr), [tk] "s"(tk), [ol] "s"(ol), [v16] "v"(v16) : "memory");
}
template <int G, int W, int WK> __device__ __forceinline__ unsigned mv_swap(v4i sl, v4i sr, unsigned tk, unsigned os, unsigned ol, unsigned v16, unsigned tab, unsigned call, unsigned nf) {
  unsigned t;
  asm volatile("s_waitcnt vmcnt(%c[w]-1)\n v_min_u32 v56, %c[ntk], v[40+%c[g]]\n v_lshl_add_u32 v56, v56, 2, %[tab]\n ds_read_b32 v56, v56\n" MV_T
               ".set k, 0\n .rept 4\n s_waitcnt vmcnt(%c[w]+%c[wk]*k)\n" MV_S MV_L ".set k, k+1\n .endr\n" MV_F
               "s_waitcnt lgkmcnt(0)\n v_readfirstlane_b32 %[t], v56\n s_nop 1"
               : [t] "=s"(t) : [g] "i"(G), [w] "i"(W), [wk] "i"(WK), [sl] "s"(sl), [sr] "s"(sr), [tk] "s"(tk), [os] "s"(os), [ol] "s"(ol), [v16] "v"(v16),
                 [tab] "s"(tab), [ntk] "i"(NTK - NST0), [vc] "v"(call), [nf] "s"(nf) : "memory");
  return t;
}
template <int G, int W> __device__ __forceinline__ void mv_store(v4i sl, unsigned os, unsigned v16, unsigned call, unsigned nf) {
  asm volatile(".set k, 0\n .rept 4\n s_waitcnt vmcnt(%c[w]-k)\n" MV_S ".set k, k+1\n .endr\n" MV_F
               ::[g] "i"(G), [w] "i"(W), [sl] "s"(sl), [os] "s"(os), [v16] "v"(v16), [vc] "v"(call), [nf] "s"(nf) : "memory");
}
template <int R> __device__ __forceinline__ void zero_acc() {
  asm volatile("v_mov_b32 v%c0, 0\n\tv_mov_b32 v%c1, 0\n\tv_mov_b32 v%c2, 0\n\tv_mov_b32 v%c3, 0" ::"i"(R), "i"(R + 1), "i"(R + 2), "i"(R + 3));
  if constexpr (R + 4 < ACC0 + 128) zero_acc<R + 4>();
}
template <int E> __device__ __forceinline__ void store_row(unsigned sc, unsigned vo, v4i ro) {
  constexpr int R = ACC0 + (E >> 2) * 16 + (E & 3);
  float f0, f1, f2, f3;
  asm volatile("v_cvt_f32_i32 %0, v%c4\n\tv_cvt_f32_i32 %1, v%c5\n\tv_cvt_f32_i32 %2, v%c6\n\tv_cvt_f32_i32 %3, v%c7\n\t"
               "v_mul_f32 %0, %8, %0\n\tv_mul_f32 %1, %8, %1\n\tv_mul_f32 %2, %8, %2\n\tv_mul_f32 %3, %8, %3"
               : "=v"(f0), "=v"(f1), "=v"(f2), "=v"(f3) : "i"(R), "i"(R + 4), "i"(R + 8), "i"(R + 12), "s"(sc));
  v4f v = {f0, f1, f2, f3};
  constexpr unsigned so = (unsigned)(128 * (E >> 4) + 16 * ((E >> 2) & 3) + 4 * (E & 3)) * (DIM * 4);
  asm volatile("buffer_store_dwordx4 %0, %1, %2, %3 offen sc1\n\ts_nop 1" ::"v"(v), "v"(vo), "s"(ro), "s"(so) : "memory");
}
template <int E0, int E1> __device__ __forceinline__ void store_rows(unsigned sc, unsigned vo, v4i ro) {
  if constexpr (E0 < E1) { store_row<E0>(sc, vo, ro); store_rows<E0 + 1, E1>(sc, vo, ro); }
}

__global__ void __launch_bounds__(768) __attribute__((amdgpu_num_vgpr(20))) fused_gemm(Params P) {
  extern __shared__ __attribute__((aligned(16))) char lds[];
  const int tid = threadIdx.x, lane = tid & 63;
  asm volatile("" ::"s"(P.ws[0]), "s"(P.x), "s"(P.w), "s"(P.wsl), "s"(P.rank));
  const int wave = (int)rfl((unsigned)tid >> 6);
  const int b = blockIdx.x;
  const int rank = P.rank;
  char* const CAS* const ktab = (char* const CAS*)__builtin_amdgcn_kernarg_segment_ptr();
  char* const wsl = P.wsl;
  unsigned cv = __hip_atomic_load((const unsigned*)(wsl + CNT_OFF + (unsigned)b * 128), __ATOMIC_RELAXED, __HIP_MEMORY_SCOPE_AGENT);
  const unsigned lds0 = (unsigned)(unsigned long long)(LAS char*)lds;
  const unsigned ctl = lds0 + LDS_C;
  const int xcd = b & 7, idx = b >> 3;
  const int ti = 2 * (idx >> 2) + (xcd >> 2), tj = 4 * (xcd & 3) + (idx & 3);
  const int sg = b & 31, so = b >> 5;
  auto plane_of = [&](int j) { return (unsigned)(4 * so + j + 28 + NPUSH) & 31u; };
  const v4i srdL = make_srd(wsl, WS_USED);
  constexpr unsigned long long O0 = order8(0), O1 = order8(1), O2 = order8(2), O3 = order8(3);
  const int mp = (int)(((idx < 8 ? O0 : idx < 16 ? O1 : idx < 24 ? O2 : O3) >> (8 * (idx & 7))) & 31u) * 8 + xcd;
  const int turn = 7 - (mp >> 5), mpair = ((mp >> 3) & 3) * 4 + (mp & 3);
  const bool last = turn == NT, mover = mpair < 14;
  const int pi = mpair % 7, sub = (mpair / 7) * 2 + ((mp >> 2) & 1), pp = (rank + 1 + pi) & 7;
  const unsigned rdy_rest = RDY_OFF + (unsigned)(pp * 64 + 32 + ((lane & 7) == 1 ? 0 : (lane & 7))) * 128;
  auto turn_end = [&](int t) { return (unsigned)((int)((TURNS >> (8 * t)) & 255u) * UEND / 128 - (LPL - 1 - sub) * (STG - (STG - STG_LAST) * t / (NT - 1))); };
  auto entry = [&](unsigned it) { return it | (flag_idx(it & 31, (unsigned)pi, it >> 5) << 16); };
  auto build_table = [&]() {
    const unsigned uend = last ? 4096u : turn_end(turn);
#pragma unroll 1
    for (unsigned it = (unsigned)(tid - 576) + 32 * NPUSH; it < 1024; it += 192) {
      const unsigned u = (unsigned)stream_pos((int)(it >> 5), (int)(it & 31));
      lds_wr(lds0 + LDS_T + u * 4, entry(it) | (u >= uend ? 0x4000u : 0u));
    }
    asm volatile("s_waitcnt lgkmcnt(0)" ::: "memory");
    lds_inc(ctl + C_TAB);
  };
  unsigned call = 0;
  auto get_call = [&]() { unsigned c; asm volatile("v_add_u32 %0, 1, %1" : "=v"(c) : "v"(cv) : "memory"); call = rfl(c); };
  auto wg_barrier = [&]() { asm volatile("s_waitcnt lgkmcnt(0)\n\ts_barrier" ::: "memory"); };
  auto wait_flag = [&](unsigned off) {
    const unsigned long long t0 = rt();
#pragma unroll 1
    for (unsigned n = 1;; ++n) {
      const unsigned v = flag_ld(srdL, off);
      if (__all((int)(v - call) >= 0)) return true;
      if ((n & 15u) == 0u) {
        const unsigned tm = rfl(flag_ld(srdL, TMO_OFF));
        if (tm == call || rt() - t0 > DEADLINE) { flag_st1(srdL, TMO_OFF, call); return false; }
      }
      __builtin_amdgcn_s_sleep(1);
    }
  };

  if (wave < 8) {
    {
      __amdgpu_buffer_rsrc_t rs = __builtin_amdgcn_make_buffer_rsrc(wsl, 0, 2 * BUF_BYTES, 0x00020000);
      const unsigned base = (unsigned)rank * SRC_BYTES;
      char* pk[7] = {};
      const bool lazy = so >= 2 && turn != 0;
      if (lazy) { wg_barrier(); __builtin_amdgcn_s_sleep(STAGE_DELAY); }
      auto load_pk = [&]() {
        asm volatile("" ::: "memory");
        const int rot = (wave + b) % 7;
#pragma unroll
        for (int k = 0; k < 7; ++k) { const int r = k + rot; pk[k] = ktab[(rank + 1 + (r >= 7 ? r - 7 : r)) & 7]; }
        asm volatile("" : "+s"(pk[0]), "+s"(pk[1]), "+s"(pk[2]), "+s"(pk[3]), "+s"(pk[4]), "+s"(pk[5]), "+s"(pk[6]));
      };
      if (sg < 16) {
        const int r = 32 * wave + (lane & 31), j = lane >> 5;
        const char* src = P.x + (size_t)(256 * sg + ((r & ~15) | ((r & 3) << 2) | ((r >> 2) & 3))) * KSH + 64 * so + 16 * j;
        v4u v0 = *(const v4u*)src, v1 = *(const v4u*)(src + 32);
        if (!lazy) wg_barrier();
        if (so == 0) load_pk();
        unsigned ncv;
        asm volatile("v_not_b32 %0, %1" : "=v"(ncv) : "v"(cv) : "memory");
        const unsigned d0 = base + (ncv & 1u) * BUF_BYTES + (unsigned)sg * 4096 + (unsigned)r * 16;
        const unsigned d = d0 + plane_of(j + 2) * PLANE_BLK;
        __builtin_amdgcn_raw_buffer_store_b128(v0, rs, d0 + plane_of(j) * PLANE_BLK, 0, 17);
        __builtin_amdgcn_raw_buffer_store_b128(v1, rs, d, 0, 17);
        if (so == 0) {
#pragma unroll
          for (int k = 0; k < 7; ++k) {
            const v4i sp = make_srd(pk[k], 2 * BUF_BYTES);
            asm volatile("s_mov_b32 exec_lo, %c3\n\ts_nop 4\n\tbuffer_store_dwordx4 %0, %1, %2, 0 offen sc0 sc1\n\ts_mov_b64 exec, -1" ::"v"(v1), "v"(d), "s"(sp), "i"(NPUSH == 2 ? -1 : 0) : "memory");
            if (k >= PUSH_WIN - 1) wait_vm<PUSH_WIN - 1>();
          }
        }
        wait_vm<0>(); lds_inc(ctl + C_STG);
      } else if (wave < 4) {
        const int gw = sg - 16;
        const char* src = P.w + (size_t)(64 * so + 16 * wave) * DIM + 256 * gw + 4 * lane;
        unsigned r[16];
#pragma unroll
        for (int k = 0; k < 16; ++k) r[k] = *(const unsigned*)(src + (size_t)k * DIM);
        if (!lazy) wg_barrier();
        if (so == 0) load_pk();
        unsigned ncv;
        asm volatile("v_not_b32 %0, %1" : "=v"(ncv) : "v"(cv) : "memory");
        v4u o[4];
#pragma unroll
        for (int g = 0; g < 4; ++g) {
          unsigned t0 = __builtin_amdgcn_perm(r[4 * g + 1], r[4 * g], 0x05010400u), t1 = __builtin_amdgcn_perm(r[4 * g + 1], r[4 * g], 0x07030602u);
          unsigned t2 = __builtin_amdgcn_perm(r[4 * g + 3], r[4 * g + 2], 0x05010400u), t3 = __builtin_amdgcn_perm(r[4 * g + 3], r[4 * g + 2], 0x07030602u);
          o[0][g] = __builtin_amdgcn_perm(t2, t0, 0x05040100u); o[1][g] = __builtin_amdgcn_perm(t2, t0, 0x07060302u);
          o[2][g] = __builtin_amdgcn_perm(t3, t1, 0x05040100u); o[3][g] = __builtin_amdgcn_perm(t3, t1, 0x07060302u);
        }
        const unsigned d = base + (ncv & 1u) * BUF_BYTES + plane_of(wave) * PLANE_BLK + 65536u + (unsigned)gw * 4096 + (unsigned)(64 * (lane >> 4) + (lane & 15)) * 16;
#pragma unroll
        for (int jj = 0; jj < 4; ++jj) __builtin_amdgcn_raw_buffer_store_b128(o[jj], rs, d + jj * 256, 0, 17);
        if (so == 0 && wave >= 4 - NPUSH) {
#pragma unroll
          for (int k = 0; k < 7; ++k) {
            const v4i sp = make_srd(pk[k], 2 * BUF_BYTES);
            asm volatile("s_nop 4\n\tbuffer_store_dwordx4 %0, %4, %5, 0 offen sc0 sc1\n\tbuffer_store_dwordx4 %1, %4, %5, 0 offen offset:256 sc0 sc1\n\t"
                         "buffer_store_dwordx4 %2, %4, %5, 0 offen offset:512 sc0 sc1\n\tbuffer_store_dwordx4 %3, %4, %5, 0 offen offset:768 sc0 sc1"
                         ::"v"(o[0]), "v"(o[1]), "v"(o[2]), "v"(o[3]), "v"(d), "s"(sp) : "memory");
            if (k >= PUSH_WIN - 1) wait_vm<4 * (PUSH_WIN - 1)>();
          }
        }
        wait_vm<0>(); lds_inc(ctl + C_STG);
      } else if (!lazy) wg_barrier();
    }
    asm volatile("" ::: "v167");
    zero_acc<ACC0>();
    const int wr = wave >> 2, wc = wave & 3;
    unsigned pa = lds0 + LDS_A + (unsigned)(lane >> 4) * 2048 + (unsigned)(64 * wr + (lane & 15)) * 16;
    unsigned pb = lds0 + LDS_B + (unsigned)(lane >> 4) * 4096 + (unsigned)(64 * wc + (lane & 15)) * 16;
    const unsigned vw = ctl + (unsigned)wave * 16 + (unsigned)(lane & 3) * 4;
    const unsigned vd = (lane & 1) ? ctl + (unsigned)(wave ^ 4) * 16 + 8 : ctl + (unsigned)wave * 16 + 12;
    unsigned vo = ((unsigned)(256 * ti + 64 * wr) + (unsigned)(lane >> 4)) * (DIM * 4) + ((unsigned)(256 * tj + 64 * wc) + 4 * (unsigned)(lane & 15)) * 4;
    asm volatile("" : "+v"(vo));
    unsigned sc = rfl(__builtin_bit_cast(unsigned, (*P.sx) * (*P.sw)));
    const v4i ro = make_srd(P.out, DIM * DIM * 4);
    unsigned vcnt = 0, vf, ia = 0, ib = 0, cnt, da, db, spin = 1u << 24;
    unsigned long long ra;
    asm volatile(
      ".macro MM h, i, j\n"
      " v_mfma_i32_16x16x64_i8 v[40+\\h*64+\\i*16+\\j*4:43+\\h*64+\\i*16+\\j*4], v[24+\\i*4:27+\\i*4], v[8+\\j*4:11+\\j*4], v[40+\\h*64+\\i*16+\\j*4:43+\\h*64+\\i*16+\\j*4]\n"
      ".endm\n"
      ".macro STEP h, nb, pm, last\n"
      " s_waitcnt lgkmcnt(4+\\pm)\n"
      " MM \\h, 0, 0\n"
      " s_cmp_eq_u32 %[ia], %c[nh1]\n"
      " s_cselect_b32 %[da], %[wa], 0x2000\n"
      " MM \\h, 1, 0\n"
      " s_cselect_b32 %[ia], -1, %[ia]\n"
      " s_add_u32 %[ia], %[ia], 1\n"
      " .if \\pm\n s_waitcnt lgkmcnt(4)\n .endif\n"
      " MM \\h, 0, 1\n"
      " v_add_u32 %[pa], %[da], %[pa]\n"
      " MM \\h, 1, 1\n"
      " .if \\nb\n s_cmp_eq_u32 %[ib], %c[nb1]\n s_cselect_b32 %[db], %[wb], 0x4000\n .endif\n"
      " .if \\pm\n s_waitcnt lgkmcnt(3)\n .endif\n"
      " MM \\h, 0, 2\n"
      " v_add_u32 %[vcnt], 1, %[vcnt]\n"
      " .if \\nb\n s_cselect_b32 %[ib], -1, %[ib]\n s_add_u32 %[ib], %[ib], 1\n .endif\n"
      " MM \\h, 1, 2\n"
      " .if \\nb\n v_add_u32 %[pb], %[db], %[pb]\n .endif\n"
      " s_waitcnt lgkmcnt(2)\n"
      " MM \\h, 0, 3\n"
      " .if \\last == 0\n v_cmp_le_u32 vcc, %[vf], %[vcnt]\n .endif\n"
      " MM \\h, 1, 3\n"
      " .if \\last == 0\n"
      " s_getpc_b64 %[ra]\n"
      " s_and_b32 %[da], vcc_lo, 3\n"
      " s_cbranch_scc1 Lslow%=\n"
      " ds_read_b128 v[24:27], %[pa]\n"
      " ds_read_b128 v[28:31], %[pa] offset:256\n"
      " .endif\n"
      " s_waitcnt lgkmcnt(2-2*\\last)\n"
      " MM \\h, 2, 0\n"
      " .if \\last == 0\n"
      " ds_write_b32 %[vd], %[vcnt]\n"
      " ds_read_b32 %[vf], %[vw]\n"
      " .endif\n"
      " MM \\h, 3, 0\n"
      " .if \\nb\n ds_read_b128 v[8:11], %[pb]\n .endif\n"
      " MM \\h, 2, 1\n"
      " .if \\last == 0\n"
      " s_bitcmp1_b32 vcc_lo, 2\n"
      " s_setprio 0\n"
      " s_cbranch_scc1 1f\n"
      " s_setprio 1\n"
      "1:\n"
      " .endif\n"
      " MM \\h, 3, 1\n"
      " .if \\nb\n ds_read_b128 v[12:15], %[pb] offset:256\n s_sub_u32 %[cnt], %[cnt], 1\n .endif\n"
      " MM \\h, 2, 2\n"
      " MM \\h, 3, 2\n"
      " .if \\nb\n ds_read_b128 v[16:19], %[pb] offset:512\n s_cmp_lg_u32 %[cnt], 0\n .endif\n"
      " MM \\h, 2, 3\n"
      " MM \\h, 3, 3\n"
      " .if \\nb\n ds_read_b128 v[20:23], %[pb] offset:768\n .endif\n"
      " .if \\last == 0\n"
      " ds_read_b128 v[32:35], %[pa] offset:512\n"
      " ds_read_b128 v[36:39], %[pa] offset:768\n"
      " .endif\n"
      ".endm\n"
      " s_mov_b64 vcc, -1\n"
      " s_getpc_b64 %[ra]\n"
      " s_and_b32 %[da], vcc_lo, 3\n"
      " s_cbranch_scc1 Lslow%=\n"
      " ds_read_b128 v[24:27], %[pa]\n ds_read_b128 v[28:31], %[pa] offset:256\n"
      " ds_read_b128 v[8:11], %[pb]\n ds_read_b128 v[12:15], %[pb] offset:256\n"
      " ds_read_b128 v[16:19], %[pb] offset:512\n ds_read_b128 v[20:23], %[pb] offset:768\n"
      " ds_read_b128 v[32:35], %[pa] offset:512\n ds_read_b128 v[36:39], %[pa] offset:768\n"
      " s_mov_b32 %[cnt], 63\n"
      "Lfull%=:\n"
      " STEP 0, 0, 1, 0\n"
      " STEP 1, 1, 0, 0\n"
      " s_cbranch_scc1 Lfull%=\n"
      " STEP 0, 0, 1, 0\n"
      " STEP 1, 0, 0, 1\n"
      " s_branch Lend%=\n"
      "Lslow%=:\n"
      " ds_read_b32 %[vf], %[vw]\n"
      " s_sub_u32 %[spin], %[spin], 1\n"
      " s_waitcnt lgkmcnt(0)\n"
      " ds_write_b32 %[vd], %[vcnt]\n"
      " v_cmp_le_u32 vcc, %[vf], %[vcnt]\n"
      " s_cbranch_scc1 Lgive%=\n"
      " s_and_b32 %[da], vcc_lo, 3\n"
      " s_cbranch_scc0 Lret%=\n"
      " s_sleep 1\n"
      " s_branch Lslow%=\n"
      "Lgive%=:\n"
      " s_mov_b32 %[spin], 0\n"
      " s_mov_b32 %[sc], 0x7fc00000\n"
      " s_mov_b64 vcc, 0\n"
      "Lret%=:\n"
      " s_setpc_b64 %[ra]\n"
      "Lend%=:\n"
      " s_nop 7\n s_nop 1\n"
      ".purgem STEP\n.purgem MM\n"
      : [pa] "+&v"(pa), [pb] "+&v"(pb), [vcnt] "+&v"(vcnt), [vf] "=&v"(vf), [ia] "+&s"(ia), [ib] "+&s"(ib), [cnt] "=&s"(cnt), [da] "=&s"(da),
        [db] "=&s"(db), [spin] "+&s"(spin), [sc] "+&s"(sc), [ra] "=&s"(ra)
      : [vw] "v"(vw), [vd] "v"(vd), [nh1] "i"(NH - 1), [nb1] "i"(NB - 1), [wa] "s"(-(NH - 1) * 8192), [wb] "s"(-(NB - 1) * 16384)
      : "v8", "v9", "v10", "v11", "v12", "v13", "v14", "v15", "v16", "v17", "v18", "v19", "v20", "v21", "v22", "v23", "v24", "v25", "v26", "v27",
        "v28", "v29", "v30", "v31", "v32", "v33", "v34", "v35", "v36", "v37", "v38", "v39", "vcc", "scc", "memory");
    {
      unsigned v, k = 0;
      do v = lds_rd(ctl + ((lane & 1) ? C_FIN : C_ABORT)); while (__builtin_amdgcn_readlane(v, 1) == 0u && ++k < (1u << 24));
      sc = (__builtin_amdgcn_readlane(v, 0) != 0u || __builtin_amdgcn_readlane(v, 1) == 0u) ? 0x7fc00000u : sc;
    }
    store_rows<0, 32>(sc, vo, ro);
  } else if (wave == 8) {
    wg_barrier();
    lds_wr(lds0 + LDS_T + (NTK + (lane & 15)) * 4, 0xffffffffu);
    get_call();
    if (mover) {
      __builtin_amdgcn_s_setprio(3);
      const v4i srdR = make_srd(ktab[pp], 2 * BUF_BYTES);
      const unsigned v16 = (unsigned)lane * 16;
      const unsigned sbase = (call & 1u) * BUF_BYTES + (unsigned)pp * SRC_BYTES;
      unsigned tk_off = TK_OFF + ((call & 1u) * 8 + (unsigned)pi) * 128;
      const unsigned tk_dead = DUM_OFF + (unsigned)b * 256 + 64;
      const unsigned go_out = DUM_OFF + (unsigned)(last ? mp : mp - 32) * 256 + 192, go_in = DUM_OFF + (unsigned)mp * 256 + 192;
      flag_st1(srdL, tk_dead, 0x1000u);
      unsigned ev;
      bool ok;
      if (turn == 0) {
        lds_wr(ctl + C_MVD, 1u);
        unsigned long long fi = 0;
        static_for<0, LPL>([&](auto S) { constexpr unsigned long long F = first_items(decltype(S)::value); fi = sub == decltype(S)::value ? F : fi; });
        const unsigned it = (unsigned)(fi >> (8 * ((lane & 7) < NGR ? (lane & 7) : NGR))) & 255u, g = it & 31u;
        ev = entry(32u * NPUSH + it);
        asm volatile("" : "+v"(ev));
        ok = wait_flag(RDY_OFF + ((unsigned)pp * 64 + g) * 128);
      } else {
        const unsigned long long t0 = rt();
        const unsigned est = turn_end(turn - 1) * 67u / 10u + EST_ADD;
        while ((unsigned)(rt() - t0) < est) __builtin_amdgcn_s_sleep(16);
        ok = wait_flag(rdy_rest) && wait_flag(go_in);
        lds_wr(ctl + C_MVD + (unsigned)(lane & 1) * 4, (lane & 1) ? 0u : 1u);
        unsigned x0;
        asm volatile("s_mov_b64 exec, 1\n\tv_mov_b32 v47, %3\n\ts_nop 4\n\tbuffer_atomic_add v47, off, %1, %2 sc0\n\ts_mov_b64 exec, -1\n\t"
                     "s_waitcnt vmcnt(0)\n\tv_readfirstlane_b32 %0, v47\n\ts_nop 1" : "=s"(x0) : "s"(srdL), "s"(ok ? tk_off : tk_dead), "s"(NGR + 1) : "memory");
        const unsigned xe = x0 + NST0 + (unsigned)(lane & 7);
        ok = lds_wait(ctl + C_TAB, 3) && ok;
        ev = lds_rd(lds0 + LDS_T + (xe < NTK ? xe : NTK) * 4);
      }
      if (ok) {
        auto relay = [&](auto NGC) {
        constexpr int NG = decltype(NGC)::value;
        const unsigned fdum = DUM_OFF + (unsigned)b * 256;
        unsigned os[NG], en[NG], pf[NG];
        unsigned e;
        unsigned rdy = turn != 0 ? 4096u : (4u + NPUSH) * 32;
        auto foff = [&](unsigned x) { return x == 0xffffffffu ? fdum : PF_OFF + (x >> 16) * 128; };
        auto offs = [&](unsigned x) {
          if (__builtin_expect((x & 0x4000u) != 0u && tk_off != tk_dead, 0)) {
            tk_off = tk_dead;
            flag_st1(srdL, go_out, call);
          }
          return sbase + (x == 0xffffffffu ? 32u * NPUSH : x & 1023u) * 4096;
        };
        auto gate = [&]() {
          if (__builtin_expect((e & 1023u) >= rdy && e != 0xffffffffu, 0)) {
            const unsigned lvl = (e & 1023u) < (28 + NPUSH) * 32 ? 2u : 3u;
            unsigned r, spins = 0;
            while ((r = rfl(lds_rd(ctl + C_RDY))) < lvl && ++spins < (1u << 24)) __builtin_amdgcn_s_sleep(2);
            if (r < lvl || r > 3u) { e = 0xffffffffu; flag_st1(srdL, TMO_OFF, call); }
            else rdy = lvl == 2u ? (28u + NPUSH) * 32 : 4096u;
          }
        };
        auto swap = [&](auto G, auto W, auto WK) {
          constexpr int g = decltype(G)::value;
          const unsigned ol = offs(e), nf = pf[g], o = os[g];
          pf[g] = en[g]; en[g] = foff(e); os[g] = ol;
          e = mv_swap<g, decltype(W)::value, decltype(WK)::value>(srdL, srdR, tk_off, o, ol, v16, lds0 + LDS_T + NST0 * 4, call, nf);
        };
        static_for<0, NG>([&](auto G) {
          constexpr int g = decltype(G)::value;
          const unsigned x = __builtin_amdgcn_readlane(ev, g);
          pf[g] = fdum; en[g] = foff(x); os[g] = offs(x);
          mv_load<g>(srdL, srdR, tk_off, os[g], v16);
        });
        e = __builtin_amdgcn_readlane(ev, NG);
        if (turn == 0) {
          if (!lds_wait(ctl + C_TAB, 3) || !lds_wait(ctl + C_RDY, 1) || rfl(lds_rd(ctl + C_RDY)) > 3u) { wait_vm<0>(); flag_st1(srdL, TMO_OFF, call); return; }
        }
        static_for<0, NG>([&](auto G) {
          constexpr int g = decltype(G)::value;
          gate();
          swap(G, std::integral_constant<int, 4 + (NG - 1 - g) * 5 + g * 10>{}, std::integral_constant<int, 1>{});
        });
        int stop = -1;
#pragma unroll 1
        while (stop < 0) {
          static_for<0, NG>([&](auto G) {
            constexpr int g = decltype(G)::value;
            if (stop < 0) {
              gate();
              if (e == 0xffffffffu) stop = g;
              else {
                swap(G, std::integral_constant<int, 8 + (NG - 1) * 10>{}, std::integral_constant<int, 0>{});
              }
            }
          });
        }
        if (tk_off != tk_dead) flag_st1(srdL, go_out, call);
        static_for<0, NG>([&](auto G) {
          constexpr int g = decltype(G)::value;
          if (stop == g) static_for<0, NG>([&](auto J) {
            constexpr int j = decltype(J)::value, h = (g + j) % NG;
            mv_store<h, 7 + (NG - 1 - j) * 10 + j * 5>(srdL, os[h], v16, call, pf[h]);
          });
        });
        wait_vm<0>();
        static_for<0, NG>([&](auto G) { flag_st1(srdL, en[decltype(G)::value], call); });
        };
        relay(std::integral_constant<int, NGR>{});
        lds_wr(ctl + C_MVD, 2u);
      }
      __builtin_amdgcn_s_setprio(0);
    }
    lds_wr(ctl + C_MVD, 2u);
    if (rfl(lds_rd(ctl + C_ABORT)) != 0u && lds_wait(ctl + C_FIN, 1)) lds_wr(ctl + C_AV, 64u);
  } else if (wave == 9) {
    __builtin_amdgcn_s_setprio(2);
    lds_wr(ctl + lane * 4, 0u);
    wg_barrier();
    const bool direct = so == 1;
    if (!direct) build_table();
    get_call();
    {
      char* pb = wsl;
#pragma unroll
      for (int k = 0; k < 8; ++k) { char* t = P.ws[k]; pb = (lane == k || (lane >= 32 && (lane & 7) == k)) ? t : pb; }
      const unsigned off = lane < 8 ? RDY_OFF + (unsigned)(rank * 64 + (direct ? sg : 32 + so)) * 128
                         : lane < 16 ? PF_OFF + flag_idx((unsigned)sg, 7u, plane_of(lane - 8)) * 128
                         : lane == 16 ? CNT_OFF + (unsigned)b * 128
                         : lane < 32 ? TK_OFF + ((~call & 1u) * 8 + (unsigned)(lane - 17)) * 128
                         : PF_OFF + flag_idx((unsigned)sg, (unsigned)(rank - lane - 1) & 7u, (unsigned)lane >> 3 & 1u) * 128;
      unsigned* dst = (unsigned*)(pb + off);
      const unsigned val = (lane > 16 && lane < 32) ? 0u : call;
      const bool on = lane < 8 ? (direct && lane != rank) : lane <= 16 ? (lane < 12 || lane == 16) : lane < 32 ? (b == 255 && lane < 24) : (so == 0 && lane < 32 + 8 * NPUSH && (lane & 7) != rank);
      asm volatile("" : "+v"(dst));
      const bool staged = lds_wait(ctl + C_STG, sg < 16 ? 8 : 4);
      if (!staged) flag_st1(srdL, TMO_OFF, call);
      if (on && (staged || (lane >= 16 && lane < 32))) __hip_atomic_store(dst, val, __ATOMIC_RELAXED, __HIP_MEMORY_SCOPE_SYSTEM);
      if (direct) build_table();
      else if (staged) {
        unsigned tk = 0;
        if (lane == 0) tk = __hip_atomic_fetch_add((unsigned*)(wsl + AGG_OFF + (unsigned)so * 128), 1u, __ATOMIC_RELAXED, __HIP_MEMORY_SCOPE_AGENT);
        if (rfl(tk) + 1u == call * 32u && lane < 8 && lane != rank) __hip_atomic_store(dst, val, __ATOMIC_RELAXED, __HIP_MEMORY_SCOPE_SYSTEM);
      }
    }
    const unsigned si = (unsigned)lane >> 3, ro = ((unsigned)lane >> 1) & 3u;
    const unsigned fbase = PF_OFF + flag_idx((lane & 1) ? 16u + tj : (unsigned)ti, si, ro) * 128;
    const unsigned pbase = (call & 1u) * BUF_BYTES + (((unsigned)rank + 1 + si) & 7u) * SRC_BYTES + ro * PLANE_BLK;
    const unsigned lateN = POLL_LATE - 7 * (unsigned)(((LAGS >> (4 * (idx >> 2))) & 15u) > ((LAGW >> (4 * (idx & 3))) & 15u) ? ((LAGS >> (4 * (idx >> 2))) & 15u) : ((LAGW >> (4 * (idx & 3))) & 15u));
    unsigned q = 0, cnt = 0, npoll = 0, win = 4;
    bool mvdone = false;
    const unsigned long long t0 = rt();
#pragma unroll 1
    while (cnt < 256 || !mvdone) {
      if (!mvdone) { const unsigned m = rfl(lds_rd(ctl + C_MVD)); mvdone = m == 2u; if (m == 1u) { lds_wr(ctl + C_AV, 0u); win = 4; __builtin_amdgcn_s_sleep(4); continue; } }
      unsigned v = call - 1;
      if (ro < win && q + ro < 32) {
        asm volatile("buffer_load_dword %0, %1, %2, 0 offen sc0 sc1\n\ts_waitcnt vmcnt(0)" : "=v"(v) : "v"(fbase + q * 128), "s"(srdL) : "memory");
      }
      const unsigned long long hm = __ballot((int)(v - call) >= 0);
      const unsigned adv = (unsigned)__builtin_ctz(~((unsigned)(hm >> (8 * si)) & 255u)) >> 1;
      const bool emit = !(lane & 1) && ro < adv;
      const unsigned long long em = __ballot(emit);
      if (emit) lds_wr(lds0 + LDS_L + (cnt + __builtin_amdgcn_mbcnt_hi((unsigned)(em >> 32), __builtin_amdgcn_mbcnt_lo((unsigned)em, 0u))) * 4, pbase + q * PLANE_BLK);
      q += adv;
      cnt += (unsigned)__builtin_popcountll(em);
      const unsigned lim = cnt < 160 ? (unsigned)(rt() - t0) / LOCAL_GAP + 1 : 255u;
      lds_wr(ctl + C_AV, (cnt >> 2) < lim ? cnt >> 2 : lim);
      const bool more = __ballot(adv == win && q < 32) != 0ull;
      win = more ? 4u : 2u;
      if (!more) { if (cnt < lateN) __builtin_amdgcn_s_sleep(POLL_SLEEP); else __builtin_amdgcn_s_sleep(POLL_SLEEP2); }
      if (em == 0ull && (++npoll & 15u) == 0u) {
        const unsigned tm = rfl(flag_ld(srdL, TMO_OFF));
        if (tm == call || rt() - t0 > DEADLINE) {
          flag_st1(srdL, TMO_OFF, call);
          lds_wr(ctl + C_ABORT, 1u);
#pragma unroll 1
          for (unsigned k = (unsigned)lane; k < 256; k += 64) lds_wr(lds0 + LDS_L + k * 4, 0u);
          asm volatile("s_waitcnt lgkmcnt(0)" ::: "memory");
          break;
        }
      }
    }
    lds_wr(ctl + C_AV, 64u);
    if (rfl(flag_ld(srdL, TMO_OFF)) == call) lds_wr(ctl + C_ABORT, 1u);
    asm volatile("s_waitcnt lgkmcnt(0)" ::: "memory");
    lds_wr(ctl + C_FIN, 1u);
  } else {
    wg_barrier();
    build_table();
    __builtin_amdgcn_s_setprio(3);
    get_call();
    if (turn == 0 && wave == 10) {
      bool ok = wait_flag(RDY_OFF + (unsigned)(pp * 64 + (lane & 31)) * 128);
      if (ok) { lds_wr(ctl + C_RDY, 1u); ok = wait_flag(RDY_OFF + (unsigned)(pp * 64 + 32 + ((lane & 7) < 2 ? 2 : (lane & 7))) * 128); }
      if (ok) { lds_wr(ctl + C_RDY, 2u); ok = wait_flag(rdy_rest); }
      lds_wr(ctl + C_RDY, ok ? 3u : 9u);
    }
    auto loader = [&](auto IA) {
      constexpr bool isA = decltype(IA)::value;
      constexpr unsigned RINGB = isA ? NH * 8192 : NB * 16384;
      const unsigned v16 = (unsigned)lane * 16 + (isA ? (unsigned)ti : 16u + tj) * 4096, v16b = v16 + 2048u;
      unsigned vchk = lane < 8 ? ctl + lane * 16 + 12 : lane < 16 ? ctl + C_AV : lds0 + LDS_L + (unsigned)(lane - 16) * 4;
      const unsigned vadv = lane < 16 ? 0u : 16u;
      const unsigned vfull = lane < 8 ? ctl + (unsigned)lane * 16 + (isA ? 0 : 4) : ctl + (isA ? 512 : 768) + lane * 4;
      const unsigned ring0 = lds0 + (isA ? LDS_A : LDS_B);
      unsigned ring = ring0, n = 0, nops = 0, spin = 1u << 25;
      const v4i srd = srdL;
      int vneed = lane < 8 ? (isA ? 2 - NH : 1 - 2 * NB) : lane < 16 ? 1 : (int)0x80000000u;
      const int vinc = lane < 8 ? 2 : lane < 16 ? 1 : 0;
      wait_vm<0>();
#define L_PUB \
      " s_getreg_b32 %[c], hwreg(HW_REG_IB_STS)\n" \
      " s_bfe_u32 %[x], %[c], 0x20016\n" \
      " s_and_b32 %[c], %[c], 15\n" \
      " s_lshl4_add_u32 %[c], %[x], %[c]\n" \
      " s_sub_u32 %[x], %[nops], %[c]\n" \
      " s_lshr_b32 %[x], %[x], 3\n" \
      " .if %c[isa] == 0\n s_and_b32 %[x], %[x], -2\n .endif\n" \
      " v_mov_b32 %[vl], %[x]\n" \
      " ds_write_b32 %[vfull], %[vl]\n"
#define L_D2(v, o, k) \
      " buffer_load_dwordx4 %[" v "], %[srd], %[" o "] offen offset:" k " sc0 sc1 lds\n" \
      " buffer_load_dwordx4 %[" v "], %[srd], %[" o "] offen offset:" k "+1024 sc0 sc1 lds\n"
#define L_A(v, o) L_D2(v, o, "0") " s_add_u32 m0, m0, 0x800\n s_nop 0\n"
#define L_B(o) L_D2("v16", o, "0") L_D2("v16", o, "2048") " s_add_u32 m0, m0, 0x1000\n s_nop 0\n"
      unsigned v, vl, o0, o1, o2, o3, c, x, j, keep;
      asm volatile(
        " s_mov_b32 %[km0], m0\n"
        "Lpoll%=:\n"
        " ds_read_b32 %[v], %[vchk]\n"
        L_PUB
        " s_mov_b32 %[j], 16\n"
        " s_sub_u32 %[spin], %[spin], 1\n"
        " s_cbranch_scc1 Lout%=\n"
        " s_waitcnt lgkmcnt(1)\n"
        "Lchk%=:\n"
        " v_cmp_lt_i32 vcc, %[v], %[vneed]\n"
        " v_readlane_b32 %[o0], %[v], %[j]\n"
        " s_add_u32 %[j], %[j], 1\n"
        " v_readlane_b32 %[o1], %[v], %[j]\n"
        " s_cmp_gt_u32 %[c], 47\n"
        " s_cbranch_scc1 Lpoll%=\n"
        " s_cbranch_vccnz Lpoll%=\n"
        " s_add_u32 %[j], %[j], 1\n"
        " v_readlane_b32 %[o2], %[v], %[j]\n"
        " s_add_u32 %[j], %[j], 1\n"
        " v_readlane_b32 %[o3], %[v], %[j]\n"
        " s_mov_b32 m0, %[ring]\n"
        " s_add_u32 %[n], %[n], 1\n"
        " .if %c[isa]\n"
        L_A("v16", "o0") L_A("v16", "o1") L_A("v16", "o2") L_A("v16", "o3")
        " s_add_u32 %[nops], %[nops], 8\n"
        L_PUB
        L_A("v16b", "o0") L_A("v16b", "o1") L_A("v16b", "o2") L_A("v16b", "o3")
        " s_add_u32 %[nops], %[nops], 8\n"
        " .else\n"
        L_B("o0") L_B("o1") L_B("o2") L_B("o3")
        " s_add_u32 %[nops], %[nops], 16\n"
        " .endif\n"
        " v_add_u32 %[vneed], %[vinc], %[vneed]\n"
        " v_add_u32 %[vchk], %[vadv], %[vchk]\n"
        " s_cmp_eq_u32 m0, %[rend]\n"
        " s_cselect_b32 %[ring], %[ring0], m0\n"
        " s_add_u32 %[j], %[j], 1\n"
        L_PUB
        " s_cmp_lt_u32 %[n], 64\n"
        " s_cbranch_scc1 Lchk%=\n"
        "Lout%=:\n"
        L_PUB
        " s_cmp_lg_u32 %[c], 0\n"
        " s_cbranch_scc1 Lout%=\n"
        " s_mov_b32 m0, %[km0]\n"
        : [v] "=&v"(v), [vneed] "+&v"(vneed), [vchk] "+&v"(vchk), [vl] "=&v"(vl), [o0] "=&s"(o0), [o1] "=&s"(o1), [o2] "=&s"(o2), [o3] "=&s"(o3), [c] "=&s"(c),
          [x] "=&s"(x), [j] "=&s"(j), [km0] "=&s"(keep), [n] "+&s"(n), [nops] "+&s"(nops), [ring] "+&s"(ring), [spin] "+&s"(spin)
        : [v16] "v"(v16), [v16b] "v"(v16b), [vfull] "v"(vfull), [vinc] "v"(vinc), [vadv] "v"(vadv),
          [srd] "s"(srd), [ring0] "s"(ring0), [rend] "s"(ring0 + RINGB), [isa] "i"(isA ? 1 : 0)
        : "vcc", "scc", "memory");
      if (n != 64u) lds_wr(ctl + C_ABORT, 1u);
      asm volatile("s_waitcnt lgkmcnt(0)" ::: "memory");
      lds_wr(vfull, 255u);
    };
    if (wave == 10) loader(std::true_type{}); else loader(std::false_type{});
  }
}

extern "C" void kernel_launch(int rank, int nranks, ncclComm_t comm, hipStream_t stream,
                              void* const* d_in, const int* in_sizes, int n_in,
                              void* d_out, int out_size, const int* in_dtypes, int out_dtype,
                              void* const* d_out_all, void* const* d_ws_all, size_t ws_bytes) {
  (void)comm; (void)in_sizes; (void)n_in; (void)out_size; (void)in_dtypes; (void)out_dtype; (void)d_out_all;
  if (nranks != NR || (unsigned)rank >= (unsigned)NR || ws_bytes < WS_USED) { fprintf(stderr, "unexpected configuration\n"); return; }
  Params p;
  const size_t woff = (ws_bytes - WS_USED) & ~(size_t)0xfffff;
  for (int i = 0; i < NR; ++i) p.ws[i] = (char*)d_ws_all[i] + woff;
  p.x = (const char*)d_in[0]; p.w = (const char*)d_in[1];
  p.sx = (const float*)d_in[2]; p.sw = (const float*)d_in[3];
  p.out = (float*)d_out; p.wsl = p.ws[rank]; p.rank = rank;
  static bool attr_set[NR] = {};
  if (!attr_set[rank]) {
    HIP_CHECK(hipFuncSetAttribute((const void*)fused_gemm, hipFuncAttributeMaxDynamicSharedMemorySize, (int)LDS_BYTES));
    attr_set[rank] = true;
  }
  fused_gemm<<<256, 768, LDS_BYTES, stream>>>(p);
  HIP_CHECK(hipGetLastError());
}
```

```cpp
#include <hip/hip_runtime.h>
#include <rccl/rccl.h>
#include <cstdio>
#include <cstdint>
#include <type_traits>

#define HIP_CHECK(expr) do { hipError_t _e = (expr); if (_e != hipSuccess) { \
    fprintf(stderr, "HIP error %s at %s:%d\n", hipGetErrorString(_e), __FILE__, __LINE__); \
    return; } } while (0)

typedef int v4i __attribute__((ext_vector_type(4)));
typedef unsigned v4u __attribute__((ext_vector_type(4)));
typedef float v4f __attribute__((ext_vector_type(4)));
#define LAS __attribute__((address_space(3)))
#define CAS __attribute__((address_space(4)))

#define LAGS 0x77420000u
#define LAGW 0x7100u
#define NGR 3
#define LAST_LEAD 32
#define TURNS 0x807a70614d3419ull
#define STG 24
#define STG_LAST 4
#define EST_ADD 150
#define NPUSH 1
#define PUSH_WIN 3
#define STAGE_DELAY 64
#define POLL_LATE 232
#define POLL_SLEEP 24
#define POLL_SLEEP2 2
#define LOCAL_GAP 100
#define LSTR 273

constexpr int NR = 8;
constexpr int DIM = 4096, KSH = 512;
constexpr unsigned PLANE_BLK = 128u << 10;
constexpr unsigned SRC_BYTES = 32 * PLANE_BLK;
constexpr unsigned BUF_BYTES = NR * SRC_BYTES;
constexpr unsigned PF_OFF = 2 * BUF_BYTES;
constexpr unsigned RDY_OFF = PF_OFF + 32 * 600 * 128;
constexpr unsigned CNT_OFF = RDY_OFF + NR * 64 * 128;
constexpr unsigned TMO_OFF = CNT_OFF + 256 * 128;
constexpr unsigned AGG_OFF = TMO_OFF + 3072;
constexpr unsigned TK_OFF = TMO_OFF + 1024;
constexpr unsigned DUM_OFF = TMO_OFF + 4096;
constexpr unsigned WS_USED = DUM_OFF + 256 * 256;
constexpr unsigned long long DEADLINE = 60000000ull;

constexpr int NH = 10, NB = 4;
constexpr unsigned LDS_A = 0, LDS_B = NH * 8192, LDS_C = LDS_B + NB * 16384, LDS_T = LDS_C + 1024, LDS_L = LDS_T + 4096, LDS_BYTES = LDS_L + 1280;
static_assert(LDS_BYTES <= 163840 && NH % 2 == 0, "LDS");
constexpr unsigned C_ABORT = 128, C_MVD = 132, C_AV = 136, C_FIN = 140, C_RDY = 144, C_STG = 148, C_TAB = 156;
constexpr unsigned NTK = 32 * (32 - NPUSH);
constexpr int LPL = 4, NT = 7;
constexpr unsigned NST0 = LPL * (NGR + 1);
constexpr unsigned long long LAG12 = (unsigned long long)LAGS | ((unsigned long long)LAGW << 32);
constexpr int lag_of(int s) { return (int)((LAG12 >> (4 * s)) & 15u); }
constexpr int late_groups() { int r = 0; for (int s = 0; s < 12; ++s) r += (s < 8 ? 2 : 4) * lag_of(s); return r; }
constexpr int UEND = (int)NTK - late_groups() - LAST_LEAD;
constexpr int tile_lag(int idx) { return lag_of(idx >> 2) > lag_of(8 + (idx & 3)) ? lag_of(idx >> 2) : lag_of(8 + (idx & 3)); }
constexpr unsigned long long order8(int k) {
  unsigned long long r = 0;
  for (int i = 8 * k; i < 8 * k + 8; ++i) {
    int c = 0;
    for (int j = 0; j < 32; ++j) c += tile_lag(j) < tile_lag(i) || (tile_lag(j) == tile_lag(i) && j < i);
    r |= (unsigned long long)c << (8 * (i & 7));
  }
  return r;
}

struct Params {
  char* ws[NR];
  const char* x;
  const char* w;
  const float* sx;
  const float* sw;
  float* out;
  char* wsl;
  int rank;
};

__device__ __forceinline__ unsigned rfl(unsigned v) { return __builtin_amdgcn_readfirstlane(v); }
__device__ __forceinline__ v4i make_srd(const void* p, unsigned bytes) {
  unsigned long long a = (unsigned long long)p;
  v4i r;
  r.x = (int)rfl((unsigned)a); r.y = (int)(rfl((unsigned)(a >> 32)) & 0xffffu); r.z = (int)bytes; r.w = 0x00020000;
  return r;
}
__device__ __forceinline__ unsigned long long rt() { return __builtin_amdgcn_s_memrealtime(); }
template <int N> __device__ __forceinline__ void wait_vm() { asm volatile("s_waitcnt vmcnt(%0)" ::"i"(N) : "memory"); }
__device__ __forceinline__ unsigned lds_rd(unsigned addr) {
  unsigned v;
  asm volatile("ds_read_b32 %0, %1\n\ts_waitcnt lgkmcnt(0)" : "=v"(v) : "v"(addr) : "memory");
  return v;
}
__device__ __forceinline__ void lds_wr(unsigned addr, unsigned v) { asm volatile("ds_write_b32 %0, %1" ::"v"(addr), "v"(v) : "memory"); }
__device__ __forceinline__ void lds_inc(unsigned a) {
  asm volatile("s_mov_b64 exec, 1\n\tds_add_u32 %0, %1\n\ts_mov_b64 exec, -1" ::"v"(a), "v"(1u) : "memory");
}
__device__ __forceinline__ bool lds_wait(unsigned a, unsigned n) { unsigned k = 0; while (rfl(lds_rd(a)) < n) if (++k == (1u << 25)) return false; return true; }
__device__ __forceinline__ unsigned flag_ld(v4i srd, unsigned voff) {
  unsigned v;
  asm volatile("s_nop 4\n\tbuffer_load_dword %0, %1, %2, 0 offen sc0 sc1\n\ts_waitcnt vmcnt(0)" : "=v"(v) : "v"(voff), "s"(srd) : "memory");
  return v;
}
__device__ __forceinline__ void flag_st1(v4i srd, unsigned soff, unsigned v) {
  asm volatile("s_mov_b64 exec, 1\n\ts_nop 4\n\tbuffer_store_dword %0, off, %1, %2 sc0 sc1\n\ts_mov_b64 exec, -1" ::"v"(v), "s"(srd), "s"(soff) : "memory");
}
__device__ __forceinline__ unsigned flag_idx(unsigned g, unsigned si, unsigned q) { return g * LSTR + si * 32 + q; }
__host__ __device__ constexpr int stream_pos(int q, int g) {
  const int me = g < 16 ? (g >> 1) : 8 + (g & 3);
  const int key = q + (int)((LAG12 >> (4 * me)) & 15u);
  int u = g < 16 ? (g & 1) : ((g - 16) >> 2);
  for (int s = 0; s < 12; ++s) {
    const int c = key - lag_of(s) - NPUSH + (s < me ? 1 : 0);
    u += (s < 8 ? 2 : 4) * (c < 0 ? 0 : (c > 32 - NPUSH ? 32 - NPUSH : c));
  }
  return u;
}
constexpr unsigned long long first_items(int sub) {
  unsigned long long r = 0;
  for (int it = 0; it < 128; ++it) { const int u = stream_pos(NPUSH + it / 32, it % 32); if (u < (int)NST0 && u % LPL == sub) r |= (unsigned long long)it << (8 * (u / LPL)); }
  return r;
}
template <int I, int N, typename F> __device__ __forceinline__ void static_for(F&& f) {
  if constexpr (I < N) { f(std::integral_constant<int, I>{}); static_for<I + 1, N>(f); }
}


constexpr int ACC0 = 40;
#define MV_T "s_mov_b64 exec, 1\n v_mov_b32 v[40+2*%c[g]], 1\n v_mov_b32 v[41+2*%c[g]], 0\n buffer_atomic_add_x2 v[40+2*%c[g]:41+2*%c[g]], off, %[sl], %[tk] sc0\n s_mov_b64 exec, -1\n"
#define MV_L "buffer_load_dwordx4 v[64+%c[g]*16+k*4:67+%c[g]*16+k*4], %[v16], %[sr], %[ol] offen offset:k*1024 sc0 sc1\n"
#define MV_S "buffer_store_dwordx4 v[64+%c[g]*16+k*4:67+%c[g]*16+k*4], %[v16], %[sl], %[os] offen offset:k*1024 sc1\n"
#define MV_F "s_mov_b64 exec, 1\n buffer_store_dword %[vc], off, %[sl], %[nf] sc1\n s_mov_b64 exec, -1\n"
template <int G> __device__ __forceinline__ void mv_load(v4i sl, v4i sr, unsigned tk, unsigned ol, unsigned v16) {
  asm volatile("s_nop 4\n" MV_T ".set k, 0\n .rept 4\n" MV_L ".set k, k+1\n .endr"
               ::[g] "i"(G), [sl] "s"(sl), [sr] "s"(sr), [tk] "s"(tk), [ol] "s"(ol), [v16] "v"(v16) : "memory");
}
template <int G, int W, int WK> __device__ __forceinline__ unsigned mv_swap(v4i sl, v4i sr, unsigned tk, unsigned os, unsigned ol, unsigned v16, unsigned tab, unsigned call, unsigned nf) {
  unsigned t;
  asm volatile("s_waitcnt vmcnt(%c[w]-1)\n v_subrev_u32 v56, %[tb], v[40+2*%c[g]]\n v_min_u32 v56, %c[ntk], v56\n v_lshl_add_u32 v56, v56, 2, %[tab]\n ds_read_b32 v56, v56\n" MV_T
               ".set k, 0\n .rept 4\n s_waitcnt vmcnt(%c[w]+%c[wk]*k)\n" MV_S MV_L ".set k, k+1\n .endr\n" MV_F
               "s_waitcnt lgkmcnt(0)\n v_readfirstlane_b32 %[t], v56\n s_nop 1"
               : [t] "=s"(t) : [g] "i"(G), [w] "i"(W), [wk] "i"(WK), [sl] "s"(sl), [sr] "s"(sr), [tk] "s"(tk), [os] "s"(os), [ol] "s"(ol), [v16] "v"(v16),
                 [tab] "s"(tab), [tb] "s"(call << 12), [ntk] "i"(NTK - NST0), [vc] "v"(call), [nf] "s"(nf) : "memory");
  return t;
}
template <int G, int W> __device__ __forceinline__ void mv_store(v4i sl, unsigned os, unsigned v16, unsigned call, unsigned nf) {
  asm volatile(".set k, 0\n .rept 4\n s_waitcnt vmcnt(%c[w]-k)\n" MV_S ".set k, k+1\n .endr\n" MV_F
               ::[g] "i"(G), [w] "i"(W), [sl] "s"(sl), [os] "s"(os), [v16] "v"(v16), [vc] "v"(call), [nf] "s"(nf) : "memory");
}
template <int R> __device__ __forceinline__ void zero_acc() {
  asm volatile("v_mov_b32 v%c0, 0\n\tv_mov_b32 v%c1, 0\n\tv_mov_b32 v%c2, 0\n\tv_mov_b32 v%c3, 0" ::"i"(R), "i"(R + 1), "i"(R + 2), "i"(R + 3));
  if constexpr (R + 4 < ACC0 + 128) zero_acc<R + 4>();
}
template <int E> __device__ __forceinline__ void store_row(unsigned sc, unsigned vo, v4i ro) {
  constexpr int R = ACC0 + (E >> 2) * 16 + (E & 3);
  float f0, f1, f2, f3;
  asm volatile("v_cvt_f32_i32 %0, v%c4\n\tv_cvt_f32_i32 %1, v%c5\n\tv_cvt_f32_i32 %2, v%c6\n\tv_cvt_f32_i32 %3, v%c7\n\t"
               "v_mul_f32 %0, %8, %0\n\tv_mul_f32 %1, %8, %1\n\tv_mul_f32 %2, %8, %2\n\tv_mul_f32 %3, %8, %3"
               : "=v"(f0), "=v"(f1), "=v"(f2), "=v"(f3) : "i"(R), "i"(R + 4), "i"(R + 8), "i"(R + 12), "s"(sc));
  v4f v = {f0, f1, f2, f3};
  constexpr unsigned so = (unsigned)(128 * (E >> 4) + 16 * ((E >> 2) & 3) + 4 * (E & 3)) * (DIM * 4);
  asm volatile("buffer_store_dwordx4 %0, %1, %2, %3 offen sc1\n\ts_nop 1" ::"v"(v), "v"(vo), "s"(ro), "s"(so) : "memory");
}
template <int E0, int E1> __device__ __forceinline__ void store_rows(unsigned sc, unsigned vo, v4i ro) {
  if constexpr (E0 < E1) { store_row<E0>(sc, vo, ro); store_rows<E0 + 1, E1>(sc, vo, ro); }
}

__global__ void __launch_bounds__(768) __attribute__((amdgpu_num_vgpr(20))) fused_gemm(Params P) {
  extern __shared__ __attribute__((aligned(16))) char lds[];
  const int tid = threadIdx.x, lane = tid & 63;
  asm volatile("" ::"s"(P.ws[0]), "s"(P.x), "s"(P.w), "s"(P.wsl), "s"(P.rank));
  const int wave = (int)rfl((unsigned)tid >> 6);
  const int b = blockIdx.x;
  const int rank = P.rank;
  char* const CAS* const ktab = (char* const CAS*)__builtin_amdgcn_kernarg_segment_ptr();
  char* const wsl = P.wsl;
  unsigned cv = __hip_atomic_load((const unsigned*)(wsl + CNT_OFF + (unsigned)b * 128), __ATOMIC_RELAXED, __HIP_MEMORY_SCOPE_AGENT);
  const unsigned lds0 = (unsigned)(unsigned long long)(LAS char*)lds;
  const unsigned ctl = lds0 + LDS_C;
  const int xcd = b & 7, idx = b >> 3;
  const int ti = 2 * (idx >> 2) + (xcd >> 2), tj = 4 * (xcd & 3) + (idx & 3);
  const int sg = b & 31, so = b >> 5;
  auto plane_of = [&](int j) { return (unsigned)(4 * so + j + 28 + NPUSH) & 31u; };
  const v4i srdL = make_srd(wsl, WS_USED);
  constexpr unsigned long long O0 = order8(0), O1 = order8(1), O2 = order8(2), O3 = order8(3);
  const int mp = (int)(((idx < 8 ? O0 : idx < 16 ? O1 : idx < 24 ? O2 : O3) >> (8 * (idx & 7))) & 31u) * 8 + xcd;
  const int turn = 7 - (mp >> 5), mpair = ((mp >> 3) & 3) * 4 + (mp & 3);
  const bool last = turn == NT, mover = mpair < 14;
  const int pi = mpair % 7, sub = (mpair / 7) * 2 + ((mp >> 2) & 1), pp = (rank + 1 + pi) & 7;
  const unsigned rdy_rest = RDY_OFF + (unsigned)(pp * 64 + 32 + ((lane & 7) == 1 ? 0 : (lane & 7))) * 128;
  auto turn_end = [&](int t) { return (unsigned)((int)((TURNS >> (8 * t)) & 255u) * UEND / 128 - (LPL - 1 - sub) * (STG - (STG - STG_LAST) * t / (NT - 1))); };
  auto entry = [&](unsigned it) { return it | (flag_idx(it & 31, (unsigned)pi, it >> 5) << 16); };
  auto build_table = [&]() {
    const unsigned uend = last ? 4096u : turn_end(turn);
#pragma unroll 1
    for (unsigned it = (unsigned)(tid - 576) + 32 * NPUSH; it < 1024; it += 192) {
      const unsigned u = (unsigned)stream_pos((int)(it >> 5), (int)(it & 31));
      lds_wr(lds0 + LDS_T + u * 4, entry(it) | (u >= uend ? 0x4000u : 0u));
    }
    asm volatile("s_waitcnt lgkmcnt(0)" ::: "memory");
    lds_inc(ctl + C_TAB);
  };
  unsigned call = 0;
  auto get_call = [&]() { unsigned c; asm volatile("v_add_u32 %0, 1, %1" : "=v"(c) : "v"(cv) : "memory"); call = rfl(c); };
  auto wg_barrier = [&]() { asm volatile("s_waitcnt lgkmcnt(0)\n\ts_barrier" ::: "memory"); };
  auto wait_flag = [&](unsigned off) {
    const unsigned long long t0 = rt();
#pragma unroll 1
    for (unsigned n = 1;; ++n) {
      const unsigned v = flag_ld(srdL, off);
      if (__all((int)(v - call) >= 0)) return true;
      if ((n & 15u) == 0u) {
        const unsigned tm = rfl(flag_ld(srdL, TMO_OFF));
        if (tm == call || rt() - t0 > DEADLINE) { flag_st1(srdL, TMO_OFF, call); return false; }
      }
      __builtin_amdgcn_s_sleep(1);
    }
  };

  if (wave < 8) {
    {
      __amdgpu_buffer_rsrc_t rs = __builtin_amdgcn_make_buffer_rsrc(wsl, 0, 2 * BUF_BYTES, 0x00020000);
      const unsigned base = (unsigned)rank * SRC_BYTES;
      char* pk[7] = {};
      const bool lazy = so >= 2 && turn != 0;
      if (lazy) { wg_barrier(); __builtin_amdgcn_s_sleep(STAGE_DELAY); }
      auto load_pk = [&]() {
        asm volatile("" ::: "memory");
        const int rot = (wave + b) % 7;
#pragma unroll
        for (int k = 0; k < 7; ++k) { const int r = k + rot; pk[k] = ktab[(rank + 1 + (r >= 7 ? r - 7 : r)) & 7]; }
        asm volatile("" : "+s"(pk[0]), "+s"(pk[1]), "+s"(pk[2]), "+s"(pk[3]), "+s"(pk[4]), "+s"(pk[5]), "+s"(pk[6]));
      };
      if (sg < 16) {
        const int r = 32 * wave + (lane & 31), j = lane >> 5;
        const char* src = P.x + (size_t)(256 * sg + ((r & ~15) | ((r & 3) << 2) | ((r >> 2) & 3))) * KSH + 64 * so + 16 * j;
        v4u v0 = *(const v4u*)src, v1 = *(const v4u*)(src + 32);
        if (!lazy) wg_barrier();
        if (so == 0) load_pk();
        unsigned ncv;
        asm volatile("v_not_b32 %0, %1" : "=v"(ncv) : "v"(cv) : "memory");
        const unsigned d0 = base + (ncv & 1u) * BUF_BYTES + (unsigned)sg * 4096 + (unsigned)r * 16;
        const unsigned d = d0 + plane_of(j + 2) * PLANE_BLK;
        __builtin_amdgcn_raw_buffer_store_b128(v0, rs, d0 + plane_of(j) * PLANE_BLK, 0, 17);
        __builtin_amdgcn_raw_buffer_store_b128(v1, rs, d, 0, 17);
        if (so == 0) {
#pragma unroll
          for (int k = 0; k < 7; ++k) {
            const v4i sp = make_srd(pk[k], 2 * BUF_BYTES);
            asm volatile("s_mov_b32 exec_lo, %c3\n\ts_nop 4\n\tbuffer_store_dwordx4 %0, %1, %2, 0 offen sc0 sc1\n\ts_mov_b64 exec, -1" ::"v"(v1), "v"(d), "s"(sp), "i"(NPUSH == 2 ? -1 : 0) : "memory");
            if (k >= PUSH_WIN - 1) wait_vm<PUSH_WIN - 1>();
          }
        }
        wait_vm<0>(); lds_inc(ctl + C_STG);
      } else if (wave < 4) {
        const int gw = sg - 16;
        const char* src = P.w + (size_t)(64 * so + 16 * wave) * DIM + 256 * gw + 4 * lane;
        unsigned r[16];
#pragma unroll
        for (int k = 0; k < 16; ++k) r[k] = *(const unsigned*)(src + (size_t)k * DIM);
        if (!lazy) wg_barrier();
        if (so == 0) load_pk();
        unsigned ncv;
        asm volatile("v_not_b32 %0, %1" : "=v"(ncv) : "v"(cv) : "memory");
        v4u o[4];
#pragma unroll
        for (int g = 0; g < 4; ++g) {
          unsigned t0 = __builtin_amdgcn_perm(r[4 * g + 1], r[4 * g], 0x05010400u), t1 = __builtin_amdgcn_perm(r[4 * g + 1], r[4 * g], 0x07030602u);
          unsigned t2 = __builtin_amdgcn_perm(r[4 * g + 3], r[4 * g + 2], 0x05010400u), t3 = __builtin_amdgcn_perm(r[4 * g + 3], r[4 * g + 2], 0x07030602u);
          o[0][g] = __builtin_amdgcn_perm(t2, t0, 0x05040100u); o[1][g] = __builtin_amdgcn_perm(t2, t0, 0x07060302u);
          o[2][g] = __builtin_amdgcn_perm(t3, t1, 0x05040100u); o[3][g] = __builtin_amdgcn_perm(t3, t1, 0x07060302u);
        }
        const unsigned d = base + (ncv & 1u) * BUF_BYTES + plane_of(wave) * PLANE_BLK + 65536u + (unsigned)gw * 4096 + (unsigned)(64 * (lane >> 4) + (lane & 15)) * 16;
#pragma unroll
        for (int jj = 0; jj < 4; ++jj) __builtin_amdgcn_raw_buffer_store_b128(o[jj], rs, d + jj * 256, 0, 17);
        if (so == 0 && wave >= 4 - NPUSH) {
#pragma unroll
          for (int k = 0; k < 7; ++k) {
            const v4i sp = make_srd(pk[k], 2 * BUF_BYTES);
            asm volatile("s_nop 4\n\tbuffer_store_dwordx4 %0, %4, %5, 0 offen sc0 sc1\n\tbuffer_store_dwordx4 %1, %4, %5, 0 offen offset:256 sc0 sc1\n\t"
                         "buffer_store_dwordx4 %2, %4, %5, 0 offen offset:512 sc0 sc1\n\tbuffer_store_dwordx4 %3, %4, %5, 0 offen offset:768 sc0 sc1"
                         ::"v"(o[0]), "v"(o[1]), "v"(o[2]), "v"(o[3]), "v"(d), "s"(sp) : "memory");
            if (k >= PUSH_WIN - 1) wait_vm<4 * (PUSH_WIN - 1)>();
          }
        }
        wait_vm<0>(); lds_inc(ctl + C_STG);
      } else if (!lazy) wg_barrier();
    }
    asm volatile("" ::: "v167");
    zero_acc<ACC0>();
    const int wr = wave >> 2, wc = wave & 3;
    unsigned pa = lds0 + LDS_A + (unsigned)(lane >> 4) * 2048 + (unsigned)(64 * wr + (lane & 15)) * 16;
    unsigned pb = lds0 + LDS_B + (unsigned)(lane >> 4) * 4096 + (unsigned)(64 * wc + (lane & 15)) * 16;
    const unsigned vw = ctl + (unsigned)wave * 16 + (unsigned)(lane & 3) * 4;
    const unsigned vd = (lane & 1) ? ctl + (unsigned)(wave ^ 4) * 16 + 8 : ctl + (unsigned)wave * 16 + 12;
    unsigned vo = ((unsigned)(256 * ti + 64 * wr) + (unsigned)(lane >> 4)) * (DIM * 4) + ((unsigned)(256 * tj + 64 * wc) + 4 * (unsigned)(lane & 15)) * 4;
    asm volatile("" : "+v"(vo));
    unsigned sc = rfl(__builtin_bit_cast(unsigned, (*P.sx) * (*P.sw)));
    const v4i ro = make_srd(P.out, DIM * DIM * 4);
    unsigned vcnt = 0, vf, ia = 0, ib = 0, cnt, da, db, spin = 1u << 24;
    unsigned long long ra;
    asm volatile(
      ".macro MM h, i, j\n"
      " v_mfma_i32_16x16x64_i8 v[40+\\h*64+\\i*16+\\j*4:43+\\h*64+\\i*16+\\j*4], v[24+\\i*4:27+\\i*4], v[8+\\j*4:11+\\j*4], v[40+\\h*64+\\i*16+\\j*4:43+\\h*64+\\i*16+\\j*4]\n"
      ".endm\n"
      ".macro STEP h, nb, pm, last\n"
      " s_waitcnt lgkmcnt(4+\\pm)\n"
      " MM \\h, 0, 0\n"
      " s_cmp_eq_u32 %[ia], %c[nh1]\n"
      " s_cselect_b32 %[da], %[wa], 0x2000\n"
      " MM \\h, 1, 0\n"
      " s_cselect_b32 %[ia], -1, %[ia]\n"
      " s_add_u32 %[ia], %[ia], 1\n"
      " .if \\pm\n s_waitcnt lgkmcnt(4)\n .endif\n"
      " MM \\h, 0, 1\n"
      " v_add_u32 %[pa], %[da], %[pa]\n"
      " MM \\h, 1, 1\n"
      " .if \\nb\n s_cmp_eq_u32 %[ib], %c[nb1]\n s_cselect_b32 %[db], %[wb], 0x4000\n .endif\n"
      " .if \\pm\n s_waitcnt lgkmcnt(3)\n .endif\n"
      " MM \\h, 0, 2\n"
      " v_add_u32 %[vcnt], 1, %[vcnt]\n"
      " .if \\nb\n s_cselect_b32 %[ib], -1, %[ib]\n s_add_u32 %[ib], %[ib], 1\n .endif\n"
      " MM \\h, 1, 2\n"
      " .if \\nb\n v_add_u32 %[pb], %[db], %[pb]\n .endif\n"
      " s_waitcnt lgkmcnt(2)\n"
      " MM \\h, 0, 3\n"
      " .if \\last == 0\n v_cmp_le_u32 vcc, %[vf], %[vcnt]\n .endif\n"
      " MM \\h, 1, 3\n"
      " .if \\last == 0\n"
      " s_getpc_b64 %[ra]\n"
      " s_and_b32 %[da], vcc_lo, 3\n"
      " s_cbranch_scc1 Lslow%=\n"
      " ds_read_b128 v[24:27], %[pa]\n"
      " ds_read_b128 v[28:31], %[pa] offset:256\n"
      " .endif\n"
      " s_waitcnt lgkmcnt(2-2*\\last)\n"
      " MM \\h, 2, 0\n"
      " .if \\last == 0\n"
      " ds_write_b32 %[vd], %[vcnt]\n"
      " ds_read_b32 %[vf], %[vw]\n"
      " .endif\n"
      " MM \\h, 3, 0\n"
      " .if \\nb\n ds_read_b128 v[8:11], %[pb]\n .endif\n"
      " MM \\h, 2, 1\n"
      " .if \\last == 0\n"
      " s_bitcmp1_b32 vcc_lo, 2\n"
      " s_setprio 0\n"
      " s_cbranch_scc1 1f\n"
      " s_setprio 1\n"
      "1:\n"
      " .endif\n"
      " MM \\h, 3, 1\n"
      " .if \\nb\n ds_read_b128 v[12:15], %[pb] offset:256\n s_sub_u32 %[cnt], %[cnt], 1\n .endif\n"
      " MM \\h, 2, 2\n"
      " MM \\h, 3, 2\n"
      " .if \\nb\n ds_read_b128 v[16:19], %[pb] offset:512\n s_cmp_lg_u32 %[cnt], 0\n .endif\n"
      " MM \\h, 2, 3\n"
      " MM \\h, 3, 3\n"
      " .if \\nb\n ds_read_b128 v[20:23], %[pb] offset:768\n .endif\n"
      " .if \\last == 0\n"
      " ds_read_b128 v[32:35], %[pa] offset:512\n"
      " ds_read_b128 v[36:39], %[pa] offset:768\n"
      " .endif\n"
      ".endm\n"
      " s_mov_b64 vcc, -1\n"
      " s_getpc_b64 %[ra]\n"
      " s_and_b32 %[da], vcc_lo, 3\n"
      " s_cbranch_scc1 Lslow%=\n"
      " ds_read_b128 v[24:27], %[pa]\n ds_read_b128 v[28:31], %[pa] offset:256\n"
      " ds_read_b128 v[8:11], %[pb]\n ds_read_b128 v[12:15], %[pb] offset:256\n"
      " ds_read_b128 v[16:19], %[pb] offset:512\n ds_read_b128 v[20:23], %[pb] offset:768\n"
      " ds_read_b128 v[32:35], %[pa] offset:512\n ds_read_b128 v[36:39], %[pa] offset:768\n"
      " s_mov_b32 %[cnt], 63\n"
      "Lfull%=:\n"
      " STEP 0, 0, 1, 0\n"
      " STEP 1, 1, 0, 0\n"
      " s_cbranch_scc1 Lfull%=\n"
      " STEP 0, 0, 1, 0\n"
      " STEP 1, 0, 0, 1\n"
      " s_branch Lend%=\n"
      "Lslow%=:\n"
      " ds_read_b32 %[vf], %[vw]\n"
      " s_sub_u32 %[spin], %[spin], 1\n"
      " s_waitcnt lgkmcnt(0)\n"
      " ds_write_b32 %[vd], %[vcnt]\n"
      " v_cmp_le_u32 vcc, %[vf], %[vcnt]\n"
      " s_cbranch_scc1 Lgive%=\n"
      " s_and_b32 %[da], vcc_lo, 3\n"
      " s_cbranch_scc0 Lret%=\n"
      " s_sleep 1\n"
      " s_branch Lslow%=\n"
      "Lgive%=:\n"
      " s_mov_b32 %[spin], 0\n"
      " s_mov_b32 %[sc], 0x7fc00000\n"
      " s_mov_b64 vcc, 0\n"
      "Lret%=:\n"
      " s_setpc_b64 %[ra]\n"
      "Lend%=:\n"
      " s_nop 7\n s_nop 1\n"
      ".purgem STEP\n.purgem MM\n"
      : [pa] "+&v"(pa), [pb] "+&v"(pb), [vcnt] "+&v"(vcnt), [vf] "=&v"(vf), [ia] "+&s"(ia), [ib] "+&s"(ib), [cnt] "=&s"(cnt), [da] "=&s"(da),
        [db] "=&s"(db), [spin] "+&s"(spin), [sc] "+&s"(sc), [ra] "=&s"(ra)
      : [vw] "v"(vw), [vd] "v"(vd), [nh1] "i"(NH - 1), [nb1] "i"(NB - 1), [wa] "s"(-(NH - 1) * 8192), [wb] "s"(-(NB - 1) * 16384)
      : "v8", "v9", "v10", "v11", "v12", "v13", "v14", "v15", "v16", "v17", "v18", "v19", "v20", "v21", "v22", "v23", "v24", "v25", "v26", "v27",
        "v28", "v29", "v30", "v31", "v32", "v33", "v34", "v35", "v36", "v37", "v38", "v39", "vcc", "scc", "memory");
    {
      unsigned v, k = 0;
      do v = lds_rd(ctl + ((lane & 1) ? C_FIN : C_ABORT)); while (__builtin_amdgcn_readlane(v, 1) == 0u && ++k < (1u << 24));
      sc = (__builtin_amdgcn_readlane(v, 0) != 0u || __builtin_amdgcn_readlane(v, 1) == 0u) ? 0x7fc00000u : sc;
    }
    store_rows<0, 32>(sc, vo, ro);
  } else if (wave == 8) {
    wg_barrier();
    lds_wr(lds0 + LDS_T + (NTK + (lane & 15)) * 4, 0xffffffffu);
    get_call();
    if (mover) {
      __builtin_amdgcn_s_setprio(3);
      const v4i srdR = make_srd(ktab[pp], 2 * BUF_BYTES);
      const unsigned v16 = (unsigned)lane * 16;
      const unsigned sbase = (call & 1u) * BUF_BYTES + (unsigned)pp * SRC_BYTES;
      unsigned tk_off = TK_OFF + (unsigned)pi * 128;
      const unsigned tk_dead = DUM_OFF + (unsigned)b * 256 + 64;
      const unsigned go_out = DUM_OFF + (unsigned)(last ? mp : mp - 32) * 256 + 192, go_in = DUM_OFF + (unsigned)mp * 256 + 192;
      flag_st1(srdL, tk_dead, (call << 12) + 0x800u);
      unsigned ev;
      bool ok;
      if (turn == 0) {
        lds_wr(ctl + C_MVD, 1u);
        asm volatile("s_mov_b64 exec, 1\n\tv_mov_b32 v58, %2\n\tv_mov_b32 v59, %3\n\ts_nop 4\n\tbuffer_atomic_umax_x2 v[58:59], off, %0, %1\n\ts_mov_b64 exec, -1"
                     ::"s"(srdL), "s"(tk_off), "s"(call << 12), "s"(call >> 20) : "memory");
        unsigned long long fi = 0;
        static_for<0, LPL>([&](auto S) { constexpr unsigned long long F = first_items(decltype(S)::value); fi = sub == decltype(S)::value ? F : fi; });
        const unsigned it = (unsigned)(fi >> (8 * ((lane & 7) < NGR ? (lane & 7) : NGR))) & 255u, g = it & 31u;
        ev = entry(32u * NPUSH + it);
        asm volatile("" : "+v"(ev));
        ok = wait_flag(RDY_OFF + ((unsigned)pp * 64 + g) * 128);
      } else {
        const unsigned long long t0 = rt();
        const unsigned est = turn_end(turn - 1) * 67u / 10u + EST_ADD;
        while ((unsigned)(rt() - t0) < est) __builtin_amdgcn_s_sleep(16);
        ok = wait_flag(rdy_rest) && wait_flag(go_in);
        lds_wr(ctl + C_MVD + (unsigned)(lane & 1) * 4, (lane & 1) ? 0u : 1u);
        unsigned x0;
        asm volatile("s_mov_b64 exec, 1\n\tv_mov_b32 v58, %3\n\tv_mov_b32 v59, 0\n\ts_nop 4\n\tbuffer_atomic_add_x2 v[58:59], off, %1, %2 sc0\n\ts_mov_b64 exec, -1\n\t"
                     "s_waitcnt vmcnt(0)\n\tv_readfirstlane_b32 %0, v58\n\ts_nop 1" : "=s"(x0) : "s"(srdL), "s"(ok ? tk_off : tk_dead), "s"(NGR + 1) : "memory");
        const unsigned xe = x0 - (call << 12) + NST0 + (unsigned)(lane & 7);
        ok = lds_wait(ctl + C_TAB, 3) && ok;
        ev = lds_rd(lds0 + LDS_T + (xe < NTK ? xe : NTK) * 4);
      }
      if (ok) {
        auto relay = [&](auto NGC) {
        constexpr int NG = decltype(NGC)::value;
        const unsigned fdum = DUM_OFF + (unsigned)b * 256;
        unsigned os[NG], en[NG], pf[NG];
        unsigned e;
        unsigned rdy = turn != 0 ? 4096u : (4u + NPUSH) * 32;
        auto foff = [&](unsigned x) { return x == 0xffffffffu ? fdum : PF_OFF + (x >> 16) * 128; };
        auto offs = [&](unsigned x) {
          if (__builtin_expect((x & 0x4000u) != 0u && tk_off != tk_dead, 0)) {
            tk_off = tk_dead;
            flag_st1(srdL, go_out, call);
          }
          return sbase + (x == 0xffffffffu ? 32u * NPUSH : x & 1023u) * 4096;
        };
        auto gate = [&]() {
          if (__builtin_expect((e & 1023u) >= rdy && e != 0xffffffffu, 0)) {
            const unsigned lvl = (e & 1023u) < (28 + NPUSH) * 32 ? 2u : 3u;
            unsigned r, spins = 0;
            while ((r = rfl(lds_rd(ctl + C_RDY))) < lvl && ++spins < (1u << 24)) __builtin_amdgcn_s_sleep(2);
            if (r < lvl || r > 3u) { e = 0xffffffffu; flag_st1(srdL, TMO_OFF, call); }
            else rdy = lvl == 2u ? (28u + NPUSH) * 32 : 4096u;
          }
        };
        auto swap = [&](auto G, auto W, auto WK) {
          constexpr int g = decltype(G)::value;
          const unsigned ol = offs(e), nf = pf[g], o = os[g];
          pf[g] = en[g]; en[g] = foff(e); os[g] = ol;
          e = mv_swap<g, decltype(W)::value, decltype(WK)::value>(srdL, srdR, tk_off, o, ol, v16, lds0 + LDS_T + NST0 * 4, call, nf);
        };
        static_for<0, NG>([&](auto G) {
          constexpr int g = decltype(G)::value;
          const unsigned x = __builtin_amdgcn_readlane(ev, g);
          pf[g] = fdum; en[g] = foff(x); os[g] = offs(x);
          mv_load<g>(srdL, srdR, tk_off, os[g], v16);
        });
        e = __builtin_amdgcn_readlane(ev, NG);
        if (turn == 0) {
          if (!lds_wait(ctl + C_TAB, 3) || !lds_wait(ctl + C_RDY, 1) || rfl(lds_rd(ctl + C_RDY)) > 3u) { wait_vm<0>(); flag_st1(srdL, TMO_OFF, call); return; }
        }
        static_for<0, NG>([&](auto G) {
          constexpr int g = decltype(G)::value;
          gate();
          swap(G, std::integral_constant<int, 4 + (NG - 1 - g) * 5 + g * 10>{}, std::integral_constant<int, 1>{});
        });
        int stop = -1;
#pragma unroll 1
        while (stop < 0) {
          static_for<0, NG>([&](auto G) {
            constexpr int g = decltype(G)::value;
            if (stop < 0) {
              gate();
              if (e == 0xffffffffu) stop = g;
              else {
                swap(G, std::integral_constant<int, 8 + (NG - 1) * 10>{}, std::integral_constant<int, 0>{});
              }
            }
          });
        }
        if (tk_off != tk_dead) flag_st1(srdL, go_out, call);
        static_for<0, NG>([&](auto G) {
          constexpr int g = decltype(G)::value;
          if (stop == g) static_for<0, NG>([&](auto J) {
            constexpr int j = decltype(J)::value, h = (g + j) % NG;
            mv_store<h, 7 + (NG - 1 - j) * 10 + j * 5>(srdL, os[h], v16, call, pf[h]);
          });
        });
        wait_vm<0>();
        static_for<0, NG>([&](auto G) { flag_st1(srdL, en[decltype(G)::value], call); });
        };
        relay(std::integral_constant<int, NGR>{});
        lds_wr(ctl + C_MVD, 2u);
      }
      __builtin_amdgcn_s_setprio(0);
    }
    lds_wr(ctl + C_MVD, 2u);
    if (rfl(lds_rd(ctl + C_ABORT)) != 0u && lds_wait(ctl + C_FIN, 1)) lds_wr(ctl + C_AV, 64u);
  } else if (wave == 9) {
    __builtin_amdgcn_s_setprio(2);
    lds_wr(ctl + lane * 4, 0u);
    wg_barrier();
    const bool direct = so == 1;
    if (!direct) build_table();
    get_call();
    {
      char* pb = wsl;
#pragma unroll
      for (int k = 0; k < 8; ++k) { char* t = P.ws[k]; pb = (lane == k || (lane >= 32 && (lane & 7) == k)) ? t : pb; }
      const unsigned off = lane < 8 ? RDY_OFF + (unsigned)(rank * 64 + (direct ? sg : 32 + so)) * 128
                         : lane < 16 ? PF_OFF + flag_idx((unsigned)sg, 7u, plane_of(lane - 8)) * 128
                         : lane < 32 ? CNT_OFF + (unsigned)b * 128
                         : PF_OFF + flag_idx((unsigned)sg, (unsigned)(rank - lane - 1) & 7u, (unsigned)lane >> 3 & 1u) * 128;
      unsigned* dst = (unsigned*)(pb + off);
      const bool on = lane < 8 ? (direct && lane != rank) : lane < 32 ? (lane < 12 || lane == 16) : (so == 0 && lane < 32 + 8 * NPUSH && (lane & 7) != rank);
      asm volatile("" : "+v"(dst));
      const bool staged = lds_wait(ctl + C_STG, sg < 16 ? 8 : 4);
      if (!staged) flag_st1(srdL, TMO_OFF, call);
      if (on && (staged || lane == 16)) __hip_atomic_store(dst, call, __ATOMIC_RELAXED, __HIP_MEMORY_SCOPE_SYSTEM);
      if (direct) build_table();
      else if (staged) {
        unsigned tk = 0;
        if (lane == 0) tk = __hip_atomic_fetch_add((unsigned*)(wsl + AGG_OFF + (unsigned)so * 128), 1u, __ATOMIC_RELAXED, __HIP_MEMORY_SCOPE_AGENT);
        if (rfl(tk) + 1u == call * 32u && lane < 8 && lane != rank) __hip_atomic_store(dst, call, __ATOMIC_RELAXED, __HIP_MEMORY_SCOPE_SYSTEM);
      }
    }
    const unsigned si = (unsigned)lane >> 3, ro = ((unsigned)lane >> 1) & 3u;
    const unsigned fbase = PF_OFF + flag_idx((lane & 1) ? 16u + tj : (unsigned)ti, si, ro) * 128;
    const unsigned pbase = (call & 1u) * BUF_BYTES + (((unsigned)rank + 1 + si) & 7u) * SRC_BYTES + ro * PLANE_BLK;
    const unsigned lateN = POLL_LATE - 7 * (unsigned)(((LAGS >> (4 * (idx >> 2))) & 15u) > ((LAGW >> (4 * (idx & 3))) & 15u) ? ((LAGS >> (4 * (idx >> 2))) & 15u) : ((LAGW >> (4 * (idx & 3))) & 15u));
    unsigned q = 0, cnt = 0, npoll = 0, win = 4;
    bool mvdone = false;
    const unsigned long long t0 = rt();
#pragma unroll 1
    while (cnt < 256 || !mvdone) {
      if (!mvdone) { const unsigned m = rfl(lds_rd(ctl + C_MVD)); mvdone = m == 2u; if (m == 1u) { lds_wr(ctl + C_AV, 0u); win = 4; __builtin_amdgcn_s_sleep(4); continue; } }
      unsigned v = call - 1;
      if (ro < win && q + ro < 32) {
        asm volatile("buffer_load_dword %0, %1, %2, 0 offen sc0 sc1\n\ts_waitcnt vmcnt(0)" : "=v"(v) : "v"(fbase + q * 128), "s"(srdL) : "memory");
      }
      const unsigned long long hm = __ballot((int)(v - call) >= 0);
      const unsigned adv = (unsigned)__builtin_ctz(~((unsigned)(hm >> (8 * si)) & 255u)) >> 1;
      const bool emit = !(lane & 1) && ro < adv;
      const unsigned long long em = __ballot(emit);
      if (emit) lds_wr(lds0 + LDS_L + (cnt + __builtin_amdgcn_mbcnt_hi((unsigned)(em >> 32), __builtin_amdgcn_mbcnt_lo((unsigned)em, 0u))) * 4, pbase + q * PLANE_BLK);
      q += adv;
      cnt += (unsigned)__builtin_popcountll(em);
      const unsigned lim = cnt < 160 ? (unsigned)(rt() - t0) / LOCAL_GAP + 1 : 255u;
      lds_wr(ctl + C_AV, (cnt >> 2) < lim ? cnt >> 2 : lim);
      const bool more = __ballot(adv == win && q < 32) != 0ull;
      win = more ? 4u : 2u;
      if (!more) { if (cnt < lateN) __builtin_amdgcn_s_sleep(POLL_SLEEP); else __builtin_amdgcn_s_sleep(POLL_SLEEP2); }
      if (em == 0ull && (++npoll & 15u) == 0u) {
        const unsigned tm = rfl(flag_ld(srdL, TMO_OFF));
        if (tm == call || rt() - t0 > DEADLINE) {
          flag_st1(srdL, TMO_OFF, call);
          lds_wr(ctl + C_ABORT, 1u);
#pragma unroll 1
          for (unsigned k = (unsigned)lane; k < 256; k += 64) lds_wr(lds0 + LDS_L + k * 4, 0u);
          asm volatile("s_waitcnt lgkmcnt(0)" ::: "memory");
          break;
        }
      }
    }
    lds_wr(ctl + C_AV, 64u);
    if (rfl(flag_ld(srdL, TMO_OFF)) == call) lds_wr(ctl + C_ABORT, 1u);
    asm volatile("s_waitcnt lgkmcnt(0)" ::: "memory");
    lds_wr(ctl + C_FIN, 1u);
  } else {
    wg_barrier();
    build_table();
    __builtin_amdgcn_s_setprio(3);
    get_call();
    if (turn == 0 && wave == 10) {
      bool ok = wait_flag(RDY_OFF + (unsigned)(pp * 64 + (lane & 31)) * 128);
      if (ok) { lds_wr(ctl + C_RDY, 1u); ok = wait_flag(RDY_OFF + (unsigned)(pp * 64 + 32 + ((lane & 7) < 2 ? 2 : (lane & 7))) * 128); }
      if (ok) { lds_wr(ctl + C_RDY, 2u); ok = wait_flag(rdy_rest); }
      lds_wr(ctl + C_RDY, ok ? 3u : 9u);
    }
    auto loader = [&](auto IA) {
      constexpr bool isA = decltype(IA)::value;
      constexpr unsigned RINGB = isA ? NH * 8192 : NB * 16384;
      const unsigned v16 = (unsigned)lane * 16 + (isA ? (unsigned)ti : 16u + tj) * 4096, v16b = v16 + 2048u;
      unsigned vchk = lane < 8 ? ctl + lane * 16 + 12 : lane < 16 ? ctl + C_AV : lds0 + LDS_L + (unsigned)(lane - 16) * 4;
      const unsigned vadv = lane < 16 ? 0u : 16u;
      const unsigned vfull = lane < 8 ? ctl + (unsigned)lane * 16 + (isA ? 0 : 4) : ctl + (isA ? 512 : 768) + lane * 4;
      const unsigned ring0 = lds0 + (isA ? LDS_A : LDS_B);
      unsigned ring = ring0, n = 0, nops = 0, spin = 1u << 25;
      const v4i srd = srdL;
      int vneed = lane < 8 ? (isA ? 2 - NH : 1 - 2 * NB) : lane < 16 ? 1 : (int)0x80000000u;
      const int vinc = lane < 8 ? 2 : lane < 16 ? 1 : 0;
      wait_vm<0>();
#define L_PUB \
      " s_getreg_b32 %[c], hwreg(HW_REG_IB_STS)\n" \
      " s_bfe_u32 %[x], %[c], 0x20016\n" \
      " s_and_b32 %[c], %[c], 15\n" \
      " s_lshl4_add_u32 %[c], %[x], %[c]\n" \
      " s_sub_u32 %[x], %[nops], %[c]\n" \
      " s_lshr_b32 %[x], %[x], 3\n" \
      " .if %c[isa] == 0\n s_and_b32 %[x], %[x], -2\n .endif\n" \
      " v_mov_b32 %[vl], %[x]\n" \
      " ds_write_b32 %[vfull], %[vl]\n"
#define L_D2(v, o, k) \
      " buffer_load_dwordx4 %[" v "], %[srd], %[" o "] offen offset:" k " sc0 sc1 lds\n" \
      " buffer_load_dwordx4 %[" v "], %[srd], %[" o "] offen offset:" k "+1024 sc0 sc1 lds\n"
#define L_A(v, o) L_D2(v, o, "0") " s_add_u32 m0, m0, 0x800\n s_nop 0\n"
#define L_B(o) L_D2("v16", o, "0") L_D2("v16", o, "2048") " s_add_u32 m0, m0, 0x1000\n s_nop 0\n"
      unsigned v, vl, o0, o1, o2, o3, c, x, j, keep;
      asm volatile(
        " s_mov_b32 %[km0], m0\n"
        "Lpoll%=:\n"
        " ds_read_b32 %[v], %[vchk]\n"
        L_PUB
        " s_mov_b32 %[j], 16\n"
        " s_sub_u32 %[spin], %[spin], 1\n"
        " s_cbranch_scc1 Lout%=\n"
        " s_waitcnt lgkmcnt(1)\n"
        "Lchk%=:\n"
        " v_cmp_lt_i32 vcc, %[v], %[vneed]\n"
        " v_readlane_b32 %[o0], %[v], %[j]\n"
        " s_add_u32 %[j], %[j], 1\n"
        " v_readlane_b32 %[o1], %[v], %[j]\n"
        " s_cmp_gt_u32 %[c], 47\n"
        " s_cbranch_scc1 Lpoll%=\n"
        " s_cbranch_vccnz Lpoll%=\n"
        " s_add_u32 %[j], %[j], 1\n"
        " v_readlane_b32 %[o2], %[v], %[j]\n"
        " s_add_u32 %[j], %[j], 1\n"
        " v_readlane_b32 %[o3], %[v], %[j]\n"
        " s_mov_b32 m0, %[ring]\n"
        " s_add_u32 %[n], %[n], 1\n"
        " .if %c[isa]\n"
        L_A("v16", "o0") L_A("v16", "o1") L_A("v16", "o2") L_A("v16", "o3")
        " s_add_u32 %[nops], %[nops], 8\n"
        L_PUB
        L_A("v16b", "o0") L_A("v16b", "o1") L_A("v16b", "o2") L_A("v16b", "o3")
        " s_add_u32 %[nops], %[nops], 8\n"
        " .else\n"
        L_B("o0") L_B("o1") L_B("o2") L_B("o3")
        " s_add_u32 %[nops], %[nops], 16\n"
        " .endif\n"
        " v_add_u32 %[vneed], %[vinc], %[vneed]\n"
        " v_add_u32 %[vchk], %[vadv], %[vchk]\n"
        " s_cmp_eq_u32 m0, %[rend]\n"
        " s_cselect_b32 %[ring], %[ring0], m0\n"
        " s_add_u32 %[j], %[j], 1\n"
        L_PUB
        " s_cmp_lt_u32 %[n], 64\n"
        " s_cbranch_scc1 Lchk%=\n"
        "Lout%=:\n"
        L_PUB
        " s_cmp_lg_u32 %[c], 0\n"
        " s_cbranch_scc1 Lout%=\n"
        " s_mov_b32 m0, %[km0]\n"
        : [v] "=&v"(v), [vneed] "+&v"(vneed), [vchk] "+&v"(vchk), [vl] "=&v"(vl), [o0] "=&s"(o0), [o1] "=&s"(o1), [o2] "=&s"(o2), [o3] "=&s"(o3), [c] "=&s"(c),
          [x] "=&s"(x), [j] "=&s"(j), [km0] "=&s"(keep), [n] "+&s"(n), [nops] "+&s"(nops), [ring] "+&s"(ring), [spin] "+&s"(spin)
        : [v16] "v"(v16), [v16b] "v"(v16b), [vfull] "v"(vfull), [vinc] "v"(vinc), [vadv] "v"(vadv),
          [srd] "s"(srd), [ring0] "s"(ring0), [rend] "s"(ring0 + RINGB), [isa] "i"(isA ? 1 : 0)
        : "vcc", "scc", "memory");
      if (n != 64u) lds_wr(ctl + C_ABORT, 1u);
      asm volatile("s_waitcnt lgkmcnt(0)" ::: "memory");
      lds_wr(vfull, 255u);
    };
    if (wave == 10) loader(std::true_type{}); else loader(std::false_type{});
  }
}

extern "C" void kernel_launch(int rank, int nranks, ncclComm_t comm, hipStream_t stream,
                              void* const* d_in, const int* in_sizes, int n_in,
                              void* d_out, int out_size, const int* in_dtypes, int out_dtype,
                              void* const* d_out_all, void* const* d_ws_all, size_t ws_bytes) {
  (void)comm; (void)in_sizes; (void)n_in; (void)out_size; (void)in_dtypes; (void)out_dtype; (void)d_out_all;
  if (nranks != NR || (unsigned)rank >= (unsigned)NR || ws_bytes < WS_USED) { fprintf(stderr, "unexpected configuration\n"); return; }
  Params p;
  const size_t woff = (ws_bytes - WS_USED) & ~(size_t)0xfffff;
  for (int i = 0; i < NR; ++i) p.ws[i] = (char*)d_ws_all[i] + woff;
  p.x = (const char*)d_in[0]; p.w = (const char*)d_in[1];
  p.sx = (const float*)d_in[2]; p.sw = (const float*)d_in[3];
  p.out = (float*)d_out; p.wsl = p.ws[rank]; p.rank = rank;
  static bool attr_set[NR] = {};
  if (!attr_set[rank]) {
    HIP_CHECK(hipFuncSetAttribute((const void*)fused_gemm, hipFuncAttributeMaxDynamicSharedMemorySize, (int)LDS_BYTES));
    attr_set[rank] = true;
  }
  fused_gemm<<<256, 768, LDS_BYTES, stream>>>(p);
  HIP_CHECK(hipGetLastError());
}
```
